```python
import numpy as np

kernel_name = "gpu_ag_gemm_m8192_k8192_n8192_bf16_relu"


ABSMAX_RTOL = 0.008

COMPUTE_DTYPE = "bfloat16"
EPILOGUE = "relu"
IN_QUANT = None
OUT_QUANT = None
IN_DTYPES = {"x": "bfloat16", "w_mat": "bfloat16"}
WORLD_SIZE = 8

def _to_bf16(x):
    u = np.ascontiguousarray(x, dtype=np.float32).view(np.uint32)
    return ((u + 0x7FFF + ((u >> 16) & 1)) & 0xFFFF0000).view(np.float32)

def setup_inputs():

    x = _to_bf16(np.random.randn(8192, 8192).astype(np.float32))
    w_mat = _to_bf16((np.random.randn(8192, 8192) / np.sqrt(8192)).astype(np.float32))
    return {"x": x, "w_mat": w_mat}

def _epilogue(y):
    return np.maximum(y, 0.0).astype(np.float32)

def reference(x, w_mat):
    y = (x @ w_mat).astype(np.float32)
    return _epilogue(y)
```

```cpp
#include <hip/hip_runtime.h>
#include <rccl/rccl.h>
#include <cstdio>

namespace ag {

typedef unsigned u32;
typedef unsigned long long u64;
typedef unsigned u32x4 __attribute__((ext_vector_type(4)));
typedef unsigned u32x2 __attribute__((ext_vector_type(2)));
typedef unsigned short u16x2 __attribute__((ext_vector_type(2)));
typedef short bf16x8 __attribute__((ext_vector_type(8)));
typedef float f32x4 __attribute__((ext_vector_type(4)));
typedef __amdgpu_buffer_rsrc_t rsrc_t;
#define LAS __attribute__((address_space(3)))
#define GAS __attribute__((address_space(1)))

#define AG_AU 21
#define AG_PW 3
#define AG_FORCE_RAW 0
#define AG_GEMM_AUX 17

constexpr int NR = 8;
constexpr int AU = AG_AU;
constexpr int AWAY = 32 - AU;
constexpr int NRT = 22, NEARLY = 11;
constexpr int UPT = 6;
constexpr int NKC = 64, NKS = 256;
constexpr int EARLY_UNITS = AU + 4 * AWAY, LATE_UNITS = 3 * AU;
static_assert(EARLY_UNITS <= NEARLY * UPT && LATE_UNITS <= (NRT - NEARLY) * UPT, "row split does not fit the tiles");
constexpr int NOUT = AWAY + 8;

constexpr int NCOMM = 80, NGEMM = 176, NWG = NCOMM + NGEMM;
constexpr int PW = AG_PW;
constexpr int NWL = NCOMM * PW / 7;
constexpr int NPK = NCOMM * (8 - PW);
constexpr int NUNITS = 4096;

constexpr u32 A_STEP = 12288, B_STEP = 16384;
constexpr u32 A_TILE = A_STEP * NKS, B_TILE = B_STEP * NKS;
constexpr u32 OFF_XU = 0;
constexpr u32 OFF_WU = OFF_XU + NRT * A_TILE;
constexpr u32 FRAG_BYTES = OFF_WU + 8 * B_TILE;
constexpr u32 SLOT = 8192, PK_BYTES = NUNITS * SLOT;
constexpr u32 OFF_PK = FRAG_BYTES;
constexpr u32 OFF_G = OFF_PK + 2 * PK_BYTES;
constexpr u32 G_BYTES = 2 * NR * NUNITS * 8;
constexpr u32 OFF_CNT = OFF_G + G_BYTES;
constexpr u32 OFF_ARR = OFF_CNT + 8192;
constexpr u32 OFF_TMO = OFF_ARR + 1024;
constexpr u32 OFF_CALL = OFF_TMO + 128;
constexpr u32 OFF_DBG = OFF_CALL + NWG * 128;
constexpr u32 WS_NEED = OFF_DBG + NWG * 8 * 64;

constexpr int A_SLOTS = 6, B_SLOTS = 5;
constexpr int A_RING = A_SLOTS * A_STEP, LDS_BYTES = A_RING + B_SLOTS * B_STEP;
constexpr int WAVE_SCRATCH = 4608;
static_assert(8 * WAVE_SCRATCH <= LDS_BYTES, "scratch");

constexpr u64 DEADLINE = 50000000ull;
constexpr int AUX_SYS = 17;

struct Params {
  const char* x;
  const char* w;
  char* ws[NR];
  char* out[NR];
  int rank;
  int pad;
};

#define STAMP(i) do {} while (0)

__device__ __forceinline__ rsrc_t make_rsrc(const void* p, u32 bytes) {
  const u64 a = (u64)p;
  const u32 lo = __builtin_amdgcn_readfirstlane((u32)a), hi = __builtin_amdgcn_readfirstlane((u32)(a >> 32));
  return __builtin_amdgcn_make_buffer_rsrc((void*)(((u64)hi << 32) | lo), 0, bytes, 0x00020000);
}
__device__ __forceinline__ char* pick(char* const (&t)[NR], u32 i) {
  char* p = t[0];
#pragma unroll
  for (u32 j = 1; j < NR; ++j) p = (i == j) ? t[j] : p;
  return p;
}
__device__ __forceinline__ u32 ld_agent(const char* p) { return __hip_atomic_load((GAS u32*)p, __ATOMIC_RELAXED, __HIP_MEMORY_SCOPE_AGENT); }
__device__ __forceinline__ u32 ld_sys(const char* p) { return __hip_atomic_load((GAS u32*)p, __ATOMIC_RELAXED, __HIP_MEMORY_SCOPE_SYSTEM); }
__device__ __forceinline__ void drain_vm() { asm volatile("s_waitcnt vmcnt(0)" ::: "memory"); }

__device__ __forceinline__ bool give_up(char* ws, u32 call, u64 t0) {
  const u32 t = __builtin_amdgcn_readfirstlane(ld_agent(ws + OFF_TMO));
  if (t == call) return true;
  if (__builtin_amdgcn_s_memrealtime() - t0 > DEADLINE) {
    __hip_atomic_store((GAS u32*)(ws + OFF_TMO), call, __ATOMIC_RELAXED, __HIP_MEMORY_SCOPE_AGENT);
    return true;
  }
  return false;
}

__device__ __forceinline__ u32 wave_sum(u32 v) {
  v += __builtin_amdgcn_update_dpp(0u, v, 0x111, 0xf, 0xf, true);
  v += __builtin_amdgcn_update_dpp(0u, v, 0x112, 0xf, 0xf, true);
  v += __builtin_amdgcn_update_dpp(0u, v, 0x114, 0xf, 0xf, true);
  v += __builtin_amdgcn_update_dpp(0u, v, 0x118, 0xf, 0xf, true);
  return __builtin_amdgcn_readlane(v, 15) + __builtin_amdgcn_readlane(v, 31) + __builtin_amdgcn_readlane(v, 47) + __builtin_amdgcn_readlane(v, 63);
}
__device__ __forceinline__ u32 lane_rank(u64 m) { return __builtin_amdgcn_mbcnt_hi((u32)(m >> 32), __builtin_amdgcn_mbcnt_lo((u32)m, 0u)); }

__device__ __forceinline__ u32 quad_swz(u32 rq) { return (0x78u >> (2 * rq)) & 3u; }

struct Dest { u32 base, s1, s0, cnt; };
__device__ __forceinline__ Dest dest_x(u32 ux, u32 kc) {
  const u32 tile = ux / UPT;
  return Dest{OFF_XU + tile * A_TILE + 4 * kc * A_STEP + (ux - tile * UPT) * 2048u, 1024u, A_STEP, tile * 64 + kc};
}
__device__ __forceinline__ Dest dest_w(u32 ct, u32 kc, u32 ks4, u32 half) {
  return Dest{OFF_WU + ct * B_TILE + (4 * kc + ks4) * B_STEP + half * 8192u, 4096u, 1024u, (NRT + ct) * 64 + kc};
}
__device__ __forceinline__ void store_frags(const u32x4 (&v)[8], rsrc_t rf, const Dest& d, u32 lane) {
#pragma unroll
  for (int f = 0; f < 8; ++f) __builtin_amdgcn_raw_buffer_store_b128(v[f], rf, lane * 16, d.base + (f >> 2) * d.s1 + (f & 3) * d.s0, AUX_SYS);
}

template <bool TO_LIST>
__device__ __forceinline__ void walk_escapes(u32 m0, u32 m1, LAS unsigned char* img, LAS unsigned char* list, u32 lane) {
  u64 m = ((u64)m1 << 32) | m0;
  u32 off = 0;
  for (;;) {
    const u64 bal = __builtin_amdgcn_ballot_w64(m != 0);
    if (bal == 0) break;
    if (m != 0) {
      const u32 idx = off + lane_rank(bal);
      const u32 p = __builtin_ctzll(m);
      m &= m - 1;
      const u32 j = ((p >> 5) << 3) | (p & 7), b = (p >> 3) & 3;
      LAS unsigned char* cell = img + (j >> 2) * 1024 + lane * 16 + (j & 3) * 4 + b;
      if (TO_LIST) list[idx] = *cell; else *cell = list[idx];
    }
    off += __builtin_popcountll(bal);
  }
}

__device__ __forceinline__ u32 pack_unit(const u32x4 (&v)[8], rsrc_t rp, u32 soff, LAS unsigned char* scr, u32 lane) {
  u32 M[16], E[16], esum = 0;
#pragma unroll
  for (int j = 0; j < 16; ++j) {
    const u32 d0 = v[j >> 1][(j & 1) * 2], d1 = v[j >> 1][(j & 1) * 2 + 1];
    const u32 r0 = __builtin_amdgcn_alignbit(d0, d0, 31), r1 = __builtin_amdgcn_alignbit(d1, d1, 31);
    M[j] = __builtin_amdgcn_perm(r1, r0, 0x06040200u);
    E[j] = __builtin_amdgcn_perm(r1, r0, 0x07050301u);
    esum = __builtin_amdgcn_sad_u8(E[j], 0u, esum);
  }
  u32 base = (wave_sum(esum) + 11059u) >> 12;
  base = base < 7u ? 7u : (base > 255u ? 255u : base);
  const u16x2 b2 = {(unsigned short)base, (unsigned short)base}, seven = {7, 7};
  u32 P[3][2] = {{0, 0}, {0, 0}, {0, 0}};
#pragma unroll
  for (int j = 0; j < 16; ++j) {
    const u16x2 ev = __builtin_bit_cast(u16x2, E[j] & 0x00ff00ffu), od = __builtin_bit_cast(u16x2, E[j]) >> (u16x2){8, 8};
    const u32 ce = __builtin_bit_cast(u32, __builtin_elementwise_min((u16x2)(b2 - ev), seven));
    const u32 co = __builtin_bit_cast(u32, __builtin_elementwise_min((u16x2)(b2 - od), seven));
    const u32 c = ce | (co << 8);
    const int s = j & 7, wd = j >> 3;
    P[0][wd] |= (c & 0x01010101u) << s;
    P[1][wd] |= ((c >> 1) & 0x01010101u) << s;
    P[2][wd] |= ((c >> 2) & 0x01010101u) << s;
  }
  const u32 m0 = P[0][0] & P[1][0] & P[2][0], m1 = P[0][1] & P[1][1] & P[2][1];
  const u32 n = wave_sum(__builtin_popcount(m0) + __builtin_popcount(m1));
  if (n > 256u || AG_FORCE_RAW) {
#pragma unroll
    for (int f = 0; f < 8; ++f) __builtin_amdgcn_raw_buffer_store_b128(v[f], rp, lane * 16, soff + f * 1024, AUX_SYS);
    return 1u << 17;
  }
#pragma unroll
  for (int i = 0; i < 4; ++i)
    __builtin_amdgcn_raw_buffer_store_b128((u32x4){M[4 * i], M[4 * i + 1], M[4 * i + 2], M[4 * i + 3]}, rp, lane * 16, soff + i * 1024, AUX_SYS);
  __builtin_amdgcn_raw_buffer_store_b128((u32x4){P[0][0], P[0][1], P[1][0], P[1][1]}, rp, lane * 16, soff + 4096, AUX_SYS);
  __builtin_amdgcn_raw_buffer_store_b64((u32x2){P[2][0], P[2][1]}, rp, lane * 8, soff + 5120, AUX_SYS);
  if (n) {
#pragma unroll
    for (int i = 0; i < 4; ++i) *(LAS u32x4*)(scr + i * 1024 + lane * 16) = (u32x4){E[4 * i], E[4 * i + 1], E[4 * i + 2], E[4 * i + 3]};
    walk_escapes<true>(m0, m1, scr, scr + 4096, lane);
    const u32 lw = *(LAS u32*)(scr + 4096 + lane * 4);
    if (lane * 4 < n) __builtin_amdgcn_raw_buffer_store_b32(lw, rp, lane * 4, soff + 5632, AUX_SYS);
  }
  return base | (n << 8);
}

__device__ __forceinline__ void unpack_unit(u32x4 (&v)[8], rsrc_t rp, u32 soff, u32 info, LAS unsigned char* scr, u32 lane) {
  if (info >> 17) {
#pragma unroll
    for (int f = 0; f < 8; ++f) v[f] = __builtin_amdgcn_raw_buffer_load_b128(rp, lane * 16, soff + f * 1024, AUX_SYS);
    return;
  }
  const u32 base = info & 255u, n = (info >> 8) & 511u;
  const u32x4 p01 = __builtin_amdgcn_raw_buffer_load_b128(rp, lane * 16, soff + 4096, AUX_SYS);
  const u32x2 p2 = __builtin_amdgcn_raw_buffer_load_b64(rp, lane * 8, soff + 5120, AUX_SYS);
  u32 lw = 0;
  if (lane * 4 < n) lw = __builtin_amdgcn_raw_buffer_load_b32(rp, lane * 4, soff + 5632, AUX_SYS);
  u32x4 mq[4];
#pragma unroll
  for (int i = 0; i < 4; ++i) mq[i] = __builtin_amdgcn_raw_buffer_load_b128(rp, lane * 16, soff + i * 1024, AUX_SYS);
  const u32 P0[2] = {p01.x, p01.y}, P1[2] = {p01.z, p01.w}, P2[2] = {p2.x, p2.y};
  const u32 b4 = base * 0x01010101u;
  u32x4 eq[4];
#pragma unroll
  for (int j = 0; j < 16; ++j) {
    const int s = j & 7, wd = j >> 3;
    const u32 c = ((P0[wd] >> s) & 0x01010101u) | (((P1[wd] >> s) & 0x01010101u) << 1) | (((P2[wd] >> s) & 0x01010101u) << 2);
    eq[j >> 2][j & 3] = b4 - c;
  }
  if (n) {
#pragma unroll
    for (int i = 0; i < 4; ++i) *(LAS u32x4*)(scr + i * 1024 + lane * 16) = eq[i];
    *(LAS u32*)(scr + 4096 + lane * 4) = lw;
    walk_escapes<false>(P0[0] & P1[0] & P2[0], P0[1] & P1[1] & P2[1], scr, scr + 4096, lane);
#pragma unroll
    for (int i = 0; i < 4; ++i) eq[i] = *(LAS u32x4*)(scr + i * 1024 + lane * 16);
  }
#pragma unroll
  for (int j = 0; j < 16; ++j) {
    const u32 e = eq[j >> 2][j & 3], m = mq[j >> 2][j & 3];
    const u32 r0 = __builtin_amdgcn_perm(e, m, 0x05010400u), r1 = __builtin_amdgcn_perm(e, m, 0x07030602u);
    v[j >> 1][(j & 1) * 2] = __builtin_amdgcn_alignbit(r0, r0, 1);
    v[j >> 1][(j & 1) * 2 + 1] = __builtin_amdgcn_alignbit(r1, r1, 1);
  }
}

__device__ __forceinline__ void packer(const Params& P, char* ws, u32 call, u32 pidx, LAS unsigned char* scr, u32 lane, u32 wid) {
  const u32 r = P.rank, g4 = r & 4u, par = call & 1u;
  const rsrc_t rx = make_rsrc(P.x, 1024u * 8192u * 2u), rw = make_rsrc(P.w, 8192u * 1024u * 2u);
  const rsrc_t rf = make_rsrc(ws, FRAG_BYTES), rp = make_rsrc(ws + OFF_PK + par * PK_BYTES, PK_BYTES);
  const u32 goff = OFF_G + (par * NR + r) * NUNITS * 8;
  char* const g_in = pick(P.ws, g4 | ((r + 1 + lane) & 3u)) + goff;
  char* const g_out = pick(P.ws, (g4 ^ 4u) | (lane & 3u)) + goff;
  const u32 kq = (lane & 3u) ^ quad_swz(lane >> 4);
  const u32 vx = (lane >> 2) * 16384u + kq * 16u, vw = kq * 8u * 2048u + (lane >> 2) * 16u;
  for (u32 q = pidx; q < NUNITS; q += NPK) {
    const u32 kc = q >> 6, t = q & 63u;
    u32x4 v[8];
    Dest d;
    bool own, send;
    char* gp;
    if (t < 32u) {
#pragma unroll
      for (int f = 0; f < 8; ++f) v[f] = __builtin_amdgcn_raw_buffer_load_b128(rx, vx, (32u * t + 16u * (f >> 2)) * 16384u + kc * 256u + (f & 3) * 64u, 0);
      own = t < AU;
      d = dest_x(t, kc);
      send = own ? lane < 3u : lane < 4u;
      gp = own ? g_in : g_out;
    } else {
      const u32 ks4 = (t - 32u) >> 3, cg = t & 7u;
      u32x4 s[8];
#pragma unroll
      for (int i = 0; i < 8; ++i) s[i] = __builtin_amdgcn_raw_buffer_load_b128(rw, vw, (128u * kc + 32u * ks4 + i) * 2048u + cg * 256u, 0);
#pragma unroll
      for (int nf = 0; nf < 8; ++nf)
#pragma unroll
        for (int tt = 0; tt < 4; ++tt) v[nf][tt] = __builtin_amdgcn_perm(s[2 * tt + 1][nf >> 1], s[2 * tt][nf >> 1], (nf & 1) ? 0x07060302u : 0x05040100u);
      own = true;
      d = dest_w(cg >> 1, kc, ks4, cg & 1u);
      send = lane == (cg >> 1);
      gp = g_out;
    }
    const u32 info = pack_unit(v, rp, q * SLOT, scr, lane);
    if (own) store_frags(v, rf, d, lane);
    drain_vm();
    if (send) __hip_atomic_store((GAS u64*)(gp + q * 8u), ((u64)call << 32) | info, __ATOMIC_RELAXED, __HIP_MEMORY_SCOPE_SYSTEM);
    if (own && lane == 0) __hip_atomic_fetch_add((GAS u32*)(ws + OFF_CNT) + d.cnt, 1u, __ATOMIC_RELAXED, __HIP_MEMORY_SCOPE_AGENT);
    if (q == pidx) STAMP(1);
  }
  STAMP(2);
}

__device__ __forceinline__ void puller(const Params& P, char* ws, u32 call, u32 gw, LAS unsigned char* scr, u32 lane, u32 wid) {
  const u32 r = P.rank, g4 = r & 4u, par = call & 1u;
  const u32 li = gw % 7u, idx = gw / 7u;
  const bool ing = li < 3u;
  const u32 src = ing ? (g4 | ((r + 1 + li) & 3u)) : ((g4 ^ 4u) | (li - 3u));
  const rsrc_t rf = make_rsrc(ws, FRAG_BYTES), rp = make_rsrc(pick(P.ws, src) + OFF_PK + par * PK_BYTES, PK_BYTES);
  char* const gbase = ws + OFF_G + (par * NR + src) * NUNITS * 8;
  const u32 total = ing ? NKC * AU : NKC * NOUT;
  for (u32 i0 = idx; i0 < total; i0 += NWL * 64) {
    u32 i = i0 + NWL * lane;
    i = i < total ? i : i0;
    u32 ql;
    if (ing) {
      const u32 kc = i / AU;
      ql = kc * 64u + (i - kc * AU);
    } else {
      const u32 kc = i / NOUT, t = i - kc * NOUT, tw = t - AWAY;
      ql = kc * 64u + (t < AWAY ? AU + t : 32u + (tw >> 1) * 8u + 2u * (r & 3u) + (tw & 1u));
    }
    u64 gr = __hip_atomic_load((GAS u64*)(gbase + ql * 8u), __ATOMIC_RELAXED, __HIP_MEMORY_SCOPE_SYSTEM);
    for (u32 t = 0; t < 64u && i0 + NWL * t < total; ++t) {
      if (__builtin_amdgcn_readlane((u32)(gr >> 32), t) != call) {
        const u64 t0 = __builtin_amdgcn_s_memrealtime();
        for (;;) {
          gr = __hip_atomic_load((GAS u64*)(gbase + ql * 8u), __ATOMIC_RELAXED, __HIP_MEMORY_SCOPE_SYSTEM);
          if (__builtin_amdgcn_readlane((u32)(gr >> 32), t) == call) break;
          if (give_up(ws, call, t0)) return;
          __builtin_amdgcn_s_sleep(2);
        }
      }
      asm volatile("" ::: "memory");
      if (t == 0 && i0 == idx) STAMP(1);
      const u32 q = __builtin_amdgcn_readlane(ql, t), info = __builtin_amdgcn_readlane((u32)gr, t);
      const u32 kc = q >> 6, tq = q & 63u;
      u32x4 v[8];
      unpack_unit(v, rp, q * SLOT, info, scr, lane);
      Dest d;
      if (tq < 32u) d = dest_x(ing ? NEARLY * UPT + li * AU + tq : AU + (li - 3u) * AWAY + (tq - AU), kc);
      else d = dest_w(4u + (li - 3u), kc, (tq - 32u) >> 3, tq & 1u);
      store_frags(v, rf, d, lane);
      drain_vm();
      if (lane == 0) __hip_atomic_fetch_add((GAS u32*)(ws + OFF_CNT) + d.cnt, 1u, __ATOMIC_RELAXED, __HIP_MEMORY_SCOPE_AGENT);
    }
  }
  STAMP(2);
}

__device__ __forceinline__ void gate(int c, int& ready, const char* cnt, u32 target, char* ws, u32 call) {
  if (c < ready) return;
  const u64 t0 = __builtin_amdgcn_s_memrealtime();
  for (u32 spins = 0;; ++spins) {
    const u64 m = __builtin_amdgcn_ballot_w64((int)(ld_agent(cnt) - target) >= 0);
    ready = ~m ? __builtin_ctzll(~m) : 64;
    if (c < ready) break;
    if ((spins & 7u) == 7u && give_up(ws, call, t0)) { ready = 64; break; }
    __builtin_amdgcn_s_sleep(3);
  }
  asm volatile("" ::: "memory");
}

template <int G>
__device__ __forceinline__ void gemm_loop(f32x4 (&acc)[3][8], LAS char* lds, rsrc_t rs, const char* cnt, u32 target, char* ws, u32 call, u32 wr, u32 lane, u32 wid) {
  constexpr int NP = G ? 4 : 3, NS = G ? B_SLOTS : A_SLOTS;
  constexpr u32 STEP = G ? B_STEP : A_STEP;
  const u32 voff = wr * NP * 1024u + lane * 16u;
  LAS char* const ring = lds + (G ? A_RING : 0) + wr * NP * 1024u;
  const u32 img = ((lane & 15u) * 4u + ((lane >> 4) ^ quad_swz((lane >> 2) & 3u))) * 16u;
  LAS char* const ra = lds + wr * 3072u + img;
  LAS char* const rb = lds + A_RING + G * 8192u + img;
#define AG_PIECE(ks, slot, imm) __builtin_amdgcn_raw_ptr_buffer_load_lds(rs, ring + (slot) * STEP, 16, voff, (ks) * STEP, imm, AG_GEMM_AUX)
#define AG_ISSUE(ks, slot) do { AG_PIECE(ks, slot, 0); AG_PIECE(ks, slot, 1024); AG_PIECE(ks, slot, 2048); if (G) AG_PIECE(ks, slot, 3072); } while (0)
#define AG_WAIT_BAR() do { if (G) asm volatile("s_waitcnt vmcnt(12)\n\ts_barrier" ::: "memory"); else asm volatile("s_waitcnt vmcnt(9)\n\ts_barrier" ::: "memory"); } while (0)
#define AG_BAR() asm volatile("s_barrier" ::: "memory")
  int ready = 0;
  gate(0, ready, cnt, target, ws, call);
  STAMP(1);
#pragma unroll
  for (int s = 0; s < 4; ++s) AG_ISSUE(s, s);
  AG_WAIT_BAR();
  if (G) AG_BAR();
  u32 sa = 0, sb = 0, sl = 4;
  for (int kc = 0; kc < NKC; ++kc) {
#pragma unroll
    for (int s = 0; s < 4; ++s) {
      bf16x8 a[3], b[8];
#pragma unroll
      for (int m = 0; m < 3; ++m) a[m] = *(const LAS bf16x8*)(ra + sa * A_STEP + m * 1024);
#pragma unroll
      for (int n = 0; n < 8; ++n) b[n] = *(const LAS bf16x8*)(rb + sb * B_STEP + n * 1024);
      if (s == 0 && kc + 1 < NKC) gate(kc + 1, ready, cnt, target, ws, call);
      AG_ISSUE(kc * 4 + s + 4, sl);
      AG_WAIT_BAR();
      __builtin_amdgcn_sched_barrier(0);
      __builtin_amdgcn_s_setprio(1);
#pragma unroll
      for (int n = 0; n < 8; ++n)
#pragma unroll
        for (int m = 0; m < 3; ++m) acc[m][n] = __builtin_amdgcn_mfma_f32_16x16x32_bf16(a[m], b[n], acc[m][n], 0, 0, 0);
      __builtin_amdgcn_s_setprio(0);
      __builtin_amdgcn_sched_barrier(0);
      AG_BAR();
      sa = sa + 1 == A_SLOTS ? 0 : sa + 1;
      sb = sb + 1 == B_SLOTS ? 0 : sb + 1;
      sl = sl + 1 == NS ? 0 : sl + 1;
    }
  }
  if (!G) AG_BAR();
  drain_vm();
#undef AG_ISSUE
#undef AG_PIECE
#undef AG_WAIT_BAR
#undef AG_BAR
}

__device__ __forceinline__ u32 cvt_pk(float lo, float hi) {
  u32 r;
  asm volatile("v_cvt_pk_bf16_f32 %0, %1, %2" : "=v"(r) : "v"(lo), "v"(hi));
  return r;
}
__device__ __forceinline__ float relu(float v) { return v < 0.f ? 0.f : v; }

__device__ __forceinline__ void gemm(const Params& P, char* ws, u32 call, u32 b, LAS char* lds, u32 lane, u32 wid) {
  const u32 r = P.rank, g4 = r & 4u;
  const u32 xcd = b & 7u, slot = b >> 3;
  const u32 late = xcd & 1u, cp = xcd >> 1;
  const u32 rt = late * NEARLY + (slot >> 1), ct = cp + 4u * (slot & 1u);
  const u32 grp = wid >> 2, wr = wid & 3u;
  const int left = (late ? LATE_UNITS : EARLY_UNITS) - (int)(slot >> 1) * UPT;
  const u32 nun = left < 0 ? 0u : (left > UPT ? (u32)UPT : (u32)left);
  f32x4 acc[3][8];
#pragma unroll
  for (int m = 0; m < 3; ++m)
#pragma unroll
    for (int n = 0; n < 8; ++n) acc[m][n] = (f32x4){0.f, 0.f, 0.f, 0.f};
  u32 ta = call * nun, tb = call * 8u;
  if (grp == 0) gemm_loop<0>(acc, lds, make_rsrc(ws + OFF_XU + rt * A_TILE, A_TILE), ws + OFF_CNT + (rt * 64 + lane) * 4, ta, ws, call, wr, lane, wid);
  else gemm_loop<1>(acc, lds, make_rsrc(ws + OFF_WU + ct * B_TILE, B_TILE), ws + OFF_CNT + ((NRT + ct) * 64 + lane) * 4, tb, ws, call, wr, lane, wid);
  STAMP(2);
  const bool bad = __builtin_amdgcn_readfirstlane(ld_agent(ws + OFF_TMO)) == call;
  const u32 owner = ct < 4u ? r : ((g4 ^ 4u) | (ct - 4u));
  const u32 col0 = (ct < 4u ? ct : (r & 3u)) * 256u + grp * 128u;
  const rsrc_t ro = make_rsrc(pick(P.out, owner), 8192u * 1024u * 2u);
  const u32 vo = (lane >> 4) * 4u * 2048u + (lane & 15u) * 16u;
#pragma unroll
  for (int m = 0; m < 3; ++m) {
    const u32 fr = wr * 3u + m;
    const u32 uc = (slot >> 1) * UPT + (fr >> 1);
    if ((fr >> 1) >= nun) continue;
    u32 src, su;
    if (late) { const u32 j = uc / AU; src = g4 | ((r + 1 + j) & 3u); su = uc - j * AU; }
    else if (uc < AU) { src = r; su = uc; }
    else { const u32 j = (uc - AU) / AWAY; src = (g4 ^ 4u) | j; su = AU + (uc - AU) - j * AWAY; }
    const u32 row0 = src * 1024u + su * 32u + (fr & 1u) * 16u;
#pragma unroll
    for (int e = 0; e < 4; ++e) {
      u32x4 o;
#pragma unroll
      for (int n = 0; n < 4; ++n) o[n] = cvt_pk(relu(acc[m][2 * n][e]), relu(acc[m][2 * n + 1][e]));
      if (bad) o = (u32x4){0x7fc07fc0u, 0x7fc07fc0u, 0x7fc07fc0u, 0x7fc07fc0u};
      __builtin_amdgcn_raw_buffer_store_b128(o, ro, vo, (row0 + e) * 2048u + col0 * 2u, AUX_SYS);
    }
  }
  drain_vm();
  if (ct >= 4u && lane == 0) __hip_atomic_fetch_add((GAS u32*)(pick(P.ws, owner) + OFF_ARR + r * 128u), 1u, __ATOMIC_RELAXED, __HIP_MEMORY_SCOPE_SYSTEM);
  STAMP(3);
}

__device__ __forceinline__ void wait_arrivals(const Params& P, char* ws, u32 call, u32 lane, u32 wid) {
  const u32 r = P.rank;
  const char* p = ws + OFF_ARR + (((r & 4u) ^ 4u) | (lane & 3u)) * 128u;
  const u64 t0 = __builtin_amdgcn_s_memrealtime();
  for (;;) {
    if (__builtin_amdgcn_ballot_w64((int)(ld_sys(p) - call * (NRT * 8u)) < 0) == 0) break;
    if (give_up(ws, call, t0)) {
      if (lane == 0) *(GAS unsigned short*)(pick(P.out, r) + (size_t)r * 1024u * 2048u) = 0x7fc0;
      break;
    }
    __builtin_amdgcn_s_sleep(4);
  }
  STAMP(3);
}

__global__ __launch_bounds__(512, 2) void ag_gemm_kernel(Params P) {
  extern __shared__ __attribute__((aligned(16))) char smem[];
  LAS char* lds = (LAS char*)smem;
  const u32 lane = threadIdx.x & 63u, wid = __builtin_amdgcn_readfirstlane(threadIdx.x >> 6);
  char* const ws = pick(P.ws, P.rank);
  STAMP(0);
  u32 bid = blockIdx.x;
  asm volatile("" : "+s"(bid));
  char* const cptr = ws + OFF_CALL + bid * 128u + wid * 4u;
  const u32 call = __builtin_amdgcn_readfirstlane(ld_agent(cptr)) + 1u;
  __hip_atomic_store((GAS u32*)cptr, call, __ATOMIC_RELAXED, __HIP_MEMORY_SCOPE_AGENT);
  if (bid >= NCOMM) {
    gemm(P, ws, call, bid - NCOMM, lds, lane, wid);
  } else {
    LAS unsigned char* scr = (LAS unsigned char*)lds + wid * WAVE_SCRATCH;
    if (wid < PW) {
      const u32 gw = bid * PW + wid;
      if (gw < NWL * 7) {
        __builtin_amdgcn_s_setprio(2);
        puller(P, ws, call, gw, scr, lane, wid);
      }
    } else {
      const u32 pidx = bid * (8 - PW) + (wid - PW);
      packer(P, ws, call, pidx, scr, lane, wid);
      if (pidx == 0) wait_arrivals(P, ws, call, lane, wid);
    }
  }
}

}

extern "C" void kernel_launch(int rank, int nranks, ncclComm_t comm, hipStream_t stream,
                              void* const* d_in, const int* in_sizes, int n_in,
                              void* d_out, int out_size, const int* in_dtypes, int out_dtype,
                              void* const* d_out_all, void* const* d_ws_all, size_t ws_bytes) {
  (void)comm; (void)in_dtypes; (void)out_dtype; (void)d_out;
  if (nranks != ag::NR || n_in != 2 || (unsigned)rank >= (unsigned)ag::NR || in_sizes[0] != 1024 * 8192 || in_sizes[1] != 8192 * 1024 ||
      out_size != 8192 * 1024 || ws_bytes < ag::WS_NEED) {
    fprintf(stderr, "ag_gemm: unexpected problem shape\n");
    return;
  }
  ag::Params p;
  p.x = (const char*)d_in[0];
  p.w = (const char*)d_in[1];
  for (int i = 0; i < ag::NR; ++i) { p.ws[i] = (char*)d_ws_all[i]; p.out[i] = (char*)d_out_all[i]; }
  p.rank = rank;
  p.pad = 0;
  hipError_t e = hipFuncSetAttribute((const void*)ag::ag_gemm_kernel, hipFuncAttributeMaxDynamicSharedMemorySize, ag::LDS_BYTES);
  if (e != hipSuccess) { fprintf(stderr, "ag_gemm: %s\n", hipGetErrorString(e)); return; }
  hipLaunchKernelGGL(ag::ag_gemm_kernel, dim3(ag::NWG), dim3(512), ag::LDS_BYTES, stream, p);
  e = hipGetLastError();
  if (e != hipSuccess) fprintf(stderr, "ag_gemm launch: %s\n", hipGetErrorString(e));
}
```

```cpp
#include <hip/hip_runtime.h>
#include <rccl/rccl.h>
#include <cstdio>

namespace ag {

typedef unsigned u32;
typedef unsigned long long u64;
typedef unsigned u32x4 __attribute__((ext_vector_type(4)));
typedef unsigned u32x2 __attribute__((ext_vector_type(2)));
typedef unsigned short u16x2 __attribute__((ext_vector_type(2)));
typedef short bf16x8 __attribute__((ext_vector_type(8)));
typedef float f32x4 __attribute__((ext_vector_type(4)));
typedef __amdgpu_buffer_rsrc_t rsrc_t;
#define LAS __attribute__((address_space(3)))
#define GAS __attribute__((address_space(1)))

#define AG_AU 21
#define AG_NCL 7
#define AG_TOK 2
#define AG_OUT_THROTTLE 0
#define AG_LOOP 2
#define AG_RPAR 0
#define AG_V0 1
#define AG_VD 6
#define AG_FORCE_RAW 0
#define AG_GEMM_AUX 17

constexpr int NR = 8;
constexpr int AU = AG_AU;
constexpr int AWAY = 32 - AU;
constexpr int NRT = 22, NEARLY = 11;
constexpr int UPT = 6;
constexpr int NKC = 64, NKS = 256;
constexpr int EARLY_UNITS = AU + 4 * AWAY, LATE_UNITS = 3 * AU;
static_assert(EARLY_UNITS <= NEARLY * UPT && LATE_UNITS <= (NRT - NEARLY) * UPT, "row split does not fit the tiles");
constexpr int NOUT = AWAY + 8;

constexpr int NCOMM = 80, NGEMM = 176, NWG = NCOMM + NGEMM;
constexpr int NCL = AG_NCL;
constexpr int NPKWG = NCOMM - 7 * NCL;
constexpr int NPK = NPKWG * 8;
constexpr u32 TOK = AG_TOK;
constexpr int NUNITS = 4096;

constexpr u32 A_STEP = 12288, B_STEP = 16384;
constexpr u32 A_TILE = A_STEP * NKS, B_TILE = B_STEP * NKS;
constexpr u32 OFF_XU = 0;
constexpr u32 OFF_WU = OFF_XU + NRT * A_TILE;
constexpr u32 FRAG_BYTES = OFF_WU + 8 * B_TILE;
constexpr u32 SLOT = 8192, PK_BYTES = NUNITS * SLOT;
constexpr u32 OFF_PK = FRAG_BYTES;
constexpr u32 OFF_G = OFF_PK + 2 * PK_BYTES;
constexpr u32 G_BYTES = 2 * NR * NUNITS * 8;
constexpr u32 OFF_CNT = OFF_G + G_BYTES;
constexpr u32 OFF_ARR = OFF_CNT + 8192;
constexpr u32 OFF_TKT = OFF_ARR + 1024;
constexpr u32 OFF_TMO = OFF_TKT + 1024;
constexpr u32 OFF_CALL = OFF_TMO + 128;
constexpr u32 OFF_DBG = OFF_CALL + NWG * 128;
constexpr u32 OFF_DBG2 = OFF_DBG + NWG * 8 * 64;
constexpr u32 WS_NEED = OFF_DBG2 + 65536;

constexpr int A_SLOTS = 6, B_SLOTS = 5;
constexpr int A_RING = A_SLOTS * A_STEP, LDS_BYTES = A_RING + B_SLOTS * B_STEP;
constexpr int WAVE_SCRATCH = 4608;
static_assert(8 * WAVE_SCRATCH + 64 <= LDS_BYTES, "scratch");

constexpr u64 DEADLINE = 50000000ull;
constexpr int AUX_SYS = 17;

struct Params {
  const char* x;
  const char* w;
  char* ws[NR];
  char* out[NR];
  int rank;
  int pad;
};

#define STAMP(i) do {} while (0)
#define STAMPC(i) do {} while (0)

__device__ __forceinline__ rsrc_t make_rsrc(const void* p, u32 bytes) {
  const u64 a = (u64)p;
  const u32 lo = __builtin_amdgcn_readfirstlane((u32)a), hi = __builtin_amdgcn_readfirstlane((u32)(a >> 32));
  return __builtin_amdgcn_make_buffer_rsrc((void*)(((u64)hi << 32) | lo), 0, bytes, 0x00020000);
}
__device__ __forceinline__ char* pick(char* const (&t)[NR], u32 i) {
  char* p = t[0];
#pragma unroll
  for (u32 j = 1; j < NR; ++j) p = (i == j) ? t[j] : p;
  return p;
}
__device__ __forceinline__ u32 ld_agent(const char* p) { return __hip_atomic_load((GAS u32*)p, __ATOMIC_RELAXED, __HIP_MEMORY_SCOPE_AGENT); }
__device__ __forceinline__ u32 ld_sys(const char* p) { return __hip_atomic_load((GAS u32*)p, __ATOMIC_RELAXED, __HIP_MEMORY_SCOPE_SYSTEM); }
__device__ __forceinline__ void drain_vm() { asm volatile("s_waitcnt vmcnt(0)" ::: "memory"); }

__device__ __forceinline__ bool give_up(char* ws, u32 call, u64 t0) {
  const u32 t = __builtin_amdgcn_readfirstlane(ld_agent(ws + OFF_TMO));
  if (t == call) return true;
  if (__builtin_amdgcn_s_memrealtime() - t0 > DEADLINE) {
    __hip_atomic_store((GAS u32*)(ws + OFF_TMO), call, __ATOMIC_RELAXED, __HIP_MEMORY_SCOPE_AGENT);
    return true;
  }
  return false;
}

__device__ __forceinline__ u32 wave_sum(u32 v) {
  v += __builtin_amdgcn_update_dpp(0u, v, 0x111, 0xf, 0xf, true);
  v += __builtin_amdgcn_update_dpp(0u, v, 0x112, 0xf, 0xf, true);
  v += __builtin_amdgcn_update_dpp(0u, v, 0x114, 0xf, 0xf, true);
  v += __builtin_amdgcn_update_dpp(0u, v, 0x118, 0xf, 0xf, true);
  return __builtin_amdgcn_readlane(v, 15) + __builtin_amdgcn_readlane(v, 31) + __builtin_amdgcn_readlane(v, 47) + __builtin_amdgcn_readlane(v, 63);
}
__device__ __forceinline__ u32 lane_rank(u64 m) { return __builtin_amdgcn_mbcnt_hi((u32)(m >> 32), __builtin_amdgcn_mbcnt_lo((u32)m, 0u)); }

__device__ __forceinline__ u32 quad_swz(u32 rq) { return (0x78u >> (2 * rq)) & 3u; }

struct Dest { u32 base, s1, s0, cnt; };
__device__ __forceinline__ Dest dest_x(u32 ux, u32 kc) {
  const u32 tile = ux / UPT;
  return Dest{OFF_XU + tile * A_TILE + 4 * kc * A_STEP + (ux - tile * UPT) * 2048u, 1024u, A_STEP, tile * 64 + kc};
}
__device__ __forceinline__ Dest dest_w(u32 ct, u32 kc, u32 ks4, u32 half) {
  return Dest{OFF_WU + ct * B_TILE + (4 * kc + ks4) * B_STEP + half * 8192u, 4096u, 1024u, (NRT + ct) * 64 + kc};
}
__device__ __forceinline__ void store_frags(const u32x4 (&v)[8], rsrc_t rf, const Dest& d, u32 lane) {
#pragma unroll
  for (int f = 0; f < 8; ++f) __builtin_amdgcn_raw_buffer_store_b128(v[f], rf, lane * 16, d.base + (f >> 2) * d.s1 + (f & 3) * d.s0, AUX_SYS);
}

template <bool TO_LIST>
__device__ __forceinline__ void walk_escapes(u32 m0, u32 m1, LAS unsigned char* img, LAS unsigned char* list, u32 lane) {
  u64 m = ((u64)m1 << 32) | m0;
  u32 off = 0;
  for (;;) {
    const u64 bal = __builtin_amdgcn_ballot_w64(m != 0);
    if (bal == 0) break;
    if (m != 0) {
      const u32 idx = off + lane_rank(bal);
      const u32 p = __builtin_ctzll(m);
      m &= m - 1;
      const u32 j = ((p >> 5) << 3) | (p & 7), b = (p >> 3) & 3;
      LAS unsigned char* cell = img + (j >> 2) * 1024 + lane * 16 + (j & 3) * 4 + b;
      if (TO_LIST) list[idx] = *cell; else *cell = list[idx];
    }
    off += __builtin_popcountll(bal);
  }
}

__device__ __forceinline__ u32 pack_unit(const u32x4 (&v)[8], rsrc_t rp, u32 soff, LAS unsigned char* scr, u32 lane) {
  u32 M[16], E[16], esum = 0;
#pragma unroll
  for (int j = 0; j < 16; ++j) {
    const u32 d0 = v[j >> 1][(j & 1) * 2], d1 = v[j >> 1][(j & 1) * 2 + 1];
    const u32 r0 = __builtin_amdgcn_alignbit(d0, d0, 31), r1 = __builtin_amdgcn_alignbit(d1, d1, 31);
    M[j] = __builtin_amdgcn_perm(r1, r0, 0x06040200u);
    E[j] = __builtin_amdgcn_perm(r1, r0, 0x07050301u);
    esum = __builtin_amdgcn_sad_u8(E[j], 0u, esum);
  }
  u32 base = (wave_sum(esum) + 11059u) >> 12;
  base = base < 7u ? 7u : (base > 255u ? 255u : base);
  const u16x2 b2 = {(unsigned short)base, (unsigned short)base}, seven = {7, 7};
  u32 P[3][2] = {{0, 0}, {0, 0}, {0, 0}};
#pragma unroll
  for (int j = 0; j < 16; ++j) {
    const u16x2 ev = __builtin_bit_cast(u16x2, E[j] & 0x00ff00ffu), od = __builtin_bit_cast(u16x2, E[j]) >> (u16x2){8, 8};
    const u32 ce = __builtin_bit_cast(u32, __builtin_elementwise_min((u16x2)(b2 - ev), seven));
    const u32 co = __builtin_bit_cast(u32, __builtin_elementwise_min((u16x2)(b2 - od), seven));
    const u32 c = ce | (co << 8);
    const int s = j & 7, wd = j >> 3;
    P[0][wd] |= (c & 0x01010101u) << s;
    P[1][wd] |= ((c >> 1) & 0x01010101u) << s;
    P[2][wd] |= ((c >> 2) & 0x01010101u) << s;
  }
  const u32 m0 = P[0][0] & P[1][0] & P[2][0], m1 = P[0][1] & P[1][1] & P[2][1];
  const u32 n = wave_sum(__builtin_popcount(m0) + __builtin_popcount(m1));
  if (n > 256u || AG_FORCE_RAW) {
#pragma unroll
    for (int f = 0; f < 8; ++f) __builtin_amdgcn_raw_buffer_store_b128(v[f], rp, lane * 16, soff + f * 1024, AUX_SYS);
    return 1u << 17;
  }
#pragma unroll
  for (int i = 0; i < 4; ++i)
    __builtin_amdgcn_raw_buffer_store_b128((u32x4){M[4 * i], M[4 * i + 1], M[4 * i + 2], M[4 * i + 3]}, rp, lane * 16, soff + i * 1024, AUX_SYS);
  __builtin_amdgcn_raw_buffer_store_b128((u32x4){P[0][0], P[0][1], P[1][0], P[1][1]}, rp, lane * 16, soff + 4096, AUX_SYS);
  __builtin_amdgcn_raw_buffer_store_b64((u32x2){P[2][0], P[2][1]}, rp, lane * 8, soff + 5120, AUX_SYS);
  if (n) {
#pragma unroll
    for (int i = 0; i < 4; ++i) *(LAS u32x4*)(scr + i * 1024 + lane * 16) = (u32x4){E[4 * i], E[4 * i + 1], E[4 * i + 2], E[4 * i + 3]};
    walk_escapes<true>(m0, m1, scr, scr + 4096, lane);
    const u32 lw = *(LAS u32*)(scr + 4096 + lane * 4);
    if (lane * 4 < n) __builtin_amdgcn_raw_buffer_store_b32(lw, rp, lane * 4, soff + 5632, AUX_SYS);
  }
  return base | (n << 8);
}

struct Packed { u32x4 mq[4], p01; u32x2 p2; u32 lw; };

__device__ __forceinline__ void fetch_packed(Packed& k, rsrc_t rp, u32 soff, u32 n, u32 lane) {
  k.p01 = __builtin_amdgcn_raw_buffer_load_b128(rp, lane * 16, soff + 4096, AUX_SYS);
  k.p2 = __builtin_amdgcn_raw_buffer_load_b64(rp, lane * 8, soff + 5120, AUX_SYS);
  k.lw = __builtin_amdgcn_raw_buffer_load_b32(rp, lane * 4 < n ? lane * 4 : 0x80000000u, soff + 5632, AUX_SYS);
#pragma unroll
  for (int i = 0; i < 4; ++i) k.mq[i] = __builtin_amdgcn_raw_buffer_load_b128(rp, lane * 16, soff + i * 1024, AUX_SYS);
}

__device__ __forceinline__ void decode_unit(u32x4 (&v)[8], const Packed& k, u32 base, u32 n, LAS unsigned char* scr, u32 lane) {
  const u32 P0[2] = {k.p01.x, k.p01.y}, P1[2] = {k.p01.z, k.p01.w}, P2[2] = {k.p2.x, k.p2.y};
  const u32 b4 = base * 0x01010101u;
  u32x4 eq[4];
#pragma unroll
  for (int j = 0; j < 16; ++j) {
    const int s = j & 7, wd = j >> 3;
    const u32 c = ((P0[wd] >> s) & 0x01010101u) | (((P1[wd] >> s) & 0x01010101u) << 1) | (((P2[wd] >> s) & 0x01010101u) << 2);
    eq[j >> 2][j & 3] = b4 - c;
  }
  if (n) {
#pragma unroll
    for (int i = 0; i < 4; ++i) *(LAS u32x4*)(scr + i * 1024 + lane * 16) = eq[i];
    *(LAS u32*)(scr + 4096 + lane * 4) = k.lw;
    walk_escapes<false>(P0[0] & P1[0] & P2[0], P0[1] & P1[1] & P2[1], scr, scr + 4096, lane);
#pragma unroll
    for (int i = 0; i < 4; ++i) eq[i] = *(LAS u32x4*)(scr + i * 1024 + lane * 16);
  }
#pragma unroll
  for (int j = 0; j < 16; ++j) {
    const u32 e = eq[j >> 2][j & 3], m = k.mq[j >> 2][j & 3];
    const u32 r0 = __builtin_amdgcn_perm(e, m, 0x05010400u), r1 = __builtin_amdgcn_perm(e, m, 0x07030602u);
    v[j >> 1][(j & 1) * 2] = __builtin_amdgcn_alignbit(r0, r0, 1);
    v[j >> 1][(j & 1) * 2 + 1] = __builtin_amdgcn_alignbit(r1, r1, 1);
  }
}

__device__ __forceinline__ void packer(const Params& P, char* ws, u32 call, u32 pidx, LAS unsigned char* scr, u32 lane, u32 wid) {
  const u32 r = P.rank, g4 = r & 4u, par = call & 1u;
  const rsrc_t rx = make_rsrc(P.x, 1024u * 8192u * 2u), rw = make_rsrc(P.w, 8192u * 1024u * 2u);
  const rsrc_t rf = make_rsrc(ws, FRAG_BYTES), rp = make_rsrc(ws + OFF_PK + par * PK_BYTES, PK_BYTES);
  const u32 goff = OFF_G + (par * NR + r) * NUNITS * 8;
  char* const g_in = pick(P.ws, g4 | ((r + 1 + lane) & 3u)) + goff;
  char* const g_out = pick(P.ws, (g4 ^ 4u) | (lane & 3u)) + goff;
  const u32 kq = (lane & 3u) ^ quad_swz(lane >> 4);
  const u32 vx = (lane >> 2) * 16384u + kq * 16u, vw = kq * 8u * 2048u + (lane >> 2) * 16u;
  STAMP(6);
  for (u32 q = pidx; q < NUNITS; q += NPK) {
    const u32 kc = q >> 6, t = q & 63u;
    u32x4 v[8];
    Dest d;
    bool own, send;
    char* gp;
    if (t < 32u) {
#pragma unroll
      for (int f = 0; f < 8; ++f) v[f] = __builtin_amdgcn_raw_buffer_load_b128(rx, vx, (32u * t + 16u * (f >> 2)) * 16384u + kc * 256u + (f & 3) * 64u, 0);
      own = t < AU;
      d = dest_x(t, kc);
      send = own ? lane < 3u : lane < 4u;
      gp = own ? g_in : g_out;
    } else {
      const u32 ks4 = (t - 32u) >> 3, cg = t & 7u;
      u32x4 s[8];
#pragma unroll
      for (int i = 0; i < 8; ++i) s[i] = __builtin_amdgcn_raw_buffer_load_b128(rw, vw, (128u * kc + 32u * ks4 + i) * 2048u + cg * 256u, 0);
#pragma unroll
      for (int nf = 0; nf < 8; ++nf)
#pragma unroll
        for (int tt = 0; tt < 4; ++tt) v[nf][tt] = __builtin_amdgcn_perm(s[2 * tt + 1][nf >> 1], s[2 * tt][nf >> 1], (nf & 1) ? 0x07060302u : 0x05040100u);
      own = true;
      d = dest_w(cg >> 1, kc, ks4, cg & 1u);
      send = lane == (cg >> 1);
      gp = g_out;
    }
    const u32 info = pack_unit(v, rp, q * SLOT, scr, lane);
    if (own) store_frags(v, rf, d, lane);
    if (q == pidx) STAMP(5);
    drain_vm();
    if (send) __hip_atomic_store((GAS u64*)(gp + q * 8u), ((u64)call << 32) | info, __ATOMIC_RELAXED, __HIP_MEMORY_SCOPE_SYSTEM);
    if (own && lane == 0) __hip_atomic_fetch_add((GAS u32*)(ws + OFF_CNT) + d.cnt, 1u, __ATOMIC_RELAXED, __HIP_MEMORY_SCOPE_AGENT);
    if (q == pidx) STAMP(1);
  }
  STAMP(2);
}

__device__ __forceinline__ u32 lds_add(LAS u32* p, u32 v) {
  return __builtin_amdgcn_readfirstlane(__hip_atomic_fetch_add(p, v, __ATOMIC_RELAXED, __HIP_MEMORY_SCOPE_WORKGROUP)) >> 6;
}

__device__ __forceinline__ void puller(const Params& P, char* ws, u32 call, u32 pb, LAS u32* ctl, LAS unsigned char* scr, u32 lane, u32 wid) {
  const u32 r = P.rank, g4 = r & 4u, par = call & 1u;
  const u32 li = pb % 7u;
  const bool ing = li < 3u;
  const u32 src = ing ? (g4 | ((r + 1 + li) & 3u)) : ((g4 ^ 4u) | (li - 3u));
  const rsrc_t rf = make_rsrc(ws, FRAG_BYTES), rp = make_rsrc(pick(P.ws, src) + OFF_PK + par * PK_BYTES, PK_BYTES);
  char* const gbase = ws + OFF_G + (par * NR + src) * NUNITS * 8;
  GAS u32* const cnt = (GAS u32*)(ws + OFF_CNT);
  const u32 total = ing ? NKC * AU : NKC * NOUT;
  constexpr u32 NONE = 0xffffffffu;
  GAS u32* const tkt = (GAS u32*)(ws + OFF_TKT + li * 128u);
  const u32 tbase = (call - 1u) * (total + NCL * 8u) * 64u;
  auto draw = [&]() -> u32 { return __hip_atomic_fetch_add(tkt, 1u, __ATOMIC_RELAXED, __HIP_MEMORY_SCOPE_AGENT); };
  auto unit_of = [&](u32 raw_ticket) -> u32 {
    const u32 i = ((u32)__builtin_amdgcn_readfirstlane(raw_ticket) - tbase) >> 6;
    if (i >= total) return NONE;
    if (ing) { const u32 kc = i / AU; return kc * 64u + (i - kc * AU); }
    const u32 kc = i / NOUT, t = i - kc * NOUT, tw = t - AWAY;
    return kc * 64u + (t < AWAY ? AU + t : 32u + (tw >> 1) * 8u + 2u * (r & 3u) + (tw & 1u));
  };
  auto granule = [&](u32 q) -> u64 { return __hip_atomic_load((GAS u64*)(gbase + (q & (NUNITS - 1)) * 8u), __ATOMIC_RELAXED, __HIP_MEMORY_SCOPE_SYSTEM); };
  auto tokens = [&]() -> u32 { return (u32)(__builtin_amdgcn_readfirstlane(__hip_atomic_load(ctl + 1, __ATOMIC_RELAXED, __HIP_MEMORY_SCOPE_WORKGROUP)) >> 6) + TOK; };
  auto dest_of = [&](u32 q) -> Dest {
    const u32 kc = q >> 6, tq = q & 63u;
    if (tq < 32u) return dest_x(ing ? NEARLY * UPT + li * AU + tq : AU + (li - 3u) * AWAY + (tq - AU), kc);
    return dest_w(4u + (li - 3u), kc, (tq - 32u) >> 3, tq & 1u);
  };
  auto publish = [&](u32 c) { if (c != NONE && lane == 0) __hip_atomic_fetch_add(cnt + c, 1u, __ATOMIC_RELAXED, __HIP_MEMORY_SCOPE_AGENT); };
  u32 tk = lds_add(ctl, 1u), q = unit_of(draw()), pend = NONE;
  u64 gr = granule(q);
  u32 tag = __builtin_amdgcn_readfirstlane((u32)(gr >> 32)), info = __builtin_amdgcn_readfirstlane((u32)gr);
  bool first = true;
#define LAP(a) do {} while (0)
  while (q != NONE) {
    if (tag != call || tokens() <= tk) {
      drain_vm();
      publish(pend);
      pend = NONE;
      if (tag != call) {
        const u64 t0 = __builtin_amdgcn_s_memrealtime();
        for (;;) {
          gr = granule(q);
          tag = __builtin_amdgcn_readfirstlane((u32)(gr >> 32));
          info = __builtin_amdgcn_readfirstlane((u32)gr);
          if (tag == call || give_up(ws, call, t0)) break;
          __builtin_amdgcn_s_sleep(2);
        }
        if (tag != call) { lds_add(ctl + 1, 1u << 20); break; }
      }
      while (tokens() <= tk) __builtin_amdgcn_s_sleep(1);
    }
    asm volatile("" ::: "memory");
    if (first) { STAMP(1); first = false; }
    LAP(acc_w);
    const Dest d = dest_of(q);
    const u32 cur = info, soff = q * SLOT;
    u32x4 v[8];
    if (cur >> 17) {
#pragma unroll
      for (int f = 0; f < 8; ++f) v[f] = __builtin_amdgcn_raw_buffer_load_b128(rp, lane * 16, soff + f * 1024, AUX_SYS);
      const u32 nt = draw();
      tk = lds_add(ctl, 1u);
      drain_vm();
      lds_add(ctl + 1, 1u);
      q = unit_of(nt);
      gr = granule(q);
      publish(pend);
      tag = __builtin_amdgcn_readfirstlane((u32)(gr >> 32));
      info = __builtin_amdgcn_readfirstlane((u32)gr);
      store_frags(v, rf, d, lane);
    } else {
      const u32 n = (cur >> 8) & 511u;
      Packed k;
      fetch_packed(k, rp, soff, n, lane);
      const u32 nt = draw();
      tk = lds_add(ctl, 1u);
      drain_vm();
      lds_add(ctl + 1, 1u);
      LAP(acc_l);
      q = unit_of(nt);
      gr = granule(q);
      publish(pend);
      decode_unit(v, k, cur & 255u, n, scr, lane);
      __builtin_amdgcn_sched_barrier(0);
      tag = __builtin_amdgcn_readfirstlane((u32)(gr >> 32));
      info = __builtin_amdgcn_readfirstlane((u32)gr);
      __builtin_amdgcn_sched_barrier(0);
      store_frags(v, rf, d, lane);
      LAP(acc_d);
    }
    pend = d.cnt;
  }
  drain_vm();
  publish(pend);
  STAMP(2);
}

__device__ __forceinline__ void gate(int c, int& ready, const char* cnt, u32 target, char* ws, u32 call) {
  if (c < ready) return;
  const u64 t0 = __builtin_amdgcn_s_memrealtime();
  for (u32 spins = 0;; ++spins) {
    const u64 m = __builtin_amdgcn_ballot_w64((int)(ld_agent(cnt) - target) >= 0);
    ready = ~m ? __builtin_ctzll(~m) : 64;
    if (c < ready) break;
    if ((spins & 7u) == 7u && give_up(ws, call, t0)) { ready = 64; break; }
    __builtin_amdgcn_s_sleep(3);
  }
  asm volatile("" ::: "memory");
}

template <int G>
__device__ __forceinline__ void gemm_loop(f32x4 (&acc)[3][8], LAS char* lds, rsrc_t rs, const char* cnt, u32 target, char* ws, u32 call, u32 wr, u32 lane, u32 wid) {
  constexpr int NP = G ? 4 : 3, NS = G ? B_SLOTS : A_SLOTS;
  constexpr u32 STEP = G ? B_STEP : A_STEP;
  const u32 voff = wr * NP * 1024u + lane * 16u;
  LAS char* const ring = lds + (G ? A_RING : 0) + wr * NP * 1024u;
  const u32 img = ((lane & 15u) * 4u + ((lane >> 4) ^ quad_swz((lane >> 2) & 3u))) * 16u;
  LAS char* const ra = lds + wr * 3072u + img;
  LAS char* const rb = lds + A_RING + G * 8192u + img;
#define AG_PIECE(ks, slot, imm) __builtin_amdgcn_raw_ptr_buffer_load_lds(rs, ring + (slot) * STEP, 16, voff, (ks) * STEP, imm, AG_GEMM_AUX)
#define AG_ISSUE(ks, slot) do { AG_PIECE(ks, slot, 0); AG_PIECE(ks, slot, 1024); AG_PIECE(ks, slot, 2048); if (G) AG_PIECE(ks, slot, 3072); } while (0)
#define AG_WAIT_BAR() do { if (G) asm volatile("s_waitcnt vmcnt(12)\n\ts_barrier" ::: "memory"); else asm volatile("s_waitcnt vmcnt(9)\n\ts_barrier" ::: "memory"); } while (0)
#define AG_BAR() asm volatile("s_barrier" ::: "memory")
  int ready = 0;
  gate(0, ready, cnt, target, ws, call);
  STAMP(1);
  STAMPC(4);
#pragma unroll
  for (int s = 0; s < 4; ++s) AG_ISSUE(s, s);
  AG_WAIT_BAR();
  if (G) AG_BAR();
  u32 sa = 0, sb = 0, sl = 4;
  for (int kc = 0; kc < NKC; ++kc) {
#pragma unroll
    for (int s = 0; s < 4; ++s) {
      bf16x8 a[3], b[8];
#pragma unroll
      for (int m = 0; m < 3; ++m) a[m] = *(const LAS bf16x8*)(ra + sa * A_STEP + m * 1024);
#pragma unroll
      for (int n = 0; n < 8; ++n) b[n] = *(const LAS bf16x8*)(rb + sb * B_STEP + n * 1024);
      if (s == 0 && kc + 1 < NKC) gate(kc + 1, ready, cnt, target, ws, call);
      AG_ISSUE(kc * 4 + s + 4, sl);
      AG_WAIT_BAR();
      __builtin_amdgcn_sched_barrier(0);
      __builtin_amdgcn_s_setprio(1);
#pragma unroll
      for (int n = 0; n < 8; ++n)
#pragma unroll
        for (int m = 0; m < 3; ++m) acc[m][n] = __builtin_amdgcn_mfma_f32_16x16x32_bf16(a[m], b[n], acc[m][n], 0, 0, 0);
      __builtin_amdgcn_s_setprio(0);
      __builtin_amdgcn_sched_barrier(0);
      AG_BAR();
      __builtin_amdgcn_sched_barrier(0);
      sa = sa + 1 == A_SLOTS ? 0 : sa + 1;
      sb = sb + 1 == B_SLOTS ? 0 : sb + 1;
      sl = sl + 1 == NS ? 0 : sl + 1;
    }
  }
  if (!G) AG_BAR();
  drain_vm();
#undef AG_ISSUE
#undef AG_PIECE
#undef AG_WAIT_BAR
#undef AG_BAR
}

template <int G>
__device__ __forceinline__ void gemm_loop2(f32x4 (&acc)[3][8], LAS char* lds, rsrc_t rs, const char* cnt, u32 target, char* ws, u32 call, u32 wr, u32 lane, u32 wid) {
  constexpr int NP = G ? 4 : 3;
  constexpr u32 STEP = G ? B_STEP : A_STEP, SLOTB = A_STEP + B_STEP;
  const u32 voff = wr * NP * 1024u + lane * 16u;
  LAS char* const ring = lds + (G ? A_STEP : 0) + wr * NP * 1024u;
  const u32 img = ((lane & 15u) * 4u + ((lane >> 4) ^ quad_swz((lane >> 2) & 3u))) * 16u;
  LAS char* const ra = lds + wr * 3072u + img;
  LAS char* const rb = lds + A_STEP + G * 8192u + img;
#define AG_PIECE(ks, off, imm) __builtin_amdgcn_raw_ptr_buffer_load_lds(rs, ring + (off), 16, voff, (ks) * STEP, imm, AG_GEMM_AUX)
#define AG_ISSUE(ks, off) do { AG_PIECE(ks, off, 0); AG_PIECE(ks, off, 1024); AG_PIECE(ks, off, 2048); if (G) AG_PIECE(ks, off, 3072); } while (0)
#define AG_WAIT_BAR() do { if (G) asm volatile("s_waitcnt vmcnt(8)\n\ts_barrier" ::: "memory"); else asm volatile("s_waitcnt vmcnt(6)\n\ts_barrier" ::: "memory"); } while (0)
#define AG_READ(set, off) do { _Pragma("unroll") for (int m = 0; m < 3; ++m) a[set][m] = *(const LAS bf16x8*)(ra + (off) + m * 1024); \
    _Pragma("unroll") for (int n = 0; n < 8; ++n) b[set][n] = *(const LAS bf16x8*)(rb + (off) + n * 1024); } while (0)
  int ready = 0;
  gate(0, ready, cnt, target, ws, call);
  STAMP(1);
  STAMPC(4);
#pragma unroll
  for (int s = 0; s < 4; ++s) AG_ISSUE(s, s * SLOTB);
  AG_WAIT_BAR();
  bf16x8 a[2][3], b[2][8];
  AG_READ(0, 0);
  u32 ord = SLOTB, owr = 4 * SLOTB;
  for (int kc = 0; kc < NKC; ++kc) {
#pragma unroll
    for (int s = 0; s < 4; ++s) {
      if (s == 0 && kc + 1 < NKC) gate(kc + 1, ready, cnt, target, ws, call);
      const int cs = s & 1, ns = cs ^ 1;
      const u32 ks = kc * 4 + s + 4;
      LAS char* const pa = ra + ord;
      LAS char* const pb = rb + ord;
      __builtin_amdgcn_sched_barrier(0);
#pragma unroll
      for (int i = 0; i < 24; ++i) {
        const int n = i / 3, m = i % 3;
        acc[m][n] = __builtin_amdgcn_mfma_f32_16x16x32_bf16(a[cs][m], b[cs][n], acc[m][n], 0, 0, 0);
        if ((i & 1) == AG_RPAR && (i >> 1) < 11) {
          const int j = i >> 1;
          __builtin_amdgcn_sched_barrier(0);
          if (j < 3) a[ns][j] = *(const LAS bf16x8*)(pa + j * 1024);
          else b[ns][j - 3] = *(const LAS bf16x8*)(pb + (j - 3) * 1024);
          __builtin_amdgcn_sched_barrier(0);
        }
        if (i == AG_V0) { __builtin_amdgcn_sched_barrier(0); AG_PIECE(ks, owr, 0); __builtin_amdgcn_sched_barrier(0); }
        if (i == AG_V0 + AG_VD) { __builtin_amdgcn_sched_barrier(0); AG_PIECE(ks, owr, 1024); __builtin_amdgcn_sched_barrier(0); }
        if (i == AG_V0 + 2 * AG_VD) { __builtin_amdgcn_sched_barrier(0); AG_PIECE(ks, owr, 2048); __builtin_amdgcn_sched_barrier(0); }
        if (G && i == AG_V0 + 3 * AG_VD) { __builtin_amdgcn_sched_barrier(0); AG_PIECE(ks, owr, 3072); __builtin_amdgcn_sched_barrier(0); }
      }
      ord = ord + SLOTB == 5 * SLOTB ? 0 : ord + SLOTB;
      owr = owr + SLOTB == 5 * SLOTB ? 0 : owr + SLOTB;
      __builtin_amdgcn_sched_barrier(0);
      AG_WAIT_BAR();
    }
  }
  drain_vm();
#undef AG_PIECE
#undef AG_ISSUE
#undef AG_WAIT_BAR
#undef AG_READ
}

__device__ __forceinline__ u32 cvt_pk(float lo, float hi) {
  u32 r;
  asm volatile("v_cvt_pk_bf16_f32 %0, %1, %2" : "=v"(r) : "v"(lo), "v"(hi));
  return r;
}
__device__ __forceinline__ float relu(float v) { return v < 0.f ? 0.f : v; }

__device__ __forceinline__ void gemm(const Params& P, char* ws, u32 call, u32 b, LAS char* lds, u32 lane, u32 wid) {
  const u32 r = P.rank, g4 = r & 4u;
  const u32 xcd = b & 7u, slot = b >> 3;
  const u32 late = xcd & 1u, ent = (xcd >> 1) * 22u + slot;
  const u32 rc = ent >> 3, rt = late * NEARLY + rc, ct = ent & 7u;
  const u32 grp = wid >> 2, wr = wid & 3u;
  const int left = (late ? LATE_UNITS : EARLY_UNITS) - (int)rc * UPT;
  const u32 nun = left < 0 ? 0u : (left > UPT ? (u32)UPT : (u32)left);
  f32x4 acc[3][8];
#pragma unroll
  for (int m = 0; m < 3; ++m)
#pragma unroll
    for (int n = 0; n < 8; ++n) acc[m][n] = (f32x4){0.f, 0.f, 0.f, 0.f};
  u32 ta = call * nun, tb = call * 8u;
#define gemm_loop gemm_loop2
  if (grp == 0) gemm_loop<0>(acc, lds, make_rsrc(ws + OFF_XU + rt * A_TILE, A_TILE), ws + OFF_CNT + (rt * 64 + lane) * 4, ta, ws, call, wr, lane, wid);
  else gemm_loop<1>(acc, lds, make_rsrc(ws + OFF_WU + ct * B_TILE, B_TILE), ws + OFF_CNT + ((NRT + ct) * 64 + lane) * 4, tb, ws, call, wr, lane, wid);
  STAMP(2);
  STAMPC(5);
  const bool bad = __builtin_amdgcn_readfirstlane(ld_agent(ws + OFF_TMO)) == call;
  const u32 owner = ct < 4u ? r : ((g4 ^ 4u) | (ct - 4u));
  const u32 col0 = (ct < 4u ? ct : (r & 3u)) * 256u + grp * 128u;
  const rsrc_t ro = make_rsrc(pick(P.out, owner), 8192u * 1024u * 2u);
  const u32 vo = (lane >> 4) * 4u * 2048u + (lane & 15u) * 16u;
#pragma unroll
  for (int m = 0; m < 3; ++m) {
    const u32 fr = wr * 3u + m;
    const u32 uc = rc * UPT + (fr >> 1);
    if ((fr >> 1) >= nun) continue;
    u32 src, su;
    if (late) { const u32 j = uc / AU; src = g4 | ((r + 1 + j) & 3u); su = uc - j * AU; }
    else if (uc < AU) { src = r; su = uc; }
    else { const u32 j = (uc - AU) / AWAY; src = (g4 ^ 4u) | j; su = AU + (uc - AU) - j * AWAY; }
    const u32 row0 = src * 1024u + su * 32u + (fr & 1u) * 16u;
#pragma unroll
    for (int e = 0; e < 4; ++e) {
      u32x4 o;
#pragma unroll
      for (int n = 0; n < 4; ++n) o[n] = cvt_pk(relu(acc[m][2 * n][e]), relu(acc[m][2 * n + 1][e]));
      if (bad) o = (u32x4){0x7fc07fc0u, 0x7fc07fc0u, 0x7fc07fc0u, 0x7fc07fc0u};
      __builtin_amdgcn_raw_buffer_store_b128(o, ro, vo, (row0 + e) * 2048u + col0 * 2u, AUX_SYS);
      if (ct >= 4u) asm volatile("s_waitcnt vmcnt(%0)" :: "i"(AG_OUT_THROTTLE) : "memory");
    }
  }
  drain_vm();
  if (ct >= 4u && lane == 0) __hip_atomic_fetch_add((GAS u32*)(pick(P.ws, owner) + OFF_ARR + r * 128u), 1u, __ATOMIC_RELAXED, __HIP_MEMORY_SCOPE_SYSTEM);
  STAMP(3);
}

__device__ __forceinline__ void wait_arrivals(const Params& P, char* ws, u32 call, u32 lane, u32 wid) {
  const u32 r = P.rank;
  const char* p = ws + OFF_ARR + (((r & 4u) ^ 4u) | (lane & 3u)) * 128u;
  const u64 t0 = __builtin_amdgcn_s_memrealtime();
  for (;;) {
    if (__builtin_amdgcn_ballot_w64((int)(ld_sys(p) - call * (NRT * 8u)) < 0) == 0) break;
    if (give_up(ws, call, t0)) {
      if (lane == 0) *(GAS unsigned short*)(pick(P.out, r) + (size_t)r * 1024u * 2048u) = 0x7fc0;
      break;
    }
    __builtin_amdgcn_s_sleep(4);
  }
  STAMP(3);
}

__global__ __launch_bounds__(512, 2) void ag_gemm_kernel(Params P) {
  extern __shared__ __attribute__((aligned(16))) char smem[];
  LAS char* lds = (LAS char*)smem;
  const u32 lane = threadIdx.x & 63u, wid = __builtin_amdgcn_readfirstlane(threadIdx.x >> 6);
  char* const ws = pick(P.ws, P.rank);
  STAMP(0);
  u32 bid = blockIdx.x;
  asm volatile("" : "+s"(bid));
  char* const cptr = ws + OFF_CALL + bid * 128u + wid * 4u;
  const u32 call = __builtin_amdgcn_readfirstlane(ld_agent(cptr)) + 1u;
  __hip_atomic_store((GAS u32*)cptr, call, __ATOMIC_RELAXED, __HIP_MEMORY_SCOPE_AGENT);
  if (bid >= NCOMM) {
    gemm(P, ws, call, bid - NCOMM, lds, lane, wid);
  } else {
    LAS unsigned char* scr = (LAS unsigned char*)lds + wid * WAVE_SCRATCH;
    if (bid >= NPKWG) {
      LAS u32* ctl = (LAS u32*)(lds + 8 * WAVE_SCRATCH);
      if (threadIdx.x < 2) ctl[threadIdx.x] = 0;
      __syncthreads();
      puller(P, ws, call, bid - NPKWG, ctl, scr, lane, wid);
    } else {
      const u32 pidx = wid * NPKWG + bid;
      packer(P, ws, call, pidx, scr, lane, wid);
      if (pidx == 0) wait_arrivals(P, ws, call, lane, wid);
    }
  }
}

}

extern "C" void kernel_launch(int rank, int nranks, ncclComm_t comm, hipStream_t stream,
                              void* const* d_in, const int* in_sizes, int n_in,
                              void* d_out, int out_size, const int* in_dtypes, int out_dtype,
                              void* const* d_out_all, void* const* d_ws_all, size_t ws_bytes) {
  (void)comm; (void)in_dtypes; (void)out_dtype; (void)d_out;
  if (nranks != ag::NR || n_in != 2 || (unsigned)rank >= (unsigned)ag::NR || in_sizes[0] != 1024 * 8192 || in_sizes[1] != 8192 * 1024 ||
      out_size != 8192 * 1024 || ws_bytes < ag::WS_NEED) {
    fprintf(stderr, "ag_gemm: unexpected problem shape\n");
    return;
  }
  ag::Params p;
  p.x = (const char*)d_in[0];
  p.w = (const char*)d_in[1];
  for (int i = 0; i < ag::NR; ++i) { p.ws[i] = (char*)d_ws_all[i]; p.out[i] = (char*)d_out_all[i]; }
  p.rank = rank;
  p.pad = 0;
  hipError_t e = hipFuncSetAttribute((const void*)ag::ag_gemm_kernel, hipFuncAttributeMaxDynamicSharedMemorySize, ag::LDS_BYTES);
  if (e != hipSuccess) { fprintf(stderr, "ag_gemm: %s\n", hipGetErrorString(e)); return; }
  hipLaunchKernelGGL(ag::ag_gemm_kernel, dim3(ag::NWG), dim3(512), ag::LDS_BYTES, stream, p);
  e = hipGetLastError();
  if (e != hipSuccess) fprintf(stderr, "ag_gemm launch: %s\n", hipGetErrorString(e));
}
```

```cpp
#include <hip/hip_runtime.h>
#include <rccl/rccl.h>
#include <cstdio>

namespace ag {

typedef unsigned u32;
typedef unsigned long long u64;
typedef unsigned u32x4 __attribute__((ext_vector_type(4)));
typedef unsigned u32x2 __attribute__((ext_vector_type(2)));
typedef unsigned short u16x2 __attribute__((ext_vector_type(2)));
typedef short bf16x8 __attribute__((ext_vector_type(8)));
typedef float f32x4 __attribute__((ext_vector_type(4)));
typedef __amdgpu_buffer_rsrc_t rsrc_t;
#define LAS __attribute__((address_space(3)))
#define GAS __attribute__((address_space(1)))

#define AG_AU 21
#define AG_NCL 5
#define AG_TOK 15
#define AG_G2 0
#define AG_OTOK 1
#define AG_SPIN 0
#define AG_EARLY 2
#define AG_GDELAY 24
#define AG_PDELAY 0
#define AG_ORDER 0x75416320u
#define AG_NEAR 1
#define AG_CTMAP 0
#define AG_FORCE_RAW 0
#define AG_GEMM_AUX 17

constexpr int NR = 8;
constexpr int AU = AG_AU;
constexpr int AWAY = 32 - AU;
constexpr int NRT = 26, NEARLY = 13;
constexpr int UPT = 5;
constexpr int NKC = 64, NKS = 256;
constexpr int EARLY_UNITS = AU + 4 * AWAY, LATE_UNITS = 3 * AU;
static_assert(EARLY_UNITS <= NEARLY * UPT && LATE_UNITS <= (NRT - NEARLY) * UPT, "row split does not fit the tiles");
constexpr int NOUT = AWAY + 8;

constexpr int NCOMM = 48, NGEMM = 208, NWG = NCOMM + NGEMM;
constexpr int NCL = AG_NCL;
constexpr int NPKWG = NCOMM - 7 * NCL;
static_assert(4 * NCL <= NCOMM / 2 && (NCOMM - 4 * NCL) % 4 == 0, "puller placement");
constexpr int NPK = NPKWG * 8;
constexpr u32 GPACK = (8 + AG_G2) * NGEMM;
constexpr int NUNITS = 4096;

constexpr u32 A_STEP = 10240, B_STEP = 16384;
constexpr u32 A_TILE = A_STEP * NKS, B_TILE = B_STEP * NKS;
constexpr u32 OFF_XU = 0;
constexpr u32 OFF_WU = OFF_XU + NRT * A_TILE;
constexpr u32 FRAG_BYTES = OFF_WU + 8 * B_TILE;
constexpr u32 SLOT = 8192, PK_BYTES = NUNITS * SLOT;
constexpr u32 OFF_PK = FRAG_BYTES;
constexpr u32 OSLOT = 14336, OUNITS = NRT * 8;
constexpr u32 OFF_OB = OFF_PK + 2 * PK_BYTES;
constexpr u32 OFF_G = OFF_OB + 4 * OUNITS * OSLOT;
constexpr u32 G_BYTES = 2 * NR * NUNITS * 8;
constexpr u32 OFF_CNT = OFF_G + G_BYTES;
constexpr u32 OFF_OQ = OFF_CNT + 16384;
constexpr u32 OFF_TKT = OFF_OQ + 4 * 2048;
constexpr u32 OFF_TMO = OFF_TKT + 2048;
constexpr u32 OFF_CALL = OFF_TMO + 128;
constexpr u32 OFF_DBG = OFF_CALL + NWG * 128;
constexpr u32 OFF_DBG2 = OFF_DBG + NWG * 8 * 64;
constexpr u32 WS_NEED = OFF_DBG2 + 65536;

constexpr u32 SLOTB = A_STEP + B_STEP, NSLOT = 5;
constexpr int LDS_BYTES = NSLOT * SLOTB;
constexpr int WAVE_SCRATCH = 13312;
static_assert(8 * WAVE_SCRATCH + 64 <= LDS_BYTES, "scratch");

constexpr u32 SPARE_CNT = 4000;
constexpr u64 DEADLINE = 50000000ull;
constexpr int AUX_SYS = 17;

struct Params {
  const char* x;
  const char* w;
  char* ws[NR];
  char* out[NR];
  int rank;
  int pad;
};

#define STAMP(i) do {} while (0)
#define STAMPC(i) do {} while (0)

__device__ __forceinline__ rsrc_t make_rsrc(const void* p, u32 bytes) {
  const u64 a = (u64)p;
  const u32 lo = __builtin_amdgcn_readfirstlane((u32)a), hi = __builtin_amdgcn_readfirstlane((u32)(a >> 32));
  return __builtin_amdgcn_make_buffer_rsrc((void*)(((u64)hi << 32) | lo), 0, bytes, 0x00020000);
}
__device__ __forceinline__ char* pick(char* const (&t)[NR], u32 i) {
  char* p = t[0];
#pragma unroll
  for (u32 j = 1; j < NR; ++j) p = (i == j) ? t[j] : p;
  return p;
}
__device__ __forceinline__ u32 ld_agent(const char* p) { return __hip_atomic_load((GAS u32*)p, __ATOMIC_RELAXED, __HIP_MEMORY_SCOPE_AGENT); }
__device__ __forceinline__ u32 ld_sys(const char* p) { return __hip_atomic_load((GAS u32*)p, __ATOMIC_RELAXED, __HIP_MEMORY_SCOPE_SYSTEM); }
__device__ __forceinline__ void store16(u32x4 v, rsrc_t rs, u32 voff, u32 soff) {
  __builtin_amdgcn_raw_buffer_store_b128(v, rs, voff, soff, AUX_SYS);
  asm volatile("s_nop 1" :: "v"(v) : "memory");
}
__device__ __forceinline__ void count_in(const char* base, u32 idx) {
  u64 keep;
  asm volatile("s_mov_b64 %0, exec\n\ts_mov_b64 exec, 1\n\tglobal_atomic_add %1, %2, %3\n\ts_mov_b64 exec, %0"
               : "=&s"(keep) : "v"(idx * 4u), "v"(1u), "s"(base) : "memory");
}
__device__ __forceinline__ void flag_store(char* p, u64 v, u64 mask) {
  u64 keep;
  asm volatile("s_mov_b64 %0, exec\n\ts_mov_b64 exec, %3\n\tglobal_store_dwordx2 %1, %2, off sc0 sc1\n\ts_mov_b64 exec, %0"
               : "=&s"(keep) : "v"(p), "v"(v), "s"(mask) : "memory");
}
__device__ __forceinline__ void drain_vm() { asm volatile("s_waitcnt vmcnt(0)" ::: "memory"); }

__device__ __forceinline__ bool give_up(char* ws, u32 call, u64 t0) {
  const u32 t = __builtin_amdgcn_readfirstlane(ld_agent(ws + OFF_TMO));
  if (t == call) return true;
  if (__builtin_amdgcn_s_memrealtime() - t0 > DEADLINE) {
    __hip_atomic_store((GAS u32*)(ws + OFF_TMO), call, __ATOMIC_RELAXED, __HIP_MEMORY_SCOPE_AGENT);
    return true;
  }
  return false;
}

__device__ __forceinline__ u32 wave_sum(u32 v) {
  v += __builtin_amdgcn_update_dpp(0u, v, 0x111, 0xf, 0xf, true);
  v += __builtin_amdgcn_update_dpp(0u, v, 0x112, 0xf, 0xf, true);
  v += __builtin_amdgcn_update_dpp(0u, v, 0x114, 0xf, 0xf, true);
  v += __builtin_amdgcn_update_dpp(0u, v, 0x118, 0xf, 0xf, true);
  return __builtin_amdgcn_readlane(v, 15) + __builtin_amdgcn_readlane(v, 31) + __builtin_amdgcn_readlane(v, 47) + __builtin_amdgcn_readlane(v, 63);
}
__device__ __forceinline__ u32 wave_scan(u32 v) {
  v += __builtin_amdgcn_update_dpp(0u, v, 0x111, 0xf, 0xf, true);
  v += __builtin_amdgcn_update_dpp(0u, v, 0x112, 0xf, 0xf, true);
  v += __builtin_amdgcn_update_dpp(0u, v, 0x114, 0xf, 0xf, true);
  v += __builtin_amdgcn_update_dpp(0u, v, 0x118, 0xf, 0xf, true);
  v += __builtin_amdgcn_update_dpp(0u, v, 0x142, 0xa, 0xf, false);
  v += __builtin_amdgcn_update_dpp(0u, v, 0x143, 0xc, 0xf, false);
  return v;
}

__device__ __forceinline__ u32 quad_swz(u32 rq) { return (0x78u >> (2 * rq)) & 3u; }

struct Dest { u32 base, s1, s0, cnt; };
__device__ __forceinline__ Dest dest_x(u32 ux, u32 kc) {
  const u32 tile = ux / UPT;
  return Dest{OFF_XU + tile * A_TILE + 4 * kc * A_STEP + (ux - tile * UPT) * 2048u, 1024u, A_STEP, tile * 64 + kc};
}
__device__ __forceinline__ Dest dest_w(u32 ct, u32 kc, u32 ks4, u32 half) {
  return Dest{OFF_WU + ct * B_TILE + (4 * kc + ks4) * B_STEP + half * 8192u, 4096u, 1024u, (NRT + ct) * 64 + kc};
}
__device__ __forceinline__ void store_frags(const u32x4 (&v)[8], rsrc_t rf, const Dest& d, u32 lane) {
#pragma unroll
  for (int f = 0; f < 8; ++f) store16(v[f], rf, lane * 16, d.base + (f >> 2) * d.s1 + (f & 3) * d.s0);
}

template <bool TO_LIST>
__device__ __forceinline__ void walk_escapes(u32 m0, u32 m1, u32 off, LAS unsigned char* img, LAS unsigned char* list, u32 lane) {
  u64 m = ((u64)m1 << 32) | m0;
  while (m != 0) {
    const u32 p = __builtin_ctzll(m);
    m &= m - 1;
    const u32 j = ((p >> 5) << 3) | (p & 7), b = (p >> 3) & 3;
    LAS unsigned char* cell = img + (j >> 2) * 1024 + lane * 16 + (j & 3) * 4 + b;
    if (TO_LIST) list[off] = *cell; else *cell = list[off];
    ++off;
  }
}

struct Packed { u32x4 mq[4], p01; u32x2 p2; u32 lw; };

__device__ __forceinline__ u32 encode_unit(Packed& k, const u32x4 (&v)[8], LAS unsigned char* scr, u32 lane) {
  u32 E[16], esum = 0;
#pragma unroll
  for (int j = 0; j < 16; ++j) {
    const u32 d0 = v[j >> 1][(j & 1) * 2], d1 = v[j >> 1][(j & 1) * 2 + 1];
    const u32 r0 = __builtin_amdgcn_alignbit(d0, d0, 31), r1 = __builtin_amdgcn_alignbit(d1, d1, 31);
    k.mq[j >> 2][j & 3] = __builtin_amdgcn_perm(r1, r0, 0x06040200u);
    E[j] = __builtin_amdgcn_perm(r1, r0, 0x07050301u);
    esum = __builtin_amdgcn_sad_u8(E[j], 0u, esum);
  }
#pragma unroll
  for (int i = 0; i < 4; ++i) *(LAS u32x4*)(scr + i * 1024 + lane * 16) = (u32x4){E[4 * i], E[4 * i + 1], E[4 * i + 2], E[4 * i + 3]};
  u32 base = (wave_sum(esum) + 11059u) >> 12;
  base = base < 7u ? 7u : (base > 255u ? 255u : base);
  const u16x2 b2 = {(unsigned short)base, (unsigned short)base}, seven = {7, 7};
  u32 P[3][2] = {{0, 0}, {0, 0}, {0, 0}};
#pragma unroll
  for (int j = 0; j < 16; ++j) {
    const u16x2 ev = __builtin_bit_cast(u16x2, E[j] & 0x00ff00ffu), od = __builtin_bit_cast(u16x2, E[j]) >> (u16x2){8, 8};
    const u32 ce = __builtin_bit_cast(u32, __builtin_elementwise_min((u16x2)(b2 - ev), seven));
    const u32 co = __builtin_bit_cast(u32, __builtin_elementwise_min((u16x2)(b2 - od), seven));
    const u32 c = ce | (co << 8);
    const int s = j & 7, wd = j >> 3;
    P[0][wd] |= (c & 0x01010101u) << s;
    P[1][wd] |= ((c >> 1) & 0x01010101u) << s;
    P[2][wd] |= ((c >> 2) & 0x01010101u) << s;
  }
  k.p01 = (u32x4){P[0][0], P[0][1], P[1][0], P[1][1]};
  k.p2 = (u32x2){P[2][0], P[2][1]};
  const u32 m0 = P[0][0] & P[1][0] & P[2][0], m1 = P[0][1] & P[1][1] & P[2][1];
  const u32 c = __builtin_popcount(m0) + __builtin_popcount(m1), incl = wave_scan(c);
  const u32 n = __builtin_amdgcn_readlane(incl, 63);
  k.lw = 0;
  if (n > 256u || AG_FORCE_RAW) return 1u << 17;
  walk_escapes<true>(m0, m1, incl - c, scr, scr + 4096, lane);
  k.lw = *(LAS u32*)(scr + 4096 + lane * 4);
  return base | (n << 8);
}

__device__ __forceinline__ void store_unit(const Packed& k, const u32x4 (&v)[8], u32 info, rsrc_t rp, u32 soff, u32 lane) {
  if (info >> 17) {
#pragma unroll
    for (int f = 0; f < 8; ++f) store16(v[f], rp, lane * 16, soff + f * 1024);
    return;
  }
#pragma unroll
  for (int i = 0; i < 4; ++i) store16(k.mq[i], rp, lane * 16, soff + i * 1024);
  store16(k.p01, rp, lane * 16, soff + 4096);
  __builtin_amdgcn_raw_buffer_store_b64(k.p2, rp, lane * 8, soff + 5120, AUX_SYS);
  __builtin_amdgcn_raw_buffer_store_b32(k.lw, rp, lane * 4 < (info >> 8) ? lane * 4 : 0x80000000u, soff + 5632, AUX_SYS);
}

__device__ __forceinline__ void fetch_packed(Packed& k, rsrc_t rp, u32 soff, u32 n, u32 lane) {
  k.p01 = __builtin_amdgcn_raw_buffer_load_b128(rp, lane * 16, soff + 4096, AUX_SYS);
  k.p2 = __builtin_amdgcn_raw_buffer_load_b64(rp, lane * 8, soff + 5120, AUX_SYS);
  k.lw = __builtin_amdgcn_raw_buffer_load_b32(rp, lane * 4 < n ? lane * 4 : 0x80000000u, soff + 5632, AUX_SYS);
#pragma unroll
  for (int i = 0; i < 4; ++i) k.mq[i] = __builtin_amdgcn_raw_buffer_load_b128(rp, lane * 16, soff + i * 1024, AUX_SYS);
}

__device__ __forceinline__ void decode_unit(u32x4 (&v)[8], const Packed& k, u32 base, LAS unsigned char* scr, u32 lane) {
  const u32 P0[2] = {k.p01.x, k.p01.y}, P1[2] = {k.p01.z, k.p01.w}, P2[2] = {k.p2.x, k.p2.y};
  const u32 b4 = base * 0x01010101u;
  u32x4 eq[4];
#pragma unroll
  for (int j = 0; j < 16; ++j) {
    const int s = j & 7, wd = j >> 3;
    const u32 c = ((P0[wd] >> s) & 0x01010101u) | (((P1[wd] >> s) & 0x01010101u) << 1) | (((P2[wd] >> s) & 0x01010101u) << 2);
    eq[j >> 2][j & 3] = b4 - c;
  }
  const u32 mk[2] = {P0[0] & P1[0] & P2[0], P0[1] & P1[1] & P2[1]};
  const u32 c0 = __builtin_popcount(mk[0]), c = c0 + __builtin_popcount(mk[1]), off = wave_scan(c) - c;
  if (__builtin_amdgcn_ballot_w64(c > 8u) == 0) {
    const int a0 = off & ~3u;
    const u32 d0 = __builtin_amdgcn_ds_bpermute(a0, k.lw), d1 = __builtin_amdgcn_ds_bpermute(a0 + 4, k.lw), d2 = __builtin_amdgcn_ds_bpermute(a0 + 8, k.lw);
    const u32 l0 = __builtin_amdgcn_alignbyte(d1, d0, off), l1 = __builtin_amdgcn_alignbyte(d2, d1, off);
#pragma unroll
    for (int wd = 0; wd < 2; ++wd) {
      u32 t = mk[wd] - ((mk[wd] >> 1) & 0x55555555u);
      t = (t & 0x33333333u) + ((t >> 2) & 0x33333333u);
      t = (t + (t >> 4)) & 0x0f0f0f0fu;
      u32 rank = (__umul24(t & 0xffffffu, 0x010101u) << 8) + (wd ? c0 * 0x01010101u : 0u);
#pragma unroll
      for (int s = 0; s < 8; ++s) {
        const int j = wd * 8 + s;
        const u32 e = (mk[wd] >> s) & 0x01010101u, ff = (e << 8) - e;
        eq[j >> 2][j & 3] = (__builtin_amdgcn_perm(l1, l0, rank) & ff) | (eq[j >> 2][j & 3] & ~ff);
        rank += e;
      }
    }
  } else {
#pragma unroll
    for (int i = 0; i < 4; ++i) *(LAS u32x4*)(scr + i * 1024 + lane * 16) = eq[i];
    *(LAS u32*)(scr + 4096 + lane * 4) = k.lw;
    walk_escapes<false>(mk[0], mk[1], off, scr, scr + 4096, lane);
#pragma unroll
    for (int i = 0; i < 4; ++i) eq[i] = *(LAS u32x4*)(scr + i * 1024 + lane * 16);
  }
#pragma unroll
  for (int j = 0; j < 16; ++j) {
    const u32 e = eq[j >> 2][j & 3], m = k.mq[j >> 2][j & 3];
    const u32 r0 = __builtin_amdgcn_perm(e, m, 0x05010400u), r1 = __builtin_amdgcn_perm(e, m, 0x07030602u);
    v[j >> 1][(j & 1) * 2] = __builtin_amdgcn_alignbit(r0, r0, 1);
    v[j >> 1][(j & 1) * 2 + 1] = __builtin_amdgcn_alignbit(r1, r1, 1);
  }
}

__device__ __forceinline__ void load_unit(u32x4 (&s)[8], rsrc_t rx, rsrc_t rw, u32 q, u32 vx, u32 vw) {
  const u32 kc = q >> 6, t = q & 63u;
  if (t < 32u) {
#pragma unroll
    for (int f = 0; f < 8; ++f) s[f] = __builtin_amdgcn_raw_buffer_load_b128(rx, vx, (32u * t + 16u * (f >> 2)) * 16384u + kc * 256u + (f & 3) * 64u, 0);
  } else {
#pragma unroll
    for (int i = 0; i < 8; ++i) s[i] = __builtin_amdgcn_raw_buffer_load_b128(rw, vw, (128u * kc + 32u * ((t - 32u) >> 3) + i) * 2048u + (t & 7u) * 256u, 0);
  }
}

__device__ __forceinline__ u32 packer(const Params& P, char* ws, u32 callv, u32 q0, u32 q1, u32 dq, LAS unsigned char* scr, u32 lane, u32 wid) {
  const rsrc_t rx = make_rsrc(P.x, 1024u * 8192u * 2u), rw = make_rsrc(P.w, 8192u * 1024u * 2u);
  const u32 kq = (lane & 3u) ^ quad_swz(lane >> 4);
  const u32 vx = (lane >> 2) * 16384u + kq * 16u, vw = (lane >> 4) * 8u * 2048u + (lane & 15u) * 16u;
  u32x4 s[8];
  load_unit(s, rx, rw, q0, vx, vw);
  __builtin_amdgcn_sched_barrier(0);
  const u32 call = __builtin_amdgcn_readfirstlane(callv) + 1u;
  const u32 r = P.rank, g4 = r & 4u, par = call & 1u;
  const rsrc_t rf = make_rsrc(ws, FRAG_BYTES), rp = make_rsrc(ws + OFF_PK + par * PK_BYTES, PK_BYTES);
  const u32 goff = OFF_G + (par * NR + r) * NUNITS * 8;
  char* g_in = pick(P.ws, g4 | ((r + 1u) & 3u));
  char* g_out = pick(P.ws, g4 ^ 4u);
#pragma unroll
  for (u32 j = 1; j < 4; ++j) {
    char* const pi = pick(P.ws, g4 | ((r + 1u + j) & 3u));
    char* const po = pick(P.ws, (g4 ^ 4u) | j);
    g_in = lane == j ? pi : g_in;
    g_out = lane == j ? po : g_out;
  }
  STAMP(6);
  u64 psend = 0;
  char* pgp = g_out;
  u32 pinfo = 0, pcnt = SPARE_CNT;
  auto announce = [&]() {
    flag_store(pgp, ((u64)call << 32) | pinfo, psend);
    if (pcnt != SPARE_CNT) count_in(ws + OFF_CNT, pcnt);
  };
  for (u32 q = q0; q < NUNITS; q = q1, q1 += dq) {
    const u32 kc = q >> 6, t = q & 63u;
    u32x4 v[8];
    Dest d;
    bool own;
    u64 send;
    char* gp;
    if (t < 32u) {
#pragma unroll
      for (int f = 0; f < 8; ++f) v[f] = s[f];
      own = t < AU;
      d = dest_x(t, kc);
      send = own ? 7u : 15u;
      gp = own ? g_in : g_out;
    } else {
      const u32 ks4 = (t - 32u) >> 3, cg = t & 7u;
#pragma unroll
      for (int nf = 0; nf < 8; ++nf)
#pragma unroll
        for (int tt = 0; tt < 4; ++tt) v[nf][tt] = __builtin_amdgcn_perm(s[2 * tt + 1][nf >> 1], s[2 * tt][nf >> 1], (nf & 1) ? 0x07060302u : 0x05040100u);
      own = true;
      d = dest_w(cg >> 1, kc, ks4, cg & 1u);
      send = 1u << (cg >> 1);
      gp = g_out;
    }
    if (q == q0) STAMP(4);
    if (q1 < NUNITS) load_unit(s, rx, rw, q1, vx, vw);
    Packed k;
    const u32 info = encode_unit(k, v, scr, lane);
    drain_vm();
#pragma unroll
    for (int f = 0; f < 8; ++f) asm volatile("" : "+v"(s[f]));
    announce();
    store_unit(k, v, info, rp, q * SLOT, lane);
    if (own) store_frags(v, rf, d, lane);
    if (q == q0) STAMP(5);
    psend = send;
    pgp = gp + goff + q * 8u;
    pinfo = info;
    pcnt = own ? d.cnt : SPARE_CNT;
  }
  drain_vm();
  announce();
  STAMP(2);
  return call;
}

__device__ __forceinline__ u32 lds_add(LAS u32* p, u32 v) {
  return __builtin_amdgcn_readfirstlane(__hip_atomic_fetch_add(p, v, __ATOMIC_RELAXED, __HIP_MEMORY_SCOPE_WORKGROUP)) >> 6;
}

__device__ __forceinline__ void puller(const Params& P, char* ws, u32 call, u32 li, u32 lidx, LAS u32* ctl, LAS unsigned char* scr, u32 lane, u32 wid) {
  const u32 r = P.rank, g4 = r & 4u, par = call & 1u;
  const u32 TOK = AG_TOK / NCL + (lidx < AG_TOK % NCL ? 1u : 0u);
  const bool ing = li < 3u;
  const u32 src = ing ? (g4 | ((r + 1 + li) & 3u)) : ((g4 ^ 4u) | (li - 3u));
  const rsrc_t rf = make_rsrc(ws, FRAG_BYTES), rp = make_rsrc(pick(P.ws, src) + OFF_PK + par * PK_BYTES, PK_BYTES);
  char* const gbase = ws + OFF_G + (par * NR + src) * NUNITS * 8;
  GAS u32* const cnt = (GAS u32*)(ws + OFF_CNT);
  const u32 total = ing ? NKC * AU : NKC * NOUT;
  constexpr u32 NONE = 0xffffffffu;
  GAS u32* const tkt = (GAS u32*)(ws + OFF_TKT + li * 128u);
  const u32 tbase = (call - 1u) * (total + NCL * 8u) * 64u;
  auto draw = [&]() -> u32 { return __hip_atomic_fetch_add(tkt, 1u, __ATOMIC_RELAXED, __HIP_MEMORY_SCOPE_AGENT); };
  auto unit_of = [&](u32 raw_ticket) -> u32 {
    const u32 i = ((u32)__builtin_amdgcn_readfirstlane(raw_ticket) - tbase) >> 6;
    if (i >= total) return NONE;
    if (ing) { const u32 kc = i / AU; return kc * 64u + (i - kc * AU); }
    const u32 kc = i / NOUT, t = i - kc * NOUT, tw = t - AWAY;
    return kc * 64u + (t < AWAY ? AU + t : 32u + (tw >> 1) * 8u + 2u * (r & 3u) + (tw & 1u));
  };
  auto granule = [&](u32 q) -> u64 { return __hip_atomic_load((GAS u64*)(gbase + (q & (NUNITS - 1)) * 8u), __ATOMIC_RELAXED, __HIP_MEMORY_SCOPE_SYSTEM); };
  auto try_take = [&]() -> bool {
    if ((u32)(__builtin_amdgcn_readfirstlane(__hip_atomic_load(ctl, __ATOMIC_RELAXED, __HIP_MEMORY_SCOPE_WORKGROUP)) >> 6) >= TOK) return false;
    if (lds_add(ctl, 1u) < TOK) return true;
    lds_add(ctl, ~0u);
    return false;
  };
  auto dest_of = [&](u32 q) -> Dest {
    const u32 kc = q >> 6, tq = q & 63u;
    if (tq < 32u) return dest_x(ing ? NEARLY * UPT + li * AU + tq : AU + (li - 3u) * AWAY + (tq - AU), kc);
    return dest_w(4u + (li - 3u), kc, (tq - 32u) >> 3, tq & 1u);
  };
  auto publish = [&](u32 c) { if (c != NONE) count_in(ws + OFF_CNT, c); };
  u32 q = unit_of(draw()), pend = NONE;
  u64 gr = granule(q);
  u32 tag = __builtin_amdgcn_readfirstlane((u32)(gr >> 32)), info = __builtin_amdgcn_readfirstlane((u32)gr);
  bool first = true;
#define LAP(a) do {} while (0)
  while (q != NONE) {
    bool got = false;
    if (tag == call)
      for (u32 spin = 0; !(got = try_take()) && spin < AG_SPIN; ++spin) __builtin_amdgcn_s_sleep(1);
    if (!got) {
      drain_vm();
      publish(pend);
      pend = NONE;
      if (tag != call) {
        const u64 t0 = __builtin_amdgcn_s_memrealtime();
        for (;;) {
          gr = granule(q);
          tag = __builtin_amdgcn_readfirstlane((u32)(gr >> 32));
          info = __builtin_amdgcn_readfirstlane((u32)gr);
          if (tag == call || give_up(ws, call, t0)) break;
          __builtin_amdgcn_s_sleep(2);
        }
        if (tag != call) break;
      }
      while (!try_take()) __builtin_amdgcn_s_sleep(1);
    }
    asm volatile("" ::: "memory");
    if (first) { STAMP(1); first = false; }
    LAP(acc_w);
    const Dest d = dest_of(q);
    const u32 cur = info, soff = q * SLOT;
    u32x4 v[8];
    if (cur >> 17) {
#pragma unroll
      for (int f = 0; f < 8; ++f) v[f] = __builtin_amdgcn_raw_buffer_load_b128(rp, lane * 16, soff + f * 1024, AUX_SYS);
      const u32 nt = draw();
      drain_vm();
      lds_add(ctl, ~0u);
      q = unit_of(nt);
      gr = granule(q);
      publish(pend);
      tag = __builtin_amdgcn_readfirstlane((u32)(gr >> 32));
      info = __builtin_amdgcn_readfirstlane((u32)gr);
      store_frags(v, rf, d, lane);
    } else {
      const u32 n = (cur >> 8) & 511u;
      Packed k;
      u32 nt;
      u64 keep;
      asm volatile("s_mov_b64 %1, exec\n\ts_mov_b64 exec, 1\n\tglobal_atomic_add %0, %2, %3, %4 sc0\n\ts_mov_b64 exec, %1"
                   : "=&v"(nt), "=&s"(keep) : "v"(0u), "v"(64u), "s"(tkt) : "memory");
      fetch_packed(k, rp, soff, n, lane);
      asm volatile("s_waitcnt vmcnt(%0)" :: "i"(AG_EARLY) : "memory");
      lds_add(ctl, ~0u);
      asm volatile("s_waitcnt vmcnt(0)" : "+v"(nt), "+v"(k.mq[0]), "+v"(k.mq[1]), "+v"(k.mq[2]), "+v"(k.mq[3]), "+v"(k.p01), "+v"(k.p2), "+v"(k.lw) :: "memory");
      LAP(acc_l);
      q = unit_of(nt);
      asm volatile("global_load_dwordx2 %0, %2, %3 sc0 sc1\n\ts_mov_b64 %1, exec\n\ts_mov_b64 exec, 1\n\tglobal_atomic_add %4, %5, %6\n\ts_mov_b64 exec, %1"
                   : "=&v"(gr), "=&s"(keep) : "v"((q & (NUNITS - 1)) * 8u), "s"(gbase), "v"((pend == NONE ? SPARE_CNT : pend) * 4u), "v"(1u), "s"(cnt) : "memory");
      decode_unit(v, k, cur & 255u, scr, lane);
      __builtin_amdgcn_sched_barrier(0);
      asm volatile("s_waitcnt vmcnt(1)" : "+v"(gr) :: "memory");
      tag = __builtin_amdgcn_readfirstlane((u32)(gr >> 32));
      info = __builtin_amdgcn_readfirstlane((u32)gr);
      __builtin_amdgcn_sched_barrier(0);
      store_frags(v, rf, d, lane);
      LAP(acc_d);
    }
    pend = d.cnt;
  }
  drain_vm();
  publish(pend);
  STAMP(2);
}

constexpr u32 NOROW = 0xffffffffu;
__device__ __forceinline__ u32 frag_row(u32 r, u32 rt, u32 fr) {
  const u32 g4 = r & 4u;
  const bool late = rt >= NEARLY;
  const u32 uc = (late ? rt - NEARLY : rt) * UPT + (fr >> 1);
  u32 src, su;
  if (late) {
    if (uc >= LATE_UNITS) return NOROW;
    const u32 j = uc / AU;
    src = g4 | ((r + 1 + j) & 3u);
    su = uc - j * AU;
  } else if (uc < AU) {
    src = r;
    su = uc;
  } else {
    if (uc >= EARLY_UNITS) return NOROW;
    const u32 j = (uc - AU) / AWAY;
    src = (g4 ^ 4u) | j;
    su = AU + (uc - AU) - j * AWAY;
  }
  return ((AG_ORDER >> (4u * src)) & 7u) * 1024u + su * 32u + (fr & 1u) * 16u;
}
__device__ __forceinline__ u32 wave_frag0(u32 wid) { return (0x74208630u >> (4 * wid)) & 15u; }

__device__ __forceinline__ void out_puller(const Params& P, char* ws, u32 call, u32 j, LAS u32* ctl, LAS unsigned char* scr, u32 lane, u32 wid) {
  const u32 r = P.rank, src = ((r & 4u) ^ 4u) | j;
  char* const ob = pick(P.ws, src) + OFF_OB + (r & 3u) * (OUNITS * OSLOT);
  const rsrc_t ro = make_rsrc(pick(P.out, r), 8192u * 1024u * 2u);
  GAS u32* const tkt = (GAS u32*)(ws + OFF_TKT + (7u + j) * 128u);
  const u32 tbase = (call - 1u) * (OUNITS + NCOMM * 2u) * 64u;
  const u32 vo = (lane >> 4) * 4u * 2048u + (lane & 15u) * 16u;
  for (;;) {
    const u32 k = ((u32)__builtin_amdgcn_readfirstlane(__hip_atomic_fetch_add(tkt, 1u, __ATOMIC_RELAXED, __HIP_MEMORY_SCOPE_AGENT)) - tbase) >> 6;
    if (k >= OUNITS) break;
    const u64 t0 = __builtin_amdgcn_s_memrealtime();
    u32 tag, info;
    for (u32 spins = 0;; ++spins) {
      const u64 gr = __hip_atomic_load((GAS u64*)(ws + OFF_OQ + j * 2048u + k * 8u), __ATOMIC_RELAXED, __HIP_MEMORY_SCOPE_SYSTEM);
      tag = __builtin_amdgcn_readfirstlane((u32)(gr >> 32));
      info = __builtin_amdgcn_readfirstlane((u32)gr);
      if (tag == call || ((spins & 7u) == 7u && give_up(ws, call, t0))) break;
      __builtin_amdgcn_s_sleep(6);
    }
    if (tag != call) {
      *(GAS unsigned short*)(pick(P.out, r) + (size_t)((AG_ORDER >> (4u * r)) & 7u) * 1024u * 2048u) = 0x7fc0;
      break;
    }
    for (;;) {
      if ((u32)(__builtin_amdgcn_readfirstlane(__hip_atomic_load(ctl + 1, __ATOMIC_RELAXED, __HIP_MEMORY_SCOPE_WORKGROUP)) >> 6) < AG_OTOK) {
        if (lds_add(ctl + 1, 1u) < AG_OTOK) break;
        lds_add(ctl + 1, ~0u);
      }
      __builtin_amdgcn_s_sleep(1);
    }
    asm volatile("" ::: "memory");
    const u32 uid = (info & 255u) < OUNITS ? (info & 255u) : 0u, sw = uid & 7u, mf = sw - 2u < 4u ? 2u : 3u;
    u32 total = (info >> 8) & 0x3fffu;
    total = total < 12544u ? total : 12544u;
    const bool raw = info >> 22;
    const rsrc_t rs = make_rsrc(ob + uid * OSLOT, 1024u + ((total + 15u) & ~15u));
    u32 mk[3];
#pragma unroll
    for (u32 m = 0; m < 3; ++m) mk[m] = __builtin_amdgcn_raw_buffer_load_b32(rs, (m < mf && !raw) ? lane * 4u : 0x80000000u, m * 256u, AUX_SYS);
    for (u32 p = 0; p < total; p += 1024u) __builtin_amdgcn_raw_ptr_buffer_load_lds(rs, scr + p, 16, lane * 16u, 1024u + p, 0, AUX_SYS);
    asm volatile("s_waitcnt vmcnt(0)" : "+v"(mk[0]), "+v"(mk[1]), "+v"(mk[2]) :: "memory");
    lds_add(ctl + 1, ~0u);
    const u32 len = 2u * ((__builtin_popcount(mk[0]) + __builtin_popcount(mk[1]) + __builtin_popcount(mk[2])) & ~1u) + 4u;
    u32 off = wave_scan(len) - len;
    const u32 col = j * 512u + (sw >> 2) * 256u, f0 = wave_frag0(sw);
    for (u32 m = 0; m < mf; ++m) {
      const u32 w = m == 0 ? mk[0] : (m == 1 ? mk[1] : mk[2]);
      u32x4 o[4];
      if (raw) {
#pragma unroll
        for (int e = 0; e < 4; ++e) o[e] = *(const LAS u32x4*)(scr + (m * 4u + e) * 1024u + lane * 16u);
      } else {
#pragma unroll
        for (int i = 0; i < 16; ++i) {
          const u32 a = off + 2u * __builtin_popcount(w & (((1u << i) - 1u) * 0x10001u));
          const LAS u32* const pw = (const LAS u32*)(scr + (a & ~3u));
          const u32 d = __builtin_amdgcn_alignbyte(pw[1], pw[0], a);
          const u32 lo = (u32)((int)(w << (31 - i)) >> 31), hi = (u32)((int)(w << (15 - i)) >> 31);
          o[i >> 2][i & 3] = ((d & lo) | ((d << 16) & ~lo)) & (hi | 0xffffu);
        }
        off += 2u * __builtin_popcount(w);
      }
      const u32 row0 = frag_row(src, uid >> 3, f0 + m);
      if (row0 != NOROW) {
#pragma unroll
        for (int e = 0; e < 4; ++e) store16(o[e], ro, vo, (row0 + e) * 2048u + col);
      }
    }
  }
  STAMP(3);
}

__device__ __forceinline__ void gate(int c, int& ready, const char* cnt, u32 target, char* ws, u32 call) {
  if (c < ready) return;
  const u64 t0 = __builtin_amdgcn_s_memrealtime();
  for (u32 spins = 0;; ++spins) {
    const u64 m = __builtin_amdgcn_ballot_w64((int)(ld_agent(cnt) - target) >= 0);
    ready = ~m ? __builtin_ctzll(~m) : 64;
    if (c < ready) break;
    if ((spins & 7u) == 7u && give_up(ws, call, t0)) { ready = 64; break; }
    __builtin_amdgcn_s_sleep(3);
  }
  asm volatile("" ::: "memory");
}

__device__ __forceinline__ u32 cvt_pk(float lo, float hi) {
  u32 r;
  asm volatile("v_cvt_pk_bf16_f32 %0, %1, %2" : "=v"(r) : "v"(lo), "v"(hi));
  return r;
}
__device__ __forceinline__ float relu(float v) { return v < 0.f ? 0.f : v; }

__device__ __forceinline__ void mfma_acc(f32x4& c, const bf16x8& a, const bf16x8& b) {
  asm volatile("v_mfma_f32_16x16x32_bf16 %0, %1, %2, %0" : "+v"(c) : "v"(a), "v"(b));
}
template <int N> __device__ __forceinline__ void wait_bar() { asm volatile("s_waitcnt vmcnt(%0)\n\ts_barrier" :: "i"(N) : "memory"); }

template <int MF, int NP>
__device__ __forceinline__ void gemm_wave(const Params& P, char* ws, u32 call, u32 b, LAS char* lds, u32 lane, u32 wid) {
  constexpr bool LA = NP < 4;
  constexpr u32 STEP = LA ? A_STEP : B_STEP;
  constexpr int NM = MF * 8, NRD = MF + 8, VD = NM / NP;
  const u32 r = P.rank, g4 = r & 4u;
  const u32 xcd = b & 7u, late = xcd & 1u, ent = (xcd >> 1) * 26u + (b >> 3);
  const u32 rc = ent >> 3, rt = late * NEARLY + rc, ct = ent & 7u;
  const u32 wc = wid >> 2, f0 = wave_frag0(wid);
  const int left = (late ? LATE_UNITS : EARLY_UNITS) - (int)rc * UPT;
  const u32 nun = left < 0 ? 0u : (left > UPT ? (u32)UPT : (u32)left);
  const u32 p0 = ((0x86c84030u >> (4 * wid)) & 15u) * 1024u;
  const rsrc_t rs = LA ? make_rsrc(ws + OFF_XU + rt * A_TILE, A_TILE) : make_rsrc(ws + OFF_WU + ct * B_TILE, B_TILE);
  const char* const cnt = ws + OFF_CNT + ((LA ? rt : NRT + ct) * 64 + lane) * 4;
  u32 target = call * (LA ? nun : 8u);
  const u32 voff = p0 + lane * 16u;
  LAS char* const ring = lds + (LA ? 0u : A_STEP) + p0;
  const u32 img = ((lane & 15u) * 4u + ((lane >> 4) ^ quad_swz((lane >> 2) & 3u))) * 16u;
  LAS char* const ra = lds + f0 * 1024u + img;
  LAS char* const rb = lds + A_STEP + wc * 8192u + lane * 16u;
#define AG_PIECE(ks, off, imm) __builtin_amdgcn_raw_ptr_buffer_load_lds(rs, ring + (off), 16, voff, (ks) * STEP, imm, AG_GEMM_AUX)
#define AG_PIN(stmt) do { __builtin_amdgcn_sched_barrier(0); stmt; __builtin_amdgcn_sched_barrier(0); } while (0)
#define AG_STEP(cs, ks, DMA, RD) do { \
    LAS char* const pa = ra + ord; \
    LAS char* const pb = rb + ord; \
    __builtin_amdgcn_sched_barrier(0); \
    _Pragma("unroll") for (int i = 0; i < NM; ++i) { \
      mfma_acc(acc[i % MF][i / MF], fa[cs][i % MF], fb[cs][i / MF]); \
      _Pragma("unroll") for (int j = 0; j < NRD; ++j) \
        if (RD && i == (MF == 3 ? 2 * j : (3 * j) / 2)) { \
          if (j < MF) AG_PIN(fa[(cs) ^ 1][j] = *(const LAS bf16x8*)(pa + j * 1024)); \
          else AG_PIN(fb[(cs) ^ 1][j - MF] = *(const LAS bf16x8*)(pb + (j - MF) * 1024)); \
        } \
      if (DMA && i == 1) AG_PIN(AG_PIECE(ks, owr, 0)); \
      if (DMA && i == 1 + VD) AG_PIN(AG_PIECE(ks, owr, 1024)); \
      if (DMA && NP > 2 && i == 1 + 2 * VD) AG_PIN(AG_PIECE(ks, owr, 2048)); \
      if (DMA && NP > 3 && i == 1 + 3 * VD) AG_PIN(AG_PIECE(ks, owr, 3072)); \
    } \
    ord = ord + SLOTB == NSLOT * SLOTB ? 0 : ord + SLOTB; \
    owr = owr + SLOTB == NSLOT * SLOTB ? 0 : owr + SLOTB; \
    __builtin_amdgcn_sched_barrier(0); \
  } while (0)
  f32x4 acc[MF][8];
#pragma unroll
  for (int m = 0; m < MF; ++m)
#pragma unroll
    for (int n = 0; n < 8; ++n) { acc[m][n] = (f32x4){0.f, 0.f, 0.f, 0.f}; asm volatile("" : "+v"(acc[m][n])); }
  int ready = 0;
  gate(0, ready, cnt, target, ws, call);
  STAMP(1);
  STAMPC(4);
#pragma unroll
  for (int s = 0; s < 4; ++s) {
    AG_PIECE(s, s * SLOTB, 0);
    AG_PIECE(s, s * SLOTB, 1024);
    if (NP > 2) AG_PIECE(s, s * SLOTB, 2048);
    if (NP > 3) AG_PIECE(s, s * SLOTB, 3072);
  }
  wait_bar<2 * NP>();
  bf16x8 fa[2][MF], fb[2][8];
#pragma unroll
  for (int m = 0; m < MF; ++m) fa[0][m] = *(const LAS bf16x8*)(ra + m * 1024);
#pragma unroll
  for (int n = 0; n < 8; ++n) fb[0][n] = *(const LAS bf16x8*)(rb + n * 1024);
  u32 ord = SLOTB, owr = 4 * SLOTB;
  for (int kc = 0; kc < NKC - 1; ++kc) {
    gate(kc + 1, ready, cnt, target, ws, call);
    AG_STEP(0, kc * 4 + 4, true, true); wait_bar<2 * NP>();
    AG_STEP(1, kc * 4 + 5, true, true); wait_bar<2 * NP>();
    AG_STEP(0, kc * 4 + 6, true, true); wait_bar<2 * NP>();
    AG_STEP(1, kc * 4 + 7, true, true); wait_bar<2 * NP>();
  }
  AG_STEP(0, 0, false, true); wait_bar<NP>();
  AG_STEP(1, 0, false, true); wait_bar<0>();
  AG_STEP(0, 0, false, true);
  AG_STEP(1, 0, false, false);
  asm volatile("s_nop 15" ::: "memory");
  __builtin_amdgcn_sched_barrier(0);
#undef AG_PIECE
#undef AG_PIN
#undef AG_STEP
  STAMP(2);
  STAMPC(5);
  const bool bad = __builtin_amdgcn_readfirstlane(ld_agent(ws + OFF_TMO)) == call;
  u32x4 o[MF][4];
#pragma unroll
  for (int m = 0; m < MF; ++m)
#pragma unroll
    for (int e = 0; e < 4; ++e) {
#pragma unroll
      for (int n = 0; n < 4; ++n) o[m][e][n] = cvt_pk(relu(acc[m][2 * n][e]), relu(acc[m][2 * n + 1][e]));
      if (bad) o[m][e] = (u32x4){0x7fc07fc0u, 0x7fc07fc0u, 0x7fc07fc0u, 0x7fc07fc0u};
    }
  if (ct < 4u) {
    const rsrc_t ro = make_rsrc(pick(P.out, r), 8192u * 1024u * 2u);
    const u32 vo = (lane >> 4) * 4u * 2048u + (lane & 15u) * 16u;
#pragma unroll
    for (int m = 0; m < MF; ++m) {
      const u32 row0 = frag_row(r, rt, f0 + m);
      if (row0 == NOROW) continue;
#pragma unroll
      for (int e = 0; e < 4; ++e) store16(o[m][e], ro, vo, (row0 + e) * 2048u + ct * 512u + wc * 256u);
    }
    STAMP(3);
    return;
  }
  const u32 j = ct - 4u, uid = rt * 8u + wid;
  u32 qi;
  u64 keep;
  asm volatile("s_mov_b64 %1, exec\n\ts_mov_b64 exec, 1\n\tglobal_atomic_add %0, %2, %3, %4 sc0\n\ts_mov_b64 exec, %1"
               : "=&v"(qi), "=&s"(keep) : "v"(0u), "v"(1u), "s"(ws + OFF_TKT + (11u + j) * 128u) : "memory");
  u32 mk[MF], len = 0;
#pragma unroll
  for (int m = 0; m < MF; ++m) {
    mk[m] = 0;
    if ((f0 + m) >> 1 >= nun) {
#pragma unroll
      for (int e = 0; e < 4; ++e) o[m][e] = (u32x4){0u, 0u, 0u, 0u};
    }
#pragma unroll
    for (int i = 0; i < 16; ++i) {
      u32 t;
      const u32 d = o[m][i >> 2][i & 3];
      asm("v_pk_min_u16 %0, %1, %2" : "=v"(t) : "v"(d), "v"(0x00010001u));
      mk[m] |= t << i;
    }
    len += __builtin_popcount(mk[m]);
  }
  len = 2u * (len & ~1u) + 4u;
  const u32 incl = wave_scan(len);
  u32 total = __builtin_amdgcn_readlane(incl, 63);
  const bool raw = total + 256u * MF >= 4096u * MF;
  const rsrc_t rb2 = make_rsrc(ws + OFF_OB + j * (OUNITS * OSLOT) + uid * OSLOT, OSLOT);
  asm volatile("s_barrier" ::: "memory");
  if (raw) {
    total = 4096u * MF;
#pragma unroll
    for (int m = 0; m < MF; ++m)
#pragma unroll
      for (int e = 0; e < 4; ++e) store16(o[m][e], rb2, lane * 16u, 1024u + (m * 4 + e) * 1024u);
  } else {
    LAS char* const scr = lds + wid * WAVE_SCRATCH;
    u32 off = incl - len, a = off, cur = 0;
#pragma unroll
    for (int m = 0; m < MF; ++m) {
      __builtin_amdgcn_raw_buffer_store_b32(mk[m], rb2, lane * 4u, m * 256u, AUX_SYS);
#pragma unroll
      for (int i = 0; i < 16; ++i) {
        u32 d = o[m][i >> 2][i & 3];
        d = (d & 0xffffu) ? d : d >> 16;
        const u64 t = (u64)d << ((a & 2u) * 8u);
        const u32 w = cur | (u32)t;
        *(LAS u32*)(scr + (a & ~3u)) = w;
        const u32 an = off + 2u * __builtin_popcount(mk[m] & (((2u << i) - 1u) * 0x10001u));
        cur = ((an ^ a) & ~3u) ? (u32)(t >> 32) : w;
        a = an;
      }
      off = a;
    }
    *(LAS u32*)(scr + (a & ~3u)) = cur;
    asm volatile("" ::: "memory");
    for (u32 p = 0; p < total; p += 1024u) store16(*(const LAS u32x4*)(scr + p + lane * 16u), rb2, p + lane * 16u < total ? lane * 16u : 0x80000000u, 1024u + p);
  }
  asm volatile("s_waitcnt vmcnt(0)" : "+v"(qi) :: "memory");
  const u32 slot = ((u32)__builtin_amdgcn_readfirstlane(qi) - (call - 1u) * OUNITS) & 255u;
  flag_store(pick(P.ws, (g4 ^ 4u) | j) + OFF_OQ + (r & 3u) * 2048u + slot * 8u, ((u64)call << 32) | uid | (total << 8) | (raw ? 1u << 22 : 0u), 1u);
  STAMP(3);
}

__global__ __launch_bounds__(512, 2) void ag_gemm_kernel(Params P) {
  extern __shared__ __attribute__((aligned(16))) char smem[];
  LAS char* lds = (LAS char*)smem;
  const u32 lane = threadIdx.x & 63u, wid = __builtin_amdgcn_readfirstlane(threadIdx.x >> 6);
  char* const ws = pick(P.ws, P.rank);
  STAMP(0);
  u32 bid = blockIdx.x;
  asm volatile("" : "+s"(bid));
  char* const cptr = ws + OFF_CALL + bid * 128u + wid * 4u;
  const u32 callv = ld_agent(cptr);
  LAS unsigned char* scr = (LAS unsigned char*)lds + wid * WAVE_SCRATCH;
  bool skip = false;
  u32 li = 7, lidx = 0, pkw = 0;
  if (bid < NCOMM) {
    const u32 h = (bid >> 2) & 1u, a = (bid >> 3) * 4u + (bid & 3u), nl = 3u + h;
    if (a < nl * NCL) { li = h * 3u + (a + h * (a >> 2)) % nl; lidx = a / nl; }
    else pkw = h * (NCOMM / 2 - 3 * NCL) + a - nl * NCL;
  }
  LAS u32* const ctl = (LAS u32*)(lds + 8 * WAVE_SCRATCH);
  if (bid < NCOMM) {
    ctl[lane & 1u] = 0;
    asm volatile("s_waitcnt lgkmcnt(0)\n\ts_barrier" ::: "memory");
  }
  if (li < 7u) {
    const u32 call = __builtin_amdgcn_readfirstlane(callv) + 1u;
    __hip_atomic_store((GAS u32*)cptr, call, __ATOMIC_RELAXED, __HIP_MEMORY_SCOPE_AGENT);
    if (!skip) puller(P, ws, call, li, lidx, ctl, scr, lane, wid);
    out_puller(P, ws, call, li >= 3u ? li - 3u : (li * NCL + lidx + NPKWG) & 3u, ctl, scr, lane, wid);
    return;
  }
  const bool isg = bid >= NCOMM;
  const u32 q0 = isg ? wid * NGEMM + (bid - NCOMM) : GPACK + wid * NPKWG + pkw;
  const u32 q1 = isg ? (wid < AG_G2 ? q0 + 8u * NGEMM : (u32)NUNITS) : q0 + NPK;
  for (u32 i = isg ? wid : (u32)AG_PDELAY; i; --i) __builtin_amdgcn_s_sleep(AG_GDELAY);
  const u32 call = packer(P, ws, callv, skip ? (u32)NUNITS : q0, q1, isg ? (u32)NUNITS : (u32)NPK, scr, lane, wid);
  __hip_atomic_store((GAS u32*)cptr, call, __ATOMIC_RELAXED, __HIP_MEMORY_SCOPE_AGENT);
  if (!isg) {
    out_puller(P, ws, call, pkw & 3u, ctl, scr, lane, wid);
    return;
  }
  asm volatile("s_waitcnt lgkmcnt(0)\n\ts_barrier" ::: "memory");
  STAMP(7);
  if (wid < 2u) gemm_wave<3, 3>(P, ws, call, bid - NCOMM, lds, lane, wid);
  else if (wid < 6u) gemm_wave<2, 4>(P, ws, call, bid - NCOMM, lds, lane, wid);
  else gemm_wave<3, 2>(P, ws, call, bid - NCOMM, lds, lane, wid);
}

}

extern "C" void kernel_launch(int rank, int nranks, ncclComm_t comm, hipStream_t stream,
                              void* const* d_in, const int* in_sizes, int n_in,
                              void* d_out, int out_size, const int* in_dtypes, int out_dtype,
                              void* const* d_out_all, void* const* d_ws_all, size_t ws_bytes) {
  (void)comm; (void)in_dtypes; (void)out_dtype; (void)d_out;
  if (nranks != ag::NR || n_in != 2 || (unsigned)rank >= (unsigned)ag::NR || in_sizes[0] != 1024 * 8192 || in_sizes[1] != 8192 * 1024 ||
      out_size != 8192 * 1024 || ws_bytes < ag::WS_NEED) {
    fprintf(stderr, "ag_gemm: unexpected problem shape\n");
    return;
  }
  ag::Params p;
  p.x = (const char*)d_in[0];
  p.w = (const char*)d_in[1];
  for (int i = 0; i < ag::NR; ++i) {
    const int phys = (AG_ORDER >> (4 * i)) & 7;
    p.ws[i] = (char*)d_ws_all[phys];
    p.out[i] = (char*)d_out_all[phys];
    if (phys == rank) p.rank = i;
  }
  p.pad = 0;
  hipError_t e = hipFuncSetAttribute((const void*)ag::ag_gemm_kernel, hipFuncAttributeMaxDynamicSharedMemorySize, ag::LDS_BYTES);
  if (e != hipSuccess) { fprintf(stderr, "ag_gemm: %s\n", hipGetErrorString(e)); return; }
  hipLaunchKernelGGL(ag::ag_gemm_kernel, dim3(ag::NWG), dim3(512), ag::LDS_BYTES, stream, p);
  e = hipGetLastError();
  if (e != hipSuccess) fprintf(stderr, "ag_gemm launch: %s\n", hipGetErrorString(e));
}
```

```cpp
#include <hip/hip_runtime.h>
#include <rccl/rccl.h>
#include <cstdio>

namespace ag {

typedef unsigned u32;
typedef unsigned long long u64;
typedef unsigned u32x4 __attribute__((ext_vector_type(4)));
typedef unsigned short u16x2 __attribute__((ext_vector_type(2)));
typedef short bf16x8 __attribute__((ext_vector_type(8)));
typedef float f32x4 __attribute__((ext_vector_type(4)));
typedef __amdgpu_buffer_rsrc_t rsrc_t;
#define LAS __attribute__((address_space(3)))
#define GAS __attribute__((address_space(1)))

#define AG_AU 21
#define AG_NCL 6
#define AG_TOK 12
#define AG_TOK_IN 12
#define AG_G2 8
#define AG_OTOK 2
#define AG_WLEAD 4
#define AG_LA 4
#define AG_OTOK_IN 2
#define AG_OTOK_PK 2
#define AG_NODRAIN 0
#define AG_LAZY2 1
#define AG_EARLY 4
#define AG_GDELAY 36
#define AG_PDELAY 0
#define AG_ORDER 0x75416320u
#define AG_NEAR 1
#define AG_CTMAP 1
#define AG_FORCE_RAW 0
#define AG_GEMM_AUX 17

constexpr int NR = 8;
constexpr int AU = AG_AU;
constexpr int AWAY = 32 - AU;
constexpr int NRT = 26, NEARLY = 13;
constexpr int UPT = 5;
constexpr int NKC = 64, NKS = 256;
constexpr int EARLY_UNITS = AU + 4 * AWAY, LATE_UNITS = 3 * AU;
static_assert(EARLY_UNITS <= NEARLY * UPT && LATE_UNITS <= (NRT - NEARLY) * UPT, "row split does not fit the tiles");
static_assert((EARLY_UNITS + UPT - 1) % UPT >= 2 && (LATE_UNITS + UPT - 1) % UPT >= 2, "a short tile keeps at least five fragments");
constexpr int NOUT = AWAY + 8;

constexpr int NCOMM = 48, NGEMM = 208, NWG = NCOMM + NGEMM;
constexpr int NCL = AG_NCL;
constexpr int NPKWG = NCOMM - 7 * NCL;
static_assert(4 * NCL <= NCOMM / 2 && (NCOMM - 4 * NCL) % 4 == 0, "puller placement");
constexpr int NPK = NPKWG * 8;
constexpr u32 GPACK = (8 + AG_G2) * NGEMM;
constexpr int NUNITS = 4096;
constexpr u32 WLEAD = AG_WLEAD;

constexpr u32 A_STEP = 10240, B_STEP = 16384;
constexpr u32 A_TILE = A_STEP * NKS, B_TILE = B_STEP * NKS;
constexpr u32 OFF_XU = 0;
constexpr u32 OFF_WU = OFF_XU + NRT * A_TILE;
constexpr u32 FRAG_BYTES = OFF_WU + 8 * B_TILE;
constexpr u32 SLOT = 8192, PK_BYTES = NUNITS * SLOT;
constexpr u32 OFF_PK = FRAG_BYTES;
constexpr u32 OSLOT = 14336, OUNITS = NRT * 8;
constexpr u32 OFF_OB = OFF_PK + 2 * PK_BYTES;
constexpr u32 OFF_G = OFF_OB + 4 * OUNITS * OSLOT;
constexpr u32 G_BYTES = 2 * NR * NUNITS * 8;
constexpr u32 OFF_CNT = OFF_G + G_BYTES;
constexpr u32 OFF_OQ = OFF_CNT + 16384;
constexpr u32 OFF_TKT = OFF_OQ + 4 * 2048;
constexpr u32 OFF_TMO = OFF_TKT + 2048;
constexpr u32 OFF_CALL = OFF_TMO + 128;
constexpr u32 OFF_DBG = OFF_CALL + NWG * 128;
constexpr u32 OFF_DBG2 = OFF_DBG + NWG * 8 * 64;
constexpr u32 OFF_DBG3 = OFF_DBG2 + 65536;
constexpr u32 OFF_DBG4 = OFF_DBG3 + 131072;
constexpr u32 WS_NEED = OFF_DBG4 + 262144;

constexpr u32 SLOTB = A_STEP + B_STEP, NSLOT = AG_LA + 1;
constexpr int LDS_BYTES = NSLOT * SLOTB;
constexpr int WAVE_SCRATCH = 13312;
static_assert(8 * WAVE_SCRATCH + 64 <= LDS_BYTES, "scratch");

constexpr u32 SPARE_CNT = 4000;
constexpr u64 DEADLINE = 50000000ull;
constexpr int AUX_SYS = 17;

struct Params {
  const char* x;
  const char* w;
  char* ws[NR];
  char* out[NR];
  int rank;
  int pad;
};

#define STAMP(i) do {} while (0)
#define STAMPC(i) do {} while (0)
#define STAMP2(i) do {} while (0)
#define STAMPQ(i) do {} while (0)

__device__ __forceinline__ rsrc_t make_rsrc(const void* p, u32 bytes) {
  const u64 a = (u64)p;
  const u32 lo = __builtin_amdgcn_readfirstlane((u32)a), hi = __builtin_amdgcn_readfirstlane((u32)(a >> 32));
  return __builtin_amdgcn_make_buffer_rsrc((void*)(((u64)hi << 32) | lo), 0, bytes, 0x00020000);
}
__device__ __forceinline__ char* pick(char* const (&t)[NR], u32 i) {
  char* p = t[0];
#pragma unroll
  for (u32 j = 1; j < NR; ++j) p = (i == j) ? t[j] : p;
  return p;
}
__device__ __forceinline__ u32 ld_agent(const char* p) { return __hip_atomic_load((GAS u32*)p, __ATOMIC_RELAXED, __HIP_MEMORY_SCOPE_AGENT); }
__device__ __forceinline__ u32 ld_sys(const char* p) { return __hip_atomic_load((GAS u32*)p, __ATOMIC_RELAXED, __HIP_MEMORY_SCOPE_SYSTEM); }
__device__ __forceinline__ void store16(u32x4 v, rsrc_t rs, u32 voff, u32 soff) {
  __builtin_amdgcn_raw_buffer_store_b128(v, rs, voff, soff, AUX_SYS);
  asm volatile("s_nop 1" :: "v"(v) : "memory");
}
__device__ __forceinline__ void count_in(const char* base, u32 idx) {
  u64 keep;
  asm volatile("s_mov_b64 %0, exec\n\ts_mov_b64 exec, 1\n\tglobal_atomic_add %1, %2, %3\n\ts_mov_b64 exec, %0"
               : "=&s"(keep) : "v"(idx * 4u), "v"(1u), "s"(base) : "memory");
}
#define STAMP_CNT(ws, idx) do {} while (0)
__device__ __forceinline__ void flag_store(char* p, u64 v, u64 mask) {
  u64 keep;
  asm volatile("s_mov_b64 %0, exec\n\ts_mov_b64 exec, %3\n\tglobal_store_dwordx2 %1, %2, off sc0 sc1\n\ts_mov_b64 exec, %0"
               : "=&s"(keep) : "v"(p), "v"(v), "s"(mask) : "memory");
}
__device__ __forceinline__ void drain_vm() { asm volatile("s_waitcnt vmcnt(0)" ::: "memory"); }

__device__ __forceinline__ bool give_up(char* ws, u32 call, u64 t0) {
  const u32 t = __builtin_amdgcn_readfirstlane(ld_agent(ws + OFF_TMO));
  if (t == call) return true;
  if (__builtin_amdgcn_s_memrealtime() - t0 > DEADLINE) {
    __hip_atomic_store((GAS u32*)(ws + OFF_TMO), call, __ATOMIC_RELAXED, __HIP_MEMORY_SCOPE_AGENT);
    return true;
  }
  return false;
}

__device__ __forceinline__ u32 wave_sum(u32 v) {
  v += __builtin_amdgcn_update_dpp(0u, v, 0x111, 0xf, 0xf, true);
  v += __builtin_amdgcn_update_dpp(0u, v, 0x112, 0xf, 0xf, true);
  v += __builtin_amdgcn_update_dpp(0u, v, 0x114, 0xf, 0xf, true);
  v += __builtin_amdgcn_update_dpp(0u, v, 0x118, 0xf, 0xf, true);
  return __builtin_amdgcn_readlane(v, 15) + __builtin_amdgcn_readlane(v, 31) + __builtin_amdgcn_readlane(v, 47) + __builtin_amdgcn_readlane(v, 63);
}
__device__ __forceinline__ u32 wave_scan(u32 v) {
  v += __builtin_amdgcn_update_dpp(0u, v, 0x111, 0xf, 0xf, true);
  v += __builtin_amdgcn_update_dpp(0u, v, 0x112, 0xf, 0xf, true);
  v += __builtin_amdgcn_update_dpp(0u, v, 0x114, 0xf, 0xf, true);
  v += __builtin_amdgcn_update_dpp(0u, v, 0x118, 0xf, 0xf, true);
  v += __builtin_amdgcn_update_dpp(0u, v, 0x142, 0xa, 0xf, false);
  v += __builtin_amdgcn_update_dpp(0u, v, 0x143, 0xc, 0xf, false);
  return v;
}

__device__ __forceinline__ u32 quad_swz(u32 rq) { return (0x78u >> (2 * rq)) & 3u; }

struct Dest { u32 base, s1, s0, cnt; };
__device__ __forceinline__ Dest dest_x(u32 ux, u32 kc) {
  const u32 tile = ux / UPT;
  return Dest{OFF_XU + tile * A_TILE + 4 * kc * A_STEP + (ux - tile * UPT) * 2048u, 1024u, A_STEP, tile * 64 + kc};
}
__device__ __forceinline__ Dest dest_w(u32 ct, u32 kc, u32 ks4, u32 half) {
  return Dest{OFF_WU + ct * B_TILE + (4 * kc + ks4) * B_STEP + half * 8192u, 4096u, 1024u, (NRT + ct) * 64 + kc};
}
__device__ __forceinline__ void store_frags(const u32x4 (&v)[8], rsrc_t rf, const Dest& d, u32 lane) {
#pragma unroll
  for (int f = 0; f < 8; ++f) store16(v[f], rf, lane * 16, d.base + (f >> 2) * d.s1 + (f & 3) * d.s0);
}

constexpr u32 RAW = 1u << 31;
constexpr u32 S_LIST = 0, S_TAIL = 2304, S_TAB = 3584;
struct Packed { u32x4 mq[4], p01, tail; };
__device__ __forceinline__ u32 groups_of(u32 info) { return (((info >> 8) & 4095u) + 31u) >> 5; }

__device__ __forceinline__ void build_tables(LAS unsigned char* scr, u32 lane) {
  u32 ct = 0x0c0c0c0cu, xt = 0x07060504u, k = 0;
#pragma unroll
  for (u32 b = 0; b < 4; ++b) {
    const u32 on = 0u - ((lane >> b) & 1u);
    ct ^= on & ((0x0cu ^ b) << (8u * k));
    xt ^= on & (((4u + b) ^ k) << (8u * b));
    k -= on;
  }
  *(LAS u32*)(scr + S_TAB + (lane & 31u) * 4u) = (lane & 16u) ? xt : ct;
}

template <bool TO_LIST>
__device__ __forceinline__ void walk_spelled(u32 m, u32 off, LAS unsigned char* cells, LAS unsigned char* list, u32 flip) {
  while (m != 0) {
    const u32 p = __builtin_ctz(m);
    m &= m - 1;
    LAS unsigned char* cell = cells + (p & 7u) * 4u + (p >> 3);
    if (TO_LIST) list[off] = *cell ^ flip; else *cell = list[off];
    ++off;
  }
}

__device__ __forceinline__ u32 encode_unit(Packed& k, const u32x4 (&v)[8], LAS unsigned char* scr, u32 lane) {
  const u32x4 zero = {0u, 0u, 0u, 0u};
  *(LAS u32x4*)(scr + S_LIST + lane * 32u) = zero;
  *(LAS u32x4*)(scr + S_LIST + lane * 32u + 16u) = zero;
  u32 E[16], esum = 0;
#pragma unroll
  for (int j = 0; j < 16; ++j) {
    const u32 d0 = v[j >> 1][(j & 1) * 2], d1 = v[j >> 1][(j & 1) * 2 + 1];
    const u32 r0 = __builtin_amdgcn_alignbit(d0, d0, 31), r1 = __builtin_amdgcn_alignbit(d1, d1, 31);
    k.mq[j >> 2][j & 3] = __builtin_amdgcn_perm(r1, r0, 0x06040200u);
    E[j] = __builtin_amdgcn_perm(r1, r0, 0x07050301u);
    esum = __builtin_amdgcn_sad_u8(E[j], 0u, esum);
  }
  u32 base = (wave_sum(esum) + 8028u) >> 12;
  base = base < 8u ? 8u : (base > 254u ? 254u : base);
  const u16x2 b2 = {(unsigned short)base, (unsigned short)base}, t2 = {(unsigned short)(base + 1u), (unsigned short)(base + 1u)};
  u32 P[2][2] = {{0, 0}, {0, 0}};
#pragma unroll
  for (int j = 0; j < 16; ++j) {
    const u16x2 ev = __builtin_bit_cast(u16x2, E[j] & 0x00ff00ffu), od = __builtin_bit_cast(u16x2, E[j]) >> (u16x2){8, 8};
    const u32 ce = __builtin_bit_cast(u32, __builtin_elementwise_min((u16x2)(b2 - ev), (u16x2){3, 3}));
    const u32 co = __builtin_bit_cast(u32, __builtin_elementwise_min((u16x2)(b2 - od), (u16x2){3, 3}));
    const u32 c = ce | (co << 8);
    P[0][j >> 3] |= (c & 0x01010101u) << (j & 7);
    P[1][j >> 3] |= ((c >> 1) & 0x01010101u) << (j & 7);
  }
  k.p01 = (u32x4){P[0][0], P[0][1], P[1][0], P[1][1]};
  k.tail = zero;
  const u32 mk[2] = {P[0][0] & P[1][0], P[0][1] & P[1][1]};
  u32 fl[16], sel[16];
#pragma unroll
  for (int j = 0; j < 16; ++j) {
    fl[j] = (mk[j >> 3] >> (j & 7)) & 0x01010101u;
    sel[j] = *(const LAS u32*)(size_t)__builtin_amdgcn_udot4(fl[j], 0x20100804u, (u32)(size_t)(scr + S_TAB), false);
  }
  __builtin_amdgcn_sched_barrier(0);
  const u32 c = __builtin_popcount(mk[0]) + __builtin_popcount(mk[1]), incl = wave_scan(c);
  const u32 n2 = __builtin_amdgcn_readlane(incl, 63);
  if (n2 > 2044u || AG_FORCE_RAW) return RAW;
  const u32 flip = (base + 1u) * 0x01010101u;
  u32 a = (u32)(size_t)(scr + S_LIST) + incl - c, cur = 0;
#pragma unroll
  for (int j = 0; j < 16; ++j) {
    const u64 t = (u64)__builtin_amdgcn_perm(0u, E[j] ^ flip, sel[j]) << ((a & 3u) * 8u);
    const u32 w = cur | (u32)t, an = __builtin_amdgcn_sad_u8(fl[j], 0u, a);
    __hip_atomic_fetch_or((LAS u32*)(size_t)(a & ~3u), w, __ATOMIC_RELAXED, __HIP_MEMORY_SCOPE_WAVEFRONT);
    cur = ((an ^ a) & 4u) ? (u32)(t >> 32) : w;
    a = an;
  }
  __hip_atomic_fetch_or((LAS u32*)(size_t)(a & ~3u), cur, __ATOMIC_RELAXED, __HIP_MEMORY_SCOPE_WAVEFRONT);
  asm volatile("" ::: "memory");
  const u32x4 l[2] = {*(const LAS u32x4*)(scr + S_LIST + lane * 32u), *(const LAS u32x4*)(scr + S_LIST + lane * 32u + 16u)};
  u32 Q[3] = {0, 0, 0};
#pragma unroll
  for (int s = 0; s < 8; ++s) {
    const u32 x = l[s >> 2][s & 3] ^ flip;
    const u16x2 ev = __builtin_bit_cast(u16x2, x & 0x00ff00ffu), od = __builtin_bit_cast(u16x2, x) >> (u16x2){8, 8};
    const u32 ce = __builtin_bit_cast(u32, __builtin_elementwise_min(__builtin_elementwise_sub_sat((u16x2)(t2 - ev), (u16x2){3, 3}), (u16x2){7, 7}));
    const u32 co = __builtin_bit_cast(u32, __builtin_elementwise_min(__builtin_elementwise_sub_sat((u16x2)(t2 - od), (u16x2){3, 3}), (u16x2){7, 7}));
    const u32 c2 = ce | (co << 8);
    Q[0] |= (c2 & 0x01010101u) << s;
    Q[1] |= ((c2 >> 1) & 0x01010101u) << s;
    Q[2] |= ((c2 >> 2) & 0x01010101u) << s;
  }
#pragma unroll
  for (int i = 0; i < 3; ++i) *(LAS u32*)(scr + S_TAIL + lane * 12u + i * 4u) = Q[i];
  const u32 m3 = Q[0] & Q[1] & Q[2], c3 = __builtin_popcount(m3), incl3 = wave_scan(c3);
  const u32 n3 = __builtin_amdgcn_readlane(incl3, 63);
  if (n3 > 256u) return RAW;
  asm volatile("" ::: "memory");
  walk_spelled<true>(m3, incl3 - c3, scr + S_LIST + lane * 32u, scr + S_TAIL + ((n2 + 31u) >> 5) * 12u, base + 1u);
  asm volatile("" ::: "memory");
  k.tail = *(const LAS u32x4*)(scr + S_TAIL + lane * 16u);
  return base | (n2 << 8) | (n3 << 20);
}

__device__ __forceinline__ void store_unit(const Packed& k, const u32x4 (&v)[8], u32 info, rsrc_t rp, u32 soff, u32 lane) {
  if (info & RAW) {
#pragma unroll
    for (int f = 0; f < 8; ++f) store16(v[f], rp, lane * 16, soff + f * 1024);
    return;
  }
#pragma unroll
  for (int i = 0; i < 4; ++i) store16(k.mq[i], rp, lane * 16, soff + i * 1024);
  store16(k.p01, rp, lane * 16, soff + 4096);
  store16(k.tail, rp, lane * 16u < groups_of(info) * 12u + (info >> 20) ? lane * 16u : 0x80000000u, soff + 5120);
}

__device__ __forceinline__ void fetch_packed(Packed& k, rsrc_t rp, u32 soff, u32 info, u32 lane) {
  k.tail = __builtin_amdgcn_raw_buffer_load_b128(rp, lane * 16u < groups_of(info) * 12u + (info >> 20) ? lane * 16u : 0x80000000u, soff + 5120, AUX_SYS);
  k.p01 = __builtin_amdgcn_raw_buffer_load_b128(rp, lane * 16, soff + 4096, AUX_SYS);
#pragma unroll
  for (int i = 0; i < 4; ++i) k.mq[i] = __builtin_amdgcn_raw_buffer_load_b128(rp, lane * 16, soff + i * 1024, AUX_SYS);
}

__device__ __forceinline__ void decode_unit(u32x4 (&v)[8], const Packed& k, u32 info, LAS unsigned char* scr, u32 lane) {
  const u32 base = info & 255u, g2 = groups_of(info);
  *(LAS u32x4*)(scr + S_TAIL + lane * 16u) = k.tail;
  asm volatile("" ::: "memory");
  const LAS u32* const pq = (const LAS u32*)(scr + S_TAIL + lane * 12u);
  const u32 live = lane < g2 ? ~0u : 0u, Q0 = pq[0] & live, Q1 = pq[1] & live, Q2 = pq[2] & live;
  LAS unsigned char* const spell = scr + S_TAIL + g2 * 12u;
  const u32 b4 = base * 0x01010101u, lut0 = b4 - 0x05040300u + 1u, lut1 = b4 - 0x00080706u;
  const u32 P0[2] = {k.p01.x, k.p01.y}, P1[2] = {k.p01.z, k.p01.w}, mk[2] = {P0[0] & P1[0], P0[1] & P1[1]};
  const u32 c = __builtin_popcount(mk[0]) + __builtin_popcount(mk[1]);
  u32 a[17], sel[16];
  a[0] = (u32)(size_t)(scr + S_LIST) + wave_scan(c) - c;
#pragma unroll
  for (int j = 0; j < 16; ++j) {
    const u32 f = (mk[j >> 3] >> (j & 7)) & 0x01010101u;
    sel[j] = *(const LAS u32*)(size_t)__builtin_amdgcn_udot4(f, 0x20100804u, (u32)(size_t)(scr + S_TAB + 64u), false);
    a[j + 1] = __builtin_amdgcn_sad_u8(f, 0u, a[j]);
  }
  __builtin_amdgcn_sched_barrier(0);
  u32x4 l[2];
#pragma unroll
  for (int s = 0; s < 8; ++s) {
    const u32 c2 = ((Q0 >> s) & 0x01010101u) | (((Q1 >> s) & 0x01010101u) << 1) | (((Q2 >> s) & 0x01010101u) << 2);
    l[s >> 2][s & 3] = __builtin_amdgcn_perm(lut1, lut0, c2);
  }
  const u32 m3 = Q0 & Q1 & Q2, c3 = __builtin_popcount(m3), off3 = wave_scan(c3) - c3;
  const bool many = __builtin_amdgcn_ballot_w64(c3 > 8u) != 0;
  if (!many) {
    const LAS u32* const ps = (const LAS u32*)(spell + (off3 & ~3u));
    const u32 d0 = ps[0], d1 = ps[1], d2 = ps[2];
    const u32 l0 = __builtin_amdgcn_alignbyte(d1, d0, off3), l1 = __builtin_amdgcn_alignbyte(d2, d1, off3);
    u32 t = m3 - ((m3 >> 1) & 0x55555555u);
    t = (t & 0x33333333u) + ((t >> 2) & 0x33333333u);
    t = (t + (t >> 4)) & 0x0f0f0f0fu;
    u32 rank = t << 8;
    rank += rank << 8;
    rank += (t << 8) << 16;
#pragma unroll
    for (int s = 0; s < 8; ++s) {
      const u32 e = (m3 >> s) & 0x01010101u;
      u32 ff = e << 8;
      asm("" : "+v"(ff));
      ff -= e;
      l[s >> 2][s & 3] = (__builtin_amdgcn_perm(l1, l0, rank) & ff) | (l[s >> 2][s & 3] & ~ff);
      rank += e;
    }
  }
  *(LAS u32x4*)(scr + S_LIST + lane * 32u) = l[0];
  *(LAS u32x4*)(scr + S_LIST + lane * 32u + 16u) = l[1];
  if (many) walk_spelled<false>(m3, off3, scr + S_LIST + lane * 32u, spell, 0u);
  asm volatile("" ::: "memory");
  u32 w0[16], w1[16];
#pragma unroll
  for (int j = 0; j < 16; ++j) {
    const LAS u32* const pw = (const LAS u32*)(size_t)(a[j] & ~3u);
    w0[j] = pw[0];
    w1[j] = pw[1];
  }
  __builtin_amdgcn_sched_barrier(0);
#pragma unroll
  for (int j = 0; j < 16; ++j) {
    const int s = j & 7, wd = j >> 3;
    const u32 c1 = ((P0[wd] >> s) & 0x01010101u) | (((P1[wd] >> s) & 0x01010101u) << 1);
    const u32 e = __builtin_amdgcn_perm(b4 - c1, __builtin_amdgcn_alignbyte(w1[j], w0[j], a[j]), sel[j]), m = k.mq[j >> 2][j & 3];
    const u32 r0 = __builtin_amdgcn_perm(e, m, 0x05010400u), r1 = __builtin_amdgcn_perm(e, m, 0x07030602u);
    v[j >> 1][(j & 1) * 2] = __builtin_amdgcn_alignbit(r0, r0, 1);
    v[j >> 1][(j & 1) * 2 + 1] = __builtin_amdgcn_alignbit(r1, r1, 1);
  }
}

__device__ __forceinline__ void load_unit(u32x4 (&s)[8], rsrc_t rx, rsrc_t rw, u32 q, u32 vx, u32 vw) {
  const u32 kc = q >> 6, t = q & 63u;
  if (t < 32u) {
#pragma unroll
    for (int f = 0; f < 8; ++f) s[f] = __builtin_amdgcn_raw_buffer_load_b128(rx, vx, (32u * t + 16u * (f >> 2)) * 16384u + kc * 256u + (f & 3) * 64u, 0);
  } else {
#pragma unroll
    for (int i = 0; i < 8; ++i) s[i] = __builtin_amdgcn_raw_buffer_load_b128(rw, vw, (128u * kc + 32u * ((t - 32u) >> 3) + i) * 2048u + (t & 7u) * 256u, 0);
  }
}

__device__ __forceinline__ u32 packer(const Params& P, char* ws, u32 callv, u32 q0, u32 q1, u32 dq, LAS unsigned char* scr, u32 lane, u32 wid) {
  const rsrc_t rx = make_rsrc(P.x, 1024u * 8192u * 2u), rw = make_rsrc(P.w, 8192u * 1024u * 2u);
  const u32 kq = (lane & 3u) ^ quad_swz(lane >> 4);
  const u32 vx = (lane >> 2) * 16384u + kq * 16u, vw = (lane >> 4) * 8u * 2048u + (lane & 15u) * 16u;
  u32x4 s[8];
  load_unit(s, rx, rw, q0, vx, vw);
  __builtin_amdgcn_sched_barrier(0);
  build_tables(scr, lane);
  const u32 call = __builtin_amdgcn_readfirstlane(callv) + 1u;
  const u32 r = P.rank, g4 = r & 4u, par = call & 1u;
  const rsrc_t rf = make_rsrc(ws, FRAG_BYTES), rp = make_rsrc(ws + OFF_PK + par * PK_BYTES, PK_BYTES);
  const u32 goff = OFF_G + (par * NR + r) * NUNITS * 8;
  char* g_in = pick(P.ws, g4 | ((r + 1u) & 3u));
  char* g_out = pick(P.ws, g4 ^ 4u);
#pragma unroll
  for (u32 j = 1; j < 4; ++j) {
    char* const pi = pick(P.ws, g4 | ((r + 1u + j) & 3u));
    char* const po = pick(P.ws, (g4 ^ 4u) | j);
    g_in = lane == j ? pi : g_in;
    g_out = lane == j ? po : g_out;
  }
  STAMP(6);
  u64 psend = 0;
  char* pgp = g_out;
  u32 pinfo = 0, pcnt = SPARE_CNT;
  auto announce = [&]() {
    flag_store(pgp, ((u64)call << 32) | pinfo, psend);
    if (pcnt != SPARE_CNT) { STAMP_CNT(ws, pcnt); count_in(ws + OFF_CNT, pcnt); }
  };
  for (u32 q = q0; q < NUNITS; q = q1, q1 += dq) {
    const u32 kc = q >> 6, t = q & 63u;
    u32x4 v[8];
    Dest d;
    bool own;
    u64 send;
    char* gp;
    if (t < 32u) {
#pragma unroll
      for (int f = 0; f < 8; ++f) v[f] = s[f];
      own = t < AU;
      d = dest_x(t, kc);
      send = own ? 7u : 15u;
      gp = own ? g_in : g_out;
    } else {
      const u32 ks4 = (t - 32u) >> 3, cg = t & 7u;
#pragma unroll
      for (int nf = 0; nf < 8; ++nf)
#pragma unroll
        for (int tt = 0; tt < 4; ++tt) v[nf][tt] = __builtin_amdgcn_perm(s[2 * tt + 1][nf >> 1], s[2 * tt][nf >> 1], (nf & 1) ? 0x07060302u : 0x05040100u);
      own = true;
      d = dest_w(cg >> 1, kc, ks4, cg & 1u);
      send = 1u << (cg >> 1);
      gp = g_out;
    }
    if (q == q0) STAMP(4);
    const bool lazy = AG_LAZY2 && dq == NUNITS;
    if (q1 < NUNITS && !lazy) load_unit(s, rx, rw, q1, vx, vw);
    Packed k;
    const u32 info = encode_unit(k, v, scr, lane);
    drain_vm();
#pragma unroll
    for (int f = 0; f < 8; ++f) asm volatile("" : "+v"(s[f]));
    announce();
    store_unit(k, v, info, rp, q * SLOT, lane);
    if (own) store_frags(v, rf, d, lane);
    if (q == q0) STAMP(5);
    psend = send;
    pgp = gp + goff + q * 8u;
    pinfo = info;
    pcnt = own ? d.cnt : SPARE_CNT;
    if (q1 < NUNITS && lazy) {
      drain_vm();
      announce();
      psend = 0;
      pcnt = SPARE_CNT;
      for (u32 i = AG_LAZY2 - 1; i; --i) __builtin_amdgcn_s_sleep(32);
      load_unit(s, rx, rw, q1, vx, vw);
    }
  }
  drain_vm();
  announce();
  STAMP(2);
  return call;
}

__device__ __forceinline__ u32 lds_add(LAS u32* p, u32 v) {
  return __builtin_amdgcn_readfirstlane(__hip_atomic_fetch_add(p, v, __ATOMIC_RELAXED, __HIP_MEMORY_SCOPE_WORKGROUP)) >> 6;
}

__device__ __forceinline__ void puller(const Params& P, char* ws, u32 call, u32 li, u32 lidx, LAS u32* ctl, LAS unsigned char* scr, u32 lane, u32 wid) {
  const u32 r = P.rank, g4 = r & 4u, par = call & 1u;
  const u32 tl = li < 3u ? AG_TOK_IN : AG_TOK, TOK = tl / NCL + (lidx < tl % NCL ? 1u : 0u);
  const bool ing = li < 3u;
  const u32 src = ing ? (g4 | ((r + 1 + li) & 3u)) : ((g4 ^ 4u) | (li - 3u));
  const rsrc_t rf = make_rsrc(ws, FRAG_BYTES), rp = make_rsrc(pick(P.ws, src) + OFF_PK + par * PK_BYTES, PK_BYTES);
  char* const gbase = ws + OFF_G + (par * NR + src) * NUNITS * 8;
  GAS u32* const cnt = (GAS u32*)(ws + OFF_CNT);
  const u32 total = ing ? NKC * AU : NKC * NOUT;
  constexpr u32 NONE = 0xffffffffu;
  GAS u32* const tkt = (GAS u32*)(ws + OFF_TKT + li * 128u);
  const u32 tbase = (call - 1u) * (total + NCL * 8u) * 64u;
  auto draw = [&]() -> u32 { return __hip_atomic_fetch_add(tkt, 1u, __ATOMIC_RELAXED, __HIP_MEMORY_SCOPE_AGENT); };
  auto unit_of = [&](u32 raw_ticket) -> u32 {
    const u32 i = ((u32)__builtin_amdgcn_readfirstlane(raw_ticket) - tbase) >> 6;
    if (i >= total) return NONE;
    if (ing) { const u32 kc = i / AU; return kc * 64u + (i - kc * AU); }
    u32 kc, t;
    if (i < 8u * WLEAD) { kc = i >> 3; t = AWAY + (i & 7u); }
    else {
      const u32 i2 = i - 8u * WLEAD, mid = (NKC - WLEAD) * NOUT;
      if (i2 < mid) { kc = i2 / NOUT; t = i2 - kc * NOUT; kc += t < AWAY ? 0u : WLEAD; }
      else { kc = (i2 - mid) / AWAY; t = i2 - mid - kc * AWAY; kc += NKC - WLEAD; }
    }
    const u32 tw = t - AWAY;
    return kc * 64u + (t < AWAY ? AU + t : 32u + (tw >> 1) * 8u + 2u * (r & 3u) + (tw & 1u));
  };
  auto granule = [&](u32 q) -> u64 { return __hip_atomic_load((GAS u64*)(gbase + (q & (NUNITS - 1)) * 8u), __ATOMIC_RELAXED, __HIP_MEMORY_SCOPE_SYSTEM); };
  auto peek = [&](LAS u32* p) -> u32 { return (u32)(__builtin_amdgcn_readfirstlane(__hip_atomic_load(p, __ATOMIC_RELAXED, __HIP_MEMORY_SCOPE_WORKGROUP)) >> 6); };
  auto take = [&]() { const u32 my = lds_add(ctl, 1u); while (peek(ctl + 2) + TOK <= my) __builtin_amdgcn_s_sleep(1); };
  auto dest_of = [&](u32 q) -> Dest {
    const u32 kc = q >> 6, tq = q & 63u;
    if (tq < 32u) return dest_x(ing ? NEARLY * UPT + li * AU + tq : AU + (li - 3u) * AWAY + (tq - AU), kc);
    return dest_w(4u + (li - 3u), kc, (tq - 32u) >> 3, tq & 1u);
  };
  auto publish = [&](u32 c) { if (c != NONE) { STAMP_CNT(ws, c); count_in(ws + OFF_CNT, c); } };
  build_tables(scr, lane);
  u32 q = unit_of(draw()), pend = NONE;
  u64 gr = granule(q);
  u32 tag = __builtin_amdgcn_readfirstlane((u32)(gr >> 32)), info = __builtin_amdgcn_readfirstlane((u32)gr);
  bool first = true;
#define LAP(a) do {} while (0)
  while (q != NONE) {
    if (tag != call || (!AG_NODRAIN && peek(ctl + 2) + TOK <= peek(ctl))) {
      drain_vm();
      publish(pend);
      pend = NONE;
      if (tag != call) {
        const u64 t0 = __builtin_amdgcn_s_memrealtime();
        for (;;) {
          gr = granule(q);
          tag = __builtin_amdgcn_readfirstlane((u32)(gr >> 32));
          info = __builtin_amdgcn_readfirstlane((u32)gr);
          if (tag == call || give_up(ws, call, t0)) break;
          __builtin_amdgcn_s_sleep(2);
        }
        if (tag != call) break;
      }
    }
    take();
    asm volatile("" ::: "memory");
    if (first) { STAMP(1); first = false; }
    LAP(acc_w);
    const Dest d = dest_of(q);
    const u32 cur = info, soff = q * SLOT;
    u32x4 v[8];
    if (cur & RAW) {
#pragma unroll
      for (int f = 0; f < 8; ++f) v[f] = __builtin_amdgcn_raw_buffer_load_b128(rp, lane * 16, soff + f * 1024, AUX_SYS);
      const u32 nt = draw();
      drain_vm();
      lds_add(ctl + 2, 1u);
      q = unit_of(nt);
      gr = granule(q);
      publish(pend);
      tag = __builtin_amdgcn_readfirstlane((u32)(gr >> 32));
      info = __builtin_amdgcn_readfirstlane((u32)gr);
      store_frags(v, rf, d, lane);
    } else {
      Packed k;
      u32 nt;
      u64 keep;
      asm volatile("s_mov_b64 %1, exec\n\ts_mov_b64 exec, 1\n\tglobal_atomic_add %0, %2, %3, %4 sc0\n\ts_mov_b64 exec, %1"
                   : "=&v"(nt), "=&s"(keep) : "v"(0u), "v"(64u), "s"(tkt) : "memory");
      fetch_packed(k, rp, soff, cur, lane);
      asm volatile("s_waitcnt vmcnt(%0)" :: "i"(AG_EARLY) : "memory");
      lds_add(ctl + 2, 1u);
      asm volatile("s_waitcnt vmcnt(0)" : "+v"(nt), "+v"(k.mq[0]), "+v"(k.mq[1]), "+v"(k.mq[2]), "+v"(k.mq[3]), "+v"(k.p01), "+v"(k.tail) :: "memory");
      LAP(acc_l);
      q = unit_of(nt);
      STAMP_CNT(ws, pend);
      asm volatile("global_load_dwordx2 %0, %2, %3 sc0 sc1\n\ts_mov_b64 %1, exec\n\ts_mov_b64 exec, 1\n\tglobal_atomic_add %4, %5, %6\n\ts_mov_b64 exec, %1"
                   : "=&v"(gr), "=&s"(keep) : "v"((q & (NUNITS - 1)) * 8u), "s"(gbase), "v"((pend == NONE ? SPARE_CNT : pend) * 4u), "v"(1u), "s"(cnt) : "memory");
      decode_unit(v, k, cur, scr, lane);
      __builtin_amdgcn_sched_barrier(0);
      asm volatile("s_waitcnt vmcnt(1)" : "+v"(gr) :: "memory");
      tag = __builtin_amdgcn_readfirstlane((u32)(gr >> 32));
      info = __builtin_amdgcn_readfirstlane((u32)gr);
      __builtin_amdgcn_sched_barrier(0);
      store_frags(v, rf, d, lane);
      LAP(acc_d);
    }
    pend = d.cnt;
  }
  drain_vm();
  publish(pend);
  STAMP(2);
}

constexpr u32 NOROW = 0xffffffffu;
__device__ __forceinline__ u32 frag_row(u32 r, u32 rt, u32 fr) {
  const u32 g4 = r & 4u;
  const bool late = rt >= NEARLY;
  const u32 uc = (late ? rt - NEARLY : rt) * UPT + (fr >> 1);
  u32 src, su;
  if (late) {
    if (uc >= LATE_UNITS) return NOROW;
    const u32 j = uc / AU;
    src = g4 | ((r + 1 + j) & 3u);
    su = uc - j * AU;
  } else if (uc < AU) {
    src = r;
    su = uc;
  } else {
    if (uc >= EARLY_UNITS) return NOROW;
    const u32 j = (uc - AU) / AWAY;
    src = (g4 ^ 4u) | j;
    su = AU + (uc - AU) - j * AWAY;
  }
  return ((AG_ORDER >> (4u * src)) & 7u) * 1024u + su * 32u + (fr & 1u) * 16u;
}
__device__ __forceinline__ u32 wave_frag0(u32 wid) { return (0x74208630u >> (4 * wid)) & 15u; }

__device__ __forceinline__ void out_puller(const Params& P, char* ws, u32 call, u32 j, u32 otok, LAS u32* ctl, LAS unsigned char* scr, u32 lane, u32 wid) {
  const u32 r = P.rank, src = ((r & 4u) ^ 4u) | j;
  char* const ob = pick(P.ws, src) + OFF_OB + (r & 3u) * (OUNITS * OSLOT);
  const rsrc_t ro = make_rsrc(pick(P.out, r), 8192u * 1024u * 2u);
  GAS u32* const tkt = (GAS u32*)(ws + OFF_TKT + (7u + j) * 128u);
  const u32 tbase = (call - 1u) * (OUNITS + NCOMM * 2u) * 64u;
  const u32 vo = (lane >> 4) * 4u * 2048u + (lane & 15u) * 16u;
  for (;;) {
    for (;;) {
      if ((u32)(__builtin_amdgcn_readfirstlane(__hip_atomic_load(ctl + 1, __ATOMIC_RELAXED, __HIP_MEMORY_SCOPE_WORKGROUP)) >> 6) < otok) {
        if (lds_add(ctl + 1, 1u) < otok) break;
        lds_add(ctl + 1, ~0u);
      }
      __builtin_amdgcn_s_sleep(2);
    }
    const u32 k = ((u32)__builtin_amdgcn_readfirstlane(__hip_atomic_fetch_add(tkt, 1u, __ATOMIC_RELAXED, __HIP_MEMORY_SCOPE_AGENT)) - tbase) >> 6;
    if (k >= OUNITS) { lds_add(ctl + 1, ~0u); break; }
    STAMPQ(0);
    const u64 t0 = __builtin_amdgcn_s_memrealtime();
    u32 tag, info;
    for (u32 spins = 0;; ++spins) {
      const u64 gr = __hip_atomic_load((GAS u64*)(ws + OFF_OQ + j * 2048u + k * 8u), __ATOMIC_RELAXED, __HIP_MEMORY_SCOPE_SYSTEM);
      tag = __builtin_amdgcn_readfirstlane((u32)(gr >> 32));
      info = __builtin_amdgcn_readfirstlane((u32)gr);
      if (tag == call || ((spins & 7u) == 7u && give_up(ws, call, t0))) break;
      __builtin_amdgcn_s_sleep(2);
    }
    if (tag != call) {
      *(GAS unsigned short*)(pick(P.out, r) + (size_t)((AG_ORDER >> (4u * r)) & 7u) * 1024u * 2048u) = 0x7fc0;
      lds_add(ctl + 1, ~0u);
      break;
    }
    asm volatile("" ::: "memory");
    STAMPQ(1);
    const u32 uid = (info & 255u) < OUNITS ? (info & 255u) : 0u, sw = uid & 7u, mf = sw - 2u < 4u ? 2u : 3u;
    u32 total = (info >> 8) & 0x3fffu;
    total = total < 12544u ? total : 12544u;
    const bool raw = info >> 22;
    const rsrc_t rs = make_rsrc(ob + uid * OSLOT, 1024u + ((total + 15u) & ~15u));
    u32 mk[3];
#pragma unroll
    for (u32 m = 0; m < 3; ++m) mk[m] = __builtin_amdgcn_raw_buffer_load_b32(rs, (m < mf && !raw) ? lane * 4u : 0x80000000u, m * 256u, AUX_SYS);
    for (u32 p = 0; p < total; p += 1024u) __builtin_amdgcn_raw_ptr_buffer_load_lds(rs, scr + p, 16, lane * 16u, 1024u + p, 0, AUX_SYS);
    asm volatile("s_waitcnt vmcnt(0)" : "+v"(mk[0]), "+v"(mk[1]), "+v"(mk[2]) :: "memory");
    lds_add(ctl + 1, ~0u);
    STAMPQ(2);
    const u32 len = 2u * ((__builtin_popcount(mk[0]) + __builtin_popcount(mk[1]) + __builtin_popcount(mk[2])) & ~1u) + 4u;
    u32 off = wave_scan(len) - len;
    const u32 col = j * 512u + (sw >> 2) * 256u, f0 = wave_frag0(sw);
    for (u32 m = 0; m < mf; ++m) {
      const u32 w = m == 0 ? mk[0] : (m == 1 ? mk[1] : mk[2]);
      u32x4 o[4];
      if (raw) {
#pragma unroll
        for (int e = 0; e < 4; ++e) o[e] = *(const LAS u32x4*)(scr + (m * 4u + e) * 1024u + lane * 16u);
      } else {
        u32 a[16], w0[16], w1[16];
#pragma unroll
        for (int i = 0; i < 16; ++i) {
          a[i] = (u32)(size_t)scr + off + 2u * __builtin_popcount(w & (((1u << i) - 1u) * 0x10001u));
          const LAS u32* const pw = (const LAS u32*)(size_t)(a[i] & ~3u);
          w0[i] = pw[0];
          w1[i] = pw[1];
        }
        __builtin_amdgcn_sched_barrier(0);
#pragma unroll
        for (int i = 0; i < 16; ++i) {
          const u32 d = __builtin_amdgcn_alignbyte(w1[i], w0[i], a[i]);
          const u32 lo = (u32)((int)(w << (31 - i)) >> 31), hi = (u32)((int)(w << (15 - i)) >> 31);
          o[i >> 2][i & 3] = ((d & lo) | ((d << 16) & ~lo)) & (hi | 0xffffu);
        }
        off += 2u * __builtin_popcount(w);
      }
      const u32 row0 = frag_row(src, uid >> 3, f0 + m);
      if (row0 != NOROW) {
#pragma unroll
        for (int e = 0; e < 4; ++e) store16(o[e], ro, vo, (row0 + e) * 2048u + col);
      }
    }
    STAMPQ(3);
  }
  STAMP(3);
}

__device__ __forceinline__ void gate(int c, int& ready, const char* cnt, u32 target, char* ws, u32 call) {
  if (c < ready) return;
  const u64 t0 = __builtin_amdgcn_s_memrealtime();
  for (u32 spins = 0;; ++spins) {
    const u64 m = __builtin_amdgcn_ballot_w64((int)(ld_agent(cnt) - target) >= 0);
    ready = ~m ? __builtin_ctzll(~m) : 64;
    if (c < ready) break;
    if ((spins & 7u) == 7u && give_up(ws, call, t0)) { ready = 64; break; }
    __builtin_amdgcn_s_sleep(3);
  }
  asm volatile("" ::: "memory");
}

__device__ __forceinline__ u32 relu_pk(float lo, float hi) {
  u32 r;
  asm("v_cvt_pk_bf16_f32 %0, %1, %2\n\tv_pk_max_i16 %0, %0, 0" : "=v"(r) : "v"(lo), "v"(hi));
  return r;
}

__device__ __forceinline__ void mfma_acc(f32x4& c, const bf16x8& a, const bf16x8& b) {
  asm volatile("v_mfma_f32_16x16x32_bf16 %0, %1, %2, %0" : "+v"(c) : "v"(a), "v"(b));
}
template <int N> __device__ __forceinline__ void wait_bar() { asm volatile("s_waitcnt vmcnt(%0)\n\ts_barrier" :: "i"(N) : "memory"); }

template <int MF, int NP>
__device__ __forceinline__ void gemm_wave(const Params& P, char* ws, u32 call, u32 b, LAS char* lds, u32 lane, u32 wid) {
  constexpr bool XL = NP == 5;
  constexpr u32 STEP = XL ? A_STEP : B_STEP;
  constexpr int NM = MF * 8;
  const u32 r = P.rank, g4 = r & 4u;
  const u32 xcd = b & 7u, late = xcd & 1u, ent = ((xcd >> 1) & 1u) * 26u + (b >> 3);
  const u32 rc = ent >> 2, rt = late * NEARLY + rc, ct = (ent & 3u) + (xcd & 4u);
  const u32 wc = wid >> 2, f0 = wave_frag0(wid);
  const int left = (late ? LATE_UNITS : EARLY_UNITS) - (int)rc * UPT;
  const u32 nun = left < 0 ? 0u : (left > UPT ? (u32)UPT : (u32)left);
  const u32 p0 = (wid & 1u) * NP * 1024u;
  constexpr u32 same = 1;
  const rsrc_t rs = XL ? make_rsrc(ws + OFF_XU + same * rt * A_TILE, A_TILE) : make_rsrc(ws + OFF_WU + same * ct * B_TILE, B_TILE);
  const char* const cnt = ws + OFF_CNT + ((XL ? rt : NRT + ct) * 64 + lane) * 4;
  u32 target = call * (XL ? nun : 8u);
  const u32 voff = p0 + lane * 16u;
  u32 vp[NP ? NP : 1];
#pragma unroll
  for (int p = 0; p < NP; ++p) vp[p] = (XL && (wid & 1u) * NP + p >= 2u * nun) ? lane * 16u : voff;
  LAS char* const ring = lds + (XL ? 0u : A_STEP) + p0;
  const u32 img = ((lane & 15u) * 4u + ((lane >> 4) ^ quad_swz((lane >> 2) & 3u))) * 16u;
  LAS char* const ra = lds + f0 * 1024u + img;
  LAS char* const rb = lds + A_STEP + wc * 8192u + lane * 16u;
#define AG_DMA(p, ks, off, imm) __builtin_amdgcn_raw_ptr_buffer_load_lds(rs, ring + (off), 16, vp[p < NP ? p : 0], (ks), imm, AG_GEMM_AUX)
#define AG_PIECE(p, ks, off) do { \
    if (p == 0) AG_DMA(0, (ks) * STEP, off, 0); if (p == 1) AG_DMA(1, (ks) * STEP, off, 1024); \
    if (p == 2) AG_DMA(2, (ks) * STEP, off, 2048); if (p == 3) AG_DMA(3, (ks) * STEP, off, 3072); \
    if (p == 4) AG_DMA(4, (ks) * STEP + 4096, (off) + 4096, 0); if (p == 5) AG_DMA(5, (ks) * STEP + 4096, (off) + 4096, 1024); \
    if (p == 6) AG_DMA(6, (ks) * STEP + 4096, (off) + 4096, 2048); if (p == 7) AG_DMA(7, (ks) * STEP + 4096, (off) + 4096, 3072); \
  } while (0)
#define AG_ALL(ks, off) do { _Pragma("unroll") for (int p = 0; p < NP; ++p) AG_PIECE(p, ks, off); } while (0)
#define AG_PIN(stmt) do { __builtin_amdgcn_sched_barrier(0); stmt; __builtin_amdgcn_sched_barrier(0); } while (0)
#define AG_STEP(cs, ks, DMA, RD) do { \
    LAS char* const pa = ra + ord; \
    LAS char* const pb = rb + ord; \
    __builtin_amdgcn_sched_barrier(0); \
    _Pragma("unroll") for (int i = 0; i < NM; ++i) { \
      mfma_acc(acc[i % MF][i / MF], fa[cs][i % MF], fb[cs][i / MF]); \
      _Pragma("unroll") for (int j = 0; j < MF + 8; ++j) \
        if (RD && i == (MF == 3 ? 2 * j : j)) { \
          if (j < MF) AG_PIN(fa[(cs) ^ 1][j] = *(const LAS bf16x8*)(pa + j * 1024)); \
          else AG_PIN(fb[(cs) ^ 1][j - MF] = *(const LAS bf16x8*)(pb + (j - MF) * 1024)); \
        } \
      _Pragma("unroll") for (int p = 0; p < NP; ++p) \
        if (DMA && i == 1 + p * (NP == 5 ? 5 : 3)) AG_PIN(AG_PIECE(p, ks, owr)); \
    } \
    ord = ord + SLOTB == NSLOT * SLOTB ? 0 : ord + SLOTB; \
    owr = owr + SLOTB == NSLOT * SLOTB ? 0 : owr + SLOTB; \
    __builtin_amdgcn_sched_barrier(0); \
  } while (0)
#define AG_GATE(c) do { if (NP) gate(c, ready, cnt, target, ws, call); } while (0)
  f32x4 acc[MF][8];
#pragma unroll
  for (int m = 0; m < MF; ++m)
#pragma unroll
    for (int n = 0; n < 8; ++n) { acc[m][n] = (f32x4){0.f, 0.f, 0.f, 0.f}; asm volatile("" : "+v"(acc[m][n])); }
  if (NP) __builtin_amdgcn_s_setprio(1);
  int ready = 0;
  AG_GATE(0);
  STAMP(1);
  STAMPC(4);
  constexpr int LA = AG_LA, W = (LA - 2) * NP;
#pragma unroll
  for (int s = 0; s < 4; ++s) AG_ALL(s, s * SLOTB);
  if (LA == 5) { AG_GATE(1); AG_ALL(4, 4 * SLOTB); }
  wait_bar<W>();
  bf16x8 fa[2][MF], fb[2][8];
#pragma unroll
  for (int m = 0; m < MF; ++m) fa[0][m] = *(const LAS bf16x8*)(ra + m * 1024);
#pragma unroll
  for (int n = 0; n < 8; ++n) fb[0][n] = *(const LAS bf16x8*)(rb + n * 1024);
  u32 ord = SLOTB, owr = LA * SLOTB;
  for (int kc = 0; kc < NKC - 3; ++kc) {
    if (LA == 4) AG_GATE(kc + 1);
    AG_STEP(0, kc * 4 + LA, true, true); wait_bar<W>();
    AG_STEP(1, kc * 4 + LA + 1, true, true); wait_bar<W>();
    AG_STEP(0, kc * 4 + LA + 2, true, true); wait_bar<W>();
    if (LA == 5) AG_GATE(kc + 2);
    AG_STEP(1, kc * 4 + LA + 3, true, true); wait_bar<W>();
  }
  if (LA == 4) AG_GATE(NKC - 2);
  AG_STEP(0, NKS - 12 + LA, true, true); wait_bar<W>();
  AG_STEP(1, NKS - 11 + LA, true, true); wait_bar<W>();
  AG_STEP(0, NKS - 10 + LA, true, true); wait_bar<W>();
  AG_STEP(1, NKS - 9 + LA, LA == 4, true); wait_bar<2 * NP>();
  AG_STEP(0, 0, false, true); wait_bar<NP>();
  AG_STEP(1, 0, false, true); wait_bar<0>();
  AG_STEP(0, 0, false, true);
  AG_STEP(1, 0, false, false);
  AG_GATE(NKC - 1);
  asm volatile("s_barrier" ::: "memory");
#pragma unroll
  for (int s = 0; s < 4; ++s) AG_ALL(NKS - 4 + s, s * SLOTB);
  u32 tmo;
  asm volatile("global_load_dword %0, %1, %2 sc1" : "=&v"(tmo) : "v"(0u), "s"(ws + OFF_TMO) : "memory");
  wait_bar<2 * NP + 1>();
#pragma unroll
  for (int m = 0; m < MF; ++m) fa[0][m] = *(const LAS bf16x8*)(ra + m * 1024);
#pragma unroll
  for (int n = 0; n < 8; ++n) fb[0][n] = *(const LAS bf16x8*)(rb + n * 1024);
  ord = SLOTB;
  AG_STEP(0, 0, false, true); wait_bar<NP + 1>();
  AG_STEP(1, 0, false, true); wait_bar<0>();
  AG_STEP(0, 0, false, true);
  AG_STEP(1, 0, false, false);
  asm volatile("" : "+v"(tmo));
  asm volatile("s_nop 15" ::: "memory");
  __builtin_amdgcn_sched_barrier(0);
#undef AG_PIECE
#undef AG_ALL
#undef AG_PIN
#undef AG_DMA
#undef AG_GATE
#undef AG_STEP
  STAMP(2);
  STAMPC(5);
  const bool bad = __builtin_amdgcn_readfirstlane(tmo) == call;
  u32x4 o[MF][4];
#pragma unroll
  for (int m = 0; m < MF; ++m)
#pragma unroll
    for (int e = 0; e < 4; ++e)
#pragma unroll
      for (int n = 0; n < 4; ++n) o[m][e][n] = relu_pk(acc[m][2 * n][e], acc[m][2 * n + 1][e]);
  if (bad) {
#pragma unroll
    for (int m = 0; m < MF; ++m)
#pragma unroll
      for (int e = 0; e < 4; ++e) o[m][e] = (u32x4){0x7fc07fc0u, 0x7fc07fc0u, 0x7fc07fc0u, 0x7fc07fc0u};
    asm volatile("" ::: "memory");
  }
  if (ct < 4u) {
    const rsrc_t ro = make_rsrc(pick(P.out, r), 8192u * 1024u * 2u);
    const u32 vo = (lane >> 4) * 4u * 2048u + (lane & 15u) * 16u;
#pragma unroll
    for (int m = 0; m < MF; ++m) {
      const u32 row0 = frag_row(r, rt, f0 + m);
      if (row0 == NOROW) continue;
#pragma unroll
      for (int e = 0; e < 4; ++e) store16(o[m][e], ro, vo, (row0 + e) * 2048u + ct * 512u + wc * 256u);
    }
    STAMP(3);
    return;
  }
  const u32 j = ct - 4u, uid = rt * 8u + wid;
  u32 qi;
  u64 keep;
  asm volatile("s_mov_b64 %1, exec\n\ts_mov_b64 exec, 1\n\tglobal_atomic_add %0, %2, %3, %4 sc0\n\ts_mov_b64 exec, %1"
               : "=&v"(qi), "=&s"(keep) : "v"(0u), "v"(1u), "s"(ws + OFF_TKT + (11u + j) * 128u) : "memory");
  u32 mk[MF], len = 0;
#pragma unroll
  for (int m = 0; m < MF; ++m) {
    mk[m] = 0;
    if ((f0 + m) >> 1 >= nun) {
#pragma unroll
      for (int e = 0; e < 4; ++e) o[m][e] = (u32x4){0u, 0u, 0u, 0u};
    }
#pragma unroll
    for (int i = 0; i < 16; ++i) {
      u32 t;
      const u32 d = o[m][i >> 2][i & 3];
      asm("v_pk_min_u16 %0, %1, %2" : "=v"(t) : "v"(d), "v"(0x00010001u));
      mk[m] |= t << i;
    }
    len += __builtin_popcount(mk[m]);
  }
  len = 2u * (len & ~1u) + 4u;
  const u32 incl = wave_scan(len);
  u32 total = __builtin_amdgcn_readlane(incl, 63);
  const bool raw = total + 256u * MF >= 4096u * MF;
  const rsrc_t rb2 = make_rsrc(ws + OFF_OB + j * (OUNITS * OSLOT) + uid * OSLOT, OSLOT);
  STAMP2(0);
  asm volatile("s_barrier" ::: "memory");
  STAMP2(1);
  if (raw) {
    total = 4096u * MF;
#pragma unroll
    for (int m = 0; m < MF; ++m)
#pragma unroll
      for (int e = 0; e < 4; ++e) store16(o[m][e], rb2, lane * 16u, 1024u + (m * 4 + e) * 1024u);
  } else {
    LAS char* const scr = lds + wid * WAVE_SCRATCH;
    u32 a = (u32)(size_t)scr + incl - len, cur = 0;
#pragma unroll
    for (int m = 0; m < MF; ++m) {
      __builtin_amdgcn_raw_buffer_store_b32(mk[m], rb2, lane * 4u, m * 256u, AUX_SYS);
#pragma unroll
      for (int i = 0; i < 16; ++i) {
        const u32 d0 = o[m][i >> 2][i & 3];
        u32 t;
        asm("v_pk_min_u16 %0, %1, %2" : "=v"(t) : "v"(d0), "v"(0x00010001u));
        const u32 an = __builtin_amdgcn_udot2(__builtin_bit_cast(u16x2, t), (u16x2){2, 2}, a, false);
        const u32 d = (d0 & 0xffffu) ? d0 : d0 >> 16;
        const u32 w = (d << ((a << 3) & 31u)) | cur;
        *(LAS u32*)(size_t)(a & ~3u) = w;
        cur = ((an ^ a) & 4u) ? ((an & 2u) ? d >> 16 : 0u) : w;
        a = an;
      }
    }
    *(LAS u32*)(size_t)(a & ~3u) = cur;
    asm volatile("" ::: "memory");
    STAMP2(2);
    for (u32 p = 0; p < total; p += 1024u) store16(*(const LAS u32x4*)(scr + p + lane * 16u), rb2, p + lane * 16u < total ? lane * 16u : 0x80000000u, 1024u + p);
  }
  STAMP2(3);
  asm volatile("s_waitcnt vmcnt(0)" : "+v"(qi) :: "memory");
  const u32 slot = ((u32)__builtin_amdgcn_readfirstlane(qi) - (call - 1u) * OUNITS) & 255u;
  flag_store(pick(P.ws, (g4 ^ 4u) | j) + OFF_OQ + (r & 3u) * 2048u + slot * 8u, ((u64)call << 32) | uid | (total << 8) | (raw ? 1u << 22 : 0u), 1u);
  STAMP(3);
}

__global__ __launch_bounds__(512, 2) void ag_gemm_kernel(Params P) {
  extern __shared__ __attribute__((aligned(16))) char smem[];
  LAS char* lds = (LAS char*)smem;
  const u32 lane = threadIdx.x & 63u, wid = __builtin_amdgcn_readfirstlane(threadIdx.x >> 6);
  char* const ws = pick(P.ws, P.rank);
  STAMP(0);
  u32 bid = blockIdx.x;
  asm volatile("" : "+s"(bid));
  char* const cptr = ws + OFF_CALL + bid * 128u + wid * 4u;
  const u32 callv = ld_agent(cptr);
  LAS unsigned char* scr = (LAS unsigned char*)lds + wid * WAVE_SCRATCH;
  bool skip = false;
  u32 li = 7, lidx = 0, pkw = 0;
  if (bid < NCOMM) {
    const u32 h = (bid >> 2) & 1u, a = (bid >> 3) * 4u + (bid & 3u), nl = 3u + h;
    if (a < nl * NCL) { li = h * 3u + (a + h * (a >> 2)) % nl; lidx = a / nl; }
    else pkw = h * (NCOMM / 2 - 3 * NCL) + a - nl * NCL;
  }
  LAS u32* const ctl = (LAS u32*)(lds + 8 * WAVE_SCRATCH);
  if (bid < NCOMM) {
    ctl[lane & 3u] = 0;
    asm volatile("s_waitcnt lgkmcnt(0)\n\ts_barrier" ::: "memory");
  }
  if (li < 7u) {
    const u32 call = __builtin_amdgcn_readfirstlane(callv) + 1u;
    __hip_atomic_store((GAS u32*)cptr, call, __ATOMIC_RELAXED, __HIP_MEMORY_SCOPE_AGENT);
    if (!skip) puller(P, ws, call, li, lidx, ctl, scr, lane, wid);
    out_puller(P, ws, call, li >= 3u ? li - 3u : (li * NCL + lidx + NPKWG) & 3u, li >= 3u ? AG_OTOK : AG_OTOK_IN, ctl, scr, lane, wid);
    return;
  }
  const bool isg = bid >= NCOMM;
  const u32 q0 = isg ? wid * NGEMM + (bid - NCOMM) : GPACK + wid * NPKWG + pkw;
  const u32 q1 = isg ? (wid < AG_G2 ? q0 + 8u * NGEMM : (u32)NUNITS) : q0 + NPK;
  for (u32 i = isg ? wid : (u32)AG_PDELAY; i; --i) __builtin_amdgcn_s_sleep(AG_GDELAY);
  const u32 call = packer(P, ws, callv, skip ? (u32)NUNITS : q0, q1, isg ? (u32)NUNITS : (u32)NPK, scr, lane, wid);
  __hip_atomic_store((GAS u32*)cptr, call, __ATOMIC_RELAXED, __HIP_MEMORY_SCOPE_AGENT);
  if (!isg) {
    out_puller(P, ws, call, pkw & 3u, AG_OTOK_PK, ctl, scr, lane, wid);
    return;
  }
  asm volatile("s_waitcnt lgkmcnt(0)\n\ts_barrier" ::: "memory");
  STAMP(7);
  if (wid < 2u) gemm_wave<3, 5>(P, ws, call, bid - NCOMM, lds, lane, wid);
  else if (wid < 6u) gemm_wave<2, 0>(P, ws, call, bid - NCOMM, lds, lane, wid);
  else gemm_wave<3, 8>(P, ws, call, bid - NCOMM, lds, lane, wid);
}

}

extern "C" void kernel_launch(int rank, int nranks, ncclComm_t comm, hipStream_t stream,
                              void* const* d_in, const int* in_sizes, int n_in,
                              void* d_out, int out_size, const int* in_dtypes, int out_dtype,
                              void* const* d_out_all, void* const* d_ws_all, size_t ws_bytes) {
  (void)comm; (void)in_dtypes; (void)out_dtype; (void)d_out;
  if (nranks != ag::NR || n_in != 2 || (unsigned)rank >= (unsigned)ag::NR || in_sizes[0] != 1024 * 8192 || in_sizes[1] != 8192 * 1024 ||
      out_size != 8192 * 1024 || ws_bytes < ag::WS_NEED) {
    fprintf(stderr, "ag_gemm: unexpected problem shape\n");
    return;
  }
  ag::Params p;
  p.x = (const char*)d_in[0];
  p.w = (const char*)d_in[1];
  for (int i = 0; i < ag::NR; ++i) {
    const int phys = (AG_ORDER >> (4 * i)) & 7;
    p.ws[i] = (char*)d_ws_all[phys];
    p.out[i] = (char*)d_out_all[phys];
    if (phys == rank) p.rank = i;
  }
  p.pad = 0;
  hipError_t e = hipFuncSetAttribute((const void*)ag::ag_gemm_kernel, hipFuncAttributeMaxDynamicSharedMemorySize, ag::LDS_BYTES);
  if (e != hipSuccess) { fprintf(stderr, "ag_gemm: %s\n", hipGetErrorString(e)); return; }
  hipLaunchKernelGGL(ag::ag_gemm_kernel, dim3(ag::NWG), dim3(512), ag::LDS_BYTES, stream, p);
  e = hipGetLastError();
  if (e != hipSuccess) fprintf(stderr, "ag_gemm launch: %s\n", hipGetErrorString(e));
}
```

```cpp
#include <hip/hip_runtime.h>
#include <rccl/rccl.h>
#include <cstdio>

namespace ag {

typedef unsigned u32;
typedef unsigned long long u64;
typedef unsigned u32x4 __attribute__((ext_vector_type(4)));
typedef unsigned short u16x2 __attribute__((ext_vector_type(2)));
typedef short bf16x8 __attribute__((ext_vector_type(8)));
typedef float f32x4 __attribute__((ext_vector_type(4)));
typedef __amdgpu_buffer_rsrc_t rsrc_t;
#define LAS __attribute__((address_space(3)))
#define GAS __attribute__((address_space(1)))

#define AG_AU 21
#define AG_NCL 6
#define AG_TOK 12
#define AG_TOK_IN 12
#define AG_G2 8
#define AG_OTOK 2
#define AG_WLEAD 4
#define AG_LA 4
#define AG_OTOK_IN 2
#define AG_OTOK_PK 2
#define AG_KP 1
#define AG_MIX 0
#define AG_KT 0
#define AG_XAUX 2
#define AG_WAUX 2
#define AG_OAUX 17
#define AG_EARLY 4
#define AG_GDELAY 36
#define AG_PDELAY 0
#define AG_ORDER 0x75416320u
#define AG_NEAR 1
#define AG_CTMAP 1
#define AG_FORCE_RAW 0
#define AG_GEMM_AUX 17

constexpr int NR = 8;
constexpr int AU = AG_AU;
constexpr int AWAY = 32 - AU;
constexpr int NRT = 26, NEARLY = 13;
constexpr int UPT = 5;
constexpr int NKC = 64, NKS = 256;
constexpr int EARLY_UNITS = AU + 4 * AWAY, LATE_UNITS = 3 * AU;
static_assert(EARLY_UNITS <= NEARLY * UPT && LATE_UNITS <= (NRT - NEARLY) * UPT, "row split does not fit the tiles");
static_assert((EARLY_UNITS + UPT - 1) % UPT >= 2 && (LATE_UNITS + UPT - 1) % UPT >= 2, "a short tile keeps at least five fragments");
constexpr int NOUT = AWAY + 8;

constexpr int NCOMM = 48, NGEMM = 208, NWG = NCOMM + NGEMM;
constexpr int NCL = AG_NCL;
constexpr int NPKWG = NCOMM - 7 * NCL;
static_assert(4 * NCL <= NCOMM / 2 && (NCOMM - 4 * NCL) % 4 == 0, "puller placement");
constexpr int PW = 8 - AG_MIX;
constexpr int NPK = NPKWG * 8 + 7 * NCL * AG_MIX;
constexpr u32 GPACK = (8 + AG_G2) * NGEMM;
constexpr int NUNITS = 4096;
constexpr u32 WLEAD = AG_WLEAD;

constexpr u32 A_STEP = 10240, B_STEP = 16384;
constexpr u32 A_TILE = A_STEP * NKS, B_TILE = B_STEP * NKS;
constexpr u32 OFF_XU = 0;
constexpr u32 OFF_WU = OFF_XU + NRT * A_TILE;
constexpr u32 FRAG_BYTES = OFF_WU + 8 * B_TILE;
constexpr u32 SLOT = 8192, PK_BYTES = NUNITS * SLOT;
constexpr u32 OFF_PK = FRAG_BYTES;
constexpr u32 OSLOT = 14336, OUNITS = NRT * 8;
constexpr u32 OFF_OB = OFF_PK + 2 * PK_BYTES;
constexpr u32 KP = AG_KP, IB_SRC = KP * 64 * SLOT, IB_BYTES = 2 * NR * IB_SRC;
constexpr u32 OFF_IB = OFF_OB + 4 * OUNITS * OSLOT;
constexpr u32 OFF_G = OFF_IB + IB_BYTES;
constexpr u32 G_BYTES = 2 * NR * NUNITS * 8;
constexpr u32 OFF_CNT = OFF_G + G_BYTES;
constexpr u32 OFF_OQ = OFF_CNT + 16384;
constexpr u32 OFF_TKT = OFF_OQ + 4 * 2048;
constexpr u32 OFF_TMO = OFF_TKT + 2048;
constexpr u32 OFF_CALL = OFF_TMO + 128;
constexpr u32 OFF_DBG = OFF_CALL + NWG * 128;
constexpr u32 OFF_DBG2 = OFF_DBG + NWG * 8 * 64;
constexpr u32 OFF_DBG3 = OFF_DBG2 + 65536;
constexpr u32 OFF_DBG4 = OFF_DBG3 + 131072;
constexpr u32 WS_NEED = OFF_DBG4 + 524288;

constexpr u32 SLOTB = A_STEP + B_STEP, NSLOT = AG_LA + 1;
constexpr int LDS_BYTES = NSLOT * SLOTB;
constexpr int WAVE_SCRATCH = 13312;
static_assert(8 * WAVE_SCRATCH + 64 <= LDS_BYTES, "scratch");

constexpr u32 SPARE_CNT = 4000;
constexpr u64 DEADLINE = 50000000ull;
constexpr int AUX_SYS = 17;

struct Params {
  const char* x;
  const char* w;
  char* ws[NR];
  char* out[NR];
  int rank;
  int pad;
};

#define STAMP(i) do {} while (0)
#define STAMPC(i) do {} while (0)
#define STAMPP(i) do {} while (0)
#define STAMP2(i) do {} while (0)
#define STAMPQ(i) do {} while (0)

__device__ __forceinline__ rsrc_t make_rsrc(const void* p, u32 bytes) {
  const u64 a = (u64)p;
  const u32 lo = __builtin_amdgcn_readfirstlane((u32)a), hi = __builtin_amdgcn_readfirstlane((u32)(a >> 32));
  return __builtin_amdgcn_make_buffer_rsrc((void*)(((u64)hi << 32) | lo), 0, bytes, 0x00020000);
}
__device__ __forceinline__ char* pick(char* const (&t)[NR], u32 i) {
  char* p = t[0];
#pragma unroll
  for (u32 j = 1; j < NR; ++j) p = (i == j) ? t[j] : p;
  return p;
}
__device__ __forceinline__ u32 ld_agent(const char* p) { return __hip_atomic_load((GAS u32*)p, __ATOMIC_RELAXED, __HIP_MEMORY_SCOPE_AGENT); }
__device__ __forceinline__ u32 ld_sys(const char* p) { return __hip_atomic_load((GAS u32*)p, __ATOMIC_RELAXED, __HIP_MEMORY_SCOPE_SYSTEM); }
template <int AUX = AUX_SYS>
__device__ __forceinline__ void store16(u32x4 v, rsrc_t rs, u32 voff, u32 soff) {
  __builtin_amdgcn_raw_buffer_store_b128(v, rs, voff, soff, AUX);
  asm volatile("s_nop 1" :: "v"(v) : "memory");
}
__device__ __forceinline__ void count_in(const char* base, u32 idx) {
  u64 keep;
  asm volatile("s_mov_b64 %0, exec\n\ts_mov_b64 exec, 1\n\tglobal_atomic_add %1, %2, %3\n\ts_mov_b64 exec, %0"
               : "=&s"(keep) : "v"(idx * 4u), "v"(1u), "s"(base) : "memory");
}
#define STAMP_CNT(ws, idx) do {} while (0)
__device__ __forceinline__ void flag_store(char* p, u64 v, u64 mask) {
  u64 keep;
  asm volatile("s_mov_b64 %0, exec\n\ts_mov_b64 exec, %3\n\tglobal_store_dwordx2 %1, %2, off sc0 sc1\n\ts_mov_b64 exec, %0"
               : "=&s"(keep) : "v"(p), "v"(v), "s"(mask) : "memory");
}
__device__ __forceinline__ void drain_vm() { asm volatile("s_waitcnt vmcnt(0)" ::: "memory"); }

__device__ __forceinline__ bool give_up(char* ws, u32 call, u64 t0) {
  const u32 t = __builtin_amdgcn_readfirstlane(ld_agent(ws + OFF_TMO));
  if (t == call) return true;
  if (__builtin_amdgcn_s_memrealtime() - t0 > DEADLINE) {
    __hip_atomic_store((GAS u32*)(ws + OFF_TMO), call, __ATOMIC_RELAXED, __HIP_MEMORY_SCOPE_AGENT);
    return true;
  }
  return false;
}

__device__ __forceinline__ u32 wave_sum(u32 v) {
  v += __builtin_amdgcn_update_dpp(0u, v, 0x111, 0xf, 0xf, true);
  v += __builtin_amdgcn_update_dpp(0u, v, 0x112, 0xf, 0xf, true);
  v += __builtin_amdgcn_update_dpp(0u, v, 0x114, 0xf, 0xf, true);
  v += __builtin_amdgcn_update_dpp(0u, v, 0x118, 0xf, 0xf, true);
  return __builtin_amdgcn_readlane(v, 15) + __builtin_amdgcn_readlane(v, 31) + __builtin_amdgcn_readlane(v, 47) + __builtin_amdgcn_readlane(v, 63);
}
__device__ __forceinline__ u32 wave_scan(u32 v) {
  v += __builtin_amdgcn_update_dpp(0u, v, 0x111, 0xf, 0xf, true);
  v += __builtin_amdgcn_update_dpp(0u, v, 0x112, 0xf, 0xf, true);
  v += __builtin_amdgcn_update_dpp(0u, v, 0x114, 0xf, 0xf, true);
  v += __builtin_amdgcn_update_dpp(0u, v, 0x118, 0xf, 0xf, true);
  v += __builtin_amdgcn_update_dpp(0u, v, 0x142, 0xa, 0xf, false);
  v += __builtin_amdgcn_update_dpp(0u, v, 0x143, 0xc, 0xf, false);
  return v;
}

__device__ __forceinline__ u32 quad_swz(u32 rq) { return (0x78u >> (2 * rq)) & 3u; }

struct Dest { u32 base, s1, s0, cnt; };
__device__ __forceinline__ Dest dest_x(u32 ux, u32 kc) {
  const u32 tile = ux / UPT;
  return Dest{OFF_XU + tile * A_TILE + 4 * kc * A_STEP + (ux - tile * UPT) * 2048u, 1024u, A_STEP, tile * 64 + kc};
}
__device__ __forceinline__ Dest dest_w(u32 ct, u32 kc, u32 ks4, u32 half) {
  return Dest{OFF_WU + ct * B_TILE + (4 * kc + ks4) * B_STEP + half * 8192u, 4096u, 1024u, (NRT + ct) * 64 + kc};
}
__device__ __forceinline__ void store_frags(const u32x4 (&v)[8], rsrc_t rf, const Dest& d, u32 lane) {
#pragma unroll
  for (int f = 0; f < 8; ++f) store16(v[f], rf, lane * 16, d.base + (f >> 2) * d.s1 + (f & 3) * d.s0);
}

constexpr u32 RAW = 1u << 31, PUSHED = 1u << 30;
constexpr u32 S_LIST = 0, S_TAIL = 2304, S_TAB = 3584;
struct Packed { u32x4 mq[4], p01, tail; };
__device__ __forceinline__ u32 groups_of(u32 info) { return (((info >> 8) & 4095u) + 31u) >> 5; }

__device__ __forceinline__ void build_tables(LAS unsigned char* scr, u32 lane) {
  u32 ct = 0x0c0c0c0cu, xt = 0x07060504u, k = 0;
#pragma unroll
  for (u32 b = 0; b < 4; ++b) {
    const u32 on = 0u - ((lane >> b) & 1u);
    ct ^= on & ((0x0cu ^ b) << (8u * k));
    xt ^= on & (((4u + b) ^ k) << (8u * b));
    k -= on;
  }
  *(LAS u32*)(scr + S_TAB + (lane & 31u) * 4u) = (lane & 16u) ? xt : ct;
}

template <bool TO_LIST>
__device__ __forceinline__ void walk_spelled(u32 m, u32 off, LAS unsigned char* cells, LAS unsigned char* list, u32 flip) {
  while (m != 0) {
    const u32 p = __builtin_ctz(m);
    m &= m - 1;
    LAS unsigned char* cell = cells + (p & 7u) * 4u + (p >> 3);
    if (TO_LIST) list[off] = *cell ^ flip; else *cell = list[off];
    ++off;
  }
}

__device__ __forceinline__ u32 encode_unit(Packed& k, const u32x4 (&v)[8], LAS unsigned char* scr, u32 lane) {
  const u32x4 zero = {0u, 0u, 0u, 0u};
  *(LAS u32x4*)(scr + S_LIST + lane * 32u) = zero;
  *(LAS u32x4*)(scr + S_LIST + lane * 32u + 16u) = zero;
  u32 E[16], esum = 0;
#pragma unroll
  for (int j = 0; j < 16; ++j) {
    const u32 d0 = v[j >> 1][(j & 1) * 2], d1 = v[j >> 1][(j & 1) * 2 + 1];
    const u32 r0 = __builtin_amdgcn_alignbit(d0, d0, 31), r1 = __builtin_amdgcn_alignbit(d1, d1, 31);
    k.mq[j >> 2][j & 3] = __builtin_amdgcn_perm(r1, r0, 0x06040200u);
    E[j] = __builtin_amdgcn_perm(r1, r0, 0x07050301u);
    esum = __builtin_amdgcn_sad_u8(E[j], 0u, esum);
  }
  u32 base = (wave_sum(esum) + 8028u) >> 12;
  base = base < 8u ? 8u : (base > 254u ? 254u : base);
  const u16x2 b2 = {(unsigned short)base, (unsigned short)base}, t2 = {(unsigned short)(base + 1u), (unsigned short)(base + 1u)};
  u32 P[2][2] = {{0, 0}, {0, 0}};
#pragma unroll
  for (int j = 0; j < 16; ++j) {
    const u16x2 ev = __builtin_bit_cast(u16x2, E[j] & 0x00ff00ffu), od = __builtin_bit_cast(u16x2, E[j]) >> (u16x2){8, 8};
    const u32 ce = __builtin_bit_cast(u32, __builtin_elementwise_min((u16x2)(b2 - ev), (u16x2){3, 3}));
    const u32 co = __builtin_bit_cast(u32, __builtin_elementwise_min((u16x2)(b2 - od), (u16x2){3, 3}));
    const u32 c = ce | (co << 8);
    P[0][j >> 3] |= (c & 0x01010101u) << (j & 7);
    P[1][j >> 3] |= ((c >> 1) & 0x01010101u) << (j & 7);
  }
  k.p01 = (u32x4){P[0][0], P[0][1], P[1][0], P[1][1]};
  k.tail = zero;
  const u32 mk[2] = {P[0][0] & P[1][0], P[0][1] & P[1][1]};
  u32 fl[16], sel[16];
#pragma unroll
  for (int j = 0; j < 16; ++j) {
    fl[j] = (mk[j >> 3] >> (j & 7)) & 0x01010101u;
    sel[j] = *(const LAS u32*)(size_t)__builtin_amdgcn_udot4(fl[j], 0x20100804u, (u32)(size_t)(scr + S_TAB), false);
  }
  __builtin_amdgcn_sched_barrier(0);
  const u32 c = __builtin_popcount(mk[0]) + __builtin_popcount(mk[1]), incl = wave_scan(c);
  const u32 n2 = __builtin_amdgcn_readlane(incl, 63);
  if (n2 > 2044u || AG_FORCE_RAW) return RAW;
  const u32 flip = (base + 1u) * 0x01010101u;
  u32 a = (u32)(size_t)(scr + S_LIST) + incl - c, cur = 0;
#pragma unroll
  for (int j = 0; j < 16; ++j) {
    const u64 t = (u64)__builtin_amdgcn_perm(0u, E[j] ^ flip, sel[j]) << ((a & 3u) * 8u);
    const u32 w = cur | (u32)t, an = __builtin_amdgcn_sad_u8(fl[j], 0u, a);
    __hip_atomic_fetch_or((LAS u32*)(size_t)(a & ~3u), w, __ATOMIC_RELAXED, __HIP_MEMORY_SCOPE_WAVEFRONT);
    cur = ((an ^ a) & 4u) ? (u32)(t >> 32) : w;
    a = an;
  }
  __hip_atomic_fetch_or((LAS u32*)(size_t)(a & ~3u), cur, __ATOMIC_RELAXED, __HIP_MEMORY_SCOPE_WAVEFRONT);
  asm volatile("" ::: "memory");
  const u32x4 l[2] = {*(const LAS u32x4*)(scr + S_LIST + lane * 32u), *(const LAS u32x4*)(scr + S_LIST + lane * 32u + 16u)};
  u32 Q[3] = {0, 0, 0};
#pragma unroll
  for (int s = 0; s < 8; ++s) {
    const u32 x = l[s >> 2][s & 3] ^ flip;
    const u16x2 ev = __builtin_bit_cast(u16x2, x & 0x00ff00ffu), od = __builtin_bit_cast(u16x2, x) >> (u16x2){8, 8};
    const u32 ce = __builtin_bit_cast(u32, __builtin_elementwise_min(__builtin_elementwise_sub_sat((u16x2)(t2 - ev), (u16x2){3, 3}), (u16x2){7, 7}));
    const u32 co = __builtin_bit_cast(u32, __builtin_elementwise_min(__builtin_elementwise_sub_sat((u16x2)(t2 - od), (u16x2){3, 3}), (u16x2){7, 7}));
    const u32 c2 = ce | (co << 8);
    Q[0] |= (c2 & 0x01010101u) << s;
    Q[1] |= ((c2 >> 1) & 0x01010101u) << s;
    Q[2] |= ((c2 >> 2) & 0x01010101u) << s;
  }
#pragma unroll
  for (int i = 0; i < 3; ++i) *(LAS u32*)(scr + S_TAIL + lane * 12u + i * 4u) = Q[i];
  const u32 m3 = Q[0] & Q[1] & Q[2], c3 = __builtin_popcount(m3), incl3 = wave_scan(c3);
  const u32 n3 = __builtin_amdgcn_readlane(incl3, 63);
  if (n3 > 256u) return RAW;
  asm volatile("" ::: "memory");
  walk_spelled<true>(m3, incl3 - c3, scr + S_LIST + lane * 32u, scr + S_TAIL + ((n2 + 31u) >> 5) * 12u, base + 1u);
  asm volatile("" ::: "memory");
  k.tail = *(const LAS u32x4*)(scr + S_TAIL + lane * 16u);
  return base | (n2 << 8) | (n3 << 20);
}

__device__ __forceinline__ void store_unit(const Packed& k, const u32x4 (&v)[8], u32 info, rsrc_t rp, u32 soff, u32 lane) {
  if (info & RAW) {
#pragma unroll
    for (int f = 0; f < 8; ++f) store16(v[f], rp, lane * 16, soff + f * 1024);
    return;
  }
#pragma unroll
  for (int i = 0; i < 4; ++i) store16(k.mq[i], rp, lane * 16, soff + i * 1024);
  store16(k.p01, rp, lane * 16, soff + 4096);
  store16(k.tail, rp, lane * 16u < groups_of(info) * 12u + (info >> 20) ? lane * 16u : 0x80000000u, soff + 5120);
}

__device__ __forceinline__ void fetch_packed(Packed& k, rsrc_t rp, u32 soff, u32 info, u32 lane) {
  k.tail = __builtin_amdgcn_raw_buffer_load_b128(rp, lane * 16u < groups_of(info) * 12u + (info >> 20) ? lane * 16u : 0x80000000u, soff + 5120, AUX_SYS);
  k.p01 = __builtin_amdgcn_raw_buffer_load_b128(rp, lane * 16, soff + 4096, AUX_SYS);
#pragma unroll
  for (int i = 0; i < 4; ++i) k.mq[i] = __builtin_amdgcn_raw_buffer_load_b128(rp, lane * 16, soff + i * 1024, AUX_SYS);
}

__device__ __forceinline__ void decode_unit(u32x4 (&v)[8], const Packed& k, u32 info, LAS unsigned char* scr, u32 lane) {
  const u32 base = info & 255u, g2 = groups_of(info);
  *(LAS u32x4*)(scr + S_TAIL + lane * 16u) = k.tail;
  asm volatile("" ::: "memory");
  const LAS u32* const pq = (const LAS u32*)(scr + S_TAIL + lane * 12u);
  const u32 live = lane < g2 ? ~0u : 0u, Q0 = pq[0] & live, Q1 = pq[1] & live, Q2 = pq[2] & live;
  LAS unsigned char* const spell = scr + S_TAIL + g2 * 12u;
  const u32 b4 = base * 0x01010101u, lut0 = b4 - 0x05040300u + 1u, lut1 = b4 - 0x00080706u;
  const u32 P0[2] = {k.p01.x, k.p01.y}, P1[2] = {k.p01.z, k.p01.w}, mk[2] = {P0[0] & P1[0], P0[1] & P1[1]};
  const u32 c = __builtin_popcount(mk[0]) + __builtin_popcount(mk[1]);
  u32 a[17], sel[16];
  a[0] = (u32)(size_t)(scr + S_LIST) + wave_scan(c) - c;
#pragma unroll
  for (int j = 0; j < 16; ++j) {
    const u32 f = (mk[j >> 3] >> (j & 7)) & 0x01010101u;
    sel[j] = *(const LAS u32*)(size_t)__builtin_amdgcn_udot4(f, 0x20100804u, (u32)(size_t)(scr + S_TAB + 64u), false);
    a[j + 1] = __builtin_amdgcn_sad_u8(f, 0u, a[j]);
  }
  __builtin_amdgcn_sched_barrier(0);
  u32x4 l[2];
#pragma unroll
  for (int s = 0; s < 8; ++s) {
    const u32 c2 = ((Q0 >> s) & 0x01010101u) | (((Q1 >> s) & 0x01010101u) << 1) | (((Q2 >> s) & 0x01010101u) << 2);
    l[s >> 2][s & 3] = __builtin_amdgcn_perm(lut1, lut0, c2);
  }
  const u32 m3 = Q0 & Q1 & Q2, c3 = __builtin_popcount(m3), off3 = wave_scan(c3) - c3;
  const bool many = __builtin_amdgcn_ballot_w64(c3 > 8u) != 0;
  if (!many) {
    const LAS u32* const ps = (const LAS u32*)(spell + (off3 & ~3u));
    const u32 d0 = ps[0], d1 = ps[1], d2 = ps[2];
    const u32 l0 = __builtin_amdgcn_alignbyte(d1, d0, off3), l1 = __builtin_amdgcn_alignbyte(d2, d1, off3);
    u32 t = m3 - ((m3 >> 1) & 0x55555555u);
    t = (t & 0x33333333u) + ((t >> 2) & 0x33333333u);
    t = (t + (t >> 4)) & 0x0f0f0f0fu;
    u32 rank = t << 8;
    rank += rank << 8;
    rank += (t << 8) << 16;
#pragma unroll
    for (int s = 0; s < 8; ++s) {
      const u32 e = (m3 >> s) & 0x01010101u;
      u32 ff = e << 8;
      asm("" : "+v"(ff));
      ff -= e;
      l[s >> 2][s & 3] = (__builtin_amdgcn_perm(l1, l0, rank) & ff) | (l[s >> 2][s & 3] & ~ff);
      rank += e;
    }
  }
  *(LAS u32x4*)(scr + S_LIST + lane * 32u) = l[0];
  *(LAS u32x4*)(scr + S_LIST + lane * 32u + 16u) = l[1];
  if (many) walk_spelled<false>(m3, off3, scr + S_LIST + lane * 32u, spell, 0u);
  asm volatile("" ::: "memory");
  u32 w0[16], w1[16];
#pragma unroll
  for (int j = 0; j < 16; ++j) {
    const LAS u32* const pw = (const LAS u32*)(size_t)(a[j] & ~3u);
    w0[j] = pw[0];
    w1[j] = pw[1];
  }
  __builtin_amdgcn_sched_barrier(0);
#pragma unroll
  for (int j = 0; j < 16; ++j) {
    const int s = j & 7, wd = j >> 3;
    const u32 c1 = ((P0[wd] >> s) & 0x01010101u) | (((P1[wd] >> s) & 0x01010101u) << 1);
    const u32 e = __builtin_amdgcn_perm(b4 - c1, __builtin_amdgcn_alignbyte(w1[j], w0[j], a[j]), sel[j]), m = k.mq[j >> 2][j & 3];
    const u32 r0 = __builtin_amdgcn_perm(e, m, 0x05010400u), r1 = __builtin_amdgcn_perm(e, m, 0x07030602u);
    v[j >> 1][(j & 1) * 2] = __builtin_amdgcn_alignbit(r0, r0, 1);
    v[j >> 1][(j & 1) * 2 + 1] = __builtin_amdgcn_alignbit(r1, r1, 1);
  }
}

__device__ __forceinline__ void load_unit(u32x4 (&s)[8], rsrc_t rx, rsrc_t rw, u32 q, u32 vx, u32 vw) {
  const u32 kc = q >> 6, t = q & 63u;
  if (t < 32u) {
#pragma unroll
    for (int f = 0; f < 8; ++f) s[f] = __builtin_amdgcn_raw_buffer_load_b128(rx, vx, (32u * t + 16u * (f >> 2)) * 16384u + kc * 256u + (f & 3) * 64u, AG_XAUX);
  } else {
#pragma unroll
    for (int i = 0; i < 8; ++i) s[i] = __builtin_amdgcn_raw_buffer_load_b128(rw, vw, (128u * kc + 32u * ((t - 32u) >> 3) + i) * 2048u + (t & 7u) * 256u, AG_WAUX);
  }
}

__device__ __forceinline__ u32 packer(const Params& P, char* ws, u32 callv, u32 q0, u32 q1, u32 dq, LAS unsigned char* scr, u32 lane, u32 wid) {
  const rsrc_t rx = make_rsrc(P.x, 1024u * 8192u * 2u), rw = make_rsrc(P.w, 8192u * 1024u * 2u);
  const u32 kq = (lane & 3u) ^ quad_swz(lane >> 4);
  const u32 vx = (lane >> 2) * 16384u + kq * 16u, vw = (lane >> 4) * 8u * 2048u + (lane & 15u) * 16u;
  u32x4 s[8];
  STAMPP(0);
  load_unit(s, rx, rw, q0, vx, vw);
  __builtin_amdgcn_sched_barrier(0);
  build_tables(scr, lane);
  const u32 call = __builtin_amdgcn_readfirstlane(callv) + 1u;
  const u32 r = P.rank, g4 = r & 4u, par = call & 1u;
  const rsrc_t rf = make_rsrc(ws, FRAG_BYTES), rp = make_rsrc(ws + OFF_PK + par * PK_BYTES, PK_BYTES);
  const u32 goff = OFF_G + (par * NR + r) * NUNITS * 8;
  char* g_in = pick(P.ws, g4 | ((r + 1u) & 3u));
  char* g_out = pick(P.ws, g4 ^ 4u);
#pragma unroll
  for (u32 j = 1; j < 4; ++j) {
    char* const pi = pick(P.ws, g4 | ((r + 1u + j) & 3u));
    char* const po = pick(P.ws, (g4 ^ 4u) | j);
    g_in = lane == j ? pi : g_in;
    g_out = lane == j ? po : g_out;
  }
  STAMPP(1);
  u64 psend = 0;
  char* pgp = g_out;
  u32 pinfo = 0, pcnt = SPARE_CNT;
  auto announce = [&]() {
    flag_store(pgp, ((u64)call << 32) | pinfo, psend);
    if (pcnt != SPARE_CNT) { STAMP_CNT(ws, pcnt); count_in(ws + OFF_CNT, pcnt); }
  };
  for (u32 q = q0; q < NUNITS; q = q1, q1 += dq) {
    const u32 kc = q >> 6, t = q & 63u;
    u32x4 v[8];
    Dest d;
    bool own;
    u64 send;
    char* gp;
    if (t < 32u) {
#pragma unroll
      for (int f = 0; f < 8; ++f) v[f] = s[f];
      own = t < AU;
      d = dest_x(t, kc);
      send = own ? 7u : 15u;
      gp = own ? g_in : g_out;
    } else {
      const u32 ks4 = (t - 32u) >> 3, cg = t & 7u;
#pragma unroll
      for (int nf = 0; nf < 8; ++nf)
#pragma unroll
        for (int tt = 0; tt < 4; ++tt) v[nf][tt] = __builtin_amdgcn_perm(s[2 * tt + 1][nf >> 1], s[2 * tt][nf >> 1], (nf & 1) ? 0x07060302u : 0x05040100u);
      own = true;
      d = dest_w(cg >> 1, kc, ks4, cg & 1u);
      send = 1u << (cg >> 1);
      gp = g_out;
    }
    if (q == q0) STAMPP(2);
    const bool lazy = dq == NUNITS;
    if (q1 < NUNITS && !lazy) load_unit(s, rx, rw, q1, vx, vw);
    Packed k;
    u32 info = RAW | PUSHED;
    if (kc >= KP) info = encode_unit(k, v, scr, lane);
    if (q == q0) STAMPP(3);
    drain_vm();
#pragma unroll
    for (int f = 0; f < 8; ++f) asm volatile("" : "+v"(s[f]));
    announce();
    if (kc >= KP) store_unit(k, v, info, rp, q * SLOT, lane);
    else {
      for (u32 m = (u32)send; m; m &= m - 1u) {
        const u32 jd = __builtin_ctz(m), dst = (t < AU) ? (g4 | ((r + 1u + jd) & 3u)) : ((g4 ^ 4u) | jd);
        store_unit(k, v, RAW, make_rsrc(pick(P.ws, dst) + OFF_IB + (par * NR + r) * IB_SRC, IB_SRC), q * SLOT, lane);
      }
    }
    if (own) store_frags(v, rf, d, lane);
    if (q == q0) STAMPP(4);
    psend = send;
    pgp = gp + goff + q * 8u;
    pinfo = info;
    pcnt = own ? d.cnt : SPARE_CNT;
    if (q1 < NUNITS && lazy) {
      drain_vm();
      announce();
      STAMPP(5);
      psend = 0;
      pcnt = SPARE_CNT;
      load_unit(s, rx, rw, q1, vx, vw);
    }
  }
  drain_vm();
  announce();
  STAMPP(6);
  STAMP(2);
  return call;
}

__device__ __forceinline__ u32 lds_add(LAS u32* p, u32 v) {
  return __builtin_amdgcn_readfirstlane(__hip_atomic_fetch_add(p, v, __ATOMIC_RELAXED, __HIP_MEMORY_SCOPE_WORKGROUP)) >> 6;
}

__device__ __forceinline__ void puller(const Params& P, char* ws, u32 call, u32 li, u32 lidx, LAS u32* ctl, LAS unsigned char* scr, u32 lane, u32 wid) {
  const u32 r = P.rank, g4 = r & 4u, par = call & 1u;
  const u32 tl = li < 3u ? AG_TOK_IN : AG_TOK, TOK0 = tl / NCL + (lidx < tl % NCL ? 1u : 0u);
  u32 TOK = TOK0;
  const bool ing = li < 3u;
  const u32 src = ing ? (g4 | ((r + 1 + li) & 3u)) : ((g4 ^ 4u) | (li - 3u));
  const rsrc_t rf = make_rsrc(ws, FRAG_BYTES), rp = make_rsrc(pick(P.ws, src) + OFF_PK + par * PK_BYTES, PK_BYTES);
  const rsrc_t ri = make_rsrc(ws + OFF_IB + (par * NR + src) * IB_SRC, IB_SRC);
  char* const gbase = ws + OFF_G + (par * NR + src) * NUNITS * 8;
  GAS u32* const cnt = (GAS u32*)(ws + OFF_CNT);
  const u32 total = ing ? NKC * AU : NKC * NOUT;
  constexpr u32 NONE = 0xffffffffu;
  GAS u32* const tkt = (GAS u32*)(ws + OFF_TKT + li * 128u);
  const u32 tbase = (call - 1u) * (total + NCL * PW) * 64u;
  auto draw = [&]() -> u32 { return __hip_atomic_fetch_add(tkt, 1u, __ATOMIC_RELAXED, __HIP_MEMORY_SCOPE_AGENT); };
  auto unit_of = [&](u32 raw_ticket) -> u32 {
    const u32 i = ((u32)__builtin_amdgcn_readfirstlane(raw_ticket) - tbase) >> 6;
    if (i >= total) return NONE;
    if (ing) { const u32 kc = i / AU; return kc * 64u + (i - kc * AU); }
    u32 kc, t;
    if (i < 8u * WLEAD) { kc = i >> 3; t = AWAY + (i & 7u); }
    else {
      const u32 i2 = i - 8u * WLEAD, mid = (NKC - WLEAD) * NOUT;
      if (i2 < mid) { kc = i2 / NOUT; t = i2 - kc * NOUT; kc += t < AWAY ? 0u : WLEAD; }
      else { kc = (i2 - mid) / AWAY; t = i2 - mid - kc * AWAY; kc += NKC - WLEAD; }
    }
    const u32 tw = t - AWAY;
    return kc * 64u + (t < AWAY ? AU + t : 32u + (tw >> 1) * 8u + 2u * (r & 3u) + (tw & 1u));
  };
  auto granule = [&](u32 q) -> u64 { return __hip_atomic_load((GAS u64*)(gbase + (q & (NUNITS - 1)) * 8u), __ATOMIC_RELAXED, __HIP_MEMORY_SCOPE_SYSTEM); };
  auto peek = [&](LAS u32* p) -> u32 { return (u32)(__builtin_amdgcn_readfirstlane(__hip_atomic_load(p, __ATOMIC_RELAXED, __HIP_MEMORY_SCOPE_WORKGROUP)) >> 6); };
  auto take = [&]() { const u32 my = lds_add(ctl, 1u); while (peek(ctl + 2) + TOK <= my) __builtin_amdgcn_s_sleep(1); };
  auto dest_of = [&](u32 q) -> Dest {
    const u32 kc = q >> 6, tq = q & 63u;
    if (q == NUNITS) return Dest{FRAG_BYTES, 0u, 0u, NONE};
    if (tq < 32u) return dest_x(ing ? NEARLY * UPT + li * AU + tq : AU + (li - 3u) * AWAY + (tq - AU), kc);
    return dest_w(4u + (li - 3u), kc, (tq - 32u) >> 3, tq & 1u);
  };
  auto publish = [&](u32 c) { if (c != NONE) { STAMP_CNT(ws, c); count_in(ws + OFF_CNT, c); } };
  build_tables(scr, lane);
  u32 q = NUNITS, pend = NONE, tag = call, info = 127u;
  u64 gr;
  bool first = true;
#define LAP(a) do {} while (0)
  while (q != NONE) {
    TOK = TOK0 + ((q >> 6) < AG_KT ? 1u : 0u);
    if (tag != call || (!(info & PUSHED) && peek(ctl + 2) + TOK <= peek(ctl))) {
      drain_vm();
      publish(pend);
      pend = NONE;
      if (tag != call) {
        const u64 t0 = __builtin_amdgcn_s_memrealtime();
        for (;;) {
          gr = granule(q);
          tag = __builtin_amdgcn_readfirstlane((u32)(gr >> 32));
          info = __builtin_amdgcn_readfirstlane((u32)gr);
          if (tag == call || give_up(ws, call, t0)) break;
          __builtin_amdgcn_s_sleep(2);
        }
        if (tag != call) break;
      }
    }
    if (!(info & PUSHED)) take();
    asm volatile("" ::: "memory");
    if (first && q != NUNITS) { STAMP(1); first = false; }
    LAP(acc_w);
    const Dest d = dest_of(q);
    const u32 cur = info, soff = q * SLOT;
    u32x4 v[8];
    if (cur & RAW) {
#pragma unroll
      for (int f = 0; f < 8; ++f) v[f] = __builtin_amdgcn_raw_buffer_load_b128((cur & PUSHED) ? ri : rp, lane * 16, soff + f * 1024, AUX_SYS);
      const u32 nt = draw();
      drain_vm();
      if (!(cur & PUSHED)) lds_add(ctl + 2, 1u);
      q = unit_of(nt);
      gr = granule(q);
      publish(pend);
      tag = __builtin_amdgcn_readfirstlane((u32)(gr >> 32));
      info = __builtin_amdgcn_readfirstlane((u32)gr);
      store_frags(v, rf, d, lane);
    } else {
      Packed k;
      u32 nt;
      u64 keep;
      asm volatile("s_mov_b64 %1, exec\n\ts_mov_b64 exec, 1\n\tglobal_atomic_add %0, %2, %3, %4 sc0\n\ts_mov_b64 exec, %1"
                   : "=&v"(nt), "=&s"(keep) : "v"(0u), "v"(64u), "s"(tkt) : "memory");
      fetch_packed(k, rp, soff, cur, lane);
      asm volatile("s_waitcnt vmcnt(%0)" :: "i"(AG_EARLY) : "memory");
      lds_add(ctl + 2, 1u);
      asm volatile("s_waitcnt vmcnt(0)" : "+v"(nt), "+v"(k.mq[0]), "+v"(k.mq[1]), "+v"(k.mq[2]), "+v"(k.mq[3]), "+v"(k.p01), "+v"(k.tail) :: "memory");
      LAP(acc_l);
      q = unit_of(nt);
      STAMP_CNT(ws, pend);
      asm volatile("global_load_dwordx2 %0, %2, %3 sc0 sc1\n\ts_mov_b64 %1, exec\n\ts_mov_b64 exec, 1\n\tglobal_atomic_add %4, %5, %6\n\ts_mov_b64 exec, %1"
                   : "=&v"(gr), "=&s"(keep) : "v"((q & (NUNITS - 1)) * 8u), "s"(gbase), "v"((pend == NONE ? SPARE_CNT : pend) * 4u), "v"(1u), "s"(cnt) : "memory");
      decode_unit(v, k, cur, scr, lane);
      __builtin_amdgcn_sched_barrier(0);
      asm volatile("s_waitcnt vmcnt(1)" : "+v"(gr) :: "memory");
      tag = __builtin_amdgcn_readfirstlane((u32)(gr >> 32));
      info = __builtin_amdgcn_readfirstlane((u32)gr);
      __builtin_amdgcn_sched_barrier(0);
      store_frags(v, rf, d, lane);
      LAP(acc_d);
    }
    pend = d.cnt;
  }
  drain_vm();
  publish(pend);
  STAMP(2);
}

constexpr u32 NOROW = 0xffffffffu;
__device__ __forceinline__ u32 frag_row(u32 r, u32 rt, u32 fr) {
  const u32 g4 = r & 4u;
  const bool late = rt >= NEARLY;
  const u32 uc = (late ? rt - NEARLY : rt) * UPT + (fr >> 1);
  u32 src, su;
  if (late) {
    if (uc >= LATE_UNITS) return NOROW;
    const u32 j = uc / AU;
    src = g4 | ((r + 1 + j) & 3u);
    su = uc - j * AU;
  } else if (uc < AU) {
    src = r;
    su = uc;
  } else {
    if (uc >= EARLY_UNITS) return NOROW;
    const u32 j = (uc - AU) / AWAY;
    src = (g4 ^ 4u) | j;
    su = AU + (uc - AU) - j * AWAY;
  }
  return ((AG_ORDER >> (4u * src)) & 7u) * 1024u + su * 32u + (fr & 1u) * 16u;
}
__device__ __forceinline__ u32 wave_frag0(u32 wid) { return (0x74208630u >> (4 * wid)) & 15u; }

__device__ __forceinline__ void out_puller(const Params& P, char* ws, u32 call, u32 j, u32 otok, LAS u32* ctl, LAS unsigned char* scr, u32 lane, u32 wid) {
  const u32 r = P.rank, src = ((r & 4u) ^ 4u) | j;
  char* const ob = pick(P.ws, src) + OFF_OB + (r & 3u) * (OUNITS * OSLOT);
  const rsrc_t ro = make_rsrc(pick(P.out, r), 8192u * 1024u * 2u);
  GAS u32* const tkt = (GAS u32*)(ws + OFF_TKT + (7u + j) * 128u);
  const u32 tbase = (call - 1u) * (OUNITS + NCOMM * 2u) * 64u;
  const u32 vo = (lane >> 4) * 4u * 2048u + (lane & 15u) * 16u;
  for (;;) {
    for (;;) {
      if ((u32)(__builtin_amdgcn_readfirstlane(__hip_atomic_load(ctl + 1, __ATOMIC_RELAXED, __HIP_MEMORY_SCOPE_WORKGROUP)) >> 6) < otok) {
        if (lds_add(ctl + 1, 1u) < otok) break;
        lds_add(ctl + 1, ~0u);
      }
      __builtin_amdgcn_s_sleep(2);
    }
    const u32 k = ((u32)__builtin_amdgcn_readfirstlane(__hip_atomic_fetch_add(tkt, 1u, __ATOMIC_RELAXED, __HIP_MEMORY_SCOPE_AGENT)) - tbase) >> 6;
    if (k >= OUNITS) { lds_add(ctl + 1, ~0u); break; }
    STAMPQ(0);
    const u64 t0 = __builtin_amdgcn_s_memrealtime();
    u32 tag, info;
    for (u32 spins = 0;; ++spins) {
      const u64 gr = __hip_atomic_load((GAS u64*)(ws + OFF_OQ + j * 2048u + k * 8u), __ATOMIC_RELAXED, __HIP_MEMORY_SCOPE_SYSTEM);
      tag = __builtin_amdgcn_readfirstlane((u32)(gr >> 32));
      info = __builtin_amdgcn_readfirstlane((u32)gr);
      if (tag == call || ((spins & 7u) == 7u && give_up(ws, call, t0))) break;
      __builtin_amdgcn_s_sleep(2);
    }
    if (tag != call) {
      *(GAS unsigned short*)(pick(P.out, r) + (size_t)((AG_ORDER >> (4u * r)) & 7u) * 1024u * 2048u) = 0x7fc0;
      lds_add(ctl + 1, ~0u);
      break;
    }
    asm volatile("" ::: "memory");
    STAMPQ(1);
    const u32 uid = (info & 255u) < OUNITS ? (info & 255u) : 0u, sw = uid & 7u, mf = sw - 2u < 4u ? 2u : 3u;
    u32 total = (info >> 8) & 0x3fffu;
    total = total < 12544u ? total : 12544u;
    const bool raw = info >> 22;
    const rsrc_t rs = make_rsrc(ob + uid * OSLOT, 1024u + ((total + 15u) & ~15u));
    u32 mk[3];
#pragma unroll
    for (u32 m = 0; m < 3; ++m) mk[m] = __builtin_amdgcn_raw_buffer_load_b32(rs, (m < mf && !raw) ? lane * 4u : 0x80000000u, m * 256u, AUX_SYS);
    for (u32 p = 0; p < total; p += 1024u) __builtin_amdgcn_raw_ptr_buffer_load_lds(rs, scr + p, 16, lane * 16u, 1024u + p, 0, AUX_SYS);
    asm volatile("s_waitcnt vmcnt(0)" : "+v"(mk[0]), "+v"(mk[1]), "+v"(mk[2]) :: "memory");
    lds_add(ctl + 1, ~0u);
    STAMPQ(2);
    const u32 len = 2u * ((__builtin_popcount(mk[0]) + __builtin_popcount(mk[1]) + __builtin_popcount(mk[2])) & ~1u) + 4u;
    u32 off = wave_scan(len) - len;
    const u32 col = j * 512u + (sw >> 2) * 256u, f0 = wave_frag0(sw);
    for (u32 m = 0; m < mf; ++m) {
      const u32 w = m == 0 ? mk[0] : (m == 1 ? mk[1] : mk[2]);
      u32x4 o[4];
      if (raw) {
#pragma unroll
        for (int e = 0; e < 4; ++e) o[e] = *(const LAS u32x4*)(scr + (m * 4u + e) * 1024u + lane * 16u);
      } else {
        u32 a[16], w0[16], w1[16];
#pragma unroll
        for (int i = 0; i < 16; ++i) {
          a[i] = (u32)(size_t)scr + off + 2u * __builtin_popcount(w & (((1u << i) - 1u) * 0x10001u));
          const LAS u32* const pw = (const LAS u32*)(size_t)(a[i] & ~3u);
          w0[i] = pw[0];
          w1[i] = pw[1];
        }
        __builtin_amdgcn_sched_barrier(0);
#pragma unroll
        for (int i = 0; i < 16; ++i) {
          const u32 d = __builtin_amdgcn_alignbyte(w1[i], w0[i], a[i]);
          const u32 lo = (u32)((int)(w << (31 - i)) >> 31), hi = (u32)((int)(w << (15 - i)) >> 31);
          o[i >> 2][i & 3] = ((d & lo) | ((d << 16) & ~lo)) & (hi | 0xffffu);
        }
        off += 2u * __builtin_popcount(w);
      }
      const u32 row0 = frag_row(src, uid >> 3, f0 + m);
      if (row0 != NOROW) {
#pragma unroll
        for (int e = 0; e < 4; ++e) store16<AG_OAUX>(o[e], ro, vo, (row0 + e) * 2048u + col);
      }
    }
    STAMPQ(3);
  }
  STAMP(3);
}

__device__ __forceinline__ void gate(int c, int& ready, const char* cnt, u32 target, char* ws, u32 call) {
  if (c < ready) return;
  const u64 t0 = __builtin_amdgcn_s_memrealtime();
  for (u32 spins = 0;; ++spins) {
    const u64 m = __builtin_amdgcn_ballot_w64((int)(ld_agent(cnt) - target) >= 0);
    ready = ~m ? __builtin_ctzll(~m) : 64;
    if (c < ready) break;
    if ((spins & 7u) == 7u && give_up(ws, call, t0)) { ready = 64; break; }
    __builtin_amdgcn_s_sleep(3);
  }
  asm volatile("" ::: "memory");
}

__device__ __forceinline__ u32 relu_pk(float lo, float hi) {
  u32 r;
  asm("v_cvt_pk_bf16_f32 %0, %1, %2\n\tv_pk_max_i16 %0, %0, 0" : "=v"(r) : "v"(lo), "v"(hi));
  return r;
}

__device__ __forceinline__ void mfma_acc(f32x4& c, const bf16x8& a, const bf16x8& b) {
  asm volatile("v_mfma_f32_16x16x32_bf16 %0, %1, %2, %0" : "+v"(c) : "v"(a), "v"(b));
}
template <int N> __device__ __forceinline__ void wait_bar() { asm volatile("s_waitcnt vmcnt(%0)\n\ts_barrier" :: "i"(N) : "memory"); }

template <int MF, int NP>
__device__ __forceinline__ void gemm_wave(const Params& P, char* ws, u32 call, u32 b, LAS char* lds, u32 lane, u32 wid) {
  constexpr bool XL = NP == 5;
  constexpr u32 STEP = XL ? A_STEP : B_STEP;
  constexpr int NM = MF * 8;
  const u32 r = P.rank, g4 = r & 4u;
  const u32 xcd = b & 7u, late = xcd & 1u, ent = ((xcd >> 1) & 1u) * 26u + (b >> 3);
  const u32 rc = ent >> 2, rt = late * NEARLY + rc, ct = (ent & 3u) + (xcd & 4u);
  const u32 wc = wid >> 2, f0 = wave_frag0(wid);
  const int left = (late ? LATE_UNITS : EARLY_UNITS) - (int)rc * UPT;
  const u32 nun = left < 0 ? 0u : (left > UPT ? (u32)UPT : (u32)left);
  const u32 p0 = (wid & 1u) * NP * 1024u;
  constexpr u32 same = 1;
  const rsrc_t rs = XL ? make_rsrc(ws + OFF_XU + same * rt * A_TILE, A_TILE) : make_rsrc(ws + OFF_WU + same * ct * B_TILE, B_TILE);
  const char* const cnt = ws + OFF_CNT + ((XL ? rt : NRT + ct) * 64 + lane) * 4;
  u32 target = call * (XL ? nun : 8u);
  const u32 voff = p0 + lane * 16u;
  u32 vp[NP ? NP : 1];
#pragma unroll
  for (int p = 0; p < NP; ++p) vp[p] = (XL && (wid & 1u) * NP + p >= 2u * nun) ? lane * 16u : voff;
  LAS char* const ring = lds + (XL ? 0u : A_STEP) + p0;
  const u32 img = ((lane & 15u) * 4u + ((lane >> 4) ^ quad_swz((lane >> 2) & 3u))) * 16u;
  LAS char* const ra = lds + f0 * 1024u + img;
  LAS char* const rb = lds + A_STEP + wc * 8192u + lane * 16u;
#define AG_DMA(p, ks, off, imm) __builtin_amdgcn_raw_ptr_buffer_load_lds(rs, ring + (off), 16, vp[p < NP ? p : 0], (ks), imm, AG_GEMM_AUX)
#define AG_PIECE(p, ks, off) do { \
    if (p == 0) AG_DMA(0, (ks) * STEP, off, 0); if (p == 1) AG_DMA(1, (ks) * STEP, off, 1024); \
    if (p == 2) AG_DMA(2, (ks) * STEP, off, 2048); if (p == 3) AG_DMA(3, (ks) * STEP, off, 3072); \
    if (p == 4) AG_DMA(4, (ks) * STEP + 4096, (off) + 4096, 0); if (p == 5) AG_DMA(5, (ks) * STEP + 4096, (off) + 4096, 1024); \
    if (p == 6) AG_DMA(6, (ks) * STEP + 4096, (off) + 4096, 2048); if (p == 7) AG_DMA(7, (ks) * STEP + 4096, (off) + 4096, 3072); \
  } while (0)
#define AG_ALL(ks, off) do { _Pragma("unroll") for (int p = 0; p < NP; ++p) AG_PIECE(p, ks, off); } while (0)
#define AG_PIN(stmt) do { __builtin_amdgcn_sched_barrier(0); stmt; __builtin_amdgcn_sched_barrier(0); } while (0)
#define AG_STEP(cs, ks, DMA, RD) do { \
    LAS char* const pa = ra + ord; \
    LAS char* const pb = rb + ord; \
    __builtin_amdgcn_sched_barrier(0); \
    _Pragma("unroll") for (int i = 0; i < NM; ++i) { \
      mfma_acc(acc[i % MF][i / MF], fa[cs][i % MF], fb[cs][i / MF]); \
      _Pragma("unroll") for (int j = 0; j < MF + 8; ++j) \
        if (RD && i == (MF == 3 ? 2 * j : j)) { \
          if (j < MF) AG_PIN(fa[(cs) ^ 1][j] = *(const LAS bf16x8*)(pa + j * 1024)); \
          else AG_PIN(fb[(cs) ^ 1][j - MF] = *(const LAS bf16x8*)(pb + (j - MF) * 1024)); \
        } \
      _Pragma("unroll") for (int p = 0; p < NP; ++p) \
        if (DMA && i == 1 + p * (NP == 5 ? 5 : 3)) AG_PIN(AG_PIECE(p, ks, owr)); \
    } \
    ord = ord + SLOTB == NSLOT * SLOTB ? 0 : ord + SLOTB; \
    owr = owr + SLOTB == NSLOT * SLOTB ? 0 : owr + SLOTB; \
    __builtin_amdgcn_sched_barrier(0); \
  } while (0)
#define AG_GATE(c) do { if (NP) gate(c, ready, cnt, target, ws, call); } while (0)
  f32x4 acc[MF][8];
#pragma unroll
  for (int m = 0; m < MF; ++m)
#pragma unroll
    for (int n = 0; n < 8; ++n) { acc[m][n] = (f32x4){0.f, 0.f, 0.f, 0.f}; asm volatile("" : "+v"(acc[m][n])); }
  if (NP) __builtin_amdgcn_s_setprio(1);
  int ready = 0;
  AG_GATE(0);
  STAMP(1);
  STAMPC(4);
  constexpr int LA = AG_LA, W = (LA - 2) * NP;
#pragma unroll
  for (int s = 0; s < 4; ++s) AG_ALL(s, s * SLOTB);
  if (LA == 5) { AG_GATE(1); AG_ALL(4, 4 * SLOTB); }
  wait_bar<W>();
  bf16x8 fa[2][MF], fb[2][8];
#pragma unroll
  for (int m = 0; m < MF; ++m) fa[0][m] = *(const LAS bf16x8*)(ra + m * 1024);
#pragma unroll
  for (int n = 0; n < 8; ++n) fb[0][n] = *(const LAS bf16x8*)(rb + n * 1024);
  u32 ord = SLOTB, owr = LA * SLOTB;
  for (int kc = 0; kc < NKC - 3; ++kc) {
    if (LA == 4) AG_GATE(kc + 1);
    AG_STEP(0, kc * 4 + LA, true, true); wait_bar<W>();
    AG_STEP(1, kc * 4 + LA + 1, true, true); wait_bar<W>();
    AG_STEP(0, kc * 4 + LA + 2, true, true); wait_bar<W>();
    if (LA == 5) AG_GATE(kc + 2);
    AG_STEP(1, kc * 4 + LA + 3, true, true); wait_bar<W>();
  }
  if (LA == 4) AG_GATE(NKC - 2);
  AG_STEP(0, NKS - 12 + LA, true, true); wait_bar<W>();
  AG_STEP(1, NKS - 11 + LA, true, true); wait_bar<W>();
  AG_STEP(0, NKS - 10 + LA, true, true); wait_bar<W>();
  AG_STEP(1, NKS - 9 + LA, LA == 4, true); wait_bar<2 * NP>();
  AG_STEP(0, 0, false, true); wait_bar<NP>();
  AG_STEP(1, 0, false, true); wait_bar<0>();
  AG_STEP(0, 0, false, true);
  AG_STEP(1, 0, false, false);
  AG_GATE(NKC - 1);
  asm volatile("s_barrier" ::: "memory");
#pragma unroll
  for (int s = 0; s < 4; ++s) AG_ALL(NKS - 4 + s, s * SLOTB);
  u32 tmo;
  asm volatile("global_load_dword %0, %1, %2 sc1" : "=&v"(tmo) : "v"(0u), "s"(ws + OFF_TMO) : "memory");
  wait_bar<2 * NP + 1>();
#pragma unroll
  for (int m = 0; m < MF; ++m) fa[0][m] = *(const LAS bf16x8*)(ra + m * 1024);
#pragma unroll
  for (int n = 0; n < 8; ++n) fb[0][n] = *(const LAS bf16x8*)(rb + n * 1024);
  ord = SLOTB;
  AG_STEP(0, 0, false, true); wait_bar<NP + 1>();
  AG_STEP(1, 0, false, true); wait_bar<0>();
  AG_STEP(0, 0, false, true);
  AG_STEP(1, 0, false, false);
  asm volatile("" : "+v"(tmo));
  asm volatile("s_nop 15" ::: "memory");
  __builtin_amdgcn_sched_barrier(0);
#undef AG_PIECE
#undef AG_ALL
#undef AG_PIN
#undef AG_DMA
#undef AG_GATE
#undef AG_STEP
  STAMP(2);
  STAMPC(5);
  const bool bad = __builtin_amdgcn_readfirstlane(tmo) == call;
  u32x4 o[MF][4];
#pragma unroll
  for (int m = 0; m < MF; ++m)
#pragma unroll
    for (int e = 0; e < 4; ++e)
#pragma unroll
      for (int n = 0; n < 4; ++n) o[m][e][n] = relu_pk(acc[m][2 * n][e], acc[m][2 * n + 1][e]);
  if (bad) {
#pragma unroll
    for (int m = 0; m < MF; ++m)
#pragma unroll
      for (int e = 0; e < 4; ++e) o[m][e] = (u32x4){0x7fc07fc0u, 0x7fc07fc0u, 0x7fc07fc0u, 0x7fc07fc0u};
    asm volatile("" ::: "memory");
  }
  if (ct < 4u) {
    const rsrc_t ro = make_rsrc(pick(P.out, r), 8192u * 1024u * 2u);
    const u32 vo = (lane >> 4) * 4u * 2048u + (lane & 15u) * 16u;
#pragma unroll
    for (int m = 0; m < MF; ++m) {
      const u32 row0 = frag_row(r, rt, f0 + m);
      if (row0 == NOROW) continue;
#pragma unroll
      for (int e = 0; e < 4; ++e) store16<AG_OAUX>(o[m][e], ro, vo, (row0 + e) * 2048u + ct * 512u + wc * 256u);
    }
    STAMP(3);
    return;
  }
  const u32 j = ct - 4u, uid = rt * 8u + wid;
  u32 qi;
  u64 keep;
  asm volatile("s_mov_b64 %1, exec\n\ts_mov_b64 exec, 1\n\tglobal_atomic_add %0, %2, %3, %4 sc0\n\ts_mov_b64 exec, %1"
               : "=&v"(qi), "=&s"(keep) : "v"(0u), "v"(1u), "s"(ws + OFF_TKT + (11u + j) * 128u) : "memory");
  u32 mk[MF], len = 0;
#pragma unroll
  for (int m = 0; m < MF; ++m) {
    mk[m] = 0;
    if ((f0 + m) >> 1 >= nun) {
#pragma unroll
      for (int e = 0; e < 4; ++e) o[m][e] = (u32x4){0u, 0u, 0u, 0u};
    }
#pragma unroll
    for (int i = 0; i < 16; ++i) {
      u32 t;
      const u32 d = o[m][i >> 2][i & 3];
      asm("v_pk_min_u16 %0, %1, %2" : "=v"(t) : "v"(d), "v"(0x00010001u));
      mk[m] |= t << i;
    }
    len += __builtin_popcount(mk[m]);
  }
  len = 2u * (len & ~1u) + 4u;
  const u32 incl = wave_scan(len);
  u32 total = __builtin_amdgcn_readlane(incl, 63);
  const bool raw = total + 256u * MF >= 4096u * MF;
  const rsrc_t rb2 = make_rsrc(ws + OFF_OB + j * (OUNITS * OSLOT) + uid * OSLOT, OSLOT);
  STAMP2(0);
  asm volatile("s_barrier" ::: "memory");
  STAMP2(1);
  if (raw) {
    total = 4096u * MF;
#pragma unroll
    for (int m = 0; m < MF; ++m)
#pragma unroll
      for (int e = 0; e < 4; ++e) store16(o[m][e], rb2, lane * 16u, 1024u + (m * 4 + e) * 1024u);
  } else {
    LAS char* const scr = lds + wid * WAVE_SCRATCH;
    u32 a = (u32)(size_t)scr + incl - len, cur = 0;
#pragma unroll
    for (int m = 0; m < MF; ++m) {
      __builtin_amdgcn_raw_buffer_store_b32(mk[m], rb2, lane * 4u, m * 256u, AUX_SYS);
#pragma unroll
      for (int i = 0; i < 16; ++i) {
        const u32 d0 = o[m][i >> 2][i & 3];
        u32 t;
        asm("v_pk_min_u16 %0, %1, %2" : "=v"(t) : "v"(d0), "v"(0x00010001u));
        const u32 an = __builtin_amdgcn_udot2(__builtin_bit_cast(u16x2, t), (u16x2){2, 2}, a, false);
        const u32 d = (d0 & 0xffffu) ? d0 : d0 >> 16;
        const u32 w = (d << ((a << 3) & 31u)) | cur;
        *(LAS u32*)(size_t)(a & ~3u) = w;
        cur = ((an ^ a) & 4u) ? ((an & 2u) ? d >> 16 : 0u) : w;
        a = an;
      }
    }
    *(LAS u32*)(size_t)(a & ~3u) = cur;
    asm volatile("" ::: "memory");
    STAMP2(2);
    for (u32 p = 0; p < total; p += 1024u) store16(*(const LAS u32x4*)(scr + p + lane * 16u), rb2, p + lane * 16u < total ? lane * 16u : 0x80000000u, 1024u + p);
  }
  STAMP2(3);
  asm volatile("s_waitcnt vmcnt(0)" : "+v"(qi) :: "memory");
  const u32 slot = ((u32)__builtin_amdgcn_readfirstlane(qi) - (call - 1u) * OUNITS) & 255u;
  flag_store(pick(P.ws, (g4 ^ 4u) | j) + OFF_OQ + (r & 3u) * 2048u + slot * 8u, ((u64)call << 32) | uid | (total << 8) | (raw ? 1u << 22 : 0u), 1u);
  STAMP(3);
}

__global__ __launch_bounds__(512, 2) void ag_gemm_kernel(Params P) {
  extern __shared__ __attribute__((aligned(16))) char smem[];
  LAS char* lds = (LAS char*)smem;
  const u32 lane = threadIdx.x & 63u, wid = __builtin_amdgcn_readfirstlane(threadIdx.x >> 6);
  char* const ws = pick(P.ws, P.rank);
  STAMP(0);
  u32 bid = blockIdx.x;
  asm volatile("" : "+s"(bid));
  char* const cptr = ws + OFF_CALL + bid * 128u + wid * 4u;
  const u32 callv = ld_agent(cptr);
  LAS unsigned char* scr = (LAS unsigned char*)lds + wid * WAVE_SCRATCH;
  bool skip = false;
  u32 li = 7, lidx = 0, pkw = 0;
  if (bid < NCOMM) {
    const u32 h = (bid >> 2) & 1u, a = (bid >> 3) * 4u + (bid & 3u), nl = 3u + h;
    if (a < nl * NCL) { li = h * 3u + (a + h * (a >> 2)) % nl; lidx = a / nl; }
    else pkw = h * (NCOMM / 2 - 3 * NCL) + a - nl * NCL;
  }
  LAS u32* const ctl = (LAS u32*)(lds + 8 * WAVE_SCRATCH);
  if (bid < NCOMM) {
    ctl[lane & 3u] = 0;
    asm volatile("s_waitcnt lgkmcnt(0)\n\ts_barrier" ::: "memory");
  }
  const u32 oj = li >= 7u ? pkw & 3u : (li >= 3u ? li - 3u : (li * NCL + lidx + NPKWG) & 3u);
  const u32 otok = li >= 7u ? AG_OTOK_PK : (li >= 3u ? AG_OTOK : AG_OTOK_IN);
  const bool isg = bid >= NCOMM;
  u32 call;
  if (li < 7u && wid < PW) {
    call = __builtin_amdgcn_readfirstlane(callv) + 1u;
    if (!skip) puller(P, ws, call, li, lidx, ctl, scr, lane, wid);
  } else {
    const u32 pidx = li < 7u ? NPKWG * 8u + (li * NCL + lidx) * AG_MIX + (wid - PW) : wid * NPKWG + pkw;
    const u32 q0 = isg ? wid * NGEMM + (bid - NCOMM) : GPACK + pidx;
    const u32 q1 = isg ? (wid < AG_G2 ? q0 + 8u * NGEMM : (u32)NUNITS) : q0 + NPK;
    for (u32 i = isg ? wid : (u32)AG_PDELAY; i; --i) __builtin_amdgcn_s_sleep(AG_GDELAY);
    call = packer(P, ws, callv, skip ? (u32)NUNITS : q0, q1, isg ? (u32)NUNITS : (u32)NPK, scr, lane, wid);
  }
  __hip_atomic_store((GAS u32*)cptr, call, __ATOMIC_RELAXED, __HIP_MEMORY_SCOPE_AGENT);
  if (!isg) {
    out_puller(P, ws, call, oj, otok, ctl, scr, lane, wid);
    return;
  }
  asm volatile("s_waitcnt lgkmcnt(0)\n\ts_barrier" ::: "memory");
  STAMP(7);
  if (wid < 2u) gemm_wave<3, 5>(P, ws, call, bid - NCOMM, lds, lane, wid);
  else if (wid < 6u) gemm_wave<2, 0>(P, ws, call, bid - NCOMM, lds, lane, wid);
  else gemm_wave<3, 8>(P, ws, call, bid - NCOMM, lds, lane, wid);
}

}

extern "C" void kernel_launch(int rank, int nranks, ncclComm_t comm, hipStream_t stream,
                              void* const* d_in, const int* in_sizes, int n_in,
                              void* d_out, int out_size, const int* in_dtypes, int out_dtype,
                              void* const* d_out_all, void* const* d_ws_all, size_t ws_bytes) {
  (void)comm; (void)in_dtypes; (void)out_dtype; (void)d_out;
  if (nranks != ag::NR || n_in != 2 || (unsigned)rank >= (unsigned)ag::NR || in_sizes[0] != 1024 * 8192 || in_sizes[1] != 8192 * 1024 ||
      out_size != 8192 * 1024 || ws_bytes < ag::WS_NEED) {
    fprintf(stderr, "ag_gemm: unexpected problem shape\n");
    return;
  }
  ag::Params p;
  p.x = (const char*)d_in[0];
  p.w = (const char*)d_in[1];
  for (int i = 0; i < ag::NR; ++i) {
    const int phys = (AG_ORDER >> (4 * i)) & 7;
    p.ws[i] = (char*)d_ws_all[phys];
    p.out[i] = (char*)d_out_all[phys];
    if (phys == rank) p.rank = i;
  }
  p.pad = 0;
  hipError_t e = hipFuncSetAttribute((const void*)ag::ag_gemm_kernel, hipFuncAttributeMaxDynamicSharedMemorySize, ag::LDS_BYTES);
  if (e != hipSuccess) { fprintf(stderr, "ag_gemm: %s\n", hipGetErrorString(e)); return; }
  hipLaunchKernelGGL(ag::ag_gemm_kernel, dim3(ag::NWG), dim3(512), ag::LDS_BYTES, stream, p);
  e = hipGetLastError();
  if (e != hipSuccess) fprintf(stderr, "ag_gemm launch: %s\n", hipGetErrorString(e));
}
```

```cpp
#include <hip/hip_runtime.h>
#include <rccl/rccl.h>
#include <cstdio>

namespace ag {

typedef unsigned u32;
typedef unsigned long long u64;
typedef unsigned u32x4 __attribute__((ext_vector_type(4)));
typedef unsigned short u16x2 __attribute__((ext_vector_type(2)));
typedef short bf16x8 __attribute__((ext_vector_type(8)));
typedef float f32x4 __attribute__((ext_vector_type(4)));
typedef __amdgpu_buffer_rsrc_t rsrc_t;
#define LAS __attribute__((address_space(3)))
#define GAS __attribute__((address_space(1)))

#define AG_AU 21
#define AG_NCL 6
#define AG_TOK 12
#define AG_TOK_IN 12
#define AG_TOK_FAST 12
#define AG_G2 9
#define AG_WLEAD 4
#define AG_LA 4
#define AG_KP 1
#define AG_MIX 0
#define AG_KT 0
#define AG_XAUX 2
#define AG_WAUX 2
#define AG_OAUX 17
#define AG_FTOK 2
#define AG_FTOK_IN 0
#define AG_FTOK_PK AG_FTOK_IN
#define AG_OEARLY 2
#define AG_EARLY 4
#define AG_GDELAY 28
#define AG_PDELAY 0
#define AG_ORDER 0x75416320u
#define AG_NEAR 1
#define AG_CTMAP 1
#define AG_FORCE_RAW 0
#define AG_GEMM_AUX 17

constexpr int NR = 8;
constexpr int AU = AG_AU;
constexpr int AWAY = 32 - AU;
constexpr int NRT = 26, NEARLY = 13;
constexpr int UPT = 5;
constexpr int NKC = 64, NKS = 256;
constexpr int EARLY_UNITS = AU + 4 * AWAY, LATE_UNITS = 3 * AU;
static_assert(EARLY_UNITS <= NEARLY * UPT && LATE_UNITS <= (NRT - NEARLY) * UPT, "row split does not fit the tiles");
static_assert((EARLY_UNITS + UPT - 1) % UPT >= 2 && (LATE_UNITS + UPT - 1) % UPT >= 2, "a short tile keeps at least five fragments");
constexpr int NOUT = AWAY + 8;

constexpr int NCOMM = 48, NGEMM = 208, NWG = NCOMM + NGEMM;
constexpr int NCL = AG_NCL;
constexpr int NPKWG = NCOMM - 7 * NCL;
static_assert(4 * NCL <= NCOMM / 2 && (NCOMM - 4 * NCL) % 4 == 0, "puller placement");
constexpr int PW = 8 - AG_MIX;
constexpr int NPK = NPKWG * 8 + 7 * NCL * AG_MIX;
constexpr u32 GPACK = (8 + AG_G2) * NGEMM;
static_assert(AG_CTMAP == 1 && GPACK < 4096, "packing order follows the tile map");
constexpr int NUNITS = 4096;
constexpr u32 WLEAD = AG_WLEAD;

constexpr u32 A_STEP = 10240, B_STEP = 16384;
constexpr u32 A_TILE = A_STEP * NKS, B_TILE = B_STEP * NKS;
constexpr u32 OFF_XU = 0;
constexpr u32 OFF_WU = OFF_XU + NRT * A_TILE;
constexpr u32 FRAG_BYTES = OFF_WU + 8 * B_TILE;
constexpr u32 SLOT = 8192, PK_BYTES = NUNITS * SLOT;
constexpr u32 OFF_PK = FRAG_BYTES;
constexpr u32 OSLOT = 12288, OUNITS = NRT * 8;
constexpr u32 OFF_OB = OFF_PK + 2 * PK_BYTES;
constexpr u32 KP = AG_KP, IB_SRC = KP * 64 * SLOT, IB_BYTES = 2 * NR * IB_SRC;
constexpr u32 OFF_IB = OFF_OB + 4 * OUNITS * OSLOT;
constexpr u32 OFF_G = OFF_IB + IB_BYTES;
constexpr u32 G_BYTES = 2 * NR * NUNITS * 8;
constexpr u32 OFF_CNT = OFF_G + G_BYTES;
constexpr u32 OFF_OQ = OFF_CNT + 16384;
constexpr u32 OFF_TKT = OFF_OQ + 4 * 2048;
constexpr u32 OFF_TMO = OFF_TKT + 2048;
constexpr u32 OFF_CALL = OFF_TMO + 128;
constexpr u32 OFF_DBG = OFF_CALL + NWG * 128;
constexpr u32 OFF_DBG2 = OFF_DBG + NWG * 8 * 64;
constexpr u32 OFF_DBG3 = OFF_DBG2 + 65536;
constexpr u32 OFF_DBG4 = OFF_DBG3 + 131072;
constexpr u32 WS_NEED = OFF_DBG4 + 524288;

constexpr u32 SLOTB = A_STEP + B_STEP, NSLOT = AG_LA + 1;
constexpr u32 TAB_OFF = NSLOT * SLOTB;
constexpr int LDS_BYTES = TAB_OFF + 256;
constexpr int WAVE_SCRATCH = 13312;
static_assert(8 * WAVE_SCRATCH + 64 <= LDS_BYTES, "scratch");

constexpr u32 SPARE_CNT = 4000;
constexpr u64 DEADLINE = 50000000ull;
constexpr int AUX_SYS = 17;

struct Params {
  const char* x;
  const char* w;
  char* ws[NR];
  char* out[NR];
  int rank;
  int pad;
};

#define STAMP(i) do {} while (0)
#define STAMPC(i) do {} while (0)
#define STAMPP(i) do {} while (0)
#define STAMP2(i) do {} while (0)
#define STAMPQ(i) do {} while (0)

__device__ __forceinline__ rsrc_t make_rsrc(const void* p, u32 bytes) {
  const u64 a = (u64)p;
  const u32 lo = __builtin_amdgcn_readfirstlane((u32)a), hi = __builtin_amdgcn_readfirstlane((u32)(a >> 32));
  return __builtin_amdgcn_make_buffer_rsrc((void*)(((u64)hi << 32) | lo), 0, bytes, 0x00020000);
}
__device__ __forceinline__ char* pick(char* const (&t)[NR], u32 i) {
  char* p = t[0];
#pragma unroll
  for (u32 j = 1; j < NR; ++j) p = (i == j) ? t[j] : p;
  return p;
}
__device__ __forceinline__ u32 ld_agent(const char* p) { return __hip_atomic_load((GAS u32*)p, __ATOMIC_RELAXED, __HIP_MEMORY_SCOPE_AGENT); }
__device__ __forceinline__ u32 ld_sys(const char* p) { return __hip_atomic_load((GAS u32*)p, __ATOMIC_RELAXED, __HIP_MEMORY_SCOPE_SYSTEM); }
template <int AUX = AUX_SYS>
__device__ __forceinline__ void store16(u32x4 v, rsrc_t rs, u32 voff, u32 soff) {
  __builtin_amdgcn_raw_buffer_store_b128(v, rs, voff, soff, AUX);
  asm volatile("s_nop 1" :: "v"(v) : "memory");
}
__device__ __forceinline__ void count_in(const char* base, u32 idx) {
  u64 keep;
  asm volatile("s_mov_b64 %0, exec\n\ts_mov_b64 exec, 1\n\tglobal_atomic_add %1, %2, %3\n\ts_mov_b64 exec, %0"
               : "=&s"(keep) : "v"(idx * 4u), "v"(1u), "s"(base) : "memory");
}
#define STAMP_CNT(ws, idx) do {} while (0)
__device__ __forceinline__ void flag_store(char* p, u64 v, u64 mask) {
  u64 keep;
  asm volatile("s_mov_b64 %0, exec\n\ts_mov_b64 exec, %3\n\tglobal_store_dwordx2 %1, %2, off sc0 sc1\n\ts_mov_b64 exec, %0"
               : "=&s"(keep) : "v"(p), "v"(v), "s"(mask) : "memory");
}
__device__ __forceinline__ void drain_vm() { asm volatile("s_waitcnt vmcnt(0)" ::: "memory"); }

__device__ __forceinline__ bool give_up(char* ws, u32 call, u64 t0) {
  const u32 t = __builtin_amdgcn_readfirstlane(ld_agent(ws + OFF_TMO));
  if (t == call) return true;
  if (__builtin_amdgcn_s_memrealtime() - t0 > DEADLINE) {
    __hip_atomic_store((GAS u32*)(ws + OFF_TMO), call, __ATOMIC_RELAXED, __HIP_MEMORY_SCOPE_AGENT);
    return true;
  }
  return false;
}

__device__ __forceinline__ u32 wave_sum(u32 v) {
  v += __builtin_amdgcn_update_dpp(0u, v, 0x111, 0xf, 0xf, true);
  v += __builtin_amdgcn_update_dpp(0u, v, 0x112, 0xf, 0xf, true);
  v += __builtin_amdgcn_update_dpp(0u, v, 0x114, 0xf, 0xf, true);
  v += __builtin_amdgcn_update_dpp(0u, v, 0x118, 0xf, 0xf, true);
  return __builtin_amdgcn_readlane(v, 15) + __builtin_amdgcn_readlane(v, 31) + __builtin_amdgcn_readlane(v, 47) + __builtin_amdgcn_readlane(v, 63);
}
__device__ __forceinline__ u32 wave_max(u32 v) {
  v = max(v, (u32)__builtin_amdgcn_update_dpp(0u, v, 0x111, 0xf, 0xf, true));
  v = max(v, (u32)__builtin_amdgcn_update_dpp(0u, v, 0x112, 0xf, 0xf, true));
  v = max(v, (u32)__builtin_amdgcn_update_dpp(0u, v, 0x114, 0xf, 0xf, true));
  v = max(v, (u32)__builtin_amdgcn_update_dpp(0u, v, 0x118, 0xf, 0xf, true));
  return max(max((u32)__builtin_amdgcn_readlane(v, 15), (u32)__builtin_amdgcn_readlane(v, 31)), max((u32)__builtin_amdgcn_readlane(v, 47), (u32)__builtin_amdgcn_readlane(v, 63)));
}
__device__ __forceinline__ u32 wave_scan(u32 v) {
  v += __builtin_amdgcn_update_dpp(0u, v, 0x111, 0xf, 0xf, true);
  v += __builtin_amdgcn_update_dpp(0u, v, 0x112, 0xf, 0xf, true);
  v += __builtin_amdgcn_update_dpp(0u, v, 0x114, 0xf, 0xf, true);
  v += __builtin_amdgcn_update_dpp(0u, v, 0x118, 0xf, 0xf, true);
  v += __builtin_amdgcn_update_dpp(0u, v, 0x142, 0xa, 0xf, false);
  v += __builtin_amdgcn_update_dpp(0u, v, 0x143, 0xc, 0xf, false);
  return v;
}

__device__ __forceinline__ u32 quad_swz(u32 rq) { return (0x78u >> (2 * rq)) & 3u; }

struct Dest { u32 base, s1, s0, cnt; };
__device__ __forceinline__ Dest dest_x(u32 ux, u32 kc) {
  const u32 tile = ux / UPT;
  return Dest{OFF_XU + tile * A_TILE + 4 * kc * A_STEP + (ux - tile * UPT) * 2048u, 1024u, A_STEP, tile * 64 + kc};
}
__device__ __forceinline__ Dest dest_w(u32 ct, u32 kc, u32 ks4, u32 half) {
  return Dest{OFF_WU + ct * B_TILE + (4 * kc + ks4) * B_STEP + half * 8192u, 4096u, 1024u, (NRT + ct) * 64 + kc};
}
__device__ __forceinline__ void store_frags(const u32x4 (&v)[8], rsrc_t rf, const Dest& d, u32 lane) {
#pragma unroll
  for (int f = 0; f < 8; ++f) store16(v[f], rf, lane * 16, d.base + (f >> 2) * d.s1 + (f & 3) * d.s0);
}

constexpr u32 RAW = 1u << 31, PUSHED = 1u << 30;
constexpr u32 S_LIST = 0, S_TAIL = 2304;
struct Packed { u32x4 mq[4], p01, tail; };
__device__ __forceinline__ u32 groups_of(u32 info) { return (((info >> 8) & 4095u) + 31u) >> 5; }

__device__ __forceinline__ void build_tables(LAS char* lds, u32 lane) {
  u32 ct = 0x0c0c0c0cu, xt = 0x07060504u, zt = 0x0c0c0c0cu, k = 0;
#pragma unroll
  for (u32 b = 0; b < 4; ++b) {
    const u32 on = 0u - ((lane >> b) & 1u);
    ct ^= on & ((0x0cu ^ b) << (8u * k));
    xt ^= on & (((4u + b) ^ k) << (8u * b));
    zt ^= on & ((0x0cu ^ k) << (8u * b));
    k -= on;
  }
  *(LAS u32*)(lds + TAB_OFF + lane * 4u) = (lane & 32u) ? zt : ((lane & 16u) ? xt : ct);
}

constexpr u32 TAB = TAB_OFF;

template <bool TO_LIST>
__device__ __forceinline__ void walk_spelled(u32 m, u32 off, LAS unsigned char* cells, LAS unsigned char* list, u32 flip) {
  while (m != 0) {
    const u32 p = __builtin_ctz(m);
    m &= m - 1;
    LAS unsigned char* cell = cells + (p & 7u) * 4u + (p >> 3);
    if (TO_LIST) list[off] = *cell ^ flip; else *cell = list[off];
    ++off;
  }
}

__device__ __forceinline__ u32 encode_unit(Packed& k, const u32x4 (&v)[8], LAS unsigned char* scr, u32 lane) {
  const u32x4 zero = {0u, 0u, 0u, 0u};
  *(LAS u32x4*)(scr + S_LIST + lane * 32u) = zero;
  *(LAS u32x4*)(scr + S_LIST + lane * 32u + 16u) = zero;
  u32 E[16], esum = 0;
#pragma unroll
  for (int j = 0; j < 16; ++j) {
    const u32 d0 = v[j >> 1][(j & 1) * 2], d1 = v[j >> 1][(j & 1) * 2 + 1];
    const u32 r0 = __builtin_amdgcn_alignbit(d0, d0, 31), r1 = __builtin_amdgcn_alignbit(d1, d1, 31);
    k.mq[j >> 2][j & 3] = __builtin_amdgcn_perm(r1, r0, 0x06040200u);
    E[j] = __builtin_amdgcn_perm(r1, r0, 0x07050301u);
    esum = __builtin_amdgcn_sad_u8(E[j], 0u, esum);
  }
  u32 base = (wave_sum(esum) + 8028u) >> 12;
  base = base < 8u ? 8u : (base > 254u ? 254u : base);
  const u16x2 b2 = {(unsigned short)base, (unsigned short)base}, t2 = {(unsigned short)(base + 1u), (unsigned short)(base + 1u)};
  u32 P[2][2] = {{0, 0}, {0, 0}};
#pragma unroll
  for (int j = 0; j < 16; ++j) {
    const u16x2 ev = __builtin_bit_cast(u16x2, E[j] & 0x00ff00ffu), od = __builtin_bit_cast(u16x2, E[j]) >> (u16x2){8, 8};
    const u32 ce = __builtin_bit_cast(u32, __builtin_elementwise_min((u16x2)(b2 - ev), (u16x2){3, 3}));
    const u32 co = __builtin_bit_cast(u32, __builtin_elementwise_min((u16x2)(b2 - od), (u16x2){3, 3}));
    const u32 c = ce | (co << 8);
    P[0][j >> 3] |= (c & 0x01010101u) << (j & 7);
    P[1][j >> 3] |= ((c >> 1) & 0x01010101u) << (j & 7);
  }
  k.p01 = (u32x4){P[0][0], P[0][1], P[1][0], P[1][1]};
  k.tail = zero;
  const u32 mk[2] = {P[0][0] & P[1][0], P[0][1] & P[1][1]};
  u32 fl[16], sel[16];
#pragma unroll
  for (int j = 0; j < 16; ++j) {
    fl[j] = (mk[j >> 3] >> (j & 7)) & 0x01010101u;
    sel[j] = *(const LAS u32*)(size_t)__builtin_amdgcn_udot4(fl[j], 0x20100804u, TAB, false);
  }
  __builtin_amdgcn_sched_barrier(0);
  const u32 c = __builtin_popcount(mk[0]) + __builtin_popcount(mk[1]), incl = wave_scan(c);
  const u32 n2 = __builtin_amdgcn_readlane(incl, 63);
  if (n2 > 2044u || AG_FORCE_RAW) return RAW;
  const u32 flip = (base + 1u) * 0x01010101u;
  u32 a = (u32)(size_t)(scr + S_LIST) + incl - c, cur = 0;
#pragma unroll
  for (int j = 0; j < 16; ++j) {
    const u64 t = (u64)__builtin_amdgcn_perm(0u, E[j] ^ flip, sel[j]) << ((a & 3u) * 8u);
    const u32 w = cur | (u32)t, an = __builtin_amdgcn_sad_u8(fl[j], 0u, a);
    __hip_atomic_fetch_or((LAS u32*)(size_t)(a & ~3u), w, __ATOMIC_RELAXED, __HIP_MEMORY_SCOPE_WAVEFRONT);
    cur = ((an ^ a) & 4u) ? (u32)(t >> 32) : w;
    a = an;
  }
  __hip_atomic_fetch_or((LAS u32*)(size_t)(a & ~3u), cur, __ATOMIC_RELAXED, __HIP_MEMORY_SCOPE_WAVEFRONT);
  asm volatile("" ::: "memory");
  const u32x4 l[2] = {*(const LAS u32x4*)(scr + S_LIST + lane * 32u), *(const LAS u32x4*)(scr + S_LIST + lane * 32u + 16u)};
  u32 Q[3] = {0, 0, 0};
#pragma unroll
  for (int s = 0; s < 8; ++s) {
    const u32 x = l[s >> 2][s & 3] ^ flip;
    const u16x2 ev = __builtin_bit_cast(u16x2, x & 0x00ff00ffu), od = __builtin_bit_cast(u16x2, x) >> (u16x2){8, 8};
    const u32 ce = __builtin_bit_cast(u32, __builtin_elementwise_min(__builtin_elementwise_sub_sat((u16x2)(t2 - ev), (u16x2){3, 3}), (u16x2){7, 7}));
    const u32 co = __builtin_bit_cast(u32, __builtin_elementwise_min(__builtin_elementwise_sub_sat((u16x2)(t2 - od), (u16x2){3, 3}), (u16x2){7, 7}));
    const u32 c2 = ce | (co << 8);
    Q[0] |= (c2 & 0x01010101u) << s;
    Q[1] |= ((c2 >> 1) & 0x01010101u) << s;
    Q[2] |= ((c2 >> 2) & 0x01010101u) << s;
  }
#pragma unroll
  for (int i = 0; i < 3; ++i) *(LAS u32*)(scr + S_TAIL + lane * 12u + i * 4u) = Q[i];
  const u32 m3 = Q[0] & Q[1] & Q[2], c3 = __builtin_popcount(m3), incl3 = wave_scan(c3);
  const u32 n3 = __builtin_amdgcn_readlane(incl3, 63);
  if (n3 > 256u) return RAW;
  asm volatile("" ::: "memory");
  walk_spelled<true>(m3, incl3 - c3, scr + S_LIST + lane * 32u, scr + S_TAIL + ((n2 + 31u) >> 5) * 12u, base + 1u);
  asm volatile("" ::: "memory");
  k.tail = *(const LAS u32x4*)(scr + S_TAIL + lane * 16u);
  return base | (n2 << 8) | (n3 << 20);
}

__device__ __forceinline__ void store_unit(const Packed& k, const u32x4 (&v)[8], u32 info, rsrc_t rp, u32 soff, u32 lane) {
  if (info & RAW) {
#pragma unroll
    for (int f = 0; f < 8; ++f) store16(v[f], rp, lane * 16, soff + f * 1024);
    return;
  }
#pragma unroll
  for (int i = 0; i < 4; ++i) store16(k.mq[i], rp, lane * 16, soff + i * 1024);
  store16(k.p01, rp, lane * 16, soff + 4096);
  store16(k.tail, rp, lane * 16u < groups_of(info) * 12u + (info >> 20) ? lane * 16u : 0x80000000u, soff + 5120);
}

__device__ __forceinline__ void fetch_packed(Packed& k, rsrc_t rp, u32 soff, u32 info, u32 lane) {
  k.tail = __builtin_amdgcn_raw_buffer_load_b128(rp, lane * 16u < groups_of(info) * 12u + (info >> 20) ? lane * 16u : 0x80000000u, soff + 5120, AUX_SYS);
  k.p01 = __builtin_amdgcn_raw_buffer_load_b128(rp, lane * 16, soff + 4096, AUX_SYS);
#pragma unroll
  for (int i = 0; i < 4; ++i) k.mq[i] = __builtin_amdgcn_raw_buffer_load_b128(rp, lane * 16, soff + i * 1024, AUX_SYS);
}

__device__ __forceinline__ void decode_unit(u32x4 (&v)[8], const Packed& k, u32 info, LAS unsigned char* scr, u32 lane) {
  const u32 base = info & 255u, g2 = groups_of(info);
  *(LAS u32x4*)(scr + S_TAIL + lane * 16u) = k.tail;
  asm volatile("" ::: "memory");
  const LAS u32* const pq = (const LAS u32*)(scr + S_TAIL + lane * 12u);
  const u32 live = lane < g2 ? ~0u : 0u, Q0 = pq[0] & live, Q1 = pq[1] & live, Q2 = pq[2] & live;
  LAS unsigned char* const spell = scr + S_TAIL + g2 * 12u;
  const u32 b4 = base * 0x01010101u, lut0 = b4 - 0x05040300u + 1u, lut1 = b4 - 0x00080706u;
  const u32 P0[2] = {k.p01.x, k.p01.y}, P1[2] = {k.p01.z, k.p01.w}, mk[2] = {P0[0] & P1[0], P0[1] & P1[1]};
  const u32 c = __builtin_popcount(mk[0]) + __builtin_popcount(mk[1]);
  u32 a[17], sel[16];
  a[0] = (u32)(size_t)(scr + S_LIST) + wave_scan(c) - c;
#pragma unroll
  for (int j = 0; j < 16; ++j) {
    const u32 f = (mk[j >> 3] >> (j & 7)) & 0x01010101u;
    sel[j] = *(const LAS u32*)(size_t)__builtin_amdgcn_udot4(f, 0x20100804u, TAB + 64u, false);
    a[j + 1] = __builtin_amdgcn_sad_u8(f, 0u, a[j]);
  }
  __builtin_amdgcn_sched_barrier(0);
  u32x4 l[2];
#pragma unroll
  for (int s = 0; s < 8; ++s) {
    const u32 c2 = ((Q0 >> s) & 0x01010101u) | (((Q1 >> s) & 0x01010101u) << 1) | (((Q2 >> s) & 0x01010101u) << 2);
    l[s >> 2][s & 3] = __builtin_amdgcn_perm(lut1, lut0, c2);
  }
  const u32 m3 = Q0 & Q1 & Q2, c3 = __builtin_popcount(m3), off3 = wave_scan(c3) - c3;
  const bool many = __builtin_amdgcn_ballot_w64(c3 > 8u) != 0;
  if (!many) {
    const LAS u32* const ps = (const LAS u32*)(spell + (off3 & ~3u));
    const u32 d0 = ps[0], d1 = ps[1], d2 = ps[2];
    const u32 l0 = __builtin_amdgcn_alignbyte(d1, d0, off3), l1 = __builtin_amdgcn_alignbyte(d2, d1, off3);
    u32 t = m3 - ((m3 >> 1) & 0x55555555u);
    t = (t & 0x33333333u) + ((t >> 2) & 0x33333333u);
    t = (t + (t >> 4)) & 0x0f0f0f0fu;
    u32 rank = t << 8;
    rank += rank << 8;
    rank += (t << 8) << 16;
#pragma unroll
    for (int s = 0; s < 8; ++s) {
      const u32 e = (m3 >> s) & 0x01010101u;
      u32 ff = e << 8;
      asm("" : "+v"(ff));
      ff -= e;
      l[s >> 2][s & 3] = (__builtin_amdgcn_perm(l1, l0, rank) & ff) | (l[s >> 2][s & 3] & ~ff);
      rank += e;
    }
  }
  *(LAS u32x4*)(scr + S_LIST + lane * 32u) = l[0];
  *(LAS u32x4*)(scr + S_LIST + lane * 32u + 16u) = l[1];
  if (many) walk_spelled<false>(m3, off3, scr + S_LIST + lane * 32u, spell, 0u);
  asm volatile("" ::: "memory");
  u32 w0[16], w1[16];
#pragma unroll
  for (int j = 0; j < 16; ++j) {
    const LAS u32* const pw = (const LAS u32*)(size_t)(a[j] & ~3u);
    w0[j] = pw[0];
    w1[j] = pw[1];
  }
  __builtin_amdgcn_sched_barrier(0);
#pragma unroll
  for (int j = 0; j < 16; ++j) {
    const int s = j & 7, wd = j >> 3;
    const u32 c1 = ((P0[wd] >> s) & 0x01010101u) | (((P1[wd] >> s) & 0x01010101u) << 1);
    const u32 e = __builtin_amdgcn_perm(b4 - c1, __builtin_amdgcn_alignbyte(w1[j], w0[j], a[j]), sel[j]), m = k.mq[j >> 2][j & 3];
    const u32 r0 = __builtin_amdgcn_perm(e, m, 0x05010400u), r1 = __builtin_amdgcn_perm(e, m, 0x07030602u);
    v[j >> 1][(j & 1) * 2] = __builtin_amdgcn_alignbit(r0, r0, 1);
    v[j >> 1][(j & 1) * 2 + 1] = __builtin_amdgcn_alignbit(r1, r1, 1);
  }
}

__device__ __forceinline__ void load_unit(u32x4 (&s)[8], rsrc_t rx, rsrc_t rw, u32 q, u32 vx, u32 vw) {
  const u32 kc = q >> 6, t = q & 63u;
  if (t < 32u) {
#pragma unroll
    for (int f = 0; f < 8; ++f) s[f] = __builtin_amdgcn_raw_buffer_load_b128(rx, vx, (32u * t + 16u * (f >> 2)) * 16384u + kc * 256u + (f & 3) * 64u, AG_XAUX);
  } else {
#pragma unroll
    for (int i = 0; i < 8; ++i) s[i] = __builtin_amdgcn_raw_buffer_load_b128(rw, vw, (128u * kc + 32u * ((t - 32u) >> 3) + i) * 2048u + (t & 7u) * 256u, AG_WAUX);
  }
}

__device__ __forceinline__ u32 packer(const Params& P, char* ws, u32 callv, u32 q0, u32 q1, u32 dq, u32 qend, LAS unsigned char* scr, u32 lane, u32 wid) {
  const rsrc_t rx = make_rsrc(P.x, 1024u * 8192u * 2u), rw = make_rsrc(P.w, 8192u * 1024u * 2u);
  const u32 kq = (lane & 3u) ^ quad_swz(lane >> 4);
  const u32 vx = (lane >> 2) * 16384u + kq * 16u, vw = (lane >> 4) * 8u * 2048u + (lane & 15u) * 16u;
  u32x4 s[8];
  STAMPP(0);
  load_unit(s, rx, rw, q0, vx, vw);
  __builtin_amdgcn_sched_barrier(0);
  const u32 call = __builtin_amdgcn_readfirstlane(callv) + 1u;
  const u32 r = P.rank, g4 = r & 4u, par = call & 1u;
  const rsrc_t rf = make_rsrc(ws, FRAG_BYTES), rp = make_rsrc(ws + OFF_PK + par * PK_BYTES, PK_BYTES);
  const u32 goff = OFF_G + (par * NR + r) * NUNITS * 8;
  char* g_in = pick(P.ws, g4 | ((r + 1u) & 3u));
  char* g_out = pick(P.ws, g4 ^ 4u);
#pragma unroll
  for (u32 j = 1; j < 4; ++j) {
    char* const pi = pick(P.ws, g4 | ((r + 1u + j) & 3u));
    char* const po = pick(P.ws, (g4 ^ 4u) | j);
    g_in = lane == j ? pi : g_in;
    g_out = lane == j ? po : g_out;
  }
  STAMPP(1);
  u64 psend = 0;
  char* pgp = g_out;
  u32 pinfo = 0, pcnt = SPARE_CNT;
  auto announce = [&]() {
    flag_store(pgp, ((u64)call << 32) | pinfo, psend);
    if (pcnt != SPARE_CNT) { STAMP_CNT(ws, pcnt); count_in(ws + OFF_CNT, pcnt); }
  };
  for (u32 q = q0; q < qend; q = q1, q1 += dq) {
    const u32 kc = q >> 6, t = q & 63u;
    u32x4 v[8];
    Dest d;
    bool own;
    u64 send;
    char* gp;
    if (t < 32u) {
#pragma unroll
      for (int f = 0; f < 8; ++f) v[f] = s[f];
      own = t < AU;
      d = dest_x(t, kc);
      send = own ? 7u : 15u;
      gp = own ? g_in : g_out;
    } else {
      const u32 ks4 = (t - 32u) >> 3, cg = t & 7u;
#pragma unroll
      for (int nf = 0; nf < 8; ++nf)
#pragma unroll
        for (int tt = 0; tt < 4; ++tt) v[nf][tt] = __builtin_amdgcn_perm(s[2 * tt + 1][nf >> 1], s[2 * tt][nf >> 1], (nf & 1) ? 0x07060302u : 0x05040100u);
      own = true;
      d = dest_w(cg >> 1, kc, ks4, cg & 1u);
      send = 1u << (cg >> 1);
      gp = g_out;
    }
    if (q == q0) STAMPP(2);
    const bool lazy = qend != NUNITS;
    if (q1 < qend && !lazy) load_unit(s, rx, rw, q1, vx, vw);
    Packed k;
    u32 info = RAW | PUSHED;
    if (kc >= KP) info = encode_unit(k, v, scr, lane);
    if (q == q0) STAMPP(3);
    drain_vm();
#pragma unroll
    for (int f = 0; f < 8; ++f) asm volatile("" : "+v"(s[f]));
    announce();
    if (kc >= KP) store_unit(k, v, info, rp, q * SLOT, lane);
    else {
      for (u32 m = (u32)send; m; m &= m - 1u) {
        const u32 jd = __builtin_ctz(m), dst = (t < AU) ? (g4 | ((r + 1u + jd) & 3u)) : ((g4 ^ 4u) | jd);
        store_unit(k, v, RAW, make_rsrc(pick(P.ws, dst) + OFF_IB + (par * NR + r) * IB_SRC, IB_SRC), q * SLOT, lane);
      }
    }
    if (own) store_frags(v, rf, d, lane);
    if (q == q0) STAMPP(4);
    psend = send;
    pgp = gp + goff + q * 8u;
    pinfo = info;
    pcnt = own ? d.cnt : SPARE_CNT;
    if (q1 < qend && lazy) {
      drain_vm();
      announce();
      STAMPP(5);
      psend = 0;
      pcnt = SPARE_CNT;
      load_unit(s, rx, rw, q1, vx, vw);
    }
  }
  drain_vm();
  announce();
  STAMPP(6);
  STAMP(2);
  return call;
}

__device__ __forceinline__ u32 lds_add(LAS u32* p, u32 v) {
  return __builtin_amdgcn_readfirstlane(__hip_atomic_fetch_add(p, v, __ATOMIC_RELAXED, __HIP_MEMORY_SCOPE_WORKGROUP)) >> 6;
}

__device__ __forceinline__ void puller(const Params& P, char* ws, u32 call, u32 li, u32 lidx, LAS u32* ctl, LAS unsigned char* scr, u32 lane, u32 wid) {
  const u32 r = P.rank, g4 = r & 4u, par = call & 1u;
  const u32 tl = li < 3u ? (((r + 1u + li) & 3u) == ((r ^ 3u) & 3u) ? AG_TOK_FAST : AG_TOK_IN) : AG_TOK, TOK0 = tl / NCL + (lidx < tl % NCL ? 1u : 0u);
  u32 TOK = TOK0;
  const bool ing = li < 3u;
  const u32 src = ing ? (g4 | ((r + 1 + li) & 3u)) : ((g4 ^ 4u) | (li - 3u));
  const rsrc_t rf = make_rsrc(ws, FRAG_BYTES), rp = make_rsrc(pick(P.ws, src) + OFF_PK + par * PK_BYTES, PK_BYTES);
  const rsrc_t ri = make_rsrc(ws + OFF_IB + (par * NR + src) * IB_SRC, IB_SRC);
  char* const gbase = ws + OFF_G + (par * NR + src) * NUNITS * 8;
  GAS u32* const cnt = (GAS u32*)(ws + OFF_CNT);
  const u32 total = ing ? NKC * AU : NKC * NOUT;
  constexpr u32 NONE = 0xffffffffu;
  GAS u32* const tkt = (GAS u32*)(ws + OFF_TKT + li * 128u);
  const u32 tbase = (call - 1u) * (total + NCL * PW) * 64u;
  auto draw = [&]() -> u32 { return __hip_atomic_fetch_add(tkt, 1u, __ATOMIC_RELAXED, __HIP_MEMORY_SCOPE_AGENT); };
  auto unit_of = [&](u32 raw_ticket) -> u32 {
    const u32 i = ((u32)__builtin_amdgcn_readfirstlane(raw_ticket) - tbase) >> 6;
    if (i >= total) return NONE;
    if (ing) { const u32 kc = i / AU; return kc * 64u + (i - kc * AU); }
    u32 kc, t;
    if (i < 8u * WLEAD) { kc = i >> 3; t = AWAY + (i & 7u); }
    else {
      const u32 i2 = i - 8u * WLEAD, mid = (NKC - WLEAD) * NOUT;
      if (i2 < mid) { kc = i2 / NOUT; t = i2 - kc * NOUT; kc += t < AWAY ? 0u : WLEAD; }
      else { kc = (i2 - mid) / AWAY; t = i2 - mid - kc * AWAY; kc += NKC - WLEAD; }
    }
    const u32 tw = t - AWAY;
    return kc * 64u + (t < AWAY ? AU + t : 32u + (tw >> 1) * 8u + 2u * (r & 3u) + (tw & 1u));
  };
  auto granule = [&](u32 q) -> u64 { return __hip_atomic_load((GAS u64*)(gbase + (q & (NUNITS - 1)) * 8u), __ATOMIC_RELAXED, __HIP_MEMORY_SCOPE_SYSTEM); };
  auto peek = [&](LAS u32* p) -> u32 { return (u32)(__builtin_amdgcn_readfirstlane(__hip_atomic_load(p, __ATOMIC_RELAXED, __HIP_MEMORY_SCOPE_WORKGROUP)) >> 6); };
  auto take = [&]() { const u32 my = lds_add(ctl, 1u); while (peek(ctl + 2) + TOK <= my) __builtin_amdgcn_s_sleep(1); };
  auto dest_of = [&](u32 q) -> Dest {
    const u32 kc = q >> 6, tq = q & 63u;
    if (q == NUNITS) return Dest{FRAG_BYTES, 0u, 0u, NONE};
    if (tq < 32u) return dest_x(ing ? NEARLY * UPT + li * AU + tq : AU + (li - 3u) * AWAY + (tq - AU), kc);
    return dest_w(4u + (li - 3u), kc, (tq - 32u) >> 3, tq & 1u);
  };
  auto publish = [&](u32 c) { if (c != NONE) { STAMP_CNT(ws, c); count_in(ws + OFF_CNT, c); } };
  u32 q = NUNITS, pend = NONE, tag = call, info = 127u;
  u64 gr;
  bool first = true;
#define LAP(a) do {} while (0)
  while (q != NONE) {
    TOK = TOK0 + ((q >> 6) < AG_KT ? 1u : 0u);
    if (tag != call || (!(info & PUSHED) && peek(ctl + 2) + TOK <= peek(ctl))) {
      drain_vm();
      publish(pend);
      pend = NONE;
      if (tag != call) {
        const u64 t0 = __builtin_amdgcn_s_memrealtime();
        for (;;) {
          gr = granule(q);
          tag = __builtin_amdgcn_readfirstlane((u32)(gr >> 32));
          info = __builtin_amdgcn_readfirstlane((u32)gr);
          if (tag == call || give_up(ws, call, t0)) break;
          __builtin_amdgcn_s_sleep(2);
        }
        if (tag != call) break;
      }
    }
    if (!(info & PUSHED)) take();
    asm volatile("" ::: "memory");
    if (first && q != NUNITS) { STAMP(1); first = false; }
    LAP(acc_w);
    const Dest d = dest_of(q);
    const u32 cur = info, soff = q * SLOT;
    u32x4 v[8];
    if (cur & RAW) {
#pragma unroll
      for (int f = 0; f < 8; ++f) v[f] = __builtin_amdgcn_raw_buffer_load_b128((cur & PUSHED) ? ri : rp, lane * 16, soff + f * 1024, AUX_SYS);
      const u32 nt = draw();
      drain_vm();
      if (!(cur & PUSHED)) lds_add(ctl + 2, 1u);
      q = unit_of(nt);
      gr = granule(q);
      publish(pend);
      tag = __builtin_amdgcn_readfirstlane((u32)(gr >> 32));
      info = __builtin_amdgcn_readfirstlane((u32)gr);
      store_frags(v, rf, d, lane);
    } else {
      Packed k;
      u32 nt;
      u64 keep;
      asm volatile("s_mov_b64 %1, exec\n\ts_mov_b64 exec, 1\n\tglobal_atomic_add %0, %2, %3, %4 sc0\n\ts_mov_b64 exec, %1"
                   : "=&v"(nt), "=&s"(keep) : "v"(0u), "v"(64u), "s"(tkt) : "memory");
      fetch_packed(k, rp, soff, cur, lane);
      asm volatile("s_waitcnt vmcnt(%0)" :: "i"(AG_EARLY) : "memory");
      lds_add(ctl + 2, 1u);
      asm volatile("s_waitcnt vmcnt(0)" : "+v"(nt), "+v"(k.mq[0]), "+v"(k.mq[1]), "+v"(k.mq[2]), "+v"(k.mq[3]), "+v"(k.p01), "+v"(k.tail) :: "memory");
      LAP(acc_l);
      q = unit_of(nt);
      STAMP_CNT(ws, pend);
      asm volatile("global_load_dwordx2 %0, %2, %3 sc0 sc1\n\ts_mov_b64 %1, exec\n\ts_mov_b64 exec, 1\n\tglobal_atomic_add %4, %5, %6\n\ts_mov_b64 exec, %1"
                   : "=&v"(gr), "=&s"(keep) : "v"((q & (NUNITS - 1)) * 8u), "s"(gbase), "v"((pend == NONE ? SPARE_CNT : pend) * 4u), "v"(1u), "s"(cnt) : "memory");
      decode_unit(v, k, cur, scr, lane);
      __builtin_amdgcn_sched_barrier(0);
      asm volatile("s_waitcnt vmcnt(1)" : "+v"(gr) :: "memory");
      tag = __builtin_amdgcn_readfirstlane((u32)(gr >> 32));
      info = __builtin_amdgcn_readfirstlane((u32)gr);
      __builtin_amdgcn_sched_barrier(0);
      store_frags(v, rf, d, lane);
      LAP(acc_d);
    }
    pend = d.cnt;
  }
  drain_vm();
  publish(pend);
  STAMP(2);
}

constexpr u32 NOROW = 0xffffffffu;
__device__ __forceinline__ u32 frag_row(u32 r, u32 rt, u32 fr) {
  const u32 g4 = r & 4u;
  const bool late = rt >= NEARLY;
  const u32 uc = (late ? rt - NEARLY : rt) * UPT + (fr >> 1);
  u32 src, su;
  if (late) {
    if (uc >= LATE_UNITS) return NOROW;
    const u32 j = uc / AU;
    src = g4 | ((r + 1 + j) & 3u);
    su = uc - j * AU;
  } else if (uc < AU) {
    src = r;
    su = uc;
  } else {
    if (uc >= EARLY_UNITS) return NOROW;
    const u32 j = (uc - AU) / AWAY;
    src = (g4 ^ 4u) | j;
    su = AU + (uc - AU) - j * AWAY;
  }
  return ((AG_ORDER >> (4u * src)) & 7u) * 1024u + su * 32u + (fr & 1u) * 16u;
}
__device__ __forceinline__ u32 wave_frag0(u32 wid) { return (0x74208630u >> (4 * wid)) & 15u; }

constexpr u32 O_HI = 7168;
__device__ __forceinline__ void out_puller(const Params& P, char* ws, u32 call, u32 j, u32 ftok, LAS u32* ctl, LAS unsigned char* scr, u32 lane, u32 wid) {
  const u32 r = P.rank, src = ((r & 4u) ^ 4u) | j;
  char* const ob = pick(P.ws, src) + OFF_OB + (r & 3u) * (OUNITS * OSLOT);
  const rsrc_t ro = make_rsrc(pick(P.out, r), 8192u * 1024u * 2u);
  GAS u32* const tkt = (GAS u32*)(ws + OFF_TKT + (7u + j) * 128u);
  const u32 tbase = (call - 1u) * (OUNITS + (AG_FTOK_IN ? NCOMM * 2u : NCL * 8u)) * 64u;
  const u32 vo = (lane >> 4) * 4u * 2048u + (lane & 15u) * 16u;
  char* const queue = ws + OFF_OQ + j * 2048u;
  u32 k = ((u32)__builtin_amdgcn_readfirstlane(__hip_atomic_fetch_add(tkt, 1u, __ATOMIC_RELAXED, __HIP_MEMORY_SCOPE_AGENT)) - tbase) >> 6;
  u32 tag = 0, info = 0;
  while (k < OUNITS) {
    STAMPQ(0);
    if (tag != call) {
      const u64 t0 = __builtin_amdgcn_s_memrealtime();
      for (u32 spins = 0;; ++spins) {
        const u64 gr = __hip_atomic_load((GAS u64*)(queue + k * 8u), __ATOMIC_RELAXED, __HIP_MEMORY_SCOPE_SYSTEM);
        tag = __builtin_amdgcn_readfirstlane((u32)(gr >> 32));
        info = __builtin_amdgcn_readfirstlane((u32)gr);
        if (tag == call || ((spins & 7u) == 7u && give_up(ws, call, t0))) break;
        __builtin_amdgcn_s_sleep(3);
      }
      if (tag != call) {
        *(GAS unsigned short*)(pick(P.out, r) + (size_t)((AG_ORDER >> (4u * r)) & 7u) * 1024u * 2048u) = 0x7fc0;
        break;
      }
    }
    {
      const u32 my = lds_add(ctl, 1u);
      while ((u32)(__builtin_amdgcn_readfirstlane(__hip_atomic_load(ctl + 2, __ATOMIC_RELAXED, __HIP_MEMORY_SCOPE_WORKGROUP)) >> 6) + ftok <= my) __builtin_amdgcn_s_sleep(1);
    }
    asm volatile("" ::: "memory");
    STAMPQ(1);
    const u32 uid = (info & 255u) < OUNITS ? (info & 255u) : 0u, sw = uid & 7u, mf = sw - 2u < 4u ? 2u : 3u;
    u32 total = (info >> 8) & 0x3fffu;
    total = total < 12288u ? total : 12288u;
    const bool raw = (info >> 22) & 1u;
    const u32 t4 = ((info >> 23) & 255u) * 0x01010101u;
    const rsrc_t rs = make_rsrc(ob + uid * OSLOT, (total + 15u) & ~15u);
    u32 nt;
    u64 keep, gr;
    asm volatile("s_mov_b64 %1, exec\n\ts_mov_b64 exec, 1\n\tglobal_atomic_add %0, %2, %3, %4 sc0\n\ts_mov_b64 exec, %1"
                 : "=&v"(nt), "=&s"(keep) : "v"(0u), "v"(64u), "s"(tkt) : "memory");
    for (u32 p = 0; p < total; p += 1024u) __builtin_amdgcn_raw_ptr_buffer_load_lds(rs, scr + p, 16, lane * 16u, p, 0, AUX_SYS);
    asm volatile("s_waitcnt vmcnt(%0)" :: "i"(AG_OEARLY) : "memory");
    lds_add(ctl + 2, 1u);
    asm volatile("s_waitcnt vmcnt(0)" : "+v"(nt) :: "memory");
    STAMPQ(2);
    const u32 kn = ((u32)__builtin_amdgcn_readfirstlane(nt) - tbase) >> 6;
    asm volatile("global_load_dwordx2 %0, %1, %2 sc0 sc1" : "=&v"(gr) : "v"((kn & 255u) * 8u), "s"(queue) : "memory");
    u32 mk[3] = {0, 0, 0}, a = 0;
    if (!raw) {
#pragma unroll
      for (u32 m = 0; m < 3; ++m) mk[m] = *(const LAS u32*)(scr + m * 256u + lane * 4u);
      mk[2] = mf == 3u ? mk[2] : 0u;
      const u32 c = ((__builtin_popcount(mk[0]) + __builtin_popcount(mk[1]) + __builtin_popcount(mk[2])) & ~3u) + 4u, incl = wave_scan(c);
      u32 n = __builtin_amdgcn_readlane(incl, 63);
      n = n < 4096u ? n : 4096u;
      LAS unsigned char* const lo = scr + mf * 256u;
      LAS unsigned char* const planes = lo + ((n + 15u) & ~15u);
      a = (u32)(size_t)lo + incl - c;
      const u32x4 nb[2] = {*(const LAS u32x4*)(planes + lane * 32u), *(const LAS u32x4*)(planes + lane * 32u + 16u)};
#pragma unroll
      for (int i = 0; i < 2; ++i) {
        *(LAS u32x4*)(lo + O_HI + lane * 64u + i * 16u) = nb[i] & 0x0f0f0f0fu;
        *(LAS u32x4*)(lo + O_HI + lane * 64u + 32u + i * 16u) = (nb[i] >> 4) & 0x0f0f0f0fu;
      }
      asm volatile("" ::: "memory");
    }
    asm volatile("s_waitcnt vmcnt(0)" : "+v"(gr) :: "memory");
    tag = __builtin_amdgcn_readfirstlane((u32)(gr >> 32));
    info = __builtin_amdgcn_readfirstlane((u32)gr);
    const u32 col = j * 512u + (sw >> 2) * 256u, f0 = wave_frag0(sw);
    for (u32 m = 0; m < mf; ++m) {
      const u32 w = m == 0 ? mk[0] : (m == 1 ? mk[1] : mk[2]);
      u32x4 o[4];
      if (raw) {
#pragma unroll
        for (int e = 0; e < 4; ++e) o[e] = *(const LAS u32x4*)(scr + (m * 4u + e) * 1024u + lane * 16u);
      } else {
        u32 at[8], sel[8], l0[8], l1[8], h0[8], h1[8];
#pragma unroll
        for (int s = 0; s < 8; ++s) {
          const u32 f = (w >> s) & 0x01010101u;
          const LAS u32* const pw = (const LAS u32*)(size_t)(a & ~3u);
          at[s] = a;
          sel[s] = *(const LAS u32*)(size_t)__builtin_amdgcn_udot4(f, 0x20100804u, TAB + 128u, false);
          l0[s] = pw[0];
          l1[s] = pw[1];
          h0[s] = pw[O_HI / 4];
          h1[s] = pw[O_HI / 4 + 1];
          a = __builtin_amdgcn_sad_u8(f, 0u, a);
        }
        __builtin_amdgcn_sched_barrier(0);
#pragma unroll
        for (int s = 0; s < 8; ++s) {
          const u32 lb = __builtin_amdgcn_perm(0u, __builtin_amdgcn_alignbyte(l1[s], l0[s], at[s]), sel[s]);
          const u32 hb = __builtin_amdgcn_perm(0u, t4 - (__builtin_amdgcn_alignbyte(h1[s], h0[s], at[s]) & 0x0f0f0f0fu), sel[s]);
          o[s >> 1][(s & 1) * 2] = __builtin_amdgcn_perm(hb, lb, 0x05010400u);
          o[s >> 1][(s & 1) * 2 + 1] = __builtin_amdgcn_perm(hb, lb, 0x07030602u);
        }
      }
      const u32 row0 = frag_row(src, uid >> 3, f0 + m);
      if (row0 != NOROW) {
#pragma unroll
        for (int e = 0; e < 4; ++e) store16<AG_OAUX>(o[e], ro, vo, (row0 + e) * 2048u + col);
      }
    }
    STAMPQ(3);
    k = kn;
  }
  STAMP(3);
}

__device__ __forceinline__ void gate(int c, int& ready, const char* cnt, u32 target, char* ws, u32 call) {
  if (c < ready) return;
  const u64 t0 = __builtin_amdgcn_s_memrealtime();
  for (u32 spins = 0;; ++spins) {
    const u64 m = __builtin_amdgcn_ballot_w64((int)(ld_agent(cnt) - target) >= 0);
    ready = ~m ? __builtin_ctzll(~m) : 64;
    if (c < ready) break;
    if ((spins & 7u) == 7u && give_up(ws, call, t0)) { ready = 64; break; }
    __builtin_amdgcn_s_sleep(3);
  }
  asm volatile("" ::: "memory");
}

__device__ __forceinline__ u32 relu_pk(float lo, float hi) {
  u32 r;
  asm("v_cvt_pk_bf16_f32 %0, %1, %2\n\tv_pk_max_i16 %0, %0, 0" : "=v"(r) : "v"(lo), "v"(hi));
  return r;
}

__device__ __forceinline__ void mfma_acc(f32x4& c, const bf16x8& a, const bf16x8& b) {
  asm volatile("v_mfma_f32_16x16x32_bf16 %0, %1, %2, %0" : "+v"(c) : "v"(a), "v"(b));
}
template <int N> __device__ __forceinline__ void wait_bar() { asm volatile("s_waitcnt vmcnt(%0)\n\ts_barrier" :: "i"(N) : "memory"); }

template <int MF, int NP>
__device__ __forceinline__ void gemm_wave(const Params& P, char* ws, u32 call, u32 b, LAS char* lds, u32 lane, u32 wid) {
  constexpr bool XL = NP == 5;
  constexpr u32 STEP = XL ? A_STEP : B_STEP;
  constexpr int NM = MF * 8;
  const u32 r = P.rank, g4 = r & 4u;
  const u32 xcd = b & 7u, late = xcd & 1u, ent = ((xcd >> 1) & 1u) * 26u + (b >> 3);
  const u32 rc = ent >> 2, rt = late * NEARLY + rc, ct = (ent & 3u) + (xcd & 4u);
  const u32 wc = wid >> 2, f0 = wave_frag0(wid);
  const int left = (late ? LATE_UNITS : EARLY_UNITS) - (int)rc * UPT;
  const u32 nun = left < 0 ? 0u : (left > UPT ? (u32)UPT : (u32)left);
  const u32 p0 = (wid & 1u) * NP * 1024u;
  constexpr u32 same = 1;
  const rsrc_t rs = XL ? make_rsrc(ws + OFF_XU + same * rt * A_TILE, A_TILE) : make_rsrc(ws + OFF_WU + same * ct * B_TILE, B_TILE);
  const char* const cnt = ws + OFF_CNT + ((XL ? rt : NRT + ct) * 64 + lane) * 4;
  u32 target = call * (XL ? nun : 8u);
  const u32 voff = p0 + lane * 16u;
  u32 vp[NP ? NP : 1];
#pragma unroll
  for (int p = 0; p < NP; ++p) vp[p] = (XL && (wid & 1u) * NP + p >= 2u * nun) ? lane * 16u : voff;
  LAS char* const ring = lds + (XL ? 0u : A_STEP) + p0;
  const u32 img = ((lane & 15u) * 4u + ((lane >> 4) ^ quad_swz((lane >> 2) & 3u))) * 16u;
  LAS char* const ra = lds + f0 * 1024u + img;
  LAS char* const rb = lds + A_STEP + wc * 8192u + lane * 16u;
#define AG_DMA(p, ks, off, imm) __builtin_amdgcn_raw_ptr_buffer_load_lds(rs, ring + (off), 16, vp[p < NP ? p : 0], (ks), imm, AG_GEMM_AUX)
#define AG_PIECE(p, ks, off) do { \
    if (p == 0) AG_DMA(0, (ks) * STEP, off, 0); if (p == 1) AG_DMA(1, (ks) * STEP, off, 1024); \
    if (p == 2) AG_DMA(2, (ks) * STEP, off, 2048); if (p == 3) AG_DMA(3, (ks) * STEP, off, 3072); \
    if (p == 4) AG_DMA(4, (ks) * STEP + 4096, (off) + 4096, 0); if (p == 5) AG_DMA(5, (ks) * STEP + 4096, (off) + 4096, 1024); \
    if (p == 6) AG_DMA(6, (ks) * STEP + 4096, (off) + 4096, 2048); if (p == 7) AG_DMA(7, (ks) * STEP + 4096, (off) + 4096, 3072); \
  } while (0)
#define AG_ALL(ks, off) do { _Pragma("unroll") for (int p = 0; p < NP; ++p) AG_PIECE(p, ks, off); } while (0)
#define AG_PIN(stmt) do { __builtin_amdgcn_sched_barrier(0); stmt; __builtin_amdgcn_sched_barrier(0); } while (0)
#define AG_STEP(cs, ks, DMA, RD) do { \
    LAS char* const pa = ra + ord; \
    LAS char* const pb = rb + ord; \
    __builtin_amdgcn_sched_barrier(0); \
    _Pragma("unroll") for (int i = 0; i < NM; ++i) { \
      mfma_acc(acc[i % MF][i / MF], fa[cs][i % MF], fb[cs][i / MF]); \
      _Pragma("unroll") for (int j = 0; j < MF + 8; ++j) \
        if (RD && i == (MF == 3 ? 2 * j : j)) { \
          if (j < MF) AG_PIN(fa[(cs) ^ 1][j] = *(const LAS bf16x8*)(pa + j * 1024)); \
          else AG_PIN(fb[(cs) ^ 1][j - MF] = *(const LAS bf16x8*)(pb + (j - MF) * 1024)); \
        } \
      _Pragma("unroll") for (int p = 0; p < NP; ++p) \
        if (DMA && i == 1 + p * (NP == 5 ? 5 : 3)) AG_PIN(AG_PIECE(p, ks, owr)); \
    } \
    ord = ord + SLOTB == NSLOT * SLOTB ? 0 : ord + SLOTB; \
    owr = owr + SLOTB == NSLOT * SLOTB ? 0 : owr + SLOTB; \
    __builtin_amdgcn_sched_barrier(0); \
  } while (0)
#define AG_GATE(c) do { if (NP) gate(c, ready, cnt, target, ws, call); } while (0)
  f32x4 acc[MF][8];
#pragma unroll
  for (int m = 0; m < MF; ++m)
#pragma unroll
    for (int n = 0; n < 8; ++n) { acc[m][n] = (f32x4){0.f, 0.f, 0.f, 0.f}; asm volatile("" : "+v"(acc[m][n])); }
  if (NP) __builtin_amdgcn_s_setprio(1);
  int ready = 0;
  AG_GATE(0);
  STAMP(1);
  STAMPC(4);
  constexpr int LA = AG_LA, W = (LA - 2) * NP;
#pragma unroll
  for (int s = 0; s < 4; ++s) AG_ALL(s, s * SLOTB);
  if (LA == 5) { AG_GATE(1); AG_ALL(4, 4 * SLOTB); }
  wait_bar<W>();
  bf16x8 fa[2][MF], fb[2][8];
#pragma unroll
  for (int m = 0; m < MF; ++m) fa[0][m] = *(const LAS bf16x8*)(ra + m * 1024);
#pragma unroll
  for (int n = 0; n < 8; ++n) fb[0][n] = *(const LAS bf16x8*)(rb + n * 1024);
  u32 ord = SLOTB, owr = LA * SLOTB;
  for (int kc = 0; kc < NKC - 3; ++kc) {
    if (LA == 4) AG_GATE(kc + 1);
    AG_STEP(0, kc * 4 + LA, true, true); wait_bar<W>();
    AG_STEP(1, kc * 4 + LA + 1, true, true); wait_bar<W>();
    AG_STEP(0, kc * 4 + LA + 2, true, true); wait_bar<W>();
    if (LA == 5) AG_GATE(kc + 2);
    AG_STEP(1, kc * 4 + LA + 3, true, true); wait_bar<W>();
  }
  if (LA == 4) AG_GATE(NKC - 2);
  AG_STEP(0, NKS - 12 + LA, true, true); wait_bar<W>();
  AG_STEP(1, NKS - 11 + LA, true, true); wait_bar<W>();
  AG_STEP(0, NKS - 10 + LA, true, true); wait_bar<W>();
  AG_STEP(1, NKS - 9 + LA, LA == 4, true); wait_bar<2 * NP>();
  AG_STEP(0, 0, false, true); wait_bar<NP>();
  AG_STEP(1, 0, false, true); wait_bar<0>();
  AG_STEP(0, 0, false, true);
  AG_STEP(1, 0, false, false);
  AG_GATE(NKC - 1);
  asm volatile("s_barrier" ::: "memory");
#pragma unroll
  for (int s = 0; s < 4; ++s) AG_ALL(NKS - 4 + s, s * SLOTB);
  u32 tmo;
  asm volatile("global_load_dword %0, %1, %2 sc1" : "=&v"(tmo) : "v"(0u), "s"(ws + OFF_TMO) : "memory");
  wait_bar<2 * NP + 1>();
#pragma unroll
  for (int m = 0; m < MF; ++m) fa[0][m] = *(const LAS bf16x8*)(ra + m * 1024);
#pragma unroll
  for (int n = 0; n < 8; ++n) fb[0][n] = *(const LAS bf16x8*)(rb + n * 1024);
  ord = SLOTB;
  AG_STEP(0, 0, false, true); wait_bar<NP + 1>();
  AG_STEP(1, 0, false, true); wait_bar<0>();
  AG_STEP(0, 0, false, true);
  AG_STEP(1, 0, false, false);
  asm volatile("" : "+v"(tmo));
  asm volatile("s_nop 15" ::: "memory");
  __builtin_amdgcn_sched_barrier(0);
#undef AG_PIECE
#undef AG_ALL
#undef AG_PIN
#undef AG_DMA
#undef AG_GATE
#undef AG_STEP
  STAMP(2);
  STAMPC(5);
  const bool bad = __builtin_amdgcn_readfirstlane(tmo) == call;
  u32x4 o[MF][4];
#pragma unroll
  for (int m = 0; m < MF; ++m)
#pragma unroll
    for (int e = 0; e < 4; ++e)
#pragma unroll
      for (int n = 0; n < 4; ++n) o[m][e][n] = relu_pk(acc[m][2 * n][e], acc[m][2 * n + 1][e]);
  if (bad) {
#pragma unroll
    for (int m = 0; m < MF; ++m)
#pragma unroll
      for (int e = 0; e < 4; ++e) o[m][e] = (u32x4){0x7fc07fc0u, 0x7fc07fc0u, 0x7fc07fc0u, 0x7fc07fc0u};
    asm volatile("" ::: "memory");
  }
  if (ct < 4u) {
    const rsrc_t ro = make_rsrc(pick(P.out, r), 8192u * 1024u * 2u);
    const u32 vo = (lane >> 4) * 4u * 2048u + (lane & 15u) * 16u;
#pragma unroll
    for (int m = 0; m < MF; ++m) {
      const u32 row0 = frag_row(r, rt, f0 + m);
      if (row0 == NOROW) continue;
#pragma unroll
      for (int e = 0; e < 4; ++e) store16<AG_OAUX>(o[m][e], ro, vo, (row0 + e) * 2048u + ct * 512u + wc * 256u);
    }
    STAMP(3);
    return;
  }
  const u32 j = ct - 4u, uid = rt * 8u + wid;
  u32 qi;
  u64 keep;
  asm volatile("s_mov_b64 %1, exec\n\ts_mov_b64 exec, 1\n\tglobal_atomic_add %0, %2, %3, %4 sc0\n\ts_mov_b64 exec, %1"
               : "=&v"(qi), "=&s"(keep) : "v"(0u), "v"(1u), "s"(ws + OFF_TKT + (11u + j) * 128u) : "memory");
  u32 mk[MF], fl[MF * 8], lb[MF * 8], hb[MF * 8], c = 0;
  u16x2 big[2] = {{0, 0}, {0, 0}};
#pragma unroll
  for (int m = 0; m < MF; ++m) {
    mk[m] = 0;
    if ((f0 + m) >> 1 >= nun) {
#pragma unroll
      for (int e = 0; e < 4; ++e) o[m][e] = (u32x4){0u, 0u, 0u, 0u};
    }
#pragma unroll
    for (int s = 0; s < 8; ++s) {
      const u32 d0 = o[m][s >> 1][(s & 1) * 2], d1 = o[m][s >> 1][(s & 1) * 2 + 1];
      u32 t0, t1;
      asm("v_pk_min_u16 %0, %1, %2" : "=v"(t0) : "v"(d0), "v"(0x00010001u));
      asm("v_pk_min_u16 %0, %1, %2" : "=v"(t1) : "v"(d1), "v"(0x00010001u));
      fl[m * 8 + s] = __builtin_amdgcn_perm(t1, t0, 0x06040200u);
      mk[m] |= fl[m * 8 + s] << s;
      big[0] = __builtin_elementwise_max(big[0], __builtin_bit_cast(u16x2, d0));
      big[1] = __builtin_elementwise_max(big[1], __builtin_bit_cast(u16x2, d1));
      lb[m * 8 + s] = __builtin_amdgcn_perm(d1, d0, 0x06040200u);
      hb[m * 8 + s] = __builtin_amdgcn_perm(d1, d0, 0x07050301u);
    }
    c += __builtin_popcount(mk[m]);
  }
  c = (c & ~3u) + 4u;
  const u32 incl = wave_scan(c), n = __builtin_amdgcn_readlane(incl, 63);
  big[0] = __builtin_elementwise_max(big[0], big[1]);
  u32 top = wave_max((u32)(big[0].x > big[0].y ? big[0].x : big[0].y) >> 8);
  top = top < 15u ? 15u : top;
  bool raw = n > 4096u;
  u32 total = 0;
  const rsrc_t rb2 = make_rsrc(ws + OFF_OB + j * (OUNITS * OSLOT) + uid * OSLOT, OSLOT);
  STAMP2(0);
  asm volatile("s_barrier" ::: "memory");
  STAMP2(1);
  LAS unsigned char* const scr = (LAS unsigned char*)lds + wid * WAVE_SCRATCH;
  if (!raw) {
    LAS unsigned char* const lo = scr + MF * 256u;
  #pragma unroll
    for (int m = 0; m < MF; ++m) *(LAS u32*)(scr + m * 256u + lane * 4u) = mk[m];
    asm volatile("" ::: "memory");
    const u32 t4 = top * 0x01010101u;
    u32 wide = 0;
    u32 a = (u32)(size_t)lo + incl - c, cl = 0, ch = 0;
#pragma unroll
    for (int m = 0; m < MF; ++m) {
      u32 sel[8];
#pragma unroll
      for (int s = 0; s < 8; ++s) sel[s] = *(const LAS u32*)(size_t)__builtin_amdgcn_udot4(fl[m * 8 + s], 0x20100804u, TAB, false);
      __builtin_amdgcn_sched_barrier(0);
#pragma unroll
      for (int s = 0; s < 8; ++s) {
        const u32 sh = (a & 3u) * 8u, an = __builtin_amdgcn_sad_u8(fl[m * 8 + s], 0u, a);
        const u32 code = __builtin_amdgcn_perm(0u, t4 - hb[m * 8 + s], sel[s]);
        const u64 tl = (u64)__builtin_amdgcn_perm(0u, lb[m * 8 + s], sel[s]) << sh, th = (u64)code << sh;
        wide |= code;
        const u32 wl = cl | (u32)tl, wh = ch | (u32)th;
        LAS u32* const pw = (LAS u32*)(size_t)(a & ~3u);
        pw[0] = wl;
        pw[O_HI / 4] = wh;
        const bool cross = (an ^ a) & 4u;
        cl = cross ? (u32)(tl >> 32) : wl;
        ch = cross ? (u32)(th >> 32) : wh;
        a = an;
      }
    }
    ((LAS u32*)(size_t)(a & ~3u))[0] = cl;
    ((LAS u32*)(size_t)(a & ~3u))[O_HI / 4] = ch;
    asm volatile("" ::: "memory");
    LAS unsigned char* const planes = lo + ((n + 15u) & ~15u);
#pragma unroll
    for (int i = 0; i < 2; ++i)
      *(LAS u32x4*)(planes + lane * 32u + i * 16u) = (*(const LAS u32x4*)(lo + O_HI + lane * 64u + i * 16u) & 0x0f0f0f0fu) |
                                                     ((*(const LAS u32x4*)(lo + O_HI + lane * 64u + 32u + i * 16u) << 4) & 0xf0f0f0f0u);
    raw = __builtin_amdgcn_ballot_w64((wide & 0xf0f0f0f0u) != 0) != 0;
    total = MF * 256u + ((n + 15u) & ~15u) + ((n + 63u) >> 6) * 32u;
    asm volatile("" ::: "memory");
  }
  STAMP2(2);
  if (raw) {
    total = 4096u * MF;
#pragma unroll
    for (int m = 0; m < MF; ++m)
#pragma unroll
      for (int i = 0; i < 4; ++i)
        store16((u32x4){__builtin_amdgcn_perm(hb[m * 8 + 2 * i], lb[m * 8 + 2 * i], 0x05010400u), __builtin_amdgcn_perm(hb[m * 8 + 2 * i], lb[m * 8 + 2 * i], 0x07030602u),
                        __builtin_amdgcn_perm(hb[m * 8 + 2 * i + 1], lb[m * 8 + 2 * i + 1], 0x05010400u), __builtin_amdgcn_perm(hb[m * 8 + 2 * i + 1], lb[m * 8 + 2 * i + 1], 0x07030602u)},
                rb2, lane * 16u, (m * 4 + i) * 1024u);
  } else {
    for (u32 p = 0; p < total; p += 1024u) store16(*(const LAS u32x4*)(scr + p + lane * 16u), rb2, p + lane * 16u < total ? lane * 16u : 0x80000000u, p);
  }
  STAMP2(3);
  asm volatile("s_waitcnt vmcnt(0)" : "+v"(qi) :: "memory");
  const u32 slot = ((u32)__builtin_amdgcn_readfirstlane(qi) - (call - 1u) * OUNITS) & 255u;
  flag_store(pick(P.ws, (g4 ^ 4u) | j) + OFF_OQ + (r & 3u) * 2048u + slot * 8u, ((u64)call << 32) | uid | (total << 8) | (raw ? 1u << 22 : 0u) | (top << 23), 1u);
  STAMP(3);
}

__global__ __launch_bounds__(512, 2) void ag_gemm_kernel(Params P) {
  extern __shared__ __attribute__((aligned(16))) char smem[];
  LAS char* lds = (LAS char*)smem;
  const u32 lane = threadIdx.x & 63u, wid = __builtin_amdgcn_readfirstlane(threadIdx.x >> 6);
  char* const ws = pick(P.ws, P.rank);
  STAMP(0);
  u32 bid = blockIdx.x;
  asm volatile("" : "+s"(bid));
  char* const cptr = ws + OFF_CALL + bid * 128u + wid * 4u;
  const u32 callv = ld_agent(cptr);
  LAS unsigned char* scr = (LAS unsigned char*)lds + wid * WAVE_SCRATCH;
  build_tables(lds, lane);
  bool skip = false;
  u32 li = 7, lidx = 0, pkw = 0;
  if (bid < NCOMM) {
    const u32 h = (bid >> 2) & 1u, a = (bid >> 3) * 4u + (bid & 3u), nl = 3u + h;
    if (a < nl * NCL) { li = h * 3u + (a + h * (a >> 2)) % nl; lidx = a / nl; }
    else pkw = h * (NCOMM / 2 - 3 * NCL) + a - nl * NCL;
  }
  LAS u32* const ctl = (LAS u32*)(lds + 8 * WAVE_SCRATCH);
  if (bid < NCOMM) {
    ctl[lane & 3u] = 0;
    asm volatile("s_waitcnt lgkmcnt(0)\n\ts_barrier" ::: "memory");
  }
  const u32 oj = li >= 7u ? pkw & 3u : (li >= 3u ? li - 3u : (li * NCL + lidx + NPKWG) & 3u);
  const u32 ftok = li >= 7u ? AG_FTOK_PK : (li >= 3u ? AG_FTOK : AG_FTOK_IN);
  const bool isg = bid >= NCOMM;
  u32 call;
  if (li < 7u && wid < PW) {
    call = __builtin_amdgcn_readfirstlane(callv) + 1u;
    if (!skip) puller(P, ws, call, li, lidx, ctl, scr, lane, wid);
  } else {
    const u32 gb = bid - NCOMM;
    const u32 pidx = li < 7u ? NPKWG * 8u + (li * NCL + lidx) * AG_MIX + (wid - PW) : wid * NPKWG + pkw;
    const u32 q0 = isg ? wid * NGEMM + gb : GPACK + pidx;
    const u32 q1 = isg ? ((gb & 4u) ? GPACK : 8u * NGEMM + wid * (NGEMM / 2) + (gb >> 3) * 4u + (gb & 3u)) : q0 + NPK;
    for (u32 i = isg ? wid : (u32)AG_PDELAY; i; --i) __builtin_amdgcn_s_sleep(AG_GDELAY);
    call = packer(P, ws, callv, skip ? (u32)NUNITS : q0, q1, isg ? 4u * NGEMM : (u32)NPK, isg ? GPACK : (u32)NUNITS, scr, lane, wid);
  }
  __hip_atomic_store((GAS u32*)cptr, call, __ATOMIC_RELAXED, __HIP_MEMORY_SCOPE_AGENT);
  if (!isg) {
    if (ftok) out_puller(P, ws, call, oj, ftok, ctl, scr, lane, wid);
    return;
  }
  asm volatile("s_waitcnt lgkmcnt(0)\n\ts_barrier" ::: "memory");
  STAMP(7);
  if (wid < 2u) gemm_wave<3, 5>(P, ws, call, bid - NCOMM, lds, lane, wid);
  else if (wid < 6u) gemm_wave<2, 0>(P, ws, call, bid - NCOMM, lds, lane, wid);
  else gemm_wave<3, 8>(P, ws, call, bid - NCOMM, lds, lane, wid);
}

}

extern "C" void kernel_launch(int rank, int nranks, ncclComm_t comm, hipStream_t stream,
                              void* const* d_in, const int* in_sizes, int n_in,
                              void* d_out, int out_size, const int* in_dtypes, int out_dtype,
                              void* const* d_out_all, void* const* d_ws_all, size_t ws_bytes) {
  (void)comm; (void)in_dtypes; (void)out_dtype; (void)d_out;
  if (nranks != ag::NR || n_in != 2 || (unsigned)rank >= (unsigned)ag::NR || in_sizes[0] != 1024 * 8192 || in_sizes[1] != 8192 * 1024 ||
      out_size != 8192 * 1024 || ws_bytes < ag::WS_NEED) {
    fprintf(stderr, "ag_gemm: unexpected problem shape\n");
    return;
  }
  ag::Params p;
  p.x = (const char*)d_in[0];
  p.w = (const char*)d_in[1];
  for (int i = 0; i < ag::NR; ++i) {
    const int phys = (AG_ORDER >> (4 * i)) & 7;
    p.ws[i] = (char*)d_ws_all[phys];
    p.out[i] = (char*)d_out_all[phys];
    if (phys == rank) p.rank = i;
  }
  p.pad = 0;
  hipError_t e = hipFuncSetAttribute((const void*)ag::ag_gemm_kernel, hipFuncAttributeMaxDynamicSharedMemorySize, ag::LDS_BYTES);
  if (e != hipSuccess) { fprintf(stderr, "ag_gemm: %s\n", hipGetErrorString(e)); return; }
  hipLaunchKernelGGL(ag::ag_gemm_kernel, dim3(ag::NWG), dim3(512), ag::LDS_BYTES, stream, p);
  e = hipGetLastError();
  if (e != hipSuccess) fprintf(stderr, "ag_gemm launch: %s\n", hipGetErrorString(e));
}
```

```cpp
#include <hip/hip_runtime.h>
#include <rccl/rccl.h>
#include <cstdio>

namespace ag {

typedef unsigned u32;
typedef unsigned long long u64;
typedef unsigned u32x4 __attribute__((ext_vector_type(4)));
typedef unsigned short u16x2 __attribute__((ext_vector_type(2)));
typedef short bf16x8 __attribute__((ext_vector_type(8)));
typedef float f32x4 __attribute__((ext_vector_type(4)));
typedef __amdgpu_buffer_rsrc_t rsrc_t;
#define LAS __attribute__((address_space(3)))
#define GAS __attribute__((address_space(1)))

#define AG_AU 21
#define AG_NCL 6
#define AG_HIN 4
#define AG_HOUT 4
#define AG_E1 5
#define AG_E2 2
#define AG_GX 2240
#define AG_WLEAD 6
#define AG_KP 1
#define AG_XAUX 2
#define AG_WAUX 2
#define AG_OAUX 17
#define AG_FTOK 3
#define AG_OEARLY 2
#define AG_APAD 0
#define AG_GDELAY 28
#define AG_PDELAY 0
#define AG_ORDER 0x75416320u
#define AG_FORCE_RAW 0
#define AG_GEMM_AUX 17

constexpr int NR = 8;
constexpr int AU = AG_AU;
constexpr int AWAY = 32 - AU;
constexpr int NKC = 64, NKS = 256;
constexpr int EARLY_UNITS = AU + 4 * AWAY;
constexpr u32 NFT = 31, NOT = 22, ROW_OWN = NFT, ROW_W = NFT + NOT;
constexpr int NOUT = AWAY + 8;

constexpr int NCOMM = 44, NGEMM = 4 * (NFT + NOT), NWG = NCOMM + NGEMM;
constexpr int NCL = AG_NCL;
constexpr int NPKWG = NCOMM - 7 * NCL;
static_assert(NWG == 256 && NPKWG > 0, "one workgroup per CU");
constexpr int PW = 8;
constexpr int NPK = NPKWG * 8;
constexpr u32 GPACK = 8 * NGEMM + AG_GX;
constexpr int NUNITS = 4096;
constexpr u32 WLEAD = AG_WLEAD;

constexpr u32 A_STEP = 262144 + AG_APAD, B_STEP = 16384;
constexpr u32 B_TILE = B_STEP * NKS;
constexpr u32 OFF_XU = 0;
constexpr u32 OFF_WU = OFF_XU + NKS * A_STEP;
constexpr u32 FRAG_BYTES = OFF_WU + 8 * B_TILE;
constexpr u32 SLOT = 8192, PK_BYTES = NUNITS * SLOT;
constexpr u32 OFF_PK = FRAG_BYTES;
constexpr u32 OSLOT = 12288, OUNITS = NFT * 8;
constexpr u32 OFF_OB = OFF_PK + 2 * PK_BYTES;
constexpr u32 KP = AG_KP, IB_SRC = KP * 64 * SLOT, IB_BYTES = 2 * NR * IB_SRC;
constexpr u32 OFF_IB = OFF_OB + 4 * OUNITS * OSLOT;
constexpr u32 OFF_G = OFF_IB + IB_BYTES;
constexpr u32 G_BYTES = 2 * NR * NUNITS * 8;
constexpr u32 OFF_CNT = OFF_G + G_BYTES;
constexpr u32 OFF_OQ = OFF_CNT + 16384;
constexpr u32 OFF_TKT = OFF_OQ + 4 * 2048;
constexpr u32 OFF_TMO = OFF_TKT + 2048;
constexpr u32 OFF_CALL = OFF_TMO + 128;
constexpr u32 OFF_DBG = OFF_CALL + NWG * 128;
constexpr u32 OFF_DBG2 = OFF_DBG + NWG * 8 * 64;
constexpr u32 OFF_DBG3 = OFF_DBG2 + 65536;
constexpr u32 OFF_DBG4 = OFF_DBG3 + 131072;
constexpr u32 WS_NEED = OFF_DBG4 + 524288;

constexpr u32 A_LDS = 12288, SLOTB = A_LDS + B_STEP, NSLOT = 5;
constexpr u32 TAB_OFF = NSLOT * SLOTB;
constexpr int LDS_BYTES = TAB_OFF + 256;
constexpr int WAVE_SCRATCH = 13312;
static_assert(8 * WAVE_SCRATCH + 64 <= LDS_BYTES, "scratch");

constexpr u32 SPARE = 4000, SPARE_CNT = SPARE | SPARE << 16;
constexpr u64 DEADLINE = 50000000ull;
constexpr int AUX_SYS = 17;

struct Params {
  const char* x;
  const char* w;
  char* ws[NR];
  char* out[NR];
  int rank;
  int pad;
};

#define STAMP(i) do {} while (0)
#define STAMPC(i) do {} while (0)
#define STAMPP(i) do {} while (0)
#define STAMP2(i) do {} while (0)
#define STAMPQ(i) do {} while (0)

__device__ __forceinline__ rsrc_t make_rsrc(const void* p, u32 bytes) {
  const u64 a = (u64)p;
  const u32 lo = __builtin_amdgcn_readfirstlane((u32)a), hi = __builtin_amdgcn_readfirstlane((u32)(a >> 32));
  return __builtin_amdgcn_make_buffer_rsrc((void*)(((u64)hi << 32) | lo), 0, bytes, 0x00020000);
}
__device__ __forceinline__ char* pick(char* const (&t)[NR], u32 i) {
  char* p = t[0];
#pragma unroll
  for (u32 j = 1; j < NR; ++j) p = (i == j) ? t[j] : p;
  return p;
}
__device__ __forceinline__ u32 ld_agent(const char* p) { return __hip_atomic_load((GAS u32*)p, __ATOMIC_RELAXED, __HIP_MEMORY_SCOPE_AGENT); }
__device__ __forceinline__ u32 ld_sys(const char* p) { return __hip_atomic_load((GAS u32*)p, __ATOMIC_RELAXED, __HIP_MEMORY_SCOPE_SYSTEM); }
template <int AUX = AUX_SYS>
__device__ __forceinline__ void store16(u32x4 v, rsrc_t rs, u32 voff, u32 soff) {
  __builtin_amdgcn_raw_buffer_store_b128(v, rs, voff, soff, AUX);
  asm volatile("s_nop 1" :: "v"(v) : "memory");
}
__device__ __forceinline__ u32 pair_offset(u32 c, u32 lane) { return ((lane & 1u) ? c >> 16 : c & 0xffffu) * 4u; }
__device__ __forceinline__ u64 pair_mask(u32 c) { return (c >> 16) == SPARE ? 1u : 3u; }
__device__ __forceinline__ void count_in(const char* base, u32 c, u32 lane) {
  u64 keep;
  asm volatile("s_mov_b64 %0, exec\n\ts_mov_b64 exec, %4\n\tglobal_atomic_add %1, %2, %3\n\ts_mov_b64 exec, %0"
               : "=&s"(keep) : "v"(pair_offset(c, lane)), "v"(1u), "s"(base), "s"(pair_mask(c)) : "memory");
}
#define STAMP_CNT(ws, idx) do {} while (0)
__device__ __forceinline__ void flag_store(char* p, u64 v, u64 mask) {
  u64 keep;
  asm volatile("s_mov_b64 %0, exec\n\ts_mov_b64 exec, %3\n\tglobal_store_dwordx2 %1, %2, off sc0 sc1\n\ts_mov_b64 exec, %0"
               : "=&s"(keep) : "v"(p), "v"(v), "s"(mask) : "memory");
}
__device__ __forceinline__ void drain_vm() { asm volatile("s_waitcnt vmcnt(0)" ::: "memory"); }

__device__ __forceinline__ bool give_up(char* ws, u32 call, u64 t0) {
  const u32 t = __builtin_amdgcn_readfirstlane(ld_agent(ws + OFF_TMO));
  if (t == call) return true;
  if (__builtin_amdgcn_s_memrealtime() - t0 > DEADLINE) {
    __hip_atomic_store((GAS u32*)(ws + OFF_TMO), call, __ATOMIC_RELAXED, __HIP_MEMORY_SCOPE_AGENT);
    return true;
  }
  return false;
}

__device__ __forceinline__ u32 wave_sum(u32 v) {
  v += __builtin_amdgcn_update_dpp(0u, v, 0x111, 0xf, 0xf, true);
  v += __builtin_amdgcn_update_dpp(0u, v, 0x112, 0xf, 0xf, true);
  v += __builtin_amdgcn_update_dpp(0u, v, 0x114, 0xf, 0xf, true);
  v += __builtin_amdgcn_update_dpp(0u, v, 0x118, 0xf, 0xf, true);
  return __builtin_amdgcn_readlane(v, 15) + __builtin_amdgcn_readlane(v, 31) + __builtin_amdgcn_readlane(v, 47) + __builtin_amdgcn_readlane(v, 63);
}
__device__ __forceinline__ u32 wave_max(u32 v) {
  v = max(v, (u32)__builtin_amdgcn_update_dpp(0u, v, 0x111, 0xf, 0xf, true));
  v = max(v, (u32)__builtin_amdgcn_update_dpp(0u, v, 0x112, 0xf, 0xf, true));
  v = max(v, (u32)__builtin_amdgcn_update_dpp(0u, v, 0x114, 0xf, 0xf, true));
  v = max(v, (u32)__builtin_amdgcn_update_dpp(0u, v, 0x118, 0xf, 0xf, true));
  return max(max((u32)__builtin_amdgcn_readlane(v, 15), (u32)__builtin_amdgcn_readlane(v, 31)), max((u32)__builtin_amdgcn_readlane(v, 47), (u32)__builtin_amdgcn_readlane(v, 63)));
}
__device__ __forceinline__ u32 wave_scan(u32 v) {
  v += __builtin_amdgcn_update_dpp(0u, v, 0x111, 0xf, 0xf, true);
  v += __builtin_amdgcn_update_dpp(0u, v, 0x112, 0xf, 0xf, true);
  v += __builtin_amdgcn_update_dpp(0u, v, 0x114, 0xf, 0xf, true);
  v += __builtin_amdgcn_update_dpp(0u, v, 0x118, 0xf, 0xf, true);
  v += __builtin_amdgcn_update_dpp(0u, v, 0x142, 0xa, 0xf, false);
  v += __builtin_amdgcn_update_dpp(0u, v, 0x143, 0xc, 0xf, false);
  return v;
}

__device__ __forceinline__ u32 quad_swz(u32 rq) { return (0x78u >> (2 * rq)) & 3u; }

struct Dest { u32 base, s1, s0, cnt; };
__device__ __forceinline__ Dest dest_x(u32 ux, u32 kc) {
  const u32 ft = ux < 20u ? ux / 5u : (ux - 4u) >> 2, ot = ROW_OWN + ux / 6u;
  return Dest{OFF_XU + 4 * kc * A_STEP + ux * 2048u, 1024u, A_STEP, (ft * 64 + kc) | (ot * 64 + kc) << 16};
}
__device__ __forceinline__ Dest dest_w(u32 ct, u32 kc, u32 ks4, u32 half) {
  return Dest{OFF_WU + ct * B_TILE + (4 * kc + ks4) * B_STEP + half * 8192u, 4096u, 1024u, ((ROW_W + ct) * 64 + kc) | SPARE << 16};
}
__device__ __forceinline__ void store_frags(const u32x4 (&v)[8], rsrc_t rf, const Dest& d, u32 lane) {
#pragma unroll
  for (int f = 0; f < 8; ++f) store16(v[f], rf, lane * 16, d.base + (f >> 2) * d.s1 + (f & 3) * d.s0);
}

constexpr u32 RAW = 1u << 31, PUSHED = 1u << 30;
constexpr u32 S_LIST = 0, S_TAIL = 2304, S_RARE = 3584;
struct Packed { u32x4 mq[4], p01, tail; };

__device__ __forceinline__ void build_tables(LAS char* lds, u32 lane) {
  u32 ct = 0x0c0c0c0cu, xt = 0x07060504u, zt = 0x0c0c0c0cu, k = 0;
#pragma unroll
  for (u32 b = 0; b < 4; ++b) {
    const u32 on = 0u - ((lane >> b) & 1u);
    ct ^= on & ((0x0cu ^ b) << (8u * k));
    xt ^= on & (((4u + b) ^ k) << (8u * b));
    zt ^= on & ((0x0cu ^ k) << (8u * b));
    k -= on;
  }
  *(LAS u32*)(lds + TAB_OFF + lane * 4u) = (lane & 32u) ? zt : ((lane & 16u) ? xt : ct);
}

constexpr u32 TAB = TAB_OFF;

template <bool TO_LIST>
__device__ __forceinline__ void walk_spelled(u32 m, u32 off, LAS unsigned char* cells, LAS unsigned char* list, u32 flip) {
  while (m != 0) {
    const u32 p = __builtin_ctz(m);
    m &= m - 1;
    LAS unsigned char* cell = cells + (p & 7u) * 4u + (p >> 3);
    if (TO_LIST) list[off] = *cell ^ flip; else *cell = list[off];
    ++off;
  }
}

__device__ __forceinline__ u32 encode_unit(Packed& k, const u32x4 (&v)[8], LAS unsigned char* scr, u32 lane) {
  const u32x4 zero = {0u, 0u, 0u, 0u};
  *(LAS u32x4*)(scr + S_LIST + lane * 32u) = zero;
  *(LAS u32x4*)(scr + S_LIST + lane * 32u + 16u) = zero;
  *(LAS u32x4*)(scr + S_RARE + lane * 16u) = zero;
  u32 E[16], esum = 0;
#pragma unroll
  for (int j = 0; j < 16; ++j) {
    const u32 d0 = v[j >> 1][(j & 1) * 2], d1 = v[j >> 1][(j & 1) * 2 + 1];
    const u32 r0 = __builtin_amdgcn_alignbit(d0, d0, 31), r1 = __builtin_amdgcn_alignbit(d1, d1, 31);
    k.mq[j >> 2][j & 3] = __builtin_amdgcn_perm(r1, r0, 0x06040200u);
    E[j] = __builtin_amdgcn_perm(r1, r0, 0x07050301u);
    esum = __builtin_amdgcn_sad_u8(E[j], 0u, esum);
  }
  esum = wave_sum(esum);
  u32 base = (esum + 8028u) >> 12;
  base = base < 19u ? 19u : (base > 254u ? 254u : base);
  const u32 mode = base * 4096u < esum + 6361u ? 1u : 0u;
  const u16x2 b2 = {(unsigned short)base, (unsigned short)base};
  u32 P[2][2] = {{0, 0}, {0, 0}};
#pragma unroll
  for (int j = 0; j < 16; ++j) {
    const u16x2 ev = __builtin_bit_cast(u16x2, E[j] & 0x00ff00ffu), od = __builtin_bit_cast(u16x2, E[j]) >> (u16x2){8, 8};
    const u32 ce = __builtin_bit_cast(u32, __builtin_elementwise_min((u16x2)(b2 - ev), (u16x2){3, 3}));
    const u32 co = __builtin_bit_cast(u32, __builtin_elementwise_min((u16x2)(b2 - od), (u16x2){3, 3}));
    const u32 c = ce | (co << 8);
    P[0][j >> 3] |= (c & 0x01010101u) << (j & 7);
    P[1][j >> 3] |= ((c >> 1) & 0x01010101u) << (j & 7);
  }
  k.p01 = (u32x4){P[0][0], P[0][1], P[1][0], P[1][1]};
  k.tail = zero;
  const u32 mk[2] = {P[0][0] & P[1][0], P[0][1] & P[1][1]};
  u32 fl[16], sel[16];
#pragma unroll
  for (int j = 0; j < 16; ++j) {
    fl[j] = (mk[j >> 3] >> (j & 7)) & 0x01010101u;
    sel[j] = *(const LAS u32*)(size_t)__builtin_amdgcn_udot4(fl[j], 0x20100804u, TAB, false);
  }
  __builtin_amdgcn_sched_barrier(0);
  const u32 c = __builtin_popcount(mk[0]) + __builtin_popcount(mk[1]), incl = wave_scan(c);
  const u32 n2 = __builtin_amdgcn_readlane(incl, 63);
  if (n2 > 2044u || AG_FORCE_RAW) return RAW;
  const u32 flip = (base - 3u) * 0x01010101u;
  u32 a = (u32)(size_t)(scr + S_LIST) + incl - c, cur = 0;
#pragma unroll
  for (int j = 0; j < 16; ++j) {
    const u64 t = (u64)__builtin_amdgcn_perm(0u, E[j] ^ flip, sel[j]) << ((a & 3u) * 8u);
    const u32 w = cur | (u32)t, an = __builtin_amdgcn_sad_u8(fl[j], 0u, a);
    __hip_atomic_fetch_or((LAS u32*)(size_t)(a & ~3u), w, __ATOMIC_RELAXED, __HIP_MEMORY_SCOPE_WAVEFRONT);
    cur = ((an ^ a) & 4u) ? (u32)(t >> 32) : w;
    a = an;
  }
  __hip_atomic_fetch_or((LAS u32*)(size_t)(a & ~3u), cur, __ATOMIC_RELAXED, __HIP_MEMORY_SCOPE_WAVEFRONT);
  asm volatile("" ::: "memory");
  const u32x4 l[2] = {*(const LAS u32x4*)(scr + S_LIST + lane * 32u), *(const LAS u32x4*)(scr + S_LIST + lane * 32u + 16u)};
  const unsigned short ta = mode ? base + 1u : base - 3u, tb = mode ? 3u : 0u, tc = base + 1u;
  u32 Q[2] = {0, 0};
#pragma unroll
  for (int s = 0; s < 8; ++s) {
    const u32 x = l[s >> 2][s & 3] ^ flip;
    const u16x2 ev = __builtin_bit_cast(u16x2, x & 0x00ff00ffu), od = __builtin_bit_cast(u16x2, x) >> (u16x2){8, 8};
    const u32 ce = __builtin_bit_cast(u32, __builtin_elementwise_min(__builtin_elementwise_sub_sat((u16x2)((u16x2){ta, ta} - ev), (u16x2){tb, tb}), (u16x2){3, 3}));
    const u32 co = __builtin_bit_cast(u32, __builtin_elementwise_min(__builtin_elementwise_sub_sat((u16x2)((u16x2){ta, ta} - od), (u16x2){tb, tb}), (u16x2){3, 3}));
    const u32 c2 = ce | (co << 8);
    Q[0] |= (c2 & 0x01010101u) << s;
    Q[1] |= ((c2 >> 1) & 0x01010101u) << s;
  }
  *(LAS u32*)(scr + S_TAIL + lane * 8u) = Q[0];
  *(LAS u32*)(scr + S_TAIL + lane * 8u + 4u) = Q[1];
  const u32 m3 = Q[0] & Q[1];
#pragma unroll
  for (int s = 0; s < 8; ++s) {
    fl[s] = (m3 >> s) & 0x01010101u;
    sel[s] = *(const LAS u32*)(size_t)__builtin_amdgcn_udot4(fl[s], 0x20100804u, TAB, false);
  }
  __builtin_amdgcn_sched_barrier(0);
  const u32 c3 = __builtin_popcount(m3), incl3 = wave_scan(c3), n3 = __builtin_amdgcn_readlane(incl3, 63);
  if (n3 > 768u) return RAW;
  const u32 turn = flip ^ (tc * 0x01010101u);
  a = (u32)(size_t)(scr + S_RARE) + incl3 - c3;
  cur = 0;
#pragma unroll
  for (int s = 0; s < 8; ++s) {
    const u64 t = (u64)__builtin_amdgcn_perm(0u, l[s >> 2][s & 3] ^ turn, sel[s]) << ((a & 3u) * 8u);
    const u32 w = cur | (u32)t, an = __builtin_amdgcn_sad_u8(fl[s], 0u, a);
    __hip_atomic_fetch_or((LAS u32*)(size_t)(a & ~3u), w, __ATOMIC_RELAXED, __HIP_MEMORY_SCOPE_WAVEFRONT);
    cur = ((an ^ a) & 4u) ? (u32)(t >> 32) : w;
    a = an;
  }
  __hip_atomic_fetch_or((LAS u32*)(size_t)(a & ~3u), cur, __ATOMIC_RELAXED, __HIP_MEMORY_SCOPE_WAVEFRONT);
  asm volatile("" ::: "memory");
  const u32x4 rq = *(const LAS u32x4*)(scr + S_RARE + lane * 16u);
  u32 cn[4], m4 = 0;
#pragma unroll
  for (int s = 0; s < 4; ++s) {
    const u32 x = rq[s] ^ (tc * 0x01010101u);
    const u16x2 ev = __builtin_bit_cast(u16x2, x & 0x00ff00ffu), od = __builtin_bit_cast(u16x2, x) >> (u16x2){8, 8};
    const u32 ce = __builtin_bit_cast(u32, __builtin_elementwise_min(__builtin_elementwise_sub_sat((u16x2)((u16x2){tc, tc} - ev), (u16x2){5, 5}), (u16x2){15, 15}));
    const u32 co = __builtin_bit_cast(u32, __builtin_elementwise_min(__builtin_elementwise_sub_sat((u16x2)((u16x2){tc, tc} - od), (u16x2){5, 5}), (u16x2){15, 15}));
    cn[s] = ce | (co << 8);
    m4 |= ((cn[s] + 0x01010101u) & 0x10101010u) >> 4 << s;
  }
  const u32 nib = ((n2 + 31u) >> 5) * 8u;
  u32 bytes = nib + ((n3 + 15u) >> 4) * 8u;
  *(LAS u32*)(scr + S_TAIL + nib + lane * 8u) = cn[0] | (cn[1] << 4);
  *(LAS u32*)(scr + S_TAIL + nib + lane * 8u + 4u) = cn[2] | (cn[3] << 4);
  if (__builtin_amdgcn_ballot_w64(m4 != 0) != 0) {
    const u32 c4 = __builtin_popcount(m4), incl4 = wave_scan(c4), n4 = __builtin_amdgcn_readlane(incl4, 63);
    if (n4 > 128u) return RAW;
    asm volatile("" ::: "memory");
    walk_spelled<true>(m4, incl4 - c4, scr + S_RARE + lane * 16u, scr + S_TAIL + bytes, tc);
    bytes += n4;
  }
  asm volatile("" ::: "memory");
  k.tail = *(const LAS u32x4*)(scr + S_TAIL + lane * 16u);
  return base | (mode << 8) | (bytes << 16);
}

__device__ __forceinline__ void store_unit(const Packed& k, const u32x4 (&v)[8], u32 info, rsrc_t rp, u32 soff, u32 lane) {
  if (info & RAW) {
#pragma unroll
    for (int f = 0; f < 8; ++f) store16(v[f], rp, lane * 16, soff + f * 1024);
    return;
  }
#pragma unroll
  for (int i = 0; i < 4; ++i) store16(k.mq[i], rp, lane * 16, soff + i * 1024);
  store16(k.p01, rp, lane * 16, soff + 4096);
  store16(k.tail, rp, lane * 16u < (info >> 16) ? lane * 16u : 0x80000000u, soff + 5120);
}

__device__ __forceinline__ void decode_unit(u32x4 (&v)[8], const Packed& k, u32 info, LAS unsigned char* scr, u32 lane) {
  *(LAS u32x4*)(scr + S_TAIL + lane * 16u) = k.tail;
  asm volatile("" ::: "memory");
  const u32 q0 = *(const LAS u32*)(scr + S_TAIL + lane * 8u), q1 = *(const LAS u32*)(scr + S_TAIL + lane * 8u + 4u);
  const u32 b4 = (info & 255u) * 0x01010101u, lut = (info & 256u) ? b4 - 0x00040300u + 1u : b4 - 0x00050403u;
  const u32 P0[2] = {k.p01.x, k.p01.y}, P1[2] = {k.p01.z, k.p01.w}, mk[2] = {P0[0] & P1[0], P0[1] & P1[1]};
  const u32 c = __builtin_popcount(mk[0]) + __builtin_popcount(mk[1]), incl = wave_scan(c);
  const u32 g2 = (((u32)__builtin_amdgcn_readlane(incl, 63) + 31u) >> 5) & 127u;
  LAS unsigned char* const nib = scr + S_TAIL + (g2 < 64u ? g2 : 64u) * 8u;
  const u32 x0 = *(const LAS u32*)(nib + lane * 8u), x1 = *(const LAS u32*)(nib + lane * 8u + 4u);
  u32 a[17], sel[16];
  a[0] = (u32)(size_t)(scr + S_LIST) + incl - c;
#pragma unroll
  for (int j = 0; j < 16; ++j) {
    const u32 f = (mk[j >> 3] >> (j & 7)) & 0x01010101u;
    sel[j] = *(const LAS u32*)(size_t)__builtin_amdgcn_udot4(f, 0x20100804u, TAB + 64u, false);
    a[j + 1] = __builtin_amdgcn_sad_u8(f, 0u, a[j]);
  }
  __builtin_amdgcn_sched_barrier(0);
  const u32 live = lane < g2 ? ~0u : 0u, Q0 = q0 & live, Q1 = q1 & live, m3 = Q0 & Q1;
  const u32 c3 = __builtin_popcount(m3), incl3 = wave_scan(c3), g3 = (((u32)__builtin_amdgcn_readlane(incl3, 63) + 15u) >> 4) & 63u;
  const u32 live3 = lane < g3 ? 0x0f0f0f0fu : 0u, cn[4] = {x0 & live3, (x0 >> 4) & live3, x1 & live3, (x1 >> 4) & live3};
  u32x4 rq;
  u32 any = 0;
#pragma unroll
  for (int s = 0; s < 4; ++s) {
    const u32 z = ~(cn[s] + 0x0f0f0f0fu) & 0x10101010u;
    rq[s] = b4 - 0x04040404u - cn[s] + (z >> 4) + (z >> 2);
    any |= cn[s] + 0x01010101u;
  }
  *(LAS u32x4*)(scr + S_RARE + lane * 16u) = rq;
  if (__builtin_amdgcn_ballot_w64((any & 0x10101010u) != 0) != 0) {
    u32 m4 = 0;
#pragma unroll
    for (int s = 0; s < 4; ++s) m4 |= ((cn[s] + 0x01010101u) & 0x10101010u) >> 4 << s;
    const u32 c4 = __builtin_popcount(m4);
    walk_spelled<false>(m4, (wave_scan(c4) - c4) & 127u, scr + S_RARE + lane * 16u, nib + g3 * 8u, 0u);
  }
  asm volatile("" ::: "memory");
  u32x4 l[2];
  {
    u32 a3 = (u32)(size_t)(scr + S_RARE) + ((incl3 - c3) & 1023u), at[8], s3[8], r0[8], r1[8];
#pragma unroll
    for (int s = 0; s < 8; ++s) {
      const u32 f = (m3 >> s) & 0x01010101u;
      const LAS u32* const pw = (const LAS u32*)(size_t)(a3 & ~3u);
      at[s] = a3;
      s3[s] = *(const LAS u32*)(size_t)__builtin_amdgcn_udot4(f, 0x20100804u, TAB + 64u, false);
      r0[s] = pw[0];
      r1[s] = pw[1];
      a3 = __builtin_amdgcn_sad_u8(f, 0u, a3);
    }
    __builtin_amdgcn_sched_barrier(0);
#pragma unroll
    for (int s = 0; s < 8; ++s) {
      const u32 c2 = ((Q0 >> s) & 0x01010101u) | (((Q1 >> s) & 0x01010101u) << 1);
      l[s >> 2][s & 3] = __builtin_amdgcn_perm(__builtin_amdgcn_perm(0u, lut, c2), __builtin_amdgcn_alignbyte(r1[s], r0[s], at[s]), s3[s]);
    }
  }
  *(LAS u32x4*)(scr + S_LIST + lane * 32u) = l[0];
  *(LAS u32x4*)(scr + S_LIST + lane * 32u + 16u) = l[1];
  asm volatile("" ::: "memory");
  u32 w0[16], w1[16];
#pragma unroll
  for (int j = 0; j < 16; ++j) {
    const LAS u32* const pw = (const LAS u32*)(size_t)(a[j] & ~3u);
    w0[j] = pw[0];
    w1[j] = pw[1];
  }
  __builtin_amdgcn_sched_barrier(0);
#pragma unroll
  for (int j = 0; j < 16; ++j) {
    const int s = j & 7, wd = j >> 3;
    const u32 c1 = ((P0[wd] >> s) & 0x01010101u) | (((P1[wd] >> s) & 0x01010101u) << 1);
    const u32 e = __builtin_amdgcn_perm(b4 - c1, __builtin_amdgcn_alignbyte(w1[j], w0[j], a[j]), sel[j]), m = k.mq[j >> 2][j & 3];
    const u32 r0 = __builtin_amdgcn_perm(e, m, 0x05010400u), r1 = __builtin_amdgcn_perm(e, m, 0x07030602u);
    v[j >> 1][(j & 1) * 2] = __builtin_amdgcn_alignbit(r0, r0, 1);
    v[j >> 1][(j & 1) * 2 + 1] = __builtin_amdgcn_alignbit(r1, r1, 1);
  }
}

__device__ __forceinline__ void load_unit(u32x4 (&s)[8], rsrc_t rx, rsrc_t rw, u32 q, u32 vx, u32 vw) {
  const u32 kc = q >> 6, t = q & 63u;
  if (t < 32u) {
#pragma unroll
    for (int f = 0; f < 8; ++f) s[f] = __builtin_amdgcn_raw_buffer_load_b128(rx, vx, (32u * t + 16u * (f >> 2)) * 16384u + kc * 256u + (f & 3) * 64u, AG_XAUX);
  } else {
#pragma unroll
    for (int i = 0; i < 8; ++i) s[i] = __builtin_amdgcn_raw_buffer_load_b128(rw, vw, (128u * kc + 32u * ((t - 32u) >> 3) + i) * 2048u + (t & 7u) * 256u, AG_WAUX);
  }
}

__device__ __forceinline__ u32 packer(const Params& P, char* ws, u32 callv, u32 q0, u32 q1, u32 dq, u32 qend, LAS unsigned char* scr, u32 lane, u32 wid) {
  const rsrc_t rx = make_rsrc(P.x, 1024u * 8192u * 2u), rw = make_rsrc(P.w, 8192u * 1024u * 2u);
  const u32 kq = (lane & 3u) ^ quad_swz(lane >> 4);
  const u32 vx = (lane >> 2) * 16384u + kq * 16u, vw = (lane >> 4) * 8u * 2048u + (lane & 15u) * 16u;
  u32x4 s[8];
  STAMPP(0);
  load_unit(s, rx, rw, q0, vx, vw);
  __builtin_amdgcn_sched_barrier(0);
  const u32 call = __builtin_amdgcn_readfirstlane(callv) + 1u;
  const u32 r = P.rank, g4 = r & 4u, par = call & 1u;
  const rsrc_t rf = make_rsrc(ws, FRAG_BYTES), rp = make_rsrc(ws + OFF_PK + par * PK_BYTES, PK_BYTES);
  const u32 goff = OFF_G + (par * NR + r) * NUNITS * 8;
  char* g_in = pick(P.ws, g4 | ((r + 1u) & 3u));
  char* g_out = pick(P.ws, g4 ^ 4u);
#pragma unroll
  for (u32 j = 1; j < 4; ++j) {
    char* const pi = pick(P.ws, g4 | ((r + 1u + j) & 3u));
    char* const po = pick(P.ws, (g4 ^ 4u) | j);
    g_in = lane == j ? pi : g_in;
    g_out = lane == j ? po : g_out;
  }
  STAMPP(1);
  u64 psend = 0;
  char* pgp = g_out;
  u32 pinfo = 0, pcnt = SPARE_CNT;
  auto announce = [&]() {
    flag_store(pgp, ((u64)call << 32) | pinfo, psend);
    if (pcnt != SPARE_CNT) { STAMP_CNT(ws, pcnt); count_in(ws + OFF_CNT, pcnt, lane); }
  };
  for (u32 q = q0; q < qend; q = q1, q1 += dq) {
    const u32 kc = q >> 6, t = q & 63u;
    u32x4 v[8];
    Dest d;
    bool own;
    u64 send;
    char* gp;
    if (t < 32u) {
#pragma unroll
      for (int f = 0; f < 8; ++f) v[f] = s[f];
      own = t < AU;
      d = dest_x(t, kc);
      send = own ? 7u : 15u;
      gp = own ? g_in : g_out;
    } else {
      const u32 ks4 = (t - 32u) >> 3, cg = t & 7u;
#pragma unroll
      for (int nf = 0; nf < 8; ++nf)
#pragma unroll
        for (int tt = 0; tt < 4; ++tt) v[nf][tt] = __builtin_amdgcn_perm(s[2 * tt + 1][nf >> 1], s[2 * tt][nf >> 1], (nf & 1) ? 0x07060302u : 0x05040100u);
      own = true;
      d = dest_w(cg >> 1, kc, ks4, cg & 1u);
      send = 1u << (cg >> 1);
      gp = g_out;
    }
    if (q == q0) STAMPP(2);
    const bool lazy = qend != NUNITS;
    if (q1 < qend && !lazy) load_unit(s, rx, rw, q1, vx, vw);
    Packed k;
    u32 info = RAW | PUSHED;
    if (kc >= KP) info = encode_unit(k, v, scr, lane);
    if (q == q0) STAMPP(3);
    drain_vm();
#pragma unroll
    for (int f = 0; f < 8; ++f) asm volatile("" : "+v"(s[f]));
    announce();
    if (kc >= KP) store_unit(k, v, info, rp, q * SLOT, lane);
    else {
      for (u32 m = (u32)send; m; m &= m - 1u) {
        const u32 jd = __builtin_ctz(m), dst = (t < AU) ? (g4 | ((r + 1u + jd) & 3u)) : ((g4 ^ 4u) | jd);
        store_unit(k, v, RAW, make_rsrc(pick(P.ws, dst) + OFF_IB + (par * NR + r) * IB_SRC, IB_SRC), q * SLOT, lane);
      }
    }
    if (own) store_frags(v, rf, d, lane);
    if (q == q0) STAMPP(4);
    psend = send;
    pgp = gp + goff + q * 8u;
    pinfo = info;
    pcnt = own ? d.cnt : SPARE_CNT;
    if (q1 < qend && lazy) {
      drain_vm();
      announce();
      STAMPP(5);
      psend = 0;
      pcnt = SPARE_CNT;
      load_unit(s, rx, rw, q1, vx, vw);
    }
  }
  drain_vm();
  announce();
  STAMPP(6);
  STAMP(2);
  return call;
}

__device__ __forceinline__ u32 lds_add(LAS u32* p, u32 v) {
  return __builtin_amdgcn_readfirstlane(__hip_atomic_fetch_add(p, v, __ATOMIC_RELAXED, __HIP_MEMORY_SCOPE_WORKGROUP)) >> 6;
}

__device__ __forceinline__ void puller(const Params& P, char* ws, u32 call, u32 li, u32 lidx, LAS u32* ctl, LAS unsigned char* scr, u32 lane, u32 wid) {
  const u32 r = P.rank, g4 = r & 4u, par = call & 1u;
  const u32 TOK = li < 3u ? AG_HIN : AG_HOUT;
  const bool ing = li < 3u;
  const u32 src = ing ? (g4 | ((r + 1 + li) & 3u)) : ((g4 ^ 4u) | (li - 3u));
  const rsrc_t rf = make_rsrc(ws, FRAG_BYTES), rp = make_rsrc(pick(P.ws, src) + OFF_PK + par * PK_BYTES, PK_BYTES);
  const rsrc_t ri = make_rsrc(ws + OFF_IB + (par * NR + src) * IB_SRC, IB_SRC);
  char* const gbase = ws + OFF_G + (par * NR + src) * NUNITS * 8;
  GAS u32* const cnt = (GAS u32*)(ws + OFF_CNT);
  const u32 total = ing ? NKC * AU : NKC * NOUT;
  constexpr u32 NONE = 0xffffffffu;
  const u32 idle = ((OFF_CALL - OFF_CNT + (lidx * 7u + li) * 128u + 64u + wid * 4u) >> 2) | SPARE << 16;
  GAS u32* const tkt = (GAS u32*)(ws + OFF_TKT + li * 128u);
  const u32 tbase = (call - 1u) * (total + NCL * PW) * 64u;
  auto draw = [&]() -> u32 { return __hip_atomic_fetch_add(tkt, 1u, __ATOMIC_RELAXED, __HIP_MEMORY_SCOPE_AGENT); };
  auto unit_of = [&](u32 raw_ticket) -> u32 {
    const u32 i = ((u32)__builtin_amdgcn_readfirstlane(raw_ticket) - tbase) >> 6;
    if (i >= total) return NONE;
    if (ing) { const u32 kc = i / AU; return kc * 64u + (i - kc * AU); }
    u32 kc, t;
    if (i < 8u * WLEAD) { kc = i >> 3; t = AWAY + (i & 7u); }
    else {
      const u32 i2 = i - 8u * WLEAD, mid = (NKC - WLEAD) * NOUT;
      if (i2 < mid) { kc = i2 / NOUT; t = i2 - kc * NOUT; kc += t < AWAY ? 0u : WLEAD; }
      else { kc = (i2 - mid) / AWAY; t = i2 - mid - kc * AWAY; kc += NKC - WLEAD; }
    }
    const u32 tw = t - AWAY;
    return kc * 64u + (t < AWAY ? AU + t : 32u + (tw >> 1) * 8u + 2u * (r & 3u) + (tw & 1u));
  };
  auto granule = [&](u32 q) -> u64 { return __hip_atomic_load((GAS u64*)(gbase + (q & (NUNITS - 1)) * 8u), __ATOMIC_RELAXED, __HIP_MEMORY_SCOPE_SYSTEM); };
  auto peek = [&](LAS u32* p) -> u32 { return (u32)(__builtin_amdgcn_readfirstlane(__hip_atomic_load(p, __ATOMIC_RELAXED, __HIP_MEMORY_SCOPE_WORKGROUP)) >> 6); };
  auto turn = [&](u32 my) { while (peek(ctl + 2) + TOK <= my) __builtin_amdgcn_s_sleep(1); };
  auto dest_of = [&](u32 q) -> Dest {
    const u32 kc = q >> 6, tq = q & 63u;
    if (q == NUNITS) return Dest{FRAG_BYTES, 0u, 0u, NONE};
    if (tq < 32u) return dest_x(ing ? EARLY_UNITS + li * AU + tq : AU + (li - 3u) * AWAY + (tq - AU), kc);
    return dest_w(4u + (li - 3u), kc, (tq - 32u) >> 3, tq & 1u);
  };
  auto publish = [&](u32 c) { if (c != NONE) { STAMP_CNT(ws, c); count_in(ws + OFF_CNT, c, lane); } };
  u32 q = NUNITS, pend = NONE, tag = call, info = 127u;
  u64 gr;
  bool first = true;
#define LAP(a) do {} while (0)
  while (q != NONE) {
    if (tag != call || (!(info & PUSHED) && peek(ctl + 2) + TOK <= peek(ctl))) {
      drain_vm();
      publish(pend);
      pend = NONE;
      if (tag != call) {
        const u64 t0 = __builtin_amdgcn_s_memrealtime();
        for (;;) {
          gr = granule(q);
          tag = __builtin_amdgcn_readfirstlane((u32)(gr >> 32));
          info = __builtin_amdgcn_readfirstlane((u32)gr);
          if (tag == call || give_up(ws, call, t0)) break;
          __builtin_amdgcn_s_sleep(2);
        }
        if (tag != call) break;
      }
    }
    u32 my = 0;
    if (!(info & PUSHED)) { my = lds_add(ctl, 2u); turn(my); }
    asm volatile("" ::: "memory");
    if (first && q != NUNITS) { STAMP(1); first = false; }
    LAP(acc_w);
    const Dest d = dest_of(q);
    const u32 cur = info, soff = q * SLOT;
    u32x4 v[8];
    if (cur & RAW) {
#pragma unroll
      for (int f = 0; f < 8; ++f) v[f] = __builtin_amdgcn_raw_buffer_load_b128((cur & PUSHED) ? ri : rp, lane * 16, soff + f * 1024, AUX_SYS);
      const u32 nt = draw();
      drain_vm();
      if (!(cur & PUSHED)) lds_add(ctl + 2, 2u);
      q = unit_of(nt);
      gr = granule(q);
      publish(pend);
      tag = __builtin_amdgcn_readfirstlane((u32)(gr >> 32));
      info = __builtin_amdgcn_readfirstlane((u32)gr);
      store_frags(v, rf, d, lane);
    } else {
      Packed k;
      u32 nt;
      u64 keep;
      asm volatile("s_mov_b64 %1, exec\n\ts_mov_b64 exec, 1\n\tglobal_atomic_add %0, %2, %3, %4 sc0\n\ts_mov_b64 exec, %1"
                   : "=&v"(nt), "=&s"(keep) : "v"(0u), "v"(64u), "s"(tkt) : "memory");
      k.tail = __builtin_amdgcn_raw_buffer_load_b128(rp, lane * 16u < (cur >> 16) ? lane * 16u : 0x80000000u, soff + 5120, AUX_SYS);
      k.p01 = __builtin_amdgcn_raw_buffer_load_b128(rp, lane * 16, soff + 4096, AUX_SYS);
      k.mq[0] = __builtin_amdgcn_raw_buffer_load_b128(rp, lane * 16, soff, AUX_SYS);
      asm volatile("" ::: "memory");
      turn(my + 1u);
      asm volatile("" ::: "memory");
#pragma unroll
      for (int i = 1; i < 4; ++i) k.mq[i] = __builtin_amdgcn_raw_buffer_load_b128(rp, lane * 16, soff + i * 1024, AUX_SYS);
      asm volatile("s_waitcnt vmcnt(%0)" :: "i"(AG_E1) : "memory");
      lds_add(ctl + 2, 1u);
      asm volatile("s_waitcnt vmcnt(%0)" :: "i"(AG_E2) : "memory");
      lds_add(ctl + 2, 1u);
      asm volatile("s_waitcnt vmcnt(0)" : "+v"(nt), "+v"(k.mq[0]), "+v"(k.mq[1]), "+v"(k.mq[2]), "+v"(k.mq[3]), "+v"(k.p01), "+v"(k.tail) :: "memory");
      LAP(acc_l);
      q = unit_of(nt);
      STAMP_CNT(ws, pend);
      asm volatile("global_load_dwordx2 %0, %2, %3 sc0 sc1\n\ts_mov_b64 %1, exec\n\ts_mov_b64 exec, %7\n\tglobal_atomic_add %4, %5, %6\n\ts_mov_b64 exec, %1"
                   : "=&v"(gr), "=&s"(keep) : "v"((q & (NUNITS - 1)) * 8u), "s"(gbase), "v"(pair_offset(pend == NONE ? idle : pend, lane)), "v"(1u), "s"(cnt),
                     "s"(pair_mask(pend == NONE ? idle : pend)) : "memory");
      decode_unit(v, k, cur, scr, lane);
      __builtin_amdgcn_sched_barrier(0);
      asm volatile("s_waitcnt vmcnt(1)" : "+v"(gr) :: "memory");
      tag = __builtin_amdgcn_readfirstlane((u32)(gr >> 32));
      info = __builtin_amdgcn_readfirstlane((u32)gr);
      __builtin_amdgcn_sched_barrier(0);
      store_frags(v, rf, d, lane);
      LAP(acc_d);
    }
    pend = d.cnt;
  }
  drain_vm();
  publish(pend);
  STAMP(2);
}

__device__ __forceinline__ u32 frag_row(u32 r, u32 gf) {
  const u32 g4 = r & 4u, ux = gf >> 1;
  u32 src, su;
  if (ux >= EARLY_UNITS) {
    const u32 j = (ux - EARLY_UNITS) / AU;
    src = g4 | ((r + 1 + j) & 3u);
    su = ux - EARLY_UNITS - j * AU;
  } else if (ux < AU) {
    src = r;
    su = ux;
  } else {
    const u32 j = (ux - AU) / AWAY;
    src = (g4 ^ 4u) | j;
    su = ux - j * AWAY;
  }
  return ((AG_ORDER >> (4u * src)) & 7u) * 1024u + su * 32u + (gf & 1u) * 16u;
}
__device__ __forceinline__ u32 foreign_frag0(u32 ft, u32 wid) { return ft < 4u ? (0x74208630u >> (4 * wid)) & 15u : (wid & 3u) * 2u; }
__device__ __forceinline__ u32 foreign_base(u32 ft) { return ft < 4u ? ft * 10u : ft * 8u + 8u; }

constexpr u32 O_HI = 7168;
__device__ __forceinline__ void out_puller(const Params& P, char* ws, u32 call, u32 j, u32 ftok, LAS u32* ctl, LAS unsigned char* scr, u32 lane, u32 wid) {
  const u32 r = P.rank, src = ((r & 4u) ^ 4u) | j;
  char* const ob = pick(P.ws, src) + OFF_OB + (r & 3u) * (OUNITS * OSLOT);
  const rsrc_t ro = make_rsrc(pick(P.out, r), 8192u * 1024u * 2u);
  GAS u32* const tkt = (GAS u32*)(ws + OFF_TKT + (7u + j) * 128u);
  const u32 tbase = (call - 1u) * (OUNITS + NCL * 8u) * 64u;
  const u32 vo = (lane >> 4) * 4u * 2048u + (lane & 15u) * 16u;
  char* const queue = ws + OFF_OQ + j * 2048u;
  u32 k = ((u32)__builtin_amdgcn_readfirstlane(__hip_atomic_fetch_add(tkt, 1u, __ATOMIC_RELAXED, __HIP_MEMORY_SCOPE_AGENT)) - tbase) >> 6;
  u32 tag = 0, info = 0;
  while (k < OUNITS) {
    STAMPQ(0);
    if (tag != call) {
      const u64 t0 = __builtin_amdgcn_s_memrealtime();
      for (u32 spins = 0;; ++spins) {
        const u64 gr = __hip_atomic_load((GAS u64*)(queue + k * 8u), __ATOMIC_RELAXED, __HIP_MEMORY_SCOPE_SYSTEM);
        tag = __builtin_amdgcn_readfirstlane((u32)(gr >> 32));
        info = __builtin_amdgcn_readfirstlane((u32)gr);
        if (tag == call || ((spins & 7u) == 7u && give_up(ws, call, t0))) break;
        __builtin_amdgcn_s_sleep(3);
      }
      if (tag != call) {
        *(GAS unsigned short*)(pick(P.out, r) + (size_t)((AG_ORDER >> (4u * r)) & 7u) * 1024u * 2048u) = 0x7fc0;
        break;
      }
    }
    {
      const u32 my = lds_add(ctl, 1u);
      while ((u32)(__builtin_amdgcn_readfirstlane(__hip_atomic_load(ctl + 2, __ATOMIC_RELAXED, __HIP_MEMORY_SCOPE_WORKGROUP)) >> 6) + ftok <= my) __builtin_amdgcn_s_sleep(1);
    }
    asm volatile("" ::: "memory");
    STAMPQ(1);
    const u32 uid = (info & 255u) < OUNITS ? (info & 255u) : 0u, sw = uid & 7u, mf = (uid < 32u && sw - 2u >= 4u) ? 3u : 2u;
    u32 total = (info >> 8) & 0x3fffu;
    total = total < 12288u ? total : 12288u;
    const bool raw = (info >> 22) & 1u;
    const u32 t4 = ((info >> 23) & 255u) * 0x01010101u;
    const rsrc_t rs = make_rsrc(ob + uid * OSLOT, (total + 15u) & ~15u);
    u32 nt;
    u64 keep, gr;
    asm volatile("s_mov_b64 %1, exec\n\ts_mov_b64 exec, 1\n\tglobal_atomic_add %0, %2, %3, %4 sc0\n\ts_mov_b64 exec, %1"
                 : "=&v"(nt), "=&s"(keep) : "v"(0u), "v"(64u), "s"(tkt) : "memory");
    for (u32 p = 0; p < total; p += 1024u) __builtin_amdgcn_raw_ptr_buffer_load_lds(rs, scr + p, 16, lane * 16u, p, 0, AUX_SYS);
    asm volatile("s_waitcnt vmcnt(%0)" :: "i"(AG_OEARLY) : "memory");
    lds_add(ctl + 2, 1u);
    asm volatile("s_waitcnt vmcnt(0)" : "+v"(nt) :: "memory");
    STAMPQ(2);
    const u32 kn = ((u32)__builtin_amdgcn_readfirstlane(nt) - tbase) >> 6;
    asm volatile("global_load_dwordx2 %0, %1, %2 sc0 sc1" : "=&v"(gr) : "v"((kn & 255u) * 8u), "s"(queue) : "memory");
    u32 mk[3] = {0, 0, 0}, a = 0;
    if (!raw) {
#pragma unroll
      for (u32 m = 0; m < 3; ++m) mk[m] = *(const LAS u32*)(scr + m * 256u + lane * 4u);
      mk[2] = mf == 3u ? mk[2] : 0u;
      const u32 c = ((__builtin_popcount(mk[0]) + __builtin_popcount(mk[1]) + __builtin_popcount(mk[2])) & ~3u) + 4u, incl = wave_scan(c);
      u32 n = __builtin_amdgcn_readlane(incl, 63);
      n = n < 4096u ? n : 4096u;
      LAS unsigned char* const lo = scr + mf * 256u;
      LAS unsigned char* const planes = lo + ((n + 15u) & ~15u);
      a = (u32)(size_t)lo + incl - c;
      const u32x4 nb[2] = {*(const LAS u32x4*)(planes + lane * 32u), *(const LAS u32x4*)(planes + lane * 32u + 16u)};
#pragma unroll
      for (int i = 0; i < 2; ++i) {
        *(LAS u32x4*)(lo + O_HI + lane * 64u + i * 16u) = nb[i] & 0x0f0f0f0fu;
        *(LAS u32x4*)(lo + O_HI + lane * 64u + 32u + i * 16u) = (nb[i] >> 4) & 0x0f0f0f0fu;
      }
      asm volatile("" ::: "memory");
    }
    asm volatile("s_waitcnt vmcnt(0)" : "+v"(gr) :: "memory");
    tag = __builtin_amdgcn_readfirstlane((u32)(gr >> 32));
    info = __builtin_amdgcn_readfirstlane((u32)gr);
    const u32 col = j * 512u + (sw >> 2) * 256u, f0 = foreign_base(uid >> 3) + foreign_frag0(uid >> 3, sw);
    for (u32 m = 0; m < mf; ++m) {
      const u32 w = m == 0 ? mk[0] : (m == 1 ? mk[1] : mk[2]);
      u32x4 o[4];
      if (raw) {
#pragma unroll
        for (int e = 0; e < 4; ++e) o[e] = *(const LAS u32x4*)(scr + (m * 4u + e) * 1024u + lane * 16u);
      } else {
        u32 at[8], sel[8], l0[8], l1[8], h0[8], h1[8];
#pragma unroll
        for (int s = 0; s < 8; ++s) {
          const u32 f = (w >> s) & 0x01010101u;
          const LAS u32* const pw = (const LAS u32*)(size_t)(a & ~3u);
          at[s] = a;
          sel[s] = *(const LAS u32*)(size_t)__builtin_amdgcn_udot4(f, 0x20100804u, TAB + 128u, false);
          l0[s] = pw[0];
          l1[s] = pw[1];
          h0[s] = pw[O_HI / 4];
          h1[s] = pw[O_HI / 4 + 1];
          a = __builtin_amdgcn_sad_u8(f, 0u, a);
        }
        __builtin_amdgcn_sched_barrier(0);
#pragma unroll
        for (int s = 0; s < 8; ++s) {
          const u32 lb = __builtin_amdgcn_perm(0u, __builtin_amdgcn_alignbyte(l1[s], l0[s], at[s]), sel[s]);
          const u32 hb = __builtin_amdgcn_perm(0u, t4 - (__builtin_amdgcn_alignbyte(h1[s], h0[s], at[s]) & 0x0f0f0f0fu), sel[s]);
          o[s >> 1][(s & 1) * 2] = __builtin_amdgcn_perm(hb, lb, 0x05010400u);
          o[s >> 1][(s & 1) * 2 + 1] = __builtin_amdgcn_perm(hb, lb, 0x07030602u);
        }
      }
      const u32 row0 = frag_row(src, f0 + m);
#pragma unroll
      for (int e = 0; e < 4; ++e) store16<AG_OAUX>(o[e], ro, vo, (row0 + e) * 2048u + col);
    }
    STAMPQ(3);
    k = kn;
  }
  STAMP(3);
}

__device__ __forceinline__ void gate(int c, int& ready, const char* cnt, u32 target, char* ws, u32 call) {
  if (c < ready) return;
  const u64 t0 = __builtin_amdgcn_s_memrealtime();
  for (u32 spins = 0;; ++spins) {
    const u64 m = __builtin_amdgcn_ballot_w64((int)(ld_agent(cnt) - target) >= 0);
    ready = ~m ? __builtin_ctzll(~m) : 64;
    if (c < ready) break;
    if ((spins & 7u) == 7u && give_up(ws, call, t0)) { ready = 64; break; }
    __builtin_amdgcn_s_sleep(3);
  }
  asm volatile("" ::: "memory");
}

__device__ __forceinline__ u32 relu_pk(float lo, float hi) {
  u32 r;
  asm("v_cvt_pk_bf16_f32 %0, %1, %2\n\tv_pk_max_i16 %0, %0, 0" : "=v"(r) : "v"(lo), "v"(hi));
  return r;
}

__device__ __forceinline__ void mfma_acc(f32x4& c, const bf16x8& a, const bf16x8& b) {
  asm volatile("v_mfma_f32_16x16x32_bf16 %0, %1, %2, %0" : "+v"(c) : "v"(a), "v"(b));
}
template <int N> __device__ __forceinline__ void wait_bar() { asm volatile("s_waitcnt vmcnt(%0)\n\ts_barrier" :: "i"(N) : "memory"); }

struct Tile { u32 rt, ct, gf0, nfr; };
template <int MF, int NP, int KIND>
__device__ __forceinline__ void gemm_wave(const Params& P, char* ws, u32 call, const Tile& T, u32 f0, u32 b, LAS char* lds, u32 lane, u32 wid) {
  constexpr bool XL = NP && NP < 8;
  constexpr u32 STEP = XL ? A_STEP : B_STEP;
  constexpr int NM = MF * 8;
  const u32 r = P.rank, g4 = r & 4u, rt = T.rt, ct = T.ct, wc = wid >> 2;
  const bool own = KIND == 1 || (KIND == 3 && ct < 4u);
  const u32 p0 = (wid & 1u) * NP;
  const rsrc_t rs = XL ? make_rsrc(ws + OFF_XU, OFF_WU - OFF_XU) : make_rsrc(ws + OFF_WU + ct * B_TILE, B_TILE);
  const char* const cnt = ws + OFF_CNT + ((XL ? (own ? ROW_OWN + rt : rt) : ROW_W + ct) * 64 + lane) * 4;
  u32 target = call * (XL ? T.nfr >> 1 : 8u);
  u32 vp[NP ? NP : 1];
#pragma unroll
  for (u32 p = 0; p < NP; ++p) vp[p] = lane * 16u + (XL ? T.gf0 + (p0 + p < T.nfr ? p0 : T.nfr - 1u - p) : p0) * 1024u;
  LAS char* const ring = lds + (XL ? 0u : A_LDS) + p0 * 1024u;
  const u32 img = ((lane & 15u) * 4u + ((lane >> 4) ^ quad_swz((lane >> 2) & 3u))) * 16u;
  LAS char* const ra = lds + f0 * 1024u + img;
  LAS char* const rb = lds + A_LDS + wc * 8192u + lane * 16u;
#define AG_DMA(p, ks, off, imm) __builtin_amdgcn_raw_ptr_buffer_load_lds(rs, ring + (off), 16, vp[p < NP ? p : 0], (ks), imm, AG_GEMM_AUX)
#define AG_PIECE(p, ks, off) do { \
    if (p == 0) AG_DMA(0, (ks) * STEP, off, 0); if (p == 1) AG_DMA(1, (ks) * STEP, off, 1024); \
    if (p == 2) AG_DMA(2, (ks) * STEP, off, 2048); if (p == 3) AG_DMA(3, (ks) * STEP, off, 3072); \
    if (p == 4) AG_DMA(4, (ks) * STEP + 4096, (off) + 4096, 0); if (p == 5) AG_DMA(5, (ks) * STEP + 4096, (off) + 4096, 1024); \
    if (p == 6) AG_DMA(6, (ks) * STEP + 4096, (off) + 4096, 2048); if (p == 7) AG_DMA(7, (ks) * STEP + 4096, (off) + 4096, 3072); \
  } while (0)
#define AG_ALL(ks, off) do { _Pragma("unroll") for (int p = 0; p < NP; ++p) AG_PIECE(p, ks, off); } while (0)
#define AG_PIN(stmt) do { __builtin_amdgcn_sched_barrier(0); stmt; __builtin_amdgcn_sched_barrier(0); } while (0)
#define AG_STEP(cs, ks, DMA, RD) do { \
    LAS char* const pa = ra + ord; \
    LAS char* const pb = rb + ord; \
    __builtin_amdgcn_sched_barrier(0); \
    _Pragma("unroll") for (int i = 0; i < NM; ++i) { \
      mfma_acc(acc[i % MF][i / MF], fa[cs][i % MF], fb[cs][i / MF]); \
      _Pragma("unroll") for (int j = 0; j < MF + 8; ++j) \
        if (RD && i == (NP ? (MF == 3 ? 2 * j : (3 * j) / 2) : j)) { \
          if (j < MF) AG_PIN(fa[(cs) ^ 1][j] = *(const LAS bf16x8*)(pa + j * 1024)); \
          else AG_PIN(fb[(cs) ^ 1][j - MF] = *(const LAS bf16x8*)(pb + (j - MF) * 1024)); \
        } \
      _Pragma("unroll") for (int p = 0; p < NP; ++p) \
        if (DMA && i == 1 + p * ((NM + 1) / (NP ? NP : 1))) AG_PIN(AG_PIECE(p, ks, owr)); \
    } \
    ord = ord + SLOTB == NSLOT * SLOTB ? 0 : ord + SLOTB; \
    owr = owr + SLOTB == NSLOT * SLOTB ? 0 : owr + SLOTB; \
    __builtin_amdgcn_sched_barrier(0); \
  } while (0)
#define AG_GATE(c) do { if (NP) gate(c, ready, cnt, target, ws, call); } while (0)
  f32x4 acc[MF][8];
#pragma unroll
  for (int m = 0; m < MF; ++m)
#pragma unroll
    for (int n = 0; n < 8; ++n) { acc[m][n] = (f32x4){0.f, 0.f, 0.f, 0.f}; asm volatile("" : "+v"(acc[m][n])); }
  if (NP) __builtin_amdgcn_s_setprio(1);
  int ready = 0;
  AG_GATE(0);
  STAMP(1);
  STAMPC(4);
  constexpr int LA = 4, W = (LA - 2) * NP;
#pragma unroll
  for (int s = 0; s < 4; ++s) AG_ALL(s, s * SLOTB);
  if (LA == 5) { AG_GATE(1); AG_ALL(4, 4 * SLOTB); }
  wait_bar<W>();
  bf16x8 fa[2][MF], fb[2][8];
#pragma unroll
  for (int m = 0; m < MF; ++m) fa[0][m] = *(const LAS bf16x8*)(ra + m * 1024);
#pragma unroll
  for (int n = 0; n < 8; ++n) fb[0][n] = *(const LAS bf16x8*)(rb + n * 1024);
  u32 ord = SLOTB, owr = LA * SLOTB;
  for (int kc = 0; kc < NKC - 3; ++kc) {
    if (LA == 4) AG_GATE(kc + 1);
    AG_STEP(0, kc * 4 + LA, true, true); wait_bar<W>();
    AG_STEP(1, kc * 4 + LA + 1, true, true); wait_bar<W>();
    AG_STEP(0, kc * 4 + LA + 2, true, true); wait_bar<W>();
    if (LA == 5) AG_GATE(kc + 2);
    AG_STEP(1, kc * 4 + LA + 3, true, true); wait_bar<W>();
  }
  if (LA == 4) AG_GATE(NKC - 2);
  AG_STEP(0, NKS - 12 + LA, true, true); wait_bar<W>();
  AG_STEP(1, NKS - 11 + LA, true, true); wait_bar<W>();
  AG_STEP(0, NKS - 10 + LA, true, true); wait_bar<W>();
  AG_STEP(1, NKS - 9 + LA, LA == 4, true); wait_bar<2 * NP>();
  AG_STEP(0, 0, false, true); wait_bar<NP>();
  AG_STEP(1, 0, false, true); wait_bar<0>();
  AG_STEP(0, 0, false, true);
  AG_STEP(1, 0, false, false);
  AG_GATE(NKC - 1);
  asm volatile("s_barrier" ::: "memory");
#pragma unroll
  for (int s = 0; s < 4; ++s) AG_ALL(NKS - 4 + s, s * SLOTB);
  u32 tmo;
  asm volatile("global_load_dword %0, %1, %2 sc1" : "=&v"(tmo) : "v"(0u), "s"(ws + OFF_TMO) : "memory");
  wait_bar<2 * NP + 1>();
#pragma unroll
  for (int m = 0; m < MF; ++m) fa[0][m] = *(const LAS bf16x8*)(ra + m * 1024);
#pragma unroll
  for (int n = 0; n < 8; ++n) fb[0][n] = *(const LAS bf16x8*)(rb + n * 1024);
  ord = SLOTB;
  AG_STEP(0, 0, false, true); wait_bar<NP + 1>();
  AG_STEP(1, 0, false, true); wait_bar<0>();
  AG_STEP(0, 0, false, true);
  AG_STEP(1, 0, false, false);
  asm volatile("" : "+v"(tmo));
  asm volatile("s_nop 15" ::: "memory");
  __builtin_amdgcn_sched_barrier(0);
#undef AG_PIECE
#undef AG_ALL
#undef AG_PIN
#undef AG_DMA
#undef AG_GATE
#undef AG_STEP
  STAMP(2);
  STAMPC(5);
  const bool bad = __builtin_amdgcn_readfirstlane(tmo) == call;
  u32x4 o[MF][4];
#pragma unroll
  for (int m = 0; m < MF; ++m)
#pragma unroll
    for (int e = 0; e < 4; ++e)
#pragma unroll
      for (int n = 0; n < 4; ++n) o[m][e][n] = relu_pk(acc[m][2 * n][e], acc[m][2 * n + 1][e]);
  if (bad) {
#pragma unroll
    for (int m = 0; m < MF; ++m)
#pragma unroll
      for (int e = 0; e < 4; ++e) o[m][e] = (u32x4){0x7fc07fc0u, 0x7fc07fc0u, 0x7fc07fc0u, 0x7fc07fc0u};
    asm volatile("" ::: "memory");
  }
  if (own) {
    const rsrc_t ro = make_rsrc(pick(P.out, r), 8192u * 1024u * 2u);
    const u32 vo = (lane >> 4) * 4u * 2048u + (lane & 15u) * 16u;
#pragma unroll
    for (int m = 0; m < MF; ++m) {
      if (f0 + m >= T.nfr) continue;
      const u32 row0 = frag_row(r, T.gf0 + f0 + m);
#pragma unroll
      for (int e = 0; e < 4; ++e) store16<AG_OAUX>(o[m][e], ro, vo, (row0 + e) * 2048u + ct * 512u + wc * 256u);
    }
    STAMP(3);
    return;
  }
  const u32 j = ct - 4u, uid = rt * 8u + wid;
  u32 qi;
  u64 keep;
  asm volatile("s_mov_b64 %1, exec\n\ts_mov_b64 exec, 1\n\tglobal_atomic_add %0, %2, %3, %4 sc0\n\ts_mov_b64 exec, %1"
               : "=&v"(qi), "=&s"(keep) : "v"(0u), "v"(1u), "s"(ws + OFF_TKT + (11u + j) * 128u) : "memory");
  u32 mk[MF], fl[MF * 8], lb[MF * 8], hb[MF * 8], c = 0;
  u16x2 big[2] = {{0, 0}, {0, 0}};
#pragma unroll
  for (int m = 0; m < MF; ++m) {
    mk[m] = 0;
#pragma unroll
    for (int s = 0; s < 8; ++s) {
      const u32 d0 = o[m][s >> 1][(s & 1) * 2], d1 = o[m][s >> 1][(s & 1) * 2 + 1];
      u32 t0, t1;
      asm("v_pk_min_u16 %0, %1, %2" : "=v"(t0) : "v"(d0), "v"(0x00010001u));
      asm("v_pk_min_u16 %0, %1, %2" : "=v"(t1) : "v"(d1), "v"(0x00010001u));
      fl[m * 8 + s] = __builtin_amdgcn_perm(t1, t0, 0x06040200u);
      mk[m] |= fl[m * 8 + s] << s;
      big[0] = __builtin_elementwise_max(big[0], __builtin_bit_cast(u16x2, d0));
      big[1] = __builtin_elementwise_max(big[1], __builtin_bit_cast(u16x2, d1));
      lb[m * 8 + s] = __builtin_amdgcn_perm(d1, d0, 0x06040200u);
      hb[m * 8 + s] = __builtin_amdgcn_perm(d1, d0, 0x07050301u);
    }
    c += __builtin_popcount(mk[m]);
  }
  c = (c & ~3u) + 4u;
  const u32 incl = wave_scan(c), n = __builtin_amdgcn_readlane(incl, 63);
  big[0] = __builtin_elementwise_max(big[0], big[1]);
  u32 top = wave_max((u32)(big[0].x > big[0].y ? big[0].x : big[0].y) >> 8);
  top = top < 15u ? 15u : top;
  bool raw = n > 4096u;
  u32 total = 0;
  const rsrc_t rb2 = make_rsrc(ws + OFF_OB + j * (OUNITS * OSLOT) + uid * OSLOT, OSLOT);
  STAMP2(0);
  asm volatile("s_barrier" ::: "memory");
  STAMP2(1);
  LAS unsigned char* const scr = (LAS unsigned char*)lds + wid * WAVE_SCRATCH;
  if (!raw) {
    LAS unsigned char* const lo = scr + MF * 256u;
  #pragma unroll
    for (int m = 0; m < MF; ++m) *(LAS u32*)(scr + m * 256u + lane * 4u) = mk[m];
    asm volatile("" ::: "memory");
    const u32 t4 = top * 0x01010101u;
    u32 wide = 0;
    u32 a = (u32)(size_t)lo + incl - c, cl = 0, ch = 0;
#pragma unroll
    for (int m = 0; m < MF; ++m) {
      u32 sel[8];
#pragma unroll
      for (int s = 0; s < 8; ++s) sel[s] = *(const LAS u32*)(size_t)__builtin_amdgcn_udot4(fl[m * 8 + s], 0x20100804u, TAB, false);
      __builtin_amdgcn_sched_barrier(0);
#pragma unroll
      for (int s = 0; s < 8; ++s) {
        const u32 sh = (a & 3u) * 8u, an = __builtin_amdgcn_sad_u8(fl[m * 8 + s], 0u, a);
        const u32 code = __builtin_amdgcn_perm(0u, t4 - hb[m * 8 + s], sel[s]);
        const u64 tl = (u64)__builtin_amdgcn_perm(0u, lb[m * 8 + s], sel[s]) << sh, th = (u64)code << sh;
        wide |= code;
        const u32 wl = cl | (u32)tl, wh = ch | (u32)th;
        LAS u32* const pw = (LAS u32*)(size_t)(a & ~3u);
        pw[0] = wl;
        pw[O_HI / 4] = wh;
        const bool cross = (an ^ a) & 4u;
        cl = cross ? (u32)(tl >> 32) : wl;
        ch = cross ? (u32)(th >> 32) : wh;
        a = an;
      }
    }
    ((LAS u32*)(size_t)(a & ~3u))[0] = cl;
    ((LAS u32*)(size_t)(a & ~3u))[O_HI / 4] = ch;
    asm volatile("" ::: "memory");
    LAS unsigned char* const planes = lo + ((n + 15u) & ~15u);
#pragma unroll
    for (int i = 0; i < 2; ++i)
      *(LAS u32x4*)(planes + lane * 32u + i * 16u) = (*(const LAS u32x4*)(lo + O_HI + lane * 64u + i * 16u) & 0x0f0f0f0fu) |
                                                     ((*(const LAS u32x4*)(lo + O_HI + lane * 64u + 32u + i * 16u) << 4) & 0xf0f0f0f0u);
    raw = __builtin_amdgcn_ballot_w64((wide & 0xf0f0f0f0u) != 0) != 0;
    total = MF * 256u + ((n + 15u) & ~15u) + ((n + 63u) >> 6) * 32u;
    asm volatile("" ::: "memory");
  }
  STAMP2(2);
  if (raw) {
    total = 4096u * MF;
#pragma unroll
    for (int m = 0; m < MF; ++m)
#pragma unroll
      for (int i = 0; i < 4; ++i)
        store16((u32x4){__builtin_amdgcn_perm(hb[m * 8 + 2 * i], lb[m * 8 + 2 * i], 0x05010400u), __builtin_amdgcn_perm(hb[m * 8 + 2 * i], lb[m * 8 + 2 * i], 0x07030602u),
                        __builtin_amdgcn_perm(hb[m * 8 + 2 * i + 1], lb[m * 8 + 2 * i + 1], 0x05010400u), __builtin_amdgcn_perm(hb[m * 8 + 2 * i + 1], lb[m * 8 + 2 * i + 1], 0x07030602u)},
                rb2, lane * 16u, (m * 4 + i) * 1024u);
  } else {
    for (u32 p = 0; p < total; p += 1024u) store16(*(const LAS u32x4*)(scr + p + lane * 16u), rb2, p + lane * 16u < total ? lane * 16u : 0x80000000u, p);
  }
  STAMP2(3);
  asm volatile("s_waitcnt vmcnt(0)" : "+v"(qi) :: "memory");
  const u32 slot = ((u32)__builtin_amdgcn_readfirstlane(qi) - (call - 1u) * OUNITS) & 255u;
  flag_store(pick(P.ws, (g4 ^ 4u) | j) + OFF_OQ + (r & 3u) * 2048u + slot * 8u, ((u64)call << 32) | uid | (total << 8) | (raw ? 1u << 22 : 0u) | (top << 23), 1u);
  STAMP(3);
}

__global__ __launch_bounds__(512, 2) void ag_gemm_kernel(Params P) {
  extern __shared__ __attribute__((aligned(16))) char smem[];
  LAS char* lds = (LAS char*)smem;
  const u32 lane = threadIdx.x & 63u, wid = __builtin_amdgcn_readfirstlane(threadIdx.x >> 6);
  char* const ws = pick(P.ws, P.rank);
  STAMP(0);
  u32 bid = blockIdx.x;
  asm volatile("" : "+s"(bid));
  char* const cptr = ws + OFF_CALL + bid * 128u + wid * 4u;
  const u32 callv = ld_agent(cptr);
  LAS unsigned char* scr = (LAS unsigned char*)lds + wid * WAVE_SCRATCH;
  build_tables(lds, lane);
  bool skip = false;
  const u32 li = bid < 7u * NCL ? bid % 7u : 7u, lidx = bid / 7u, pkw = bid - 7u * NCL;
  const u32 xcd = bid & 7u, pos = xcd * 26u + (xcd > 4u ? xcd - 4u : 0u) + (bid >> 3) - (xcd < 4u ? 6u : 5u);
  const bool foreign = pos >= 4u * NOT;
  const u32 ent = foreign ? pos - 4u * NOT : pos;
  Tile T;
  T.rt = ent >> 2;
  T.ct = (ent & 3u) + (foreign ? 4u : 0u);
  T.gf0 = foreign ? foreign_base(T.rt) : T.rt * 12u;
  T.nfr = foreign ? (T.rt < 4u ? 10u : 8u) : (T.rt == NOT - 1u ? 4u : 12u);
  LAS u32* const ctl = (LAS u32*)(lds + 8 * WAVE_SCRATCH);
  if (bid < NCOMM) {
    ctl[lane & 3u] = 0;
    asm volatile("s_waitcnt lgkmcnt(0)\n\ts_barrier" ::: "memory");
  }
  const bool isg = bid >= NCOMM;
  u32 call;
  if (li < 7u && wid < PW) {
    call = __builtin_amdgcn_readfirstlane(callv) + 1u;
    if (!skip) puller(P, ws, call, li, lidx, ctl, scr, lane, wid);
  } else {
    const u32 q0 = isg ? wid * NGEMM + (bid - NCOMM) : GPACK + wid * NPKWG + pkw;
    const u32 q1 = isg ? (T.nfr == 8u ? 8u * NGEMM + wid * 108u + ent - 16u : GPACK) : q0 + NPK;
    for (u32 i = isg ? wid : (u32)AG_PDELAY; i; --i) __builtin_amdgcn_s_sleep(AG_GDELAY);
    call = packer(P, ws, callv, skip ? (u32)NUNITS : q0, q1, isg ? 864u : (u32)NPK, isg ? GPACK : (u32)NUNITS, scr, lane, wid);
  }
  __hip_atomic_store((GAS u32*)cptr, call, __ATOMIC_RELAXED, __HIP_MEMORY_SCOPE_AGENT);
  if (!isg) {
    if (li - 3u < 4u) out_puller(P, ws, call, li - 3u, AG_FTOK, ctl, scr, lane, wid);
    return;
  }
  asm volatile("s_waitcnt lgkmcnt(0)\n\ts_barrier" ::: "memory");
  STAMP(7);
  const u32 b = bid - NCOMM;
  if (T.nfr == 8u) {
    if (wid < 2u) gemm_wave<2, 4, 2>(P, ws, call, T, (wid & 3u) * 2u, b, lds, lane, wid);
    else if (wid < 4u) gemm_wave<2, 8, 2>(P, ws, call, T, (wid & 3u) * 2u, b, lds, lane, wid);
  } else if (T.nfr == 10u) {
    if (wid < 2u) gemm_wave<3, 5, 2>(P, ws, call, T, foreign_frag0(0u, wid), b, lds, lane, wid);
  } else {
    if (wid < 2u) gemm_wave<3, 6, 1>(P, ws, call, T, (wid & 3u) * 3u, b, lds, lane, wid);
    else if (wid >= 4u) gemm_wave<3, 0, 1>(P, ws, call, T, (wid & 3u) * 3u, b, lds, lane, wid);
  }
  if (T.nfr == 10u ? wid >= 6u : (T.ct < 4u && wid - 2u < 2u)) gemm_wave<3, 8, 3>(P, ws, call, T, T.ct < 4u ? (wid & 3u) * 3u : foreign_frag0(0u, wid), b, lds, lane, wid);
  else if (T.nfr == 10u ? wid - 2u < 4u : (T.nfr == 8u && wid >= 4u)) gemm_wave<2, 0, 2>(P, ws, call, T, foreign_frag0(T.rt, wid), b, lds, lane, wid);
}

}

extern "C" void kernel_launch(int rank, int nranks, ncclComm_t comm, hipStream_t stream,
                              void* const* d_in, const int* in_sizes, int n_in,
                              void* d_out, int out_size, const int* in_dtypes, int out_dtype,
                              void* const* d_out_all, void* const* d_ws_all, size_t ws_bytes) {
  (void)comm; (void)in_dtypes; (void)out_dtype; (void)d_out;
  if (nranks != ag::NR || n_in != 2 || (unsigned)rank >= (unsigned)ag::NR || in_sizes[0] != 1024 * 8192 || in_sizes[1] != 8192 * 1024 ||
      out_size != 8192 * 1024 || ws_bytes < ag::WS_NEED) {
    fprintf(stderr, "ag_gemm: unexpected problem shape\n");
    return;
  }
  ag::Params p;
  p.x = (const char*)d_in[0];
  p.w = (const char*)d_in[1];
  for (int i = 0; i < ag::NR; ++i) {
    const int phys = (AG_ORDER >> (4 * i)) & 7;
    p.ws[i] = (char*)d_ws_all[phys];
    p.out[i] = (char*)d_out_all[phys];
    if (phys == rank) p.rank = i;
  }
  p.pad = 0;
  hipError_t e = hipFuncSetAttribute((const void*)ag::ag_gemm_kernel, hipFuncAttributeMaxDynamicSharedMemorySize, ag::LDS_BYTES);
  if (e != hipSuccess) { fprintf(stderr, "ag_gemm: %s\n", hipGetErrorString(e)); return; }
  hipLaunchKernelGGL(ag::ag_gemm_kernel, dim3(ag::NWG), dim3(512), ag::LDS_BYTES, stream, p);
  e = hipGetLastError();
  if (e != hipSuccess) fprintf(stderr, "ag_gemm launch: %s\n", hipGetErrorString(e));
}
```

```cpp
#include <hip/hip_runtime.h>
#include <rccl/rccl.h>
#include <cstdio>

namespace ag {

typedef unsigned u32;
typedef unsigned long long u64;
typedef unsigned u32x4 __attribute__((ext_vector_type(4)));
typedef unsigned short u16x2 __attribute__((ext_vector_type(2)));
typedef short bf16x8 __attribute__((ext_vector_type(8)));
typedef float f32x4 __attribute__((ext_vector_type(4)));
typedef __amdgpu_buffer_rsrc_t rsrc_t;
#define LAS __attribute__((address_space(3)))
#define GAS __attribute__((address_space(1)))

constexpr int NR = 8;
constexpr u32 ORDER = 0x75416320u;
constexpr int AU = 21;
constexpr int AWAY = 32 - AU;
constexpr int NKC = 64, NKS = 256;
constexpr int EARLY_UNITS = AU + 4 * AWAY;
constexpr u32 NFT = 31, NOT = 22, ROW_OWN = NFT, ROW_W = NFT + NOT;
constexpr int NOUT = AWAY + 8;

constexpr int NCOMM = 44, NGEMM = 4 * (NFT + NOT), NWG = NCOMM + NGEMM;
constexpr int NCL = 6;
constexpr int NPKWG = NCOMM - 7 * NCL;
static_assert(NWG == 256 && NPKWG > 0, "one workgroup per CU");
constexpr int NPK = NPKWG * 8;
constexpr u32 GPACK = 8 * NGEMM + 2240;
constexpr int NUNITS = 4096;
constexpr u32 LEAD = 6, RAMP = 2;
constexpr u32 HALVES = 4, OUT_TOK = 2;
constexpr u32 HELP = 4;

constexpr u32 A_STEP = 262144, B_STEP = 16384;
constexpr u32 B_TILE = B_STEP * NKS;
constexpr u32 OFF_XU = 0;
constexpr u32 OFF_WU = OFF_XU + NKS * A_STEP;
constexpr u32 FRAG_BYTES = OFF_WU + 8 * B_TILE;
constexpr u32 SLOT = 8192, PK_BYTES = NUNITS * SLOT;
constexpr u32 OFF_PK = FRAG_BYTES;
constexpr u32 OSLOT = 12288, OUNITS = NFT * 8;
constexpr u32 OFF_OB = OFF_PK + 2 * PK_BYTES;
constexpr u32 IB_SRC = 64 * SLOT, IB_BYTES = 2 * NR * IB_SRC;
constexpr u32 OFF_IB = OFF_OB + 4 * OUNITS * OSLOT;
constexpr u32 OFF_G = OFF_IB + IB_BYTES;
constexpr u32 G_BYTES = 2 * NR * NUNITS * 8;
constexpr u32 OFF_CNT = OFF_G + G_BYTES;
constexpr u32 OFF_OQ = OFF_CNT + 16384;
constexpr u32 OFF_TKT = OFF_OQ + 4 * 2048;
constexpr u32 OFF_TMO = OFF_TKT + 2048;
constexpr u32 OFF_CALL = OFF_TMO + 128;
constexpr u32 WS_NEED = OFF_CALL + NWG * 128;

constexpr u32 A_LDS = 12288, SLOTB = A_LDS + B_STEP, NSLOT = 5;
constexpr u32 TAB_OFF = 163840 - 256;
static_assert(NSLOT * SLOTB <= TAB_OFF && 4 * SLOTB + 4 * 12160 <= TAB_OFF && 4 * 12288 <= 2 * SLOTB, "LDS map");
constexpr int LDS_BYTES = TAB_OFF + 256;
constexpr int WAVE_SCRATCH = 13312;
static_assert(8 * WAVE_SCRATCH + 64 <= LDS_BYTES, "scratch");

constexpr u32 SPARE = 4000, SPARE_CNT = SPARE | SPARE << 16;
constexpr u64 DEADLINE = 50000000ull;
constexpr int AUX_SYS = 17, AUX_NT = 2;

struct Params {
  const char* x;
  const char* w;
  char* ws[NR];
  char* out[NR];
  int rank;
  int pad;
};

__device__ __forceinline__ rsrc_t make_rsrc(const void* p, u32 bytes) {
  const u64 a = (u64)p;
  const u32 lo = __builtin_amdgcn_readfirstlane((u32)a), hi = __builtin_amdgcn_readfirstlane((u32)(a >> 32));
  return __builtin_amdgcn_make_buffer_rsrc((void*)(((u64)hi << 32) | lo), 0, bytes, 0x00020000);
}
__device__ __forceinline__ char* pick(char* const (&t)[NR], u32 i) {
  char* p = t[0];
#pragma unroll
  for (u32 j = 1; j < NR; ++j) p = (i == j) ? t[j] : p;
  return p;
}
__device__ __forceinline__ u32 ld_agent(const char* p) { return __hip_atomic_load((GAS u32*)p, __ATOMIC_RELAXED, __HIP_MEMORY_SCOPE_AGENT); }
__device__ __forceinline__ void store16(u32x4 v, rsrc_t rs, u32 voff, u32 soff) {
  __builtin_amdgcn_raw_buffer_store_b128(v, rs, voff, soff, AUX_SYS);
  asm volatile("s_nop 1" :: "v"(v) : "memory");
}
__device__ __forceinline__ u32 pair_offset(u32 c, u32 lane) { return ((lane & 1u) ? c >> 16 : c & 0xffffu) * 4u; }
__device__ __forceinline__ u64 pair_mask(u32 c) { return (c >> 16) == SPARE ? 1u : 3u; }
__device__ __forceinline__ void count_in(const char* base, u32 c, u32 lane) {
  u64 keep;
  asm volatile("s_mov_b64 %0, exec\n\ts_mov_b64 exec, %4\n\tglobal_atomic_add %1, %2, %3\n\ts_mov_b64 exec, %0"
               : "=&s"(keep) : "v"(pair_offset(c, lane)), "v"(1u), "s"(base), "s"(pair_mask(c)) : "memory");
}
__device__ __forceinline__ void flag_store(char* p, u64 v, u64 mask) {
  u64 keep;
  asm volatile("s_mov_b64 %0, exec\n\ts_mov_b64 exec, %3\n\tglobal_store_dwordx2 %1, %2, off sc0 sc1\n\ts_mov_b64 exec, %0"
               : "=&s"(keep) : "v"(p), "v"(v), "s"(mask) : "memory");
}
__device__ __forceinline__ void drain_vm() { asm volatile("s_waitcnt vmcnt(0)" ::: "memory"); }

__device__ __forceinline__ bool give_up(char* ws, u32 call, u64 t0) {
  const u32 t = __builtin_amdgcn_readfirstlane(ld_agent(ws + OFF_TMO));
  if (t == call) return true;
  if (__builtin_amdgcn_s_memrealtime() - t0 > DEADLINE) {
    __hip_atomic_store((GAS u32*)(ws + OFF_TMO), call, __ATOMIC_RELAXED, __HIP_MEMORY_SCOPE_AGENT);
    return true;
  }
  return false;
}

__device__ __forceinline__ u32 wave_sum(u32 v) {
  v += __builtin_amdgcn_update_dpp(0u, v, 0x111, 0xf, 0xf, true);
  v += __builtin_amdgcn_update_dpp(0u, v, 0x112, 0xf, 0xf, true);
  v += __builtin_amdgcn_update_dpp(0u, v, 0x114, 0xf, 0xf, true);
  v += __builtin_amdgcn_update_dpp(0u, v, 0x118, 0xf, 0xf, true);
  return __builtin_amdgcn_readlane(v, 15) + __builtin_amdgcn_readlane(v, 31) + __builtin_amdgcn_readlane(v, 47) + __builtin_amdgcn_readlane(v, 63);
}
__device__ __forceinline__ u32 wave_max(u32 v) {
  v = max(v, (u32)__builtin_amdgcn_update_dpp(0u, v, 0x111, 0xf, 0xf, true));
  v = max(v, (u32)__builtin_amdgcn_update_dpp(0u, v, 0x112, 0xf, 0xf, true));
  v = max(v, (u32)__builtin_amdgcn_update_dpp(0u, v, 0x114, 0xf, 0xf, true));
  v = max(v, (u32)__builtin_amdgcn_update_dpp(0u, v, 0x118, 0xf, 0xf, true));
  return max(max((u32)__builtin_amdgcn_readlane(v, 15), (u32)__builtin_amdgcn_readlane(v, 31)), max((u32)__builtin_amdgcn_readlane(v, 47), (u32)__builtin_amdgcn_readlane(v, 63)));
}
__device__ __forceinline__ u32 wave_scan(u32 v) {
  v += __builtin_amdgcn_update_dpp(0u, v, 0x111, 0xf, 0xf, true);
  v += __builtin_amdgcn_update_dpp(0u, v, 0x112, 0xf, 0xf, true);
  v += __builtin_amdgcn_update_dpp(0u, v, 0x114, 0xf, 0xf, true);
  v += __builtin_amdgcn_update_dpp(0u, v, 0x118, 0xf, 0xf, true);
  v += __builtin_amdgcn_update_dpp(0u, v, 0x142, 0xa, 0xf, false);
  v += __builtin_amdgcn_update_dpp(0u, v, 0x143, 0xc, 0xf, false);
  return v;
}

__device__ __forceinline__ u32 quad_swz(u32 rq) { return (0x78u >> (2 * rq)) & 3u; }

struct Dest { u32 base, s1, s0, cnt; };
__device__ __forceinline__ Dest dest_x(u32 ux, u32 kc) {
  const u32 ft = ux < 20u ? ux / 5u : (ux - 4u) >> 2, ot = ROW_OWN + ux / 6u;
  return Dest{OFF_XU + 4 * kc * A_STEP + ux * 2048u, 1024u, A_STEP, (ft * 64 + kc) | (ot * 64 + kc) << 16};
}
__device__ __forceinline__ Dest dest_w(u32 ct, u32 kc, u32 ks4, u32 half) {
  return Dest{OFF_WU + ct * B_TILE + (4 * kc + ks4) * B_STEP + half * 8192u, 4096u, 1024u, ((ROW_W + ct) * 64 + kc) | SPARE << 16};
}
__device__ __forceinline__ void store_frags(const u32x4 (&v)[8], rsrc_t rf, const Dest& d, u32 lane) {
#pragma unroll
  for (int f = 0; f < 8; ++f) store16(v[f], rf, lane * 16, d.base + (f >> 2) * d.s1 + (f & 3) * d.s0);
}

constexpr u32 RAW = 1u << 31, PUSHED = 1u << 30;
constexpr u32 S_LIST = 0, S_TAIL = 2304, S_RARE = 3584;
struct Packed { u32x4 mq[4], p01, tail; };

__device__ __forceinline__ void build_tables(LAS char* lds, u32 lane) {
  u32 ct = 0x0c0c0c0cu, xt = 0x07060504u, zt = 0x0c0c0c0cu, k = 0;
#pragma unroll
  for (u32 b = 0; b < 4; ++b) {
    const u32 on = 0u - ((lane >> b) & 1u);
    ct ^= on & ((0x0cu ^ b) << (8u * k));
    xt ^= on & (((4u + b) ^ k) << (8u * b));
    zt ^= on & ((0x0cu ^ k) << (8u * b));
    k -= on;
  }
  *(LAS u32*)(lds + TAB_OFF + lane * 4u) = (lane & 32u) ? zt : ((lane & 16u) ? xt : ct);
}

constexpr u32 TAB = TAB_OFF;

template <bool TO_LIST>
__device__ __forceinline__ void walk_spelled(u32 m, u32 off, LAS unsigned char* cells, LAS unsigned char* list, u32 flip) {
  while (m != 0) {
    const u32 p = __builtin_ctz(m);
    m &= m - 1;
    LAS unsigned char* cell = cells + (p & 7u) * 4u + (p >> 3);
    if (TO_LIST) list[off] = *cell ^ flip; else *cell = list[off];
    ++off;
  }
}

__device__ __forceinline__ u32 encode_unit(u32x4& tail, const u32x4 (&v)[8], rsrc_t rp, u32 soff, LAS unsigned char* scr, u32 lane) {
  u32x4 mq[4];
  const u32x4 zero = {0u, 0u, 0u, 0u};
  *(LAS u32x4*)(scr + S_LIST + lane * 32u) = zero;
  *(LAS u32x4*)(scr + S_LIST + lane * 32u + 16u) = zero;
  *(LAS u32x4*)(scr + S_RARE + lane * 16u) = zero;
  u32 E[16], esum = 0;
#pragma unroll
  for (int j = 0; j < 16; ++j) {
    const u32 d0 = v[j >> 1][(j & 1) * 2], d1 = v[j >> 1][(j & 1) * 2 + 1];
    const u32 r0 = __builtin_amdgcn_alignbit(d0, d0, 31), r1 = __builtin_amdgcn_alignbit(d1, d1, 31);
    mq[j >> 2][j & 3] = __builtin_amdgcn_perm(r1, r0, 0x06040200u);
    E[j] = __builtin_amdgcn_perm(r1, r0, 0x07050301u);
    esum = __builtin_amdgcn_sad_u8(E[j], 0u, esum);
  }
#pragma unroll
  for (int i = 0; i < 4; ++i) store16(mq[i], rp, lane * 16, soff + i * 1024);
  esum = wave_sum(esum);
  u32 base = (esum + 8028u) >> 12;
  base = base < 19u ? 19u : (base > 254u ? 254u : base);
  const u32 mode = base * 4096u < esum + 6361u ? 1u : 0u;
  const u16x2 b2 = {(unsigned short)base, (unsigned short)base};
  u32 P[2][2] = {{0, 0}, {0, 0}};
#pragma unroll
  for (int j = 0; j < 16; ++j) {
    const u16x2 ev = __builtin_bit_cast(u16x2, E[j] & 0x00ff00ffu), od = __builtin_bit_cast(u16x2, E[j]) >> (u16x2){8, 8};
    const u32 ce = __builtin_bit_cast(u32, __builtin_elementwise_min((u16x2)(b2 - ev), (u16x2){3, 3}));
    const u32 co = __builtin_bit_cast(u32, __builtin_elementwise_min((u16x2)(b2 - od), (u16x2){3, 3}));
    const u32 c = ce | (co << 8);
    P[0][j >> 3] |= (c & 0x01010101u) << (j & 7);
    P[1][j >> 3] |= ((c >> 1) & 0x01010101u) << (j & 7);
  }
  store16((u32x4){P[0][0], P[0][1], P[1][0], P[1][1]}, rp, lane * 16, soff + 4096);
  tail = zero;
  const u32 mk[2] = {P[0][0] & P[1][0], P[0][1] & P[1][1]};
  u32 fl[16], sel[16];
#pragma unroll
  for (int j = 0; j < 16; ++j) {
    fl[j] = (mk[j >> 3] >> (j & 7)) & 0x01010101u;
    sel[j] = *(const LAS u32*)(size_t)__builtin_amdgcn_udot4(fl[j], 0x20100804u, TAB, false);
  }
  __builtin_amdgcn_sched_barrier(0);
  const u32 c = __builtin_popcount(mk[0]) + __builtin_popcount(mk[1]), incl = wave_scan(c);
  const u32 n2 = __builtin_amdgcn_readlane(incl, 63);
  if (n2 > 2044u) return RAW;
  const u32 flip = (base - 3u) * 0x01010101u;
  u32 a = (u32)(size_t)(scr + S_LIST) + incl - c, cur = 0;
#pragma unroll
  for (int j = 0; j < 16; ++j) {
    const u64 t = (u64)__builtin_amdgcn_perm(0u, E[j] ^ flip, sel[j]) << ((a & 3u) * 8u);
    const u32 w = cur | (u32)t, an = __builtin_amdgcn_sad_u8(fl[j], 0u, a);
    __hip_atomic_fetch_or((LAS u32*)(size_t)(a & ~3u), w, __ATOMIC_RELAXED, __HIP_MEMORY_SCOPE_WAVEFRONT);
    cur = ((an ^ a) & 4u) ? (u32)(t >> 32) : w;
    a = an;
  }
  __hip_atomic_fetch_or((LAS u32*)(size_t)(a & ~3u), cur, __ATOMIC_RELAXED, __HIP_MEMORY_SCOPE_WAVEFRONT);
  asm volatile("" ::: "memory");
  const u32x4 l[2] = {*(const LAS u32x4*)(scr + S_LIST + lane * 32u), *(const LAS u32x4*)(scr + S_LIST + lane * 32u + 16u)};
  const unsigned short ta = mode ? base + 1u : base - 3u, tb = mode ? 3u : 0u, tc = base + 1u;
  u32 Q[2] = {0, 0};
#pragma unroll
  for (int s = 0; s < 8; ++s) {
    const u32 x = l[s >> 2][s & 3] ^ flip;
    const u16x2 ev = __builtin_bit_cast(u16x2, x & 0x00ff00ffu), od = __builtin_bit_cast(u16x2, x) >> (u16x2){8, 8};
    const u32 ce = __builtin_bit_cast(u32, __builtin_elementwise_min(__builtin_elementwise_sub_sat((u16x2)((u16x2){ta, ta} - ev), (u16x2){tb, tb}), (u16x2){3, 3}));
    const u32 co = __builtin_bit_cast(u32, __builtin_elementwise_min(__builtin_elementwise_sub_sat((u16x2)((u16x2){ta, ta} - od), (u16x2){tb, tb}), (u16x2){3, 3}));
    const u32 c2 = ce | (co << 8);
    Q[0] |= (c2 & 0x01010101u) << s;
    Q[1] |= ((c2 >> 1) & 0x01010101u) << s;
  }
  *(LAS u32*)(scr + S_TAIL + lane * 8u) = Q[0];
  *(LAS u32*)(scr + S_TAIL + lane * 8u + 4u) = Q[1];
  const u32 m3 = Q[0] & Q[1];
#pragma unroll
  for (int s = 0; s < 8; ++s) {
    fl[s] = (m3 >> s) & 0x01010101u;
    sel[s] = *(const LAS u32*)(size_t)__builtin_amdgcn_udot4(fl[s], 0x20100804u, TAB, false);
  }
  __builtin_amdgcn_sched_barrier(0);
  const u32 c3 = __builtin_popcount(m3), incl3 = wave_scan(c3), n3 = __builtin_amdgcn_readlane(incl3, 63);
  if (n3 > 768u) return RAW;
  const u32 turn = flip ^ (tc * 0x01010101u);
  a = (u32)(size_t)(scr + S_RARE) + incl3 - c3;
  cur = 0;
#pragma unroll
  for (int s = 0; s < 8; ++s) {
    const u64 t = (u64)__builtin_amdgcn_perm(0u, l[s >> 2][s & 3] ^ turn, sel[s]) << ((a & 3u) * 8u);
    const u32 w = cur | (u32)t, an = __builtin_amdgcn_sad_u8(fl[s], 0u, a);
    __hip_atomic_fetch_or((LAS u32*)(size_t)(a & ~3u), w, __ATOMIC_RELAXED, __HIP_MEMORY_SCOPE_WAVEFRONT);
    cur = ((an ^ a) & 4u) ? (u32)(t >> 32) : w;
    a = an;
  }
  __hip_atomic_fetch_or((LAS u32*)(size_t)(a & ~3u), cur, __ATOMIC_RELAXED, __HIP_MEMORY_SCOPE_WAVEFRONT);
  asm volatile("" ::: "memory");
  const u32x4 rq = *(const LAS u32x4*)(scr + S_RARE + lane * 16u);
  u32 cn[4], m4 = 0;
#pragma unroll
  for (int s = 0; s < 4; ++s) {
    const u32 x = rq[s] ^ (tc * 0x01010101u);
    const u16x2 ev = __builtin_bit_cast(u16x2, x & 0x00ff00ffu), od = __builtin_bit_cast(u16x2, x) >> (u16x2){8, 8};
    const u32 ce = __builtin_bit_cast(u32, __builtin_elementwise_min(__builtin_elementwise_sub_sat((u16x2)((u16x2){tc, tc} - ev), (u16x2){5, 5}), (u16x2){15, 15}));
    const u32 co = __builtin_bit_cast(u32, __builtin_elementwise_min(__builtin_elementwise_sub_sat((u16x2)((u16x2){tc, tc} - od), (u16x2){5, 5}), (u16x2){15, 15}));
    cn[s] = ce | (co << 8);
    m4 |= ((cn[s] + 0x01010101u) & 0x10101010u) >> 4 << s;
  }
  const u32 nib = ((n2 + 31u) >> 5) * 8u;
  u32 bytes = nib + ((n3 + 15u) >> 4) * 8u;
  *(LAS u32*)(scr + S_TAIL + nib + lane * 8u) = cn[0] | (cn[1] << 4);
  *(LAS u32*)(scr + S_TAIL + nib + lane * 8u + 4u) = cn[2] | (cn[3] << 4);
  if (__builtin_amdgcn_ballot_w64(m4 != 0) != 0) {
    const u32 c4 = __builtin_popcount(m4), incl4 = wave_scan(c4), n4 = __builtin_amdgcn_readlane(incl4, 63);
    if (n4 > 128u) return RAW;
    asm volatile("" ::: "memory");
    walk_spelled<true>(m4, incl4 - c4, scr + S_RARE + lane * 16u, scr + S_TAIL + bytes, tc);
    bytes += n4;
  }
  asm volatile("" ::: "memory");
  tail = *(const LAS u32x4*)(scr + S_TAIL + lane * 16u);
  return base | (mode << 8) | (bytes << 16);
}

__device__ __forceinline__ void store_raw(const u32x4 (&v)[8], rsrc_t rp, u32 soff, u32 lane) {
#pragma unroll
  for (int f = 0; f < 8; ++f) store16(v[f], rp, lane * 16, soff + f * 1024);
}

__device__ __forceinline__ void decode_unit(u32x4 (&v)[8], const Packed& k, u32 info, LAS unsigned char* scr, u32 lane) {
  *(LAS u32x4*)(scr + S_TAIL + lane * 16u) = k.tail;
  asm volatile("" ::: "memory");
  const u32 q0 = *(const LAS u32*)(scr + S_TAIL + lane * 8u), q1 = *(const LAS u32*)(scr + S_TAIL + lane * 8u + 4u);
  const u32 b4 = (info & 255u) * 0x01010101u, lut = (info & 256u) ? b4 - 0x00040300u + 1u : b4 - 0x00050403u;
  const u32 P0[2] = {k.p01.x, k.p01.y}, P1[2] = {k.p01.z, k.p01.w}, mk[2] = {P0[0] & P1[0], P0[1] & P1[1]};
  const u32 c = __builtin_popcount(mk[0]) + __builtin_popcount(mk[1]), incl = wave_scan(c);
  const u32 g2 = (((u32)__builtin_amdgcn_readlane(incl, 63) + 31u) >> 5) & 127u;
  LAS unsigned char* const nib = scr + S_TAIL + (g2 < 64u ? g2 : 64u) * 8u;
  const u32 x0 = *(const LAS u32*)(nib + lane * 8u), x1 = *(const LAS u32*)(nib + lane * 8u + 4u);
  u32 a[17], sel[16];
  a[0] = (u32)(size_t)(scr + S_LIST) + incl - c;
#pragma unroll
  for (int j = 0; j < 16; ++j) {
    const u32 f = (mk[j >> 3] >> (j & 7)) & 0x01010101u;
    sel[j] = *(const LAS u32*)(size_t)__builtin_amdgcn_udot4(f, 0x20100804u, TAB + 64u, false);
    a[j + 1] = __builtin_amdgcn_sad_u8(f, 0u, a[j]);
  }
  __builtin_amdgcn_sched_barrier(0);
  const u32 live = lane < g2 ? ~0u : 0u, Q0 = q0 & live, Q1 = q1 & live, m3 = Q0 & Q1;
  const u32 c3 = __builtin_popcount(m3), incl3 = wave_scan(c3), g3 = (((u32)__builtin_amdgcn_readlane(incl3, 63) + 15u) >> 4) & 63u;
  const u32 live3 = lane < g3 ? 0x0f0f0f0fu : 0u, cn[4] = {x0 & live3, (x0 >> 4) & live3, x1 & live3, (x1 >> 4) & live3};
  u32x4 rq;
  u32 any = 0;
#pragma unroll
  for (int s = 0; s < 4; ++s) {
    const u32 z = ~(cn[s] + 0x0f0f0f0fu) & 0x10101010u;
    rq[s] = b4 - 0x04040404u - cn[s] + (z >> 4) + (z >> 2);
    any |= cn[s] + 0x01010101u;
  }
  *(LAS u32x4*)(scr + S_RARE + lane * 16u) = rq;
  if (__builtin_amdgcn_ballot_w64((any & 0x10101010u) != 0) != 0) {
    u32 m4 = 0;
#pragma unroll
    for (int s = 0; s < 4; ++s) m4 |= ((cn[s] + 0x01010101u) & 0x10101010u) >> 4 << s;
    const u32 c4 = __builtin_popcount(m4);
    walk_spelled<false>(m4, (wave_scan(c4) - c4) & 127u, scr + S_RARE + lane * 16u, nib + g3 * 8u, 0u);
  }
  asm volatile("" ::: "memory");
  u32x4 l[2];
  {
    u32 a3 = (u32)(size_t)(scr + S_RARE) + ((incl3 - c3) & 1023u), at[8], s3[8], r0[8], r1[8];
#pragma unroll
    for (int s = 0; s < 8; ++s) {
      const u32 f = (m3 >> s) & 0x01010101u;
      const LAS u32* const pw = (const LAS u32*)(size_t)(a3 & ~3u);
      at[s] = a3;
      s3[s] = *(const LAS u32*)(size_t)__builtin_amdgcn_udot4(f, 0x20100804u, TAB + 64u, false);
      r0[s] = pw[0];
      r1[s] = pw[1];
      a3 = __builtin_amdgcn_sad_u8(f, 0u, a3);
    }
    __builtin_amdgcn_sched_barrier(0);
#pragma unroll
    for (int s = 0; s < 8; ++s) {
      const u32 c2 = ((Q0 >> s) & 0x01010101u) | (((Q1 >> s) & 0x01010101u) << 1);
      l[s >> 2][s & 3] = __builtin_amdgcn_perm(__builtin_amdgcn_perm(0u, lut, c2), __builtin_amdgcn_alignbyte(r1[s], r0[s], at[s]), s3[s]);
    }
  }
  *(LAS u32x4*)(scr + S_LIST + lane * 32u) = l[0];
  *(LAS u32x4*)(scr + S_LIST + lane * 32u + 16u) = l[1];
  asm volatile("" ::: "memory");
  u32 w0[16], w1[16];
#pragma unroll
  for (int j = 0; j < 16; ++j) {
    const LAS u32* const pw = (const LAS u32*)(size_t)(a[j] & ~3u);
    w0[j] = pw[0];
    w1[j] = pw[1];
  }
  __builtin_amdgcn_sched_barrier(0);
#pragma unroll
  for (int j = 0; j < 16; ++j) {
    const int s = j & 7, wd = j >> 3;
    const u32 c1 = ((P0[wd] >> s) & 0x01010101u) | (((P1[wd] >> s) & 0x01010101u) << 1);
    const u32 e = __builtin_amdgcn_perm(b4 - c1, __builtin_amdgcn_alignbyte(w1[j], w0[j], a[j]), sel[j]), m = k.mq[j >> 2][j & 3];
    const u32 r0 = __builtin_amdgcn_perm(e, m, 0x05010400u), r1 = __builtin_amdgcn_perm(e, m, 0x07030602u);
    v[j >> 1][(j & 1) * 2] = __builtin_amdgcn_alignbit(r0, r0, 1);
    v[j >> 1][(j & 1) * 2 + 1] = __builtin_amdgcn_alignbit(r1, r1, 1);
  }
}

__device__ __forceinline__ void load_unit(u32x4 (&s)[8], rsrc_t rx, rsrc_t rw, u32 q, u32 vx, u32 vw) {
  const u32 kc = q >> 6, t = q & 63u;
  if (t < 32u) {
#pragma unroll
    for (int f = 0; f < 8; ++f) s[f] = __builtin_amdgcn_raw_buffer_load_b128(rx, vx, (32u * t + 16u * (f >> 2)) * 16384u + kc * 256u + (f & 3) * 64u, AUX_NT);
  } else {
#pragma unroll
    for (int i = 0; i < 8; ++i) s[i] = __builtin_amdgcn_raw_buffer_load_b128(rw, vw, (128u * kc + 32u * ((t - 32u) >> 3) + i) * 2048u + (t & 7u) * 256u, AUX_NT);
  }
}

__device__ __forceinline__ u32 packer(const Params& P, char* ws, u32 callv, u32 q0, u32 q1, u32 dq, u32 qend, LAS unsigned char* scr, u32 lane) {
  const rsrc_t rx = make_rsrc(P.x, 1024u * 8192u * 2u), rw = make_rsrc(P.w, 8192u * 1024u * 2u);
  const u32 kq = (lane & 3u) ^ quad_swz(lane >> 4);
  const u32 vx = (lane >> 2) * 16384u + kq * 16u, vw = (lane >> 4) * 8u * 2048u + (lane & 15u) * 16u;
  u32x4 s[8];
  load_unit(s, rx, rw, q0, vx, vw);
  __builtin_amdgcn_sched_barrier(0);
  const u32 call = __builtin_amdgcn_readfirstlane(callv) + 1u;
  const u32 r = P.rank, g4 = r & 4u, par = call & 1u;
  const rsrc_t rf = make_rsrc(ws, FRAG_BYTES), rp = make_rsrc(ws + OFF_PK + par * PK_BYTES, PK_BYTES);
  const u32 goff = OFF_G + (par * NR + r) * NUNITS * 8;
  char* g_in = pick(P.ws, g4 | ((r + 1u) & 3u));
  char* g_out = pick(P.ws, g4 ^ 4u);
#pragma unroll
  for (u32 j = 1; j < 4; ++j) {
    char* const pi = pick(P.ws, g4 | ((r + 1u + j) & 3u));
    char* const po = pick(P.ws, (g4 ^ 4u) | j);
    g_in = lane == j ? pi : g_in;
    g_out = lane == j ? po : g_out;
  }
  u64 psend = 0;
  char* pgp = g_out;
  u32 pinfo = 0, pcnt = SPARE_CNT;
  auto announce = [&]() {
    flag_store(pgp, ((u64)call << 32) | pinfo, psend);
    if (pcnt != SPARE_CNT) count_in(ws + OFF_CNT, pcnt, lane);
  };
  for (u32 q = q0; q < qend; q = q1, q1 += dq) {
    const u32 kc = q >> 6, t = q & 63u;
    u32x4 v[8];
    Dest d;
    bool own;
    u64 send;
    char* gp;
    if (t < 32u) {
#pragma unroll
      for (int f = 0; f < 8; ++f) v[f] = s[f];
      own = t < AU;
      d = dest_x(t, kc);
      send = own ? 7u : 15u;
      gp = own ? g_in : g_out;
    } else {
      const u32 ks4 = (t - 32u) >> 3, cg = t & 7u;
#pragma unroll
      for (int nf = 0; nf < 8; ++nf)
#pragma unroll
        for (int tt = 0; tt < 4; ++tt) v[nf][tt] = __builtin_amdgcn_perm(s[2 * tt + 1][nf >> 1], s[2 * tt][nf >> 1], (nf & 1) ? 0x07060302u : 0x05040100u);
      own = true;
      d = dest_w(cg >> 1, kc, ks4, cg & 1u);
      send = 1u << (cg >> 1);
      gp = g_out;
    }
    const bool lazy = qend != NUNITS;
    if (q1 < qend && !lazy) load_unit(s, rx, rw, q1, vx, vw);
    u32 info = RAW | PUSHED;
    if (kc == 0u && (t & 3u) < 3u) {
      for (u32 m = (u32)send; m; m &= m - 1u) {
        const u32 jd = __builtin_ctz(m), dst = (t < AU) ? (g4 | ((r + 1u + jd) & 3u)) : ((g4 ^ 4u) | jd);
        store_raw(v, make_rsrc(pick(P.ws, dst) + OFF_IB + (par * NR + r) * IB_SRC, IB_SRC), q * SLOT, lane);
      }
      if (own) store_frags(v, rf, d, lane);
    } else {
      if (own) store_frags(v, rf, d, lane);
      u32x4 tail;
      info = encode_unit(tail, v, rp, q * SLOT, scr, lane);
      if (!lazy || (info & RAW)) {
        drain_vm();
#pragma unroll
        for (int f = 0; f < 8; ++f) asm volatile("" : "+v"(s[f]));
        announce();
      }
      if (info & RAW) store_raw(v, rp, q * SLOT, lane);
      else store16(tail, rp, lane * 16u < (info >> 16) ? lane * 16u : 0x80000000u, q * SLOT + 5120u);
    }
    psend = send;
    pgp = gp + goff + q * 8u;
    pinfo = info;
    pcnt = own ? d.cnt : SPARE_CNT;
    if (q1 < qend && lazy) {
      drain_vm();
      announce();
      psend = 0;
      pcnt = SPARE_CNT;
      load_unit(s, rx, rw, q1, vx, vw);
    }
  }
  drain_vm();
  announce();
  return call;
}

__device__ __forceinline__ u32 lds_add(LAS u32* p, u32 v) {
  return __builtin_amdgcn_readfirstlane(__hip_atomic_fetch_add(p, v, __ATOMIC_RELAXED, __HIP_MEMORY_SCOPE_WORKGROUP)) >> 6;
}

__device__ __forceinline__ void puller(const Params& P, char* ws, u32 call, u32 li, u32 lidx, LAS u32* ctl, LAS unsigned char* scr, u32 lane, u32 wid) {
  const u32 r = P.rank, g4 = r & 4u, par = call & 1u;
  const bool ing = li < 3u;
  const u32 src = ing ? (g4 | ((r + 1 + li) & 3u)) : ((g4 ^ 4u) | (li - 3u));
  const rsrc_t rf = make_rsrc(ws, FRAG_BYTES), rp = make_rsrc(pick(P.ws, src) + OFF_PK + par * PK_BYTES, PK_BYTES);
  const rsrc_t ri = make_rsrc(ws + OFF_IB + (par * NR + src) * IB_SRC, IB_SRC);
  char* const gbase = ws + OFF_G + (par * NR + src) * NUNITS * 8;
  GAS u32* const cnt = (GAS u32*)(ws + OFF_CNT);
  const u32 total = ing ? NKC * AU : NKC * NOUT;
  constexpr u32 NONE = 0xffffffffu;
  const u32 idle = ((OFF_CALL - OFF_CNT + (lidx * 7u + li) * 128u + 64u + wid * 4u) >> 2) | SPARE << 16;
  GAS u32* const tkt = (GAS u32*)(ws + OFF_TKT + li * 128u);
  const u32 tbase = (call - 1u) * (total + NCL * 8u) * 64u;
  auto draw = [&]() -> u32 { return __hip_atomic_fetch_add(tkt, 1u, __ATOMIC_RELAXED, __HIP_MEMORY_SCOPE_AGENT); };
  auto unit_of = [&](u32 raw_ticket) -> u32 {
    const u32 i = ((u32)__builtin_amdgcn_readfirstlane(raw_ticket) - tbase) >> 6;
    if (i >= total) return NONE;
    if (ing) { const u32 kc = i / AU; return kc * 64u + (i - kc * AU); }
    constexpr u32 C1 = 8u * LEAD / RAMP, N1 = NOUT + RAMP, E1 = C1 * N1, E2 = E1 + (NKC - LEAD - C1) * NOUT;
    u32 kc, t, wi;
    if (i < E1) { kc = i / N1; t = i - kc * N1; wi = kc * (8u + RAMP) + t - AWAY; }
    else if (i < E2) { kc = (i - E1) / NOUT; t = i - E1 - kc * NOUT; kc += C1; wi = (kc + LEAD) * 8u + t - AWAY; }
    else { kc = (i - E2) / AWAY; t = i - E2 - kc * AWAY; kc += NKC - LEAD; wi = 0; }
    return t < AWAY ? kc * 64u + AU + t : (wi >> 3) * 64u + 32u + ((wi >> 1) & 3u) * 8u + 2u * (r & 3u) + (wi & 1u);
  };
  auto granule = [&](u32 q) -> u64 { return __hip_atomic_load((GAS u64*)(gbase + (q & (NUNITS - 1)) * 8u), __ATOMIC_RELAXED, __HIP_MEMORY_SCOPE_SYSTEM); };
  auto peek = [&](LAS u32* p) -> u32 { return (u32)(__builtin_amdgcn_readfirstlane(__hip_atomic_load(p, __ATOMIC_RELAXED, __HIP_MEMORY_SCOPE_WORKGROUP)) >> 6); };
  auto turn = [&](u32 my) { while (peek(ctl + 2) + HALVES <= my) __builtin_amdgcn_s_sleep(1); };
  auto dest_of = [&](u32 q) -> Dest {
    const u32 kc = q >> 6, tq = q & 63u;
    if (q == NUNITS) return Dest{FRAG_BYTES, 0u, 0u, NONE};
    if (tq < 32u) return dest_x(ing ? EARLY_UNITS + li * AU + tq : AU + (li - 3u) * AWAY + (tq - AU), kc);
    return dest_w(4u + (li - 3u), kc, (tq - 32u) >> 3, tq & 1u);
  };
  auto publish = [&](u32 c) { if (c != NONE) count_in(ws + OFF_CNT, c, lane); };
  u32 q = NUNITS, pend = NONE, tag = call, info = 127u;
  u64 gr;
  while (q != NONE) {
    if (tag != call || (!(info & PUSHED) && peek(ctl + 2) + HALVES <= peek(ctl))) {
      drain_vm();
      publish(pend);
      pend = NONE;
      if (tag != call) {
        const u64 t0 = __builtin_amdgcn_s_memrealtime();
        for (;;) {
          gr = granule(q);
          tag = __builtin_amdgcn_readfirstlane((u32)(gr >> 32));
          info = __builtin_amdgcn_readfirstlane((u32)gr);
          if (tag == call || give_up(ws, call, t0)) break;
          __builtin_amdgcn_s_sleep(2);
        }
        if (tag != call) break;
      }
    }
    u32 my = 0;
    if (!(info & PUSHED)) { my = lds_add(ctl, 2u); turn(my); }
    asm volatile("" ::: "memory");
    const Dest d = dest_of(q);
    const u32 cur = info, soff = q * SLOT;
    u32x4 v[8];
    if (cur & RAW) {
#pragma unroll
      for (int f = 0; f < 8; ++f) v[f] = __builtin_amdgcn_raw_buffer_load_b128((cur & PUSHED) ? ri : rp, lane * 16, soff + f * 1024, AUX_SYS);
      const u32 nt = draw();
      drain_vm();
      if (!(cur & PUSHED)) lds_add(ctl + 2, 2u);
      q = unit_of(nt);
      gr = granule(q);
      publish(pend);
      tag = __builtin_amdgcn_readfirstlane((u32)(gr >> 32));
      info = __builtin_amdgcn_readfirstlane((u32)gr);
      store_frags(v, rf, d, lane);
    } else {
      Packed k;
      u32 nt;
      u64 keep;
      asm volatile("s_mov_b64 %1, exec\n\ts_mov_b64 exec, 1\n\tglobal_atomic_add %0, %2, %3, %4 sc0\n\ts_mov_b64 exec, %1"
                   : "=&v"(nt), "=&s"(keep) : "v"(0u), "v"(64u), "s"(tkt) : "memory");
      k.tail = __builtin_amdgcn_raw_buffer_load_b128(rp, lane * 16u < (cur >> 16) ? lane * 16u : 0x80000000u, soff + 5120, AUX_SYS);
      k.p01 = __builtin_amdgcn_raw_buffer_load_b128(rp, lane * 16, soff + 4096, AUX_SYS);
      k.mq[0] = __builtin_amdgcn_raw_buffer_load_b128(rp, lane * 16, soff, AUX_SYS);
      asm volatile("" ::: "memory");
      turn(my + 1u);
      asm volatile("" ::: "memory");
#pragma unroll
      for (int i = 1; i < 4; ++i) k.mq[i] = __builtin_amdgcn_raw_buffer_load_b128(rp, lane * 16, soff + i * 1024, AUX_SYS);
      asm volatile("s_waitcnt vmcnt(5)" ::: "memory");
      lds_add(ctl + 2, 1u);
      asm volatile("s_waitcnt vmcnt(2)" ::: "memory");
      lds_add(ctl + 2, 1u);
      asm volatile("s_waitcnt vmcnt(0)" : "+v"(nt), "+v"(k.mq[0]), "+v"(k.mq[1]), "+v"(k.mq[2]), "+v"(k.mq[3]), "+v"(k.p01), "+v"(k.tail) :: "memory");
      q = unit_of(nt);
      asm volatile("global_load_dwordx2 %0, %2, %3 sc0 sc1\n\ts_mov_b64 %1, exec\n\ts_mov_b64 exec, %7\n\tglobal_atomic_add %4, %5, %6\n\ts_mov_b64 exec, %1"
                   : "=&v"(gr), "=&s"(keep) : "v"((q & (NUNITS - 1)) * 8u), "s"(gbase), "v"(pair_offset(pend == NONE ? idle : pend, lane)), "v"(1u), "s"(cnt),
                     "s"(pair_mask(pend == NONE ? idle : pend)) : "memory");
      decode_unit(v, k, cur, scr, lane);
      __builtin_amdgcn_sched_barrier(0);
      asm volatile("s_waitcnt vmcnt(1)" : "+v"(gr) :: "memory");
      tag = __builtin_amdgcn_readfirstlane((u32)(gr >> 32));
      info = __builtin_amdgcn_readfirstlane((u32)gr);
      __builtin_amdgcn_sched_barrier(0);
      store_frags(v, rf, d, lane);
    }
    pend = d.cnt;
  }
  drain_vm();
  publish(pend);
}

__device__ __forceinline__ u32 frag_row(u32 r, u32 gf) {
  const u32 g4 = r & 4u, ux = gf >> 1;
  u32 src, su;
  if (ux >= EARLY_UNITS) {
    const u32 j = (ux - EARLY_UNITS) / AU;
    src = g4 | ((r + 1 + j) & 3u);
    su = ux - EARLY_UNITS - j * AU;
  } else if (ux < AU) {
    src = r;
    su = ux;
  } else {
    const u32 j = (ux - AU) / AWAY;
    src = (g4 ^ 4u) | j;
    su = ux - j * AWAY;
  }
  return ((ORDER >> (4u * src)) & 7u) * 1024u + su * 32u + (gf & 1u) * 16u;
}
__device__ __forceinline__ u32 foreign_frag0(u32 ft, u32 wid) { return ft < 4u ? (0x74208630u >> (4 * wid)) & 15u : (wid & 3u) * 2u; }
__device__ __forceinline__ u32 foreign_base(u32 ft) { return ft < 4u ? ft * 10u : ft * 8u + 8u; }

constexpr u32 O_HI = 7168;
__device__ __forceinline__ void out_puller(const Params& P, char* ws, u32 call, u32 j, LAS u32* ctl, LAS unsigned char* scr, u32 lane) {
  const u32 r = P.rank, src = ((r & 4u) ^ 4u) | j;
  char* const ob = pick(P.ws, src) + OFF_OB + (r & 3u) * (OUNITS * OSLOT);
  const rsrc_t ro = make_rsrc(pick(P.out, r), 8192u * 1024u * 2u);
  GAS u32* const tkt = (GAS u32*)(ws + OFF_TKT + (7u + j) * 128u);
  const u32 tbase = (call - 1u) * (OUNITS + (NCL + HELP) * 8u) * 64u;
  const u32 vo = (lane >> 4) * 4u * 2048u + (lane & 15u) * 16u;
  char* const queue = ws + OFF_OQ + j * 2048u;
  u32 k = ((u32)__builtin_amdgcn_readfirstlane(__hip_atomic_fetch_add(tkt, 1u, __ATOMIC_RELAXED, __HIP_MEMORY_SCOPE_AGENT)) - tbase) >> 6;
  u32 tag = 0, info = 0;
  while (k < OUNITS) {
    if (tag != call) {
      const u64 t0 = __builtin_amdgcn_s_memrealtime();
      for (u32 spins = 0;; ++spins) {
        const u64 gr = __hip_atomic_load((GAS u64*)(queue + k * 8u), __ATOMIC_RELAXED, __HIP_MEMORY_SCOPE_SYSTEM);
        tag = __builtin_amdgcn_readfirstlane((u32)(gr >> 32));
        info = __builtin_amdgcn_readfirstlane((u32)gr);
        if (tag == call || ((spins & 7u) == 7u && give_up(ws, call, t0))) break;
        __builtin_amdgcn_s_sleep(3);
      }
      if (tag != call) {
        *(GAS unsigned short*)(pick(P.out, r) + (size_t)((ORDER >> (4u * r)) & 7u) * 1024u * 2048u) = 0x7fc0;
        break;
      }
    }
    {
      const u32 my = lds_add(ctl, 1u);
      while ((u32)(__builtin_amdgcn_readfirstlane(__hip_atomic_load(ctl + 2, __ATOMIC_RELAXED, __HIP_MEMORY_SCOPE_WORKGROUP)) >> 6) + OUT_TOK <= my) __builtin_amdgcn_s_sleep(1);
    }
    asm volatile("" ::: "memory");
    const u32 uid = (info & 255u) < OUNITS ? (info & 255u) : 0u, sw = uid & 7u, mf = (uid < 32u && sw - 2u >= 4u) ? 3u : 2u;
    u32 total = (info >> 8) & 0x3fffu;
    total = total < 12288u ? total : 12288u;
    const bool raw = (info >> 22) & 1u, tri = info >> 31;
    const u32 t4 = ((info >> 23) & 255u) * 0x01010101u;
    const rsrc_t rs = make_rsrc(ob + uid * OSLOT, (total + 15u) & ~15u);
    u32 nt;
    u64 keep, gr;
    asm volatile("s_mov_b64 %1, exec\n\ts_mov_b64 exec, 1\n\tglobal_atomic_add %0, %2, %3, %4 sc0\n\ts_mov_b64 exec, %1"
                 : "=&v"(nt), "=&s"(keep) : "v"(0u), "v"(64u), "s"(tkt) : "memory");
    for (u32 p = 0; p < total; p += 1024u) __builtin_amdgcn_raw_ptr_buffer_load_lds(rs, scr + p, 16, lane * 16u, p, 0, AUX_SYS);
    asm volatile("s_waitcnt vmcnt(2)" ::: "memory");
    lds_add(ctl + 2, 1u);
    asm volatile("s_waitcnt vmcnt(0)" : "+v"(nt) :: "memory");
    const u32 kn = ((u32)__builtin_amdgcn_readfirstlane(nt) - tbase) >> 6;
    asm volatile("global_load_dwordx2 %0, %1, %2 sc0 sc1" : "=&v"(gr) : "v"((kn & 255u) * 8u), "s"(queue) : "memory");
    u32 mk[3] = {0, 0, 0}, a = 0;
    if (!raw) {
#pragma unroll
      for (u32 m = 0; m < 3; ++m) mk[m] = *(const LAS u32*)(scr + m * 256u + lane * 4u);
      mk[2] = mf == 3u ? mk[2] : 0u;
      const u32 c = ((__builtin_popcount(mk[0]) + __builtin_popcount(mk[1]) + __builtin_popcount(mk[2])) & ~3u) + 4u, incl = wave_scan(c);
      u32 n = __builtin_amdgcn_readlane(incl, 63);
      n = n < 4096u ? n : 4096u;
      LAS unsigned char* const lo = scr + mf * 256u;
      LAS unsigned char* const planes = lo + ((n + 15u) & ~15u);
      a = (u32)(size_t)lo + incl - c;
      if (tri) {
#pragma unroll
        for (int h = 0; h < 2; ++h) {
          const LAS u32* const pw = (const LAS u32*)(planes + lane * 24u + h * 12u);
          const u32 w0 = pw[0], w1 = pw[1], w2 = pw[2];
          *(LAS u32x4*)(lo + O_HI + lane * 64u + h * 32u) =
              (u32x4){w0 & 0x07070707u, (w0 >> 3) & 0x07070707u, ((w0 >> 6) & 0x03030303u) | ((w2 >> 4) & 0x04040404u), w1 & 0x07070707u};
          *(LAS u32x4*)(lo + O_HI + lane * 64u + h * 32u + 16u) =
              (u32x4){(w1 >> 3) & 0x07070707u, ((w1 >> 6) & 0x03030303u) | ((w2 >> 5) & 0x04040404u), w2 & 0x07070707u, (w2 >> 3) & 0x07070707u};
        }
      } else {
        const u32x4 nb[2] = {*(const LAS u32x4*)(planes + lane * 32u), *(const LAS u32x4*)(planes + lane * 32u + 16u)};
#pragma unroll
        for (int i = 0; i < 2; ++i) {
          *(LAS u32x4*)(lo + O_HI + lane * 64u + i * 16u) = nb[i] & 0x0f0f0f0fu;
          *(LAS u32x4*)(lo + O_HI + lane * 64u + 32u + i * 16u) = (nb[i] >> 4) & 0x0f0f0f0fu;
        }
      }
      asm volatile("" ::: "memory");
    }
    asm volatile("s_waitcnt vmcnt(0)" : "+v"(gr) :: "memory");
    tag = __builtin_amdgcn_readfirstlane((u32)(gr >> 32));
    info = __builtin_amdgcn_readfirstlane((u32)gr);
    const u32 col = j * 512u + (sw >> 2) * 256u, f0 = foreign_base(uid >> 3) + foreign_frag0(uid >> 3, sw);
    for (u32 m = 0; m < mf; ++m) {
      const u32 w = m == 0 ? mk[0] : (m == 1 ? mk[1] : mk[2]);
      u32x4 o[4];
      if (raw) {
#pragma unroll
        for (int e = 0; e < 4; ++e) o[e] = *(const LAS u32x4*)(scr + (m * 4u + e) * 1024u + lane * 16u);
      } else {
        u32 at[8], sel[8], l0[8], l1[8], h0[8], h1[8];
#pragma unroll
        for (int s = 0; s < 8; ++s) {
          const u32 f = (w >> s) & 0x01010101u;
          const LAS u32* const pw = (const LAS u32*)(size_t)(a & ~3u);
          at[s] = a;
          sel[s] = *(const LAS u32*)(size_t)__builtin_amdgcn_udot4(f, 0x20100804u, TAB + 128u, false);
          l0[s] = pw[0];
          l1[s] = pw[1];
          h0[s] = pw[O_HI / 4];
          h1[s] = pw[O_HI / 4 + 1];
          a = __builtin_amdgcn_sad_u8(f, 0u, a);
        }
        __builtin_amdgcn_sched_barrier(0);
#pragma unroll
        for (int s = 0; s < 8; ++s) {
          const u32 lb = __builtin_amdgcn_perm(0u, __builtin_amdgcn_alignbyte(l1[s], l0[s], at[s]), sel[s]);
          const u32 hb = __builtin_amdgcn_perm(0u, t4 - (__builtin_amdgcn_alignbyte(h1[s], h0[s], at[s]) & 0x0f0f0f0fu), sel[s]);
          o[s >> 1][(s & 1) * 2] = __builtin_amdgcn_perm(hb, lb, 0x05010400u);
          o[s >> 1][(s & 1) * 2 + 1] = __builtin_amdgcn_perm(hb, lb, 0x07030602u);
        }
      }
      const u32 row0 = frag_row(src, f0 + m);
#pragma unroll
      for (int e = 0; e < 4; ++e) store16(o[e], ro, vo, (row0 + e) * 2048u + col);
    }
    k = kn;
  }
}

__device__ __forceinline__ void gate(int c, int& ready, const char* cnt, u32 target, char* ws, u32 call) {
  if (c < ready) return;
  const u64 t0 = __builtin_amdgcn_s_memrealtime();
  for (u32 spins = 0;; ++spins) {
    const u64 m = __builtin_amdgcn_ballot_w64((int)(ld_agent(cnt) - target) >= 0);
    ready = ~m ? __builtin_ctzll(~m) : 64;
    if (c < ready) break;
    if ((spins & 7u) == 7u && give_up(ws, call, t0)) { ready = 64; break; }
    __builtin_amdgcn_s_sleep(3);
  }
  asm volatile("" ::: "memory");
}

__device__ __forceinline__ u32 relu_pk(float lo, float hi) {
  u32 r;
  asm("v_cvt_pk_bf16_f32 %0, %1, %2\n\tv_pk_max_i16 %0, %0, 0" : "=v"(r) : "v"(lo), "v"(hi));
  return r;
}

__device__ __forceinline__ void mfma_acc(f32x4& c, const bf16x8& a, const bf16x8& b) {
  asm volatile("v_mfma_f32_16x16x32_bf16 %0, %1, %2, %0" : "+v"(c) : "v"(a), "v"(b));
}
template <int N> __device__ __forceinline__ void wait_bar() { asm volatile("s_waitcnt vmcnt(%0)\n\ts_barrier" :: "i"(N) : "memory"); }

struct Tile { u32 rt, ct, gf0, nfr; };
template <int MF, int NP, int KIND>
__device__ __forceinline__ void gemm_wave(const Params& P, char* ws, u32 call, const Tile& T, u32 f0, LAS char* lds, u32 lane, u32 wid) {
  constexpr bool XL = NP && NP < 8;
  constexpr u32 STEP = XL ? A_STEP : B_STEP;
  constexpr int NM = MF * 8;
  const u32 r = P.rank, g4 = r & 4u, rt = T.rt, ct = T.ct, wc = wid >> 2;
  const bool own = KIND == 1 || (KIND == 3 && ct < 4u);
  const u32 p0 = (wid & 1u) * NP;
  const rsrc_t rs = XL ? make_rsrc(ws + OFF_XU, OFF_WU - OFF_XU) : make_rsrc(ws + OFF_WU + ct * B_TILE, B_TILE);
  const char* const cnt = ws + OFF_CNT + ((XL ? (own ? ROW_OWN + rt : rt) : ROW_W + ct) * 64 + lane) * 4;
  const u32 target = call * (XL ? T.nfr >> 1 : 8u);
  u32 vp[NP ? NP : 1];
#pragma unroll
  for (u32 p = 0; p < NP; ++p) vp[p] = lane * 16u + (XL ? T.gf0 + (p0 + p < T.nfr ? p0 : T.nfr - 1u - p) : p0) * 1024u;
  LAS char* const ring = lds + (XL ? 0u : A_LDS) + p0 * 1024u;
  const u32 img = ((lane & 15u) * 4u + ((lane >> 4) ^ quad_swz((lane >> 2) & 3u))) * 16u;
  LAS char* const ra = lds + f0 * 1024u + img;
  LAS char* const rb = lds + A_LDS + wc * 8192u + lane * 16u;
#define AG_DMA(p, ks, off, imm) __builtin_amdgcn_raw_ptr_buffer_load_lds(rs, ring + (off), 16, vp[p < NP ? p : 0], (ks), imm, AUX_SYS)
#define AG_PIECE(p, ks, off) do { \
    if (p == 0) AG_DMA(0, (ks) * STEP, off, 0); if (p == 1) AG_DMA(1, (ks) * STEP, off, 1024); \
    if (p == 2) AG_DMA(2, (ks) * STEP, off, 2048); if (p == 3) AG_DMA(3, (ks) * STEP, off, 3072); \
    if (p == 4) AG_DMA(4, (ks) * STEP + 4096, (off) + 4096, 0); if (p == 5) AG_DMA(5, (ks) * STEP + 4096, (off) + 4096, 1024); \
    if (p == 6) AG_DMA(6, (ks) * STEP + 4096, (off) + 4096, 2048); if (p == 7) AG_DMA(7, (ks) * STEP + 4096, (off) + 4096, 3072); \
  } while (0)
#define AG_ALL(ks, off) do { _Pragma("unroll") for (int p = 0; p < NP; ++p) AG_PIECE(p, ks, off); } while (0)
#define AG_PIN(stmt) do { __builtin_amdgcn_sched_barrier(0); stmt; __builtin_amdgcn_sched_barrier(0); } while (0)
#define AG_STEP(cs, ks, DMA, RD) do { \
    LAS char* const pa = ra + ord; \
    LAS char* const pb = rb + ord; \
    __builtin_amdgcn_sched_barrier(0); \
    _Pragma("unroll") for (int i = 0; i < NM; ++i) { \
      mfma_acc(acc[i % MF][i / MF], fa[cs][i % MF], fb[cs][i / MF]); \
      _Pragma("unroll") for (int j = 0; j < MF + 8; ++j) \
        if (RD && i == (NP ? (MF == 3 ? 2 * j : (3 * j) / 2) : j)) { \
          if (j < MF) AG_PIN(fa[(cs) ^ 1][j] = *(const LAS bf16x8*)(pa + j * 1024)); \
          else AG_PIN(fb[(cs) ^ 1][j - MF] = *(const LAS bf16x8*)(pb + (j - MF) * 1024)); \
        } \
      _Pragma("unroll") for (int p = 0; p < NP; ++p) \
        if (DMA && i == 1 + p * ((NM + 1) / (NP ? NP : 1))) AG_PIN(AG_PIECE(p, ks, owr)); \
    } \
    ord = ord + SLOTB == NSLOT * SLOTB ? 0 : ord + SLOTB; \
    owr = owr + SLOTB == NSLOT * SLOTB ? 0 : owr + SLOTB; \
    __builtin_amdgcn_sched_barrier(0); \
  } while (0)
#define AG_GATE(c) do { if (NP) gate(c, ready, cnt, target, ws, call); } while (0)
  f32x4 acc[MF][8];
#pragma unroll
  for (int m = 0; m < MF; ++m)
#pragma unroll
    for (int n = 0; n < 8; ++n) { acc[m][n] = (f32x4){0.f, 0.f, 0.f, 0.f}; asm volatile("" : "+v"(acc[m][n])); }
  __builtin_amdgcn_s_setprio(NP ? 1 : 0);
  int ready = 0;
  AG_GATE(0);
  constexpr int LA = 4, W = (LA - 2) * NP;
#pragma unroll
  for (int s = 0; s < 4; ++s) AG_ALL(s, s * SLOTB);
  wait_bar<W>();
  bf16x8 fa[2][MF], fb[2][8];
#pragma unroll
  for (int m = 0; m < MF; ++m) fa[0][m] = *(const LAS bf16x8*)(ra + m * 1024);
#pragma unroll
  for (int n = 0; n < 8; ++n) fb[0][n] = *(const LAS bf16x8*)(rb + n * 1024);
  u32 ord = SLOTB, owr = LA * SLOTB;
  for (int kc = 0; kc < NKC - 3; ++kc) {
    AG_GATE(kc + 1);
    AG_STEP(0, kc * 4 + LA, true, true); wait_bar<W>();
    AG_STEP(1, kc * 4 + LA + 1, true, true); wait_bar<W>();
    AG_STEP(0, kc * 4 + LA + 2, true, true); wait_bar<W>();
    AG_STEP(1, kc * 4 + LA + 3, true, true); wait_bar<W>();
  }
  AG_GATE(NKC - 2);
  AG_STEP(0, NKS - 12 + LA, true, true); wait_bar<W>();
  AG_STEP(1, NKS - 11 + LA, true, true); wait_bar<W>();
  AG_STEP(0, NKS - 10 + LA, true, true); wait_bar<W>();
  AG_STEP(1, NKS - 9 + LA, true, true); wait_bar<2 * NP>();
  AG_STEP(0, 0, false, true); wait_bar<NP>();
  AG_STEP(1, 0, false, true); wait_bar<0>();
  AG_STEP(0, 0, false, true);
  AG_STEP(1, 0, false, false);
  asm volatile("s_barrier" ::: "memory");
  AG_GATE(NKC - 1);
#pragma unroll
  for (int s = 0; s < 4; ++s) AG_ALL(NKS - 4 + s, s * SLOTB);
  wait_bar<2 * NP>();
  u32 tmo;
  asm volatile("global_load_dword %0, %1, %2 sc1" : "=&v"(tmo) : "v"(0u), "s"(ws + OFF_TMO) : "memory");
#pragma unroll
  for (int m = 0; m < MF; ++m) fa[0][m] = *(const LAS bf16x8*)(ra + m * 1024);
#pragma unroll
  for (int n = 0; n < 8; ++n) fb[0][n] = *(const LAS bf16x8*)(rb + n * 1024);
  ord = SLOTB;
  AG_STEP(0, 0, false, true); wait_bar<NP + 1>();
  AG_STEP(1, 0, false, true); wait_bar<0>();
  AG_STEP(0, 0, false, true);
  AG_STEP(1, 0, false, false);
  asm volatile("" : "+v"(tmo));
  asm volatile("s_nop 15" ::: "memory");
  __builtin_amdgcn_sched_barrier(0);
#undef AG_PIECE
#undef AG_ALL
#undef AG_PIN
#undef AG_DMA
#undef AG_GATE
#undef AG_STEP
  const bool bad = __builtin_amdgcn_readfirstlane(tmo) == call;
  u32x4 o[MF][4];
#pragma unroll
  for (int m = 0; m < MF; ++m)
#pragma unroll
    for (int e = 0; e < 4; ++e)
#pragma unroll
      for (int n = 0; n < 4; ++n) o[m][e][n] = relu_pk(acc[m][2 * n][e], acc[m][2 * n + 1][e]);
  if (bad) {
#pragma unroll
    for (int m = 0; m < MF; ++m)
#pragma unroll
      for (int e = 0; e < 4; ++e) o[m][e] = (u32x4){0x7fc07fc0u, 0x7fc07fc0u, 0x7fc07fc0u, 0x7fc07fc0u};
    asm volatile("" ::: "memory");
  }
  if (own) {
    const rsrc_t ro = make_rsrc(pick(P.out, r), 8192u * 1024u * 2u);
    const u32 vo = (lane >> 4) * 4u * 2048u + (lane & 15u) * 16u;
#pragma unroll
    for (int m = 0; m < MF; ++m) {
      if (f0 + m >= T.nfr) continue;
      const u32 row0 = frag_row(r, T.gf0 + f0 + m);
#pragma unroll
      for (int e = 0; e < 4; ++e) store16(o[m][e], ro, vo, (row0 + e) * 2048u + ct * 512u + wc * 256u);
    }
    return;
  }
  const u32 j = ct - 4u, uid = rt * 8u + wid;
  u32 qi;
  u64 keep;
  asm volatile("s_mov_b64 %1, exec\n\ts_mov_b64 exec, 1\n\tglobal_atomic_add %0, %2, %3, %4 sc0\n\ts_mov_b64 exec, %1"
               : "=&v"(qi), "=&s"(keep) : "v"(0u), "v"(1u), "s"(ws + OFF_TKT + (11u + j) * 128u) : "memory");
  u32 mk[MF], fl[MF * 8], lb[MF * 8], hb[MF * 8], c = 0;
  u16x2 big[2] = {{0, 0}, {0, 0}};
#pragma unroll
  for (int m = 0; m < MF; ++m) {
    mk[m] = 0;
#pragma unroll
    for (int s = 0; s < 8; ++s) {
      const u32 d0 = o[m][s >> 1][(s & 1) * 2], d1 = o[m][s >> 1][(s & 1) * 2 + 1];
      u32 t0, t1;
      asm("v_pk_min_u16 %0, %1, %2" : "=v"(t0) : "v"(d0), "v"(0x00010001u));
      asm("v_pk_min_u16 %0, %1, %2" : "=v"(t1) : "v"(d1), "v"(0x00010001u));
      fl[m * 8 + s] = __builtin_amdgcn_perm(t1, t0, 0x06040200u);
      mk[m] |= fl[m * 8 + s] << s;
      big[0] = __builtin_elementwise_max(big[0], __builtin_bit_cast(u16x2, d0));
      big[1] = __builtin_elementwise_max(big[1], __builtin_bit_cast(u16x2, d1));
      lb[m * 8 + s] = __builtin_amdgcn_perm(d1, d0, 0x06040200u);
      hb[m * 8 + s] = __builtin_amdgcn_perm(d1, d0, 0x07050301u);
    }
    c += __builtin_popcount(mk[m]);
  }
  c = (c & ~3u) + 4u;
  const u32 incl = wave_scan(c), n = __builtin_amdgcn_readlane(incl, 63);
  big[0] = __builtin_elementwise_max(big[0], big[1]);
  u32 top = wave_max((u32)(big[0].x > big[0].y ? big[0].x : big[0].y) >> 8);
  top = top < 15u ? 15u : top;
  bool raw = n > 4096u, tri = false;
  u32 total = 0;
  const rsrc_t rb2 = make_rsrc(ws + OFF_OB + j * (OUNITS * OSLOT) + uid * OSLOT, OSLOT);
  LAS unsigned char* const scr = (LAS unsigned char*)lds + (wid < 4u ? wid * 12288u : 4u * SLOTB + (wid - 4u) * 12160u);
  if (!raw) {
    LAS unsigned char* const lo = scr + MF * 256u;
  #pragma unroll
    for (int m = 0; m < MF; ++m) *(LAS u32*)(scr + m * 256u + lane * 4u) = mk[m];
    asm volatile("" ::: "memory");
    *(LAS u32*)(lo + O_HI + n + (lane & 15u) * 4u) = 0u;
    const u32 t4 = top * 0x01010101u;
    u32 wide = 0;
    u32 a = (u32)(size_t)lo + incl - c, cl = 0, ch = 0;
#pragma unroll
    for (int m = 0; m < MF; ++m) {
      u32 sel[8];
#pragma unroll
      for (int s = 0; s < 8; ++s) sel[s] = *(const LAS u32*)(size_t)__builtin_amdgcn_udot4(fl[m * 8 + s], 0x20100804u, TAB, false);
      __builtin_amdgcn_sched_barrier(0);
#pragma unroll
      for (int s = 0; s < 8; ++s) {
        const u32 sh = (a & 3u) * 8u, an = __builtin_amdgcn_sad_u8(fl[m * 8 + s], 0u, a);
        const u32 code = __builtin_amdgcn_perm(0u, t4 - hb[m * 8 + s], sel[s]);
        const u64 tl = (u64)__builtin_amdgcn_perm(0u, lb[m * 8 + s], sel[s]) << sh, th = (u64)code << sh;
        wide |= code;
        const u32 wl = cl | (u32)tl, wh = ch | (u32)th;
        LAS u32* const pw = (LAS u32*)(size_t)(a & ~3u);
        pw[0] = wl;
        pw[O_HI / 4] = wh;
        const bool cross = (an ^ a) & 4u;
        cl = cross ? (u32)(tl >> 32) : wl;
        ch = cross ? (u32)(th >> 32) : wh;
        a = an;
      }
    }
    ((LAS u32*)(size_t)(a & ~3u))[0] = cl;
    ((LAS u32*)(size_t)(a & ~3u))[O_HI / 4] = ch;
    asm volatile("" ::: "memory");
    LAS unsigned char* const planes = lo + ((n + 15u) & ~15u);
    u32x4 hq[4];
#pragma unroll
    for (int i = 0; i < 4; ++i) hq[i] = *(const LAS u32x4*)(lo + O_HI + lane * 64u + i * 16u);
    raw = __builtin_amdgcn_ballot_w64((wide & 0xf0f0f0f0u) != 0) != 0;
    tri = __builtin_amdgcn_ballot_w64((wide & 0xf8f8f8f8u) != 0) == 0;
    if (tri) {
#pragma unroll
      for (int h = 0; h < 2; ++h) {
        const u32x4 x = hq[2 * h], y = hq[2 * h + 1];
        LAS u32* const pw = (LAS u32*)(planes + lane * 24u + h * 12u);
        pw[0] = x[0] | (x[1] << 3) | ((x[2] & 0x03030303u) << 6);
        pw[1] = x[3] | (y[0] << 3) | ((y[1] & 0x03030303u) << 6);
        pw[2] = y[2] | (y[3] << 3) | ((x[2] & 0x04040404u) << 4) | ((y[1] & 0x04040404u) << 5);
      }
    } else {
#pragma unroll
      for (int i = 0; i < 2; ++i) *(LAS u32x4*)(planes + lane * 32u + i * 16u) = hq[i] | (hq[2 + i] << 4);
    }
    total = MF * 256u + ((n + 15u) & ~15u) + ((n + 63u) >> 6) * (tri ? 24u : 32u);
    asm volatile("" ::: "memory");
  }
  if (raw) {
    total = 4096u * MF;
#pragma unroll
    for (int m = 0; m < MF; ++m)
#pragma unroll
      for (int i = 0; i < 4; ++i)
        store16((u32x4){__builtin_amdgcn_perm(hb[m * 8 + 2 * i], lb[m * 8 + 2 * i], 0x05010400u), __builtin_amdgcn_perm(hb[m * 8 + 2 * i], lb[m * 8 + 2 * i], 0x07030602u),
                        __builtin_amdgcn_perm(hb[m * 8 + 2 * i + 1], lb[m * 8 + 2 * i + 1], 0x05010400u), __builtin_amdgcn_perm(hb[m * 8 + 2 * i + 1], lb[m * 8 + 2 * i + 1], 0x07030602u)},
                rb2, lane * 16u, (m * 4 + i) * 1024u);
  } else {
    for (u32 p = 0; p < total; p += 1024u) store16(*(const LAS u32x4*)(scr + p + lane * 16u), rb2, p + lane * 16u < total ? lane * 16u : 0x80000000u, p);
  }
  asm volatile("s_waitcnt vmcnt(0)" : "+v"(qi) :: "memory");
  const u32 slot = ((u32)__builtin_amdgcn_readfirstlane(qi) - (call - 1u) * OUNITS) & 255u;
  flag_store(pick(P.ws, (g4 ^ 4u) | j) + OFF_OQ + (r & 3u) * 2048u + slot * 8u, ((u64)call << 32) | uid | (total << 8) | (raw ? 1u << 22 : 0u) | (top << 23) | (tri ? 1u << 31 : 0u), 1u);
}

__global__ __launch_bounds__(512, 2) void ag_gemm_kernel(Params P) {
  extern __shared__ __attribute__((aligned(16))) char smem[];
  LAS char* lds = (LAS char*)smem;
  const u32 lane = threadIdx.x & 63u, wid = __builtin_amdgcn_readfirstlane(threadIdx.x >> 6);
  char* const ws = pick(P.ws, P.rank);
  u32 bid = blockIdx.x;
  asm volatile("" : "+s"(bid));
  char* const cptr = ws + OFF_CALL + bid * 128u + wid * 4u;
  const u32 callv = ld_agent(cptr);
  LAS unsigned char* scr = (LAS unsigned char*)lds + wid * WAVE_SCRATCH;
  build_tables(lds, lane);
  const u32 li = bid < 7u * NCL ? bid % 7u : 7u, lidx = bid / 7u, pkw = bid - 7u * NCL;
  const u32 xcd = bid & 7u, pos = xcd * 26u + (xcd > 4u ? xcd - 4u : 0u) + (bid >> 3) - (xcd < 4u ? 6u : 5u);
  const bool foreign = pos >= 4u * NOT;
  const u32 ent = foreign ? pos - 4u * NOT : pos;
  Tile T;
  T.rt = ent >> 2;
  T.ct = (ent & 3u) + (foreign ? 4u : 0u);
  T.gf0 = foreign ? foreign_base(T.rt) : T.rt * 12u;
  T.nfr = foreign ? (T.rt < 4u ? 10u : 8u) : (T.rt == NOT - 1u ? 4u : 12u);
  LAS u32* const ctl = (LAS u32*)(lds + 8 * WAVE_SCRATCH);
  if (bid < NCOMM) {
    ctl[lane & 3u] = 0;
    asm volatile("s_waitcnt lgkmcnt(0)\n\ts_barrier" ::: "memory");
  }
  const bool isg = bid >= NCOMM;
  u32 call;
  if (li < 7u) {
    call = __builtin_amdgcn_readfirstlane(callv) + 1u;
    puller(P, ws, call, li, lidx, ctl, scr, lane, wid);
  } else {
    const u32 q0 = isg ? wid * NGEMM + (bid - NCOMM) : GPACK + wid * NPKWG + pkw;
    const u32 q1 = isg ? (T.nfr == 8u ? 8u * NGEMM + wid * 108u + ent - 16u : GPACK) : q0 + NPK;
    for (u32 i = isg ? wid : 0u; i; --i) __builtin_amdgcn_s_sleep(28);
    call = packer(P, ws, callv, q0, q1, isg ? 864u : (u32)NPK, isg ? GPACK : (u32)NUNITS, scr, lane);
  }
  __hip_atomic_store((GAS u32*)cptr, call, __ATOMIC_RELAXED, __HIP_MEMORY_SCOPE_AGENT);
  if (!isg) {
    const u32 hi = li < 3u ? li * NCL + lidx : 3u * NCL + pkw;
    if (li - 3u < 4u || hi < 4u * HELP) out_puller(P, ws, call, li - 3u < 4u ? li - 3u : hi & 3u, ctl, scr, lane);
    return;
  }
  asm volatile("s_waitcnt lgkmcnt(0)\n\ts_barrier" ::: "memory");
  if (T.nfr == 8u) {
    if (wid < 2u) gemm_wave<2, 4, 2>(P, ws, call, T, (wid & 3u) * 2u, lds, lane, wid);
    else if (wid < 4u) gemm_wave<2, 8, 2>(P, ws, call, T, (wid & 3u) * 2u, lds, lane, wid);
  } else if (T.nfr == 10u) {
    if (wid < 2u) gemm_wave<3, 5, 2>(P, ws, call, T, foreign_frag0(0u, wid), lds, lane, wid);
  } else {
    if (wid < 2u) gemm_wave<3, 6, 1>(P, ws, call, T, (wid & 3u) * 3u, lds, lane, wid);
    else if (wid >= 4u) gemm_wave<3, 0, 1>(P, ws, call, T, (wid & 3u) * 3u, lds, lane, wid);
  }
  if (T.nfr == 10u ? wid >= 6u : (T.ct < 4u && wid - 2u < 2u)) gemm_wave<3, 8, 3>(P, ws, call, T, T.ct < 4u ? (wid & 3u) * 3u : foreign_frag0(0u, wid), lds, lane, wid);
  else if (T.nfr == 10u ? wid - 2u < 4u : (T.nfr == 8u && wid >= 4u)) gemm_wave<2, 0, 2>(P, ws, call, T, foreign_frag0(T.rt, wid), lds, lane, wid);
}

}

extern "C" void kernel_launch(int rank, int nranks, ncclComm_t comm, hipStream_t stream,
                              void* const* d_in, const int* in_sizes, int n_in,
                              void* d_out, int out_size, const int* in_dtypes, int out_dtype,
                              void* const* d_out_all, void* const* d_ws_all, size_t ws_bytes) {
  (void)comm; (void)in_dtypes; (void)out_dtype; (void)d_out;
  if (nranks != ag::NR || n_in != 2 || (unsigned)rank >= (unsigned)ag::NR || in_sizes[0] != 1024 * 8192 || in_sizes[1] != 8192 * 1024 ||
      out_size != 8192 * 1024 || ws_bytes < ag::WS_NEED) {
    fprintf(stderr, "ag_gemm: unexpected problem shape\n");
    return;
  }
  ag::Params p;
  p.x = (const char*)d_in[0];
  p.w = (const char*)d_in[1];
  for (int i = 0; i < ag::NR; ++i) {
    const int phys = (ag::ORDER >> (4 * i)) & 7;
    p.ws[i] = (char*)d_ws_all[phys];
    p.out[i] = (char*)d_out_all[phys];
    if (phys == rank) p.rank = i;
  }
  p.pad = 0;
  hipError_t e = hipFuncSetAttribute((const void*)ag::ag_gemm_kernel, hipFuncAttributeMaxDynamicSharedMemorySize, ag::LDS_BYTES);
  if (e != hipSuccess) { fprintf(stderr, "ag_gemm: %s\n", hipGetErrorString(e)); return; }
  hipLaunchKernelGGL(ag::ag_gemm_kernel, dim3(ag::NWG), dim3(512), ag::LDS_BYTES, stream, p);
  e = hipGetLastError();
  if (e != hipSuccess) fprintf(stderr, "ag_gemm launch: %s\n", hipGetErrorString(e));
}
```

```cpp
#include <hip/hip_runtime.h>
#include <rccl/rccl.h>
#include <cstdio>

namespace ag {

typedef unsigned u32;
typedef unsigned long long u64;
typedef unsigned u32x4 __attribute__((ext_vector_type(4)));
typedef unsigned short u16x2 __attribute__((ext_vector_type(2)));
typedef short bf16x8 __attribute__((ext_vector_type(8)));
typedef float f32x4 __attribute__((ext_vector_type(4)));
typedef __amdgpu_buffer_rsrc_t rsrc_t;
#define LAS __attribute__((address_space(3)))
#define GAS __attribute__((address_space(1)))

constexpr int NR = 8;
constexpr u32 ORDER = 0x75416320u;
constexpr int AU = 21;
constexpr int AWAY = 32 - AU;
constexpr int NKC = 64, NKS = 256;
constexpr int EARLY_UNITS = AU + 4 * AWAY;
constexpr u32 NFT = 31, NOT = 22, ROW_OWN = NFT, ROW_W = NFT + NOT;
constexpr int NOUT = AWAY + 8;

constexpr int NCOMM = 44, NGEMM = 4 * (NFT + NOT), NWG = NCOMM + NGEMM;
constexpr int NCL = 6;
constexpr int NPKWG = NCOMM - 7 * NCL;
static_assert(NWG == 256 && NPKWG > 0, "one workgroup per CU");
constexpr int NPK = NPKWG * 8;
constexpr u32 GPACK = 8 * NGEMM + 2240;
constexpr int NUNITS = 4096;
constexpr u32 LEAD = 6, RAMP = 2;
constexpr u32 HALVES = 4, OUT_TOK = 2;
constexpr u32 HELP = 4;

constexpr u32 A_STEP = 262144, B_STEP = 16384;
constexpr u32 B_TILE = B_STEP * NKS;
constexpr u32 OFF_XU = 0;
constexpr u32 OFF_WU = OFF_XU + NKS * A_STEP;
constexpr u32 FRAG_BYTES = OFF_WU + 8 * B_TILE;
constexpr u32 SLOT = 8192, PK_BYTES = NUNITS * SLOT;
constexpr u32 OFF_PK = FRAG_BYTES;
constexpr u32 OSLOT = 12288, OUNITS = NFT * 8;
constexpr u32 OFF_OB = OFF_PK + 2 * PK_BYTES;
constexpr u32 IB_SRC = 64 * SLOT, IB_BYTES = 2 * NR * IB_SRC;
constexpr u32 OFF_IB = OFF_OB + 4 * OUNITS * OSLOT;
constexpr u32 OFF_G = OFF_IB + IB_BYTES;
constexpr u32 G_BYTES = 2 * NR * NUNITS * 8;
constexpr u32 OFF_CNT = OFF_G + G_BYTES;
constexpr u32 OFF_OQ = OFF_CNT + 16384;
constexpr u32 OFF_TKT = OFF_OQ + 4 * 2048;
constexpr u32 OFF_TMO = OFF_TKT + 2048;
constexpr u32 OFF_CALL = OFF_TMO + 128;
constexpr u32 WS_NEED = OFF_CALL + NWG * 128;

constexpr u32 A_LDS = 12288, SLOTB = A_LDS + B_STEP, NSLOT = 5;
constexpr u32 TAB_OFF = 163840 - 256;
static_assert(NSLOT * SLOTB + 12160 <= TAB_OFF && 7 * 12160 <= 3 * SLOTB, "LDS map");
constexpr int LDS_BYTES = TAB_OFF + 256;
constexpr int WAVE_SCRATCH = 13312;
static_assert(8 * WAVE_SCRATCH + 64 <= LDS_BYTES, "scratch");

constexpr u32 SPARE = 4000, SPARE_CNT = SPARE | SPARE << 16;
constexpr u64 DEADLINE = 50000000ull;
constexpr int AUX_SYS = 17, AUX_NT = 2;

struct Params {
  const char* x;
  const char* w;
  char* ws[NR];
  char* out[NR];
  int rank;
  int pad;
};

__device__ __forceinline__ rsrc_t make_rsrc(const void* p, u32 bytes) {
  const u64 a = (u64)p;
  const u32 lo = __builtin_amdgcn_readfirstlane((u32)a), hi = __builtin_amdgcn_readfirstlane((u32)(a >> 32));
  return __builtin_amdgcn_make_buffer_rsrc((void*)(((u64)hi << 32) | lo), 0, bytes, 0x00020000);
}
__device__ __forceinline__ char* pick(char* const (&t)[NR], u32 i) {
  char* p = t[0];
#pragma unroll
  for (u32 j = 1; j < NR; ++j) p = (i == j) ? t[j] : p;
  return p;
}
__device__ __forceinline__ u32 ld_agent(const char* p) { return __hip_atomic_load((GAS u32*)p, __ATOMIC_RELAXED, __HIP_MEMORY_SCOPE_AGENT); }
__device__ __forceinline__ void store16(u32x4 v, rsrc_t rs, u32 voff, u32 soff) {
  __builtin_amdgcn_raw_buffer_store_b128(v, rs, voff, soff, AUX_SYS);
  asm volatile("s_nop 1" :: "v"(v) : "memory");
}
__device__ __forceinline__ u32 pair_offset(u32 c, u32 lane) { return ((lane & 1u) ? c >> 16 : c & 0xffffu) * 4u; }
__device__ __forceinline__ u64 pair_mask(u32 c) { return (c >> 16) == SPARE ? 1u : 3u; }
__device__ __forceinline__ void count_in(const char* base, u32 c, u32 lane) {
  u64 keep;
  asm volatile("s_mov_b64 %0, exec\n\ts_mov_b64 exec, %4\n\tglobal_atomic_add %1, %2, %3\n\ts_mov_b64 exec, %0"
               : "=&s"(keep) : "v"(pair_offset(c, lane)), "v"(1u), "s"(base), "s"(pair_mask(c)) : "memory");
}
__device__ __forceinline__ void flag_store(char* p, u64 v, u64 mask) {
  u64 keep;
  asm volatile("s_mov_b64 %0, exec\n\ts_mov_b64 exec, %3\n\tglobal_store_dwordx2 %1, %2, off sc0 sc1\n\ts_mov_b64 exec, %0"
               : "=&s"(keep) : "v"(p), "v"(v), "s"(mask) : "memory");
}
__device__ __forceinline__ void drain_vm() { asm volatile("s_waitcnt vmcnt(0)" ::: "memory"); }

__device__ __forceinline__ bool give_up(char* ws, u32 call, u64 t0) {
  const u32 t = __builtin_amdgcn_readfirstlane(ld_agent(ws + OFF_TMO));
  if (t == call) return true;
  if (__builtin_amdgcn_s_memrealtime() - t0 > DEADLINE) {
    __hip_atomic_store((GAS u32*)(ws + OFF_TMO), call, __ATOMIC_RELAXED, __HIP_MEMORY_SCOPE_AGENT);
    return true;
  }
  return false;
}

__device__ __forceinline__ u32 wave_sum(u32 v) {
  v += __builtin_amdgcn_update_dpp(0u, v, 0x111, 0xf, 0xf, true);
  v += __builtin_amdgcn_update_dpp(0u, v, 0x112, 0xf, 0xf, true);
  v += __builtin_amdgcn_update_dpp(0u, v, 0x114, 0xf, 0xf, true);
  v += __builtin_amdgcn_update_dpp(0u, v, 0x118, 0xf, 0xf, true);
  return __builtin_amdgcn_readlane(v, 15) + __builtin_amdgcn_readlane(v, 31) + __builtin_amdgcn_readlane(v, 47) + __builtin_amdgcn_readlane(v, 63);
}
__device__ __forceinline__ u32 wave_max(u32 v) {
  v = max(v, (u32)__builtin_amdgcn_update_dpp(0u, v, 0x111, 0xf, 0xf, true));
  v = max(v, (u32)__builtin_amdgcn_update_dpp(0u, v, 0x112, 0xf, 0xf, true));
  v = max(v, (u32)__builtin_amdgcn_update_dpp(0u, v, 0x114, 0xf, 0xf, true));
  v = max(v, (u32)__builtin_amdgcn_update_dpp(0u, v, 0x118, 0xf, 0xf, true));
  return max(max((u32)__builtin_amdgcn_readlane(v, 15), (u32)__builtin_amdgcn_readlane(v, 31)), max((u32)__builtin_amdgcn_readlane(v, 47), (u32)__builtin_amdgcn_readlane(v, 63)));
}
__device__ __forceinline__ u32 wave_scan(u32 v) {
  v += __builtin_amdgcn_update_dpp(0u, v, 0x111, 0xf, 0xf, true);
  v += __builtin_amdgcn_update_dpp(0u, v, 0x112, 0xf, 0xf, true);
  v += __builtin_amdgcn_update_dpp(0u, v, 0x114, 0xf, 0xf, true);
  v += __builtin_amdgcn_update_dpp(0u, v, 0x118, 0xf, 0xf, true);
  v += __builtin_amdgcn_update_dpp(0u, v, 0x142, 0xa, 0xf, false);
  v += __builtin_amdgcn_update_dpp(0u, v, 0x143, 0xc, 0xf, false);
  return v;
}

__device__ __forceinline__ u32 quad_swz(u32 rq) { return (0x78u >> (2 * rq)) & 3u; }

struct Dest { u32 base, s1, s0, cnt; };
__device__ __forceinline__ Dest dest_x(u32 ux, u32 kc) {
  const u32 ft = ux < 20u ? ux / 5u : (ux - 4u) >> 2, ot = ROW_OWN + ux / 6u;
  return Dest{OFF_XU + 4 * kc * A_STEP + ux * 2048u, 1024u, A_STEP, (ft * 64 + kc) | (ot * 64 + kc) << 16};
}
__device__ __forceinline__ Dest dest_w(u32 ct, u32 kc, u32 ks4, u32 half) {
  return Dest{OFF_WU + ct * B_TILE + (4 * kc + ks4) * B_STEP + half * 8192u, 4096u, 1024u, ((ROW_W + ct) * 64 + kc) | SPARE << 16};
}
__device__ __forceinline__ void store_frags(const u32x4 (&v)[8], rsrc_t rf, const Dest& d, u32 lane) {
#pragma unroll
  for (int f = 0; f < 8; ++f) store16(v[f], rf, lane * 16, d.base + (f >> 2) * d.s1 + (f & 3) * d.s0);
}

constexpr u32 RAW = 1u << 31, PUSHED = 1u << 30;
constexpr u32 S_LIST = 0, S_TAIL = 2304, S_RARE = 3584;
struct Packed { u32x4 mq[4], p01, tail; };

__device__ __forceinline__ void build_tables(LAS char* lds, u32 lane) {
  u32 ct = 0x0c0c0c0cu, xt = 0x07060504u, zt = 0x0c0c0c0cu, k = 0;
#pragma unroll
  for (u32 b = 0; b < 4; ++b) {
    const u32 on = 0u - ((lane >> b) & 1u);
    ct ^= on & ((0x0cu ^ b) << (8u * k));
    xt ^= on & (((4u + b) ^ k) << (8u * b));
    zt ^= on & ((0x0cu ^ k) << (8u * b));
    k -= on;
  }
  *(LAS u32*)(lds + TAB_OFF + lane * 4u) = (lane & 32u) ? zt : ((lane & 16u) ? xt : ct);
}

constexpr u32 TAB = TAB_OFF;

template <bool TO_LIST>
__device__ __forceinline__ void walk_spelled(u32 m, u32 off, LAS unsigned char* cells, LAS unsigned char* list, u32 flip) {
  while (m != 0) {
    const u32 p = __builtin_ctz(m);
    m &= m - 1;
    LAS unsigned char* cell = cells + (p & 7u) * 4u + (p >> 3);
    if (TO_LIST) list[off] = *cell ^ flip; else *cell = list[off];
    ++off;
  }
}

__device__ __forceinline__ u32 encode_unit(u32x4& tail, const u32x4 (&v)[8], rsrc_t rp, u32 soff, LAS unsigned char* scr, u32 lane) {
  u32x4 mq[4];
  const u32x4 zero = {0u, 0u, 0u, 0u};
  *(LAS u32x4*)(scr + S_LIST + lane * 32u) = zero;
  *(LAS u32x4*)(scr + S_LIST + lane * 32u + 16u) = zero;
  *(LAS u32x4*)(scr + S_RARE + lane * 16u) = zero;
  u32 E[16], esum = 0;
#pragma unroll
  for (int j = 0; j < 16; ++j) {
    const u32 d0 = v[j >> 1][(j & 1) * 2], d1 = v[j >> 1][(j & 1) * 2 + 1];
    const u32 r0 = __builtin_amdgcn_alignbit(d0, d0, 31), r1 = __builtin_amdgcn_alignbit(d1, d1, 31);
    mq[j >> 2][j & 3] = __builtin_amdgcn_perm(r1, r0, 0x06040200u);
    E[j] = __builtin_amdgcn_perm(r1, r0, 0x07050301u);
    esum = __builtin_amdgcn_sad_u8(E[j], 0u, esum);
  }
#pragma unroll
  for (int i = 0; i < 4; ++i) store16(mq[i], rp, lane * 16, soff + i * 1024);
  esum = wave_sum(esum);
  u32 base = (esum + 8028u) >> 12;
  base = base < 19u ? 19u : (base > 254u ? 254u : base);
  const u32 mode = base * 4096u < esum + 6361u ? 1u : 0u;
  const u16x2 b2 = {(unsigned short)base, (unsigned short)base};
  u32 P[2][2] = {{0, 0}, {0, 0}};
#pragma unroll
  for (int j = 0; j < 16; ++j) {
    const u16x2 ev = __builtin_bit_cast(u16x2, E[j] & 0x00ff00ffu), od = __builtin_bit_cast(u16x2, E[j]) >> (u16x2){8, 8};
    const u32 ce = __builtin_bit_cast(u32, __builtin_elementwise_min((u16x2)(b2 - ev), (u16x2){3, 3}));
    const u32 co = __builtin_bit_cast(u32, __builtin_elementwise_min((u16x2)(b2 - od), (u16x2){3, 3}));
    const u32 c = ce | (co << 8);
    P[0][j >> 3] |= (c & 0x01010101u) << (j & 7);
    P[1][j >> 3] |= ((c >> 1) & 0x01010101u) << (j & 7);
  }
  store16((u32x4){P[0][0], P[0][1], P[1][0], P[1][1]}, rp, lane * 16, soff + 4096);
  tail = zero;
  const u32 mk[2] = {P[0][0] & P[1][0], P[0][1] & P[1][1]};
  u32 fl[16], sel[16];
#pragma unroll
  for (int j = 0; j < 16; ++j) {
    fl[j] = (mk[j >> 3] >> (j & 7)) & 0x01010101u;
    sel[j] = *(const LAS u32*)(size_t)__builtin_amdgcn_udot4(fl[j], 0x20100804u, TAB, false);
  }
  __builtin_amdgcn_sched_barrier(0);
  const u32 c = __builtin_popcount(mk[0]) + __builtin_popcount(mk[1]), incl = wave_scan(c);
  const u32 n2 = __builtin_amdgcn_readlane(incl, 63);
  if (n2 > 2044u) return RAW;
  const u32 flip = (base - 3u) * 0x01010101u;
  u32 a = (u32)(size_t)(scr + S_LIST) + incl - c, cur = 0;
#pragma unroll
  for (int j = 0; j < 16; ++j) {
    const u64 t = (u64)__builtin_amdgcn_perm(0u, E[j] ^ flip, sel[j]) << ((a & 3u) * 8u);
    const u32 w = cur | (u32)t, an = __builtin_amdgcn_sad_u8(fl[j], 0u, a);
    __hip_atomic_fetch_or((LAS u32*)(size_t)(a & ~3u), w, __ATOMIC_RELAXED, __HIP_MEMORY_SCOPE_WAVEFRONT);
    cur = ((an ^ a) & 4u) ? (u32)(t >> 32) : w;
    a = an;
  }
  __hip_atomic_fetch_or((LAS u32*)(size_t)(a & ~3u), cur, __ATOMIC_RELAXED, __HIP_MEMORY_SCOPE_WAVEFRONT);
  asm volatile("" ::: "memory");
  const u32x4 l[2] = {*(const LAS u32x4*)(scr + S_LIST + lane * 32u), *(const LAS u32x4*)(scr + S_LIST + lane * 32u + 16u)};
  const unsigned short ta = mode ? base + 1u : base - 3u, tb = mode ? 3u : 0u, tc = base + 1u;
  u32 Q[2] = {0, 0};
#pragma unroll
  for (int s = 0; s < 8; ++s) {
    const u32 x = l[s >> 2][s & 3] ^ flip;
    const u16x2 ev = __builtin_bit_cast(u16x2, x & 0x00ff00ffu), od = __builtin_bit_cast(u16x2, x) >> (u16x2){8, 8};
    const u32 ce = __builtin_bit_cast(u32, __builtin_elementwise_min(__builtin_elementwise_sub_sat((u16x2)((u16x2){ta, ta} - ev), (u16x2){tb, tb}), (u16x2){3, 3}));
    const u32 co = __builtin_bit_cast(u32, __builtin_elementwise_min(__builtin_elementwise_sub_sat((u16x2)((u16x2){ta, ta} - od), (u16x2){tb, tb}), (u16x2){3, 3}));
    const u32 c2 = ce | (co << 8);
    Q[0] |= (c2 & 0x01010101u) << s;
    Q[1] |= ((c2 >> 1) & 0x01010101u) << s;
  }
  *(LAS u32*)(scr + S_TAIL + lane * 8u) = Q[0];
  *(LAS u32*)(scr + S_TAIL + lane * 8u + 4u) = Q[1];
  const u32 m3 = Q[0] & Q[1];
#pragma unroll
  for (int s = 0; s < 8; ++s) {
    fl[s] = (m3 >> s) & 0x01010101u;
    sel[s] = *(const LAS u32*)(size_t)__builtin_amdgcn_udot4(fl[s], 0x20100804u, TAB, false);
  }
  __builtin_amdgcn_sched_barrier(0);
  const u32 c3 = __builtin_popcount(m3), incl3 = wave_scan(c3), n3 = __builtin_amdgcn_readlane(incl3, 63);
  if (n3 > 768u) return RAW;
  const u32 turn = flip ^ (tc * 0x01010101u);
  a = (u32)(size_t)(scr + S_RARE) + incl3 - c3;
  cur = 0;
#pragma unroll
  for (int s = 0; s < 8; ++s) {
    const u64 t = (u64)__builtin_amdgcn_perm(0u, l[s >> 2][s & 3] ^ turn, sel[s]) << ((a & 3u) * 8u);
    const u32 w = cur | (u32)t, an = __builtin_amdgcn_sad_u8(fl[s], 0u, a);
    __hip_atomic_fetch_or((LAS u32*)(size_t)(a & ~3u), w, __ATOMIC_RELAXED, __HIP_MEMORY_SCOPE_WAVEFRONT);
    cur = ((an ^ a) & 4u) ? (u32)(t >> 32) : w;
    a = an;
  }
  __hip_atomic_fetch_or((LAS u32*)(size_t)(a & ~3u), cur, __ATOMIC_RELAXED, __HIP_MEMORY_SCOPE_WAVEFRONT);
  asm volatile("" ::: "memory");
  const u32x4 rq = *(const LAS u32x4*)(scr + S_RARE + lane * 16u);
  u32 cn[4], m4 = 0;
#pragma unroll
  for (int s = 0; s < 4; ++s) {
    const u32 x = rq[s] ^ (tc * 0x01010101u);
    const u16x2 ev = __builtin_bit_cast(u16x2, x & 0x00ff00ffu), od = __builtin_bit_cast(u16x2, x) >> (u16x2){8, 8};
    const u32 ce = __builtin_bit_cast(u32, __builtin_elementwise_min(__builtin_elementwise_sub_sat((u16x2)((u16x2){tc, tc} - ev), (u16x2){5, 5}), (u16x2){15, 15}));
    const u32 co = __builtin_bit_cast(u32, __builtin_elementwise_min(__builtin_elementwise_sub_sat((u16x2)((u16x2){tc, tc} - od), (u16x2){5, 5}), (u16x2){15, 15}));
    cn[s] = ce | (co << 8);
    m4 |= ((cn[s] + 0x01010101u) & 0x10101010u) >> 4 << s;
  }
  const u32 nib = ((n2 + 31u) >> 5) * 8u;
  u32 bytes = nib + ((n3 + 15u) >> 4) * 8u;
  *(LAS u32*)(scr + S_TAIL + nib + lane * 8u) = cn[0] | (cn[1] << 4);
  *(LAS u32*)(scr + S_TAIL + nib + lane * 8u + 4u) = cn[2] | (cn[3] << 4);
  if (__builtin_amdgcn_ballot_w64(m4 != 0) != 0) {
    const u32 c4 = __builtin_popcount(m4), incl4 = wave_scan(c4), n4 = __builtin_amdgcn_readlane(incl4, 63);
    if (n4 > 128u) return RAW;
    asm volatile("" ::: "memory");
    walk_spelled<true>(m4, incl4 - c4, scr + S_RARE + lane * 16u, scr + S_TAIL + bytes, tc);
    bytes += n4;
  }
  asm volatile("" ::: "memory");
  tail = *(const LAS u32x4*)(scr + S_TAIL + lane * 16u);
  return base | (mode << 8) | (bytes << 16);
}

__device__ __forceinline__ void store_raw(const u32x4 (&v)[8], rsrc_t rp, u32 soff, u32 lane) {
#pragma unroll
  for (int f = 0; f < 8; ++f) store16(v[f], rp, lane * 16, soff + f * 1024);
}

__device__ __forceinline__ void decode_unit(u32x4 (&v)[8], const Packed& k, u32 info, LAS unsigned char* scr, u32 lane) {
  *(LAS u32x4*)(scr + S_TAIL + lane * 16u) = k.tail;
  asm volatile("" ::: "memory");
  const u32 q0 = *(const LAS u32*)(scr + S_TAIL + lane * 8u), q1 = *(const LAS u32*)(scr + S_TAIL + lane * 8u + 4u);
  const u32 b4 = (info & 255u) * 0x01010101u, lut = (info & 256u) ? b4 - 0x00040300u + 1u : b4 - 0x00050403u;
  const u32 P0[2] = {k.p01.x, k.p01.y}, P1[2] = {k.p01.z, k.p01.w}, mk[2] = {P0[0] & P1[0], P0[1] & P1[1]};
  const u32 c = __builtin_popcount(mk[0]) + __builtin_popcount(mk[1]), incl = wave_scan(c);
  const u32 g2 = (((u32)__builtin_amdgcn_readlane(incl, 63) + 31u) >> 5) & 127u;
  LAS unsigned char* const nib = scr + S_TAIL + (g2 < 64u ? g2 : 64u) * 8u;
  const u32 x0 = *(const LAS u32*)(nib + lane * 8u), x1 = *(const LAS u32*)(nib + lane * 8u + 4u);
  u32 a[17], sel[16];
  a[0] = (u32)(size_t)(scr + S_LIST) + incl - c;
#pragma unroll
  for (int j = 0; j < 16; ++j) {
    const u32 f = (mk[j >> 3] >> (j & 7)) & 0x01010101u;
    sel[j] = *(const LAS u32*)(size_t)__builtin_amdgcn_udot4(f, 0x20100804u, TAB + 64u, false);
    a[j + 1] = __builtin_amdgcn_sad_u8(f, 0u, a[j]);
  }
  __builtin_amdgcn_sched_barrier(0);
  const u32 live = lane < g2 ? ~0u : 0u, Q0 = q0 & live, Q1 = q1 & live, m3 = Q0 & Q1;
  const u32 c3 = __builtin_popcount(m3), incl3 = wave_scan(c3), g3 = (((u32)__builtin_amdgcn_readlane(incl3, 63) + 15u) >> 4) & 63u;
  const u32 live3 = lane < g3 ? 0x0f0f0f0fu : 0u, cn[4] = {x0 & live3, (x0 >> 4) & live3, x1 & live3, (x1 >> 4) & live3};
  u32x4 rq;
  u32 any = 0;
#pragma unroll
  for (int s = 0; s < 4; ++s) {
    const u32 z = ~(cn[s] + 0x0f0f0f0fu) & 0x10101010u;
    rq[s] = b4 - 0x04040404u - cn[s] + (z >> 4) + (z >> 2);
    any |= cn[s] + 0x01010101u;
  }
  *(LAS u32x4*)(scr + S_RARE + lane * 16u) = rq;
  if (__builtin_amdgcn_ballot_w64((any & 0x10101010u) != 0) != 0) {
    u32 m4 = 0;
#pragma unroll
    for (int s = 0; s < 4; ++s) m4 |= ((cn[s] + 0x01010101u) & 0x10101010u) >> 4 << s;
    const u32 c4 = __builtin_popcount(m4);
    walk_spelled<false>(m4, (wave_scan(c4) - c4) & 127u, scr + S_RARE + lane * 16u, nib + g3 * 8u, 0u);
  }
  asm volatile("" ::: "memory");
  u32x4 l[2];
  {
    u32 a3 = (u32)(size_t)(scr + S_RARE) + ((incl3 - c3) & 1023u), at[8], s3[8], r0[8], r1[8];
#pragma unroll
    for (int s = 0; s < 8; ++s) {
      const u32 f = (m3 >> s) & 0x01010101u;
      const LAS u32* const pw = (const LAS u32*)(size_t)(a3 & ~3u);
      at[s] = a3;
      s3[s] = *(const LAS u32*)(size_t)__builtin_amdgcn_udot4(f, 0x20100804u, TAB + 64u, false);
      r0[s] = pw[0];
      r1[s] = pw[1];
      a3 = __builtin_amdgcn_sad_u8(f, 0u, a3);
    }
    __builtin_amdgcn_sched_barrier(0);
#pragma unroll
    for (int s = 0; s < 8; ++s) {
      const u32 c2 = ((Q0 >> s) & 0x01010101u) | (((Q1 >> s) & 0x01010101u) << 1);
      l[s >> 2][s & 3] = __builtin_amdgcn_perm(__builtin_amdgcn_perm(0u, lut, c2), __builtin_amdgcn_alignbyte(r1[s], r0[s], at[s]), s3[s]);
    }
  }
  *(LAS u32x4*)(scr + S_LIST + lane * 32u) = l[0];
  *(LAS u32x4*)(scr + S_LIST + lane * 32u + 16u) = l[1];
  asm volatile("" ::: "memory");
  u32 w0[16], w1[16];
#pragma unroll
  for (int j = 0; j < 16; ++j) {
    const LAS u32* const pw = (const LAS u32*)(size_t)(a[j] & ~3u);
    w0[j] = pw[0];
    w1[j] = pw[1];
  }
  __builtin_amdgcn_sched_barrier(0);
#pragma unroll
  for (int j = 0; j < 16; ++j) {
    const int s = j & 7, wd = j >> 3;
    const u32 c1 = ((P0[wd] >> s) & 0x01010101u) | (((P1[wd] >> s) & 0x01010101u) << 1);
    const u32 e = __builtin_amdgcn_perm(b4 - c1, __builtin_amdgcn_alignbyte(w1[j], w0[j], a[j]), sel[j]), m = k.mq[j >> 2][j & 3];
    const u32 r0 = __builtin_amdgcn_perm(e, m, 0x05010400u), r1 = __builtin_amdgcn_perm(e, m, 0x07030602u);
    v[j >> 1][(j & 1) * 2] = __builtin_amdgcn_alignbit(r0, r0, 1);
    v[j >> 1][(j & 1) * 2 + 1] = __builtin_amdgcn_alignbit(r1, r1, 1);
  }
}

__device__ __forceinline__ void load_unit(u32x4 (&s)[8], rsrc_t rx, rsrc_t rw, u32 q, u32 vx, u32 vw) {
  const u32 kc = q >> 6, t = q & 63u;
  if (t < 32u) {
#pragma unroll
    for (int f = 0; f < 8; ++f) s[f] = __builtin_amdgcn_raw_buffer_load_b128(rx, vx, (32u * t + 16u * (f >> 2)) * 16384u + kc * 256u + (f & 3) * 64u, AUX_NT);
  } else {
#pragma unroll
    for (int i = 0; i < 8; ++i) s[i] = __builtin_amdgcn_raw_buffer_load_b128(rw, vw, (128u * kc + 32u * ((t - 32u) >> 3) + i) * 2048u + (t & 7u) * 256u, AUX_NT);
  }
}

__device__ __forceinline__ u32 packer(const Params& P, char* ws, u32 callv, u32 q0, u32 q1, u32 dq, u32 qend, LAS unsigned char* scr, u32 lane) {
  const rsrc_t rx = make_rsrc(P.x, 1024u * 8192u * 2u), rw = make_rsrc(P.w, 8192u * 1024u * 2u);
  const u32 kq = (lane & 3u) ^ quad_swz(lane >> 4);
  const u32 vx = (lane >> 2) * 16384u + kq * 16u, vw = (lane >> 4) * 8u * 2048u + (lane & 15u) * 16u;
  u32x4 s[8];
  load_unit(s, rx, rw, q0, vx, vw);
  __builtin_amdgcn_sched_barrier(0);
  const u32 call = __builtin_amdgcn_readfirstlane(callv) + 1u;
  const u32 r = P.rank, g4 = r & 4u, par = call & 1u;
  const rsrc_t rf = make_rsrc(ws, FRAG_BYTES), rp = make_rsrc(ws + OFF_PK + par * PK_BYTES, PK_BYTES);
  const u32 goff = OFF_G + (par * NR + r) * NUNITS * 8;
  char* g_in = pick(P.ws, g4 | ((r + 1u) & 3u));
  char* g_out = pick(P.ws, g4 ^ 4u);
#pragma unroll
  for (u32 j = 1; j < 4; ++j) {
    char* const pi = pick(P.ws, g4 | ((r + 1u + j) & 3u));
    char* const po = pick(P.ws, (g4 ^ 4u) | j);
    g_in = lane == j ? pi : g_in;
    g_out = lane == j ? po : g_out;
  }
  u64 psend = 0;
  char* pgp = g_out;
  u32 pinfo = 0, pcnt = SPARE_CNT;
  auto announce = [&]() {
    flag_store(pgp, ((u64)call << 32) | pinfo, psend);
    if (pcnt != SPARE_CNT) count_in(ws + OFF_CNT, pcnt, lane);
  };
  for (u32 q = q0; q < qend; q = q1, q1 += dq) {
    const u32 kc = q >> 6, t = q & 63u;
    u32x4 v[8];
    Dest d;
    bool own;
    u64 send;
    char* gp;
    if (t < 32u) {
#pragma unroll
      for (int f = 0; f < 8; ++f) v[f] = s[f];
      own = t < AU;
      d = dest_x(t, kc);
      send = own ? 7u : 15u;
      gp = own ? g_in : g_out;
    } else {
      const u32 ks4 = (t - 32u) >> 3, cg = t & 7u;
#pragma unroll
      for (int nf = 0; nf < 8; ++nf)
#pragma unroll
        for (int tt = 0; tt < 4; ++tt) v[nf][tt] = __builtin_amdgcn_perm(s[2 * tt + 1][nf >> 1], s[2 * tt][nf >> 1], (nf & 1) ? 0x07060302u : 0x05040100u);
      own = true;
      d = dest_w(cg >> 1, kc, ks4, cg & 1u);
      send = 1u << (cg >> 1);
      gp = g_out;
    }
    const bool lazy = qend != NUNITS;
    if (q1 < qend && !lazy) load_unit(s, rx, rw, q1, vx, vw);
    u32 info = RAW | PUSHED;
    if (kc == 0u && (t & 3u) < 3u) {
      for (u32 m = (u32)send; m; m &= m - 1u) {
        const u32 jd = __builtin_ctz(m), dst = (t < AU) ? (g4 | ((r + 1u + jd) & 3u)) : ((g4 ^ 4u) | jd);
        store_raw(v, make_rsrc(pick(P.ws, dst) + OFF_IB + (par * NR + r) * IB_SRC, IB_SRC), q * SLOT, lane);
      }
      if (own) store_frags(v, rf, d, lane);
    } else {
      if (own) store_frags(v, rf, d, lane);
      u32x4 tail;
      info = encode_unit(tail, v, rp, q * SLOT, scr, lane);
      if (!lazy || (info & RAW)) {
        drain_vm();
#pragma unroll
        for (int f = 0; f < 8; ++f) asm volatile("" : "+v"(s[f]));
        announce();
      }
      if (info & RAW) store_raw(v, rp, q * SLOT, lane);
      else store16(tail, rp, lane * 16u < (info >> 16) ? lane * 16u : 0x80000000u, q * SLOT + 5120u);
    }
    psend = send;
    pgp = gp + goff + q * 8u;
    pinfo = info;
    pcnt = own ? d.cnt : SPARE_CNT;
    if (q1 < qend && lazy) {
      drain_vm();
      announce();
      psend = 0;
      pcnt = SPARE_CNT;
      load_unit(s, rx, rw, q1, vx, vw);
    }
  }
  drain_vm();
  announce();
  return call;
}

__device__ __forceinline__ u32 lds_add(LAS u32* p, u32 v) {
  return __builtin_amdgcn_readfirstlane(__hip_atomic_fetch_add(p, v, __ATOMIC_RELAXED, __HIP_MEMORY_SCOPE_WORKGROUP)) >> 6;
}

__device__ __forceinline__ void puller(const Params& P, char* ws, u32 call, u32 li, u32 lidx, LAS u32* ctl, LAS unsigned char* scr, u32 lane, u32 wid) {
  const u32 r = P.rank, g4 = r & 4u, par = call & 1u;
  const bool ing = li < 3u;
  const u32 src = ing ? (g4 | ((r + 1 + li) & 3u)) : ((g4 ^ 4u) | (li - 3u));
  const rsrc_t rf = make_rsrc(ws, FRAG_BYTES), rp = make_rsrc(pick(P.ws, src) + OFF_PK + par * PK_BYTES, PK_BYTES);
  const rsrc_t ri = make_rsrc(ws + OFF_IB + (par * NR + src) * IB_SRC, IB_SRC);
  char* const gbase = ws + OFF_G + (par * NR + src) * NUNITS * 8;
  GAS u32* const cnt = (GAS u32*)(ws + OFF_CNT);
  const u32 total = ing ? NKC * AU : NKC * NOUT;
  constexpr u32 NONE = 0xffffffffu;
  const u32 idle = ((OFF_CALL - OFF_CNT + (lidx * 7u + li) * 128u + 64u + wid * 4u) >> 2) | SPARE << 16;
  GAS u32* const tkt = (GAS u32*)(ws + OFF_TKT + li * 128u);
  const u32 tbase = (call - 1u) * (total + NCL * 8u) * 64u;
  auto draw = [&]() -> u32 { return __hip_atomic_fetch_add(tkt, 1u, __ATOMIC_RELAXED, __HIP_MEMORY_SCOPE_AGENT); };
  auto unit_of = [&](u32 raw_ticket) -> u32 {
    const u32 i = ((u32)__builtin_amdgcn_readfirstlane(raw_ticket) - tbase) >> 6;
    if (i >= total) return NONE;
    if (ing) { const u32 kc = i / AU; return kc * 64u + (i - kc * AU); }
    constexpr u32 C1 = 8u * LEAD / RAMP, N1 = NOUT + RAMP, E1 = C1 * N1, E2 = E1 + (NKC - LEAD - C1) * NOUT;
    u32 kc, t, wi;
    if (i < E1) { kc = i / N1; t = i - kc * N1; wi = kc * (8u + RAMP) + t - AWAY; }
    else if (i < E2) { kc = (i - E1) / NOUT; t = i - E1 - kc * NOUT; kc += C1; wi = (kc + LEAD) * 8u + t - AWAY; }
    else { kc = (i - E2) / AWAY; t = i - E2 - kc * AWAY; kc += NKC - LEAD; wi = 0; }
    return t < AWAY ? kc * 64u + AU + t : (wi >> 3) * 64u + 32u + ((wi >> 1) & 3u) * 8u + 2u * (r & 3u) + (wi & 1u);
  };
  auto granule = [&](u32 q) -> u64 { return __hip_atomic_load((GAS u64*)(gbase + (q & (NUNITS - 1)) * 8u), __ATOMIC_RELAXED, __HIP_MEMORY_SCOPE_SYSTEM); };
  auto peek = [&](LAS u32* p) -> u32 { return (u32)(__builtin_amdgcn_readfirstlane(__hip_atomic_load(p, __ATOMIC_RELAXED, __HIP_MEMORY_SCOPE_WORKGROUP)) >> 6); };
  auto turn = [&](u32 my) { while (peek(ctl + 2) + HALVES <= my) __builtin_amdgcn_s_sleep(1); };
  auto dest_of = [&](u32 q) -> Dest {
    const u32 kc = q >> 6, tq = q & 63u;
    if (q == NUNITS) return Dest{FRAG_BYTES, 0u, 0u, NONE};
    if (tq < 32u) return dest_x(ing ? EARLY_UNITS + li * AU + tq : AU + (li - 3u) * AWAY + (tq - AU), kc);
    return dest_w(4u + (li - 3u), kc, (tq - 32u) >> 3, tq & 1u);
  };
  auto publish = [&](u32 c) { if (c != NONE) count_in(ws + OFF_CNT, c, lane); };
  u32 q = NUNITS, pend = NONE, tag = call, info = 127u;
  u64 gr;
  while (q != NONE) {
    if (tag != call || (!(info & PUSHED) && peek(ctl + 2) + HALVES <= peek(ctl))) {
      drain_vm();
      publish(pend);
      pend = NONE;
      if (tag != call) {
        const u64 t0 = __builtin_amdgcn_s_memrealtime();
        for (;;) {
          gr = granule(q);
          tag = __builtin_amdgcn_readfirstlane((u32)(gr >> 32));
          info = __builtin_amdgcn_readfirstlane((u32)gr);
          if (tag == call || give_up(ws, call, t0)) break;
          __builtin_amdgcn_s_sleep(2);
        }
        if (tag != call) break;
      }
    }
    u32 my = 0;
    if (!(info & PUSHED)) { my = lds_add(ctl, 2u); turn(my); }
    asm volatile("" ::: "memory");
    const Dest d = dest_of(q);
    const u32 cur = info, soff = q * SLOT;
    u32x4 v[8];
    if (cur & RAW) {
#pragma unroll
      for (int f = 0; f < 8; ++f) v[f] = __builtin_amdgcn_raw_buffer_load_b128((cur & PUSHED) ? ri : rp, lane * 16, soff + f * 1024, AUX_SYS);
      const u32 nt = draw();
      drain_vm();
      if (!(cur & PUSHED)) lds_add(ctl + 2, 2u);
      q = unit_of(nt);
      gr = granule(q);
      publish(pend);
      tag = __builtin_amdgcn_readfirstlane((u32)(gr >> 32));
      info = __builtin_amdgcn_readfirstlane((u32)gr);
      store_frags(v, rf, d, lane);
    } else {
      Packed k;
      u32 nt;
      u64 keep;
      asm volatile("s_mov_b64 %1, exec\n\ts_mov_b64 exec, 1\n\tglobal_atomic_add %0, %2, %3, %4 sc0\n\ts_mov_b64 exec, %1"
                   : "=&v"(nt), "=&s"(keep) : "v"(0u), "v"(64u), "s"(tkt) : "memory");
      k.tail = __builtin_amdgcn_raw_buffer_load_b128(rp, lane * 16u < (cur >> 16) ? lane * 16u : 0x80000000u, soff + 5120, AUX_SYS);
      k.p01 = __builtin_amdgcn_raw_buffer_load_b128(rp, lane * 16, soff + 4096, AUX_SYS);
      k.mq[0] = __builtin_amdgcn_raw_buffer_load_b128(rp, lane * 16, soff, AUX_SYS);
      asm volatile("" ::: "memory");
      turn(my + 1u);
      asm volatile("" ::: "memory");
#pragma unroll
      for (int i = 1; i < 4; ++i) k.mq[i] = __builtin_amdgcn_raw_buffer_load_b128(rp, lane * 16, soff + i * 1024, AUX_SYS);
      asm volatile("s_waitcnt vmcnt(5)" ::: "memory");
      lds_add(ctl + 2, 1u);
      asm volatile("s_waitcnt vmcnt(2)" ::: "memory");
      lds_add(ctl + 2, 1u);
      asm volatile("s_waitcnt vmcnt(0)" : "+v"(nt), "+v"(k.mq[0]), "+v"(k.mq[1]), "+v"(k.mq[2]), "+v"(k.mq[3]), "+v"(k.p01), "+v"(k.tail) :: "memory");
      q = unit_of(nt);
      asm volatile("global_load_dwordx2 %0, %2, %3 sc0 sc1\n\ts_mov_b64 %1, exec\n\ts_mov_b64 exec, %7\n\tglobal_atomic_add %4, %5, %6\n\ts_mov_b64 exec, %1"
                   : "=&v"(gr), "=&s"(keep) : "v"((q & (NUNITS - 1)) * 8u), "s"(gbase), "v"(pair_offset(pend == NONE ? idle : pend, lane)), "v"(1u), "s"(cnt),
                     "s"(pair_mask(pend == NONE ? idle : pend)) : "memory");
      decode_unit(v, k, cur, scr, lane);
      __builtin_amdgcn_sched_barrier(0);
      asm volatile("s_waitcnt vmcnt(1)" : "+v"(gr) :: "memory");
      tag = __builtin_amdgcn_readfirstlane((u32)(gr >> 32));
      info = __builtin_amdgcn_readfirstlane((u32)gr);
      __builtin_amdgcn_sched_barrier(0);
      store_frags(v, rf, d, lane);
    }
    pend = d.cnt;
  }
  drain_vm();
  publish(pend);
}

__device__ __forceinline__ u32 frag_row(u32 r, u32 gf) {
  const u32 g4 = r & 4u, ux = gf >> 1;
  u32 src, su;
  if (ux >= EARLY_UNITS) {
    const u32 j = (ux - EARLY_UNITS) / AU;
    src = g4 | ((r + 1 + j) & 3u);
    su = ux - EARLY_UNITS - j * AU;
  } else if (ux < AU) {
    src = r;
    su = ux;
  } else {
    const u32 j = (ux - AU) / AWAY;
    src = (g4 ^ 4u) | j;
    su = ux - j * AWAY;
  }
  return ((ORDER >> (4u * src)) & 7u) * 1024u + su * 32u + (gf & 1u) * 16u;
}
__device__ __forceinline__ u32 foreign_frag0(u32 ft, u32 wid) { return ft < 4u ? (0x74208630u >> (4 * wid)) & 15u : (wid & 3u) * 2u; }
__device__ __forceinline__ u32 foreign_base(u32 ft) { return ft < 4u ? ft * 10u : ft * 8u + 8u; }

constexpr u32 O_HI = 7168;
__device__ __forceinline__ void out_puller(const Params& P, char* ws, u32 call, u32 j, LAS u32* ctl, LAS unsigned char* scr, u32 lane) {
  const u32 r = P.rank, src = ((r & 4u) ^ 4u) | j;
  char* const ob = pick(P.ws, src) + OFF_OB + (r & 3u) * (OUNITS * OSLOT);
  const rsrc_t ro = make_rsrc(pick(P.out, r), 8192u * 1024u * 2u);
  GAS u32* const tkt = (GAS u32*)(ws + OFF_TKT + (7u + j) * 128u);
  const u32 tbase = (call - 1u) * (OUNITS + (NCL + HELP) * 8u) * 64u;
  const u32 vo = (lane >> 4) * 4u * 2048u + (lane & 15u) * 16u;
  char* const queue = ws + OFF_OQ + j * 2048u;
  u32 k = ((u32)__builtin_amdgcn_readfirstlane(__hip_atomic_fetch_add(tkt, 1u, __ATOMIC_RELAXED, __HIP_MEMORY_SCOPE_AGENT)) - tbase) >> 6;
  u32 tag = 0, info = 0;
  while (k < OUNITS) {
    if (tag != call) {
      const u64 t0 = __builtin_amdgcn_s_memrealtime();
      for (u32 spins = 0;; ++spins) {
        const u64 gr = __hip_atomic_load((GAS u64*)(queue + k * 8u), __ATOMIC_RELAXED, __HIP_MEMORY_SCOPE_SYSTEM);
        tag = __builtin_amdgcn_readfirstlane((u32)(gr >> 32));
        info = __builtin_amdgcn_readfirstlane((u32)gr);
        if (tag == call || ((spins & 7u) == 7u && give_up(ws, call, t0))) break;
        __builtin_amdgcn_s_sleep(3);
      }
      if (tag != call) {
        *(GAS unsigned short*)(pick(P.out, r) + (size_t)((ORDER >> (4u * r)) & 7u) * 1024u * 2048u) = 0x7fc0;
        break;
      }
    }
    {
      const u32 my = lds_add(ctl, 1u);
      while ((u32)(__builtin_amdgcn_readfirstlane(__hip_atomic_load(ctl + 2, __ATOMIC_RELAXED, __HIP_MEMORY_SCOPE_WORKGROUP)) >> 6) + OUT_TOK <= my) __builtin_amdgcn_s_sleep(1);
    }
    asm volatile("" ::: "memory");
    const u32 uid = (info & 255u) < OUNITS ? (info & 255u) : 0u, sw = uid & 7u, mf = (uid < 32u && sw - 2u >= 4u) ? 3u : 2u;
    u32 total = (info >> 8) & 0x3fffu;
    total = total < 12288u ? total : 12288u;
    const bool raw = (info >> 22) & 1u, tri = info >> 31;
    const u32 t4 = ((info >> 23) & 255u) * 0x01010101u;
    const rsrc_t rs = make_rsrc(ob + uid * OSLOT, (total + 15u) & ~15u);
    u32 nt;
    u64 keep, gr;
    asm volatile("s_mov_b64 %1, exec\n\ts_mov_b64 exec, 1\n\tglobal_atomic_add %0, %2, %3, %4 sc0\n\ts_mov_b64 exec, %1"
                 : "=&v"(nt), "=&s"(keep) : "v"(0u), "v"(64u), "s"(tkt) : "memory");
    for (u32 p = 0; p < total; p += 1024u) __builtin_amdgcn_raw_ptr_buffer_load_lds(rs, scr + p, 16, lane * 16u, p, 0, AUX_SYS);
    asm volatile("s_waitcnt vmcnt(2)" ::: "memory");
    lds_add(ctl + 2, 1u);
    asm volatile("s_waitcnt vmcnt(0)" : "+v"(nt) :: "memory");
    const u32 kn = ((u32)__builtin_amdgcn_readfirstlane(nt) - tbase) >> 6;
    asm volatile("global_load_dwordx2 %0, %1, %2 sc0 sc1" : "=&v"(gr) : "v"((kn & 255u) * 8u), "s"(queue) : "memory");
    u32 mk[3] = {0, 0, 0}, a = 0;
    if (!raw) {
#pragma unroll
      for (u32 m = 0; m < 3; ++m) mk[m] = *(const LAS u32*)(scr + m * 256u + lane * 4u);
      mk[2] = mf == 3u ? mk[2] : 0u;
      const u32 c = ((__builtin_popcount(mk[0]) + __builtin_popcount(mk[1]) + __builtin_popcount(mk[2])) & ~3u) + 4u, incl = wave_scan(c);
      u32 n = __builtin_amdgcn_readlane(incl, 63);
      n = n < 4096u ? n : 4096u;
      LAS unsigned char* const lo = scr + mf * 256u;
      LAS unsigned char* const planes = lo + ((n + 15u) & ~15u);
      a = (u32)(size_t)lo + incl - c;
      if (tri) {
#pragma unroll
        for (int h = 0; h < 2; ++h) {
          const LAS u32* const pw = (const LAS u32*)(planes + lane * 24u + h * 12u);
          const u32 w0 = pw[0], w1 = pw[1], w2 = pw[2];
          *(LAS u32x4*)(lo + O_HI + lane * 64u + h * 32u) =
              (u32x4){w0 & 0x07070707u, (w0 >> 3) & 0x07070707u, ((w0 >> 6) & 0x03030303u) | ((w2 >> 4) & 0x04040404u), w1 & 0x07070707u};
          *(LAS u32x4*)(lo + O_HI + lane * 64u + h * 32u + 16u) =
              (u32x4){(w1 >> 3) & 0x07070707u, ((w1 >> 6) & 0x03030303u) | ((w2 >> 5) & 0x04040404u), w2 & 0x07070707u, (w2 >> 3) & 0x07070707u};
        }
      } else {
        const u32x4 nb[2] = {*(const LAS u32x4*)(planes + lane * 32u), *(const LAS u32x4*)(planes + lane * 32u + 16u)};
#pragma unroll
        for (int i = 0; i < 2; ++i) {
          *(LAS u32x4*)(lo + O_HI + lane * 64u + i * 16u) = nb[i] & 0x0f0f0f0fu;
          *(LAS u32x4*)(lo + O_HI + lane * 64u + 32u + i * 16u) = (nb[i] >> 4) & 0x0f0f0f0fu;
        }
      }
      asm volatile("" ::: "memory");
    }
    asm volatile("s_waitcnt vmcnt(0)" : "+v"(gr) :: "memory");
    tag = __builtin_amdgcn_readfirstlane((u32)(gr >> 32));
    info = __builtin_amdgcn_readfirstlane((u32)gr);
    const u32 col = j * 512u + (sw >> 2) * 256u, f0 = foreign_base(uid >> 3) + foreign_frag0(uid >> 3, sw);
    for (u32 m = 0; m < mf; ++m) {
      const u32 w = m == 0 ? mk[0] : (m == 1 ? mk[1] : mk[2]);
      u32x4 o[4];
      if (raw) {
#pragma unroll
        for (int e = 0; e < 4; ++e) o[e] = *(const LAS u32x4*)(scr + (m * 4u + e) * 1024u + lane * 16u);
      } else {
        u32 at[8], sel[8], l0[8], l1[8], h0[8], h1[8];
#pragma unroll
        for (int s = 0; s < 8; ++s) {
          const u32 f = (w >> s) & 0x01010101u;
          const LAS u32* const pw = (const LAS u32*)(size_t)(a & ~3u);
          at[s] = a;
          sel[s] = *(const LAS u32*)(size_t)__builtin_amdgcn_udot4(f, 0x20100804u, TAB + 128u, false);
          l0[s] = pw[0];
          l1[s] = pw[1];
          h0[s] = pw[O_HI / 4];
          h1[s] = pw[O_HI / 4 + 1];
          a = __builtin_amdgcn_sad_u8(f, 0u, a);
        }
        __builtin_amdgcn_sched_barrier(0);
#pragma unroll
        for (int s = 0; s < 8; ++s) {
          const u32 lb = __builtin_amdgcn_perm(0u, __builtin_amdgcn_alignbyte(l1[s], l0[s], at[s]), sel[s]);
          const u32 hb = __builtin_amdgcn_perm(0u, t4 - (__builtin_amdgcn_alignbyte(h1[s], h0[s], at[s]) & 0x0f0f0f0fu), sel[s]);
          o[s >> 1][(s & 1) * 2] = __builtin_amdgcn_perm(hb, lb, 0x05010400u);
          o[s >> 1][(s & 1) * 2 + 1] = __builtin_amdgcn_perm(hb, lb, 0x07030602u);
        }
      }
      const u32 row0 = frag_row(src, f0 + m);
#pragma unroll
      for (int e = 0; e < 4; ++e) store16(o[e], ro, vo, (row0 + e) * 2048u + col);
    }
    k = kn;
  }
}

__device__ __forceinline__ void gate(int c, int& ready, const char* cnt, u32 target, char* ws, u32 call) {
  if (c < ready) return;
  const u64 t0 = __builtin_amdgcn_s_memrealtime();
  for (u32 spins = 0;; ++spins) {
    const u64 m = __builtin_amdgcn_ballot_w64((int)(ld_agent(cnt) - target) >= 0);
    ready = ~m ? __builtin_ctzll(~m) : 64;
    if (c < ready) break;
    if ((spins & 7u) == 7u && give_up(ws, call, t0)) { ready = 64; break; }
    __builtin_amdgcn_s_sleep(3);
  }
  asm volatile("" ::: "memory");
}

__device__ __forceinline__ u32 relu_pk(float lo, float hi) {
  u32 r;
  asm("v_cvt_pk_bf16_f32 %0, %1, %2\n\tv_pk_max_i16 %0, %0, 0" : "=v"(r) : "v"(lo), "v"(hi));
  return r;
}

__device__ __forceinline__ void mfma_acc(f32x4& c, const bf16x8& a, const bf16x8& b) {
  asm volatile("v_mfma_f32_16x16x32_bf16 %0, %1, %2, %0" : "+v"(c) : "v"(a), "v"(b));
}
template <int N> __device__ __forceinline__ void wait_bar() { asm volatile("s_waitcnt vmcnt(%0)\n\ts_barrier" :: "i"(N) : "memory"); }

struct Tile { u32 rt, ct, gf0, nfr; };
template <int MF, int NP>
__device__ __forceinline__ void gemm_wave(u32x4 (&o)[MF][4], char* ws, u32 call, const Tile& T, u32 f0, LAS char* lds, u32 lane, u32 wid) {
  constexpr bool XL = NP && NP < 8;
  constexpr u32 STEP = XL ? A_STEP : B_STEP;
  constexpr int NM = MF * 8;
  const u32 rt = T.rt, ct = T.ct, wc = wid >> 2;
  const bool own = ct < 4u;
  const u32 p0 = (wid & 1u) * NP;
  const rsrc_t rs = XL ? make_rsrc(ws + OFF_XU, OFF_WU - OFF_XU) : make_rsrc(ws + OFF_WU + ct * B_TILE, B_TILE);
  const char* const cnt0 = ws + OFF_CNT + (XL ? (own ? ROW_OWN + rt : rt) : ROW_W + ct) * 256;
  const char* const cnt = cnt0 + lane * 4;
  const u32 target = call * (XL ? T.nfr >> 1 : 8u);
  u32 vp[NP ? NP : 1];
#pragma unroll
  for (u32 p = 0; p < NP; ++p) vp[p] = lane * 16u + (XL ? T.gf0 + (p0 + p < T.nfr ? p0 : T.nfr - 1u - p) : p0) * 1024u;
  LAS char* const ring = lds + (XL ? 0u : A_LDS) + p0 * 1024u;
  const u32 img = ((lane & 15u) * 4u + ((lane >> 4) ^ quad_swz((lane >> 2) & 3u))) * 16u;
  LAS char* const ra = lds + f0 * 1024u + img;
  LAS char* const rb = lds + A_LDS + wc * 8192u + lane * 16u;
#define AG_DMA(p, ks, off, imm) __builtin_amdgcn_raw_ptr_buffer_load_lds(rs, ring + (off), 16, vp[p < NP ? p : 0], (ks), imm, AUX_SYS)
#define AG_PIECE(p, ks, off) do { \
    if (p == 0) AG_DMA(0, (ks) * STEP, off, 0); if (p == 1) AG_DMA(1, (ks) * STEP, off, 1024); \
    if (p == 2) AG_DMA(2, (ks) * STEP, off, 2048); if (p == 3) AG_DMA(3, (ks) * STEP, off, 3072); \
    if (p == 4) AG_DMA(4, (ks) * STEP + 4096, (off) + 4096, 0); if (p == 5) AG_DMA(5, (ks) * STEP + 4096, (off) + 4096, 1024); \
    if (p == 6) AG_DMA(6, (ks) * STEP + 4096, (off) + 4096, 2048); if (p == 7) AG_DMA(7, (ks) * STEP + 4096, (off) + 4096, 3072); \
  } while (0)
#define AG_ALL(ks, off) do { _Pragma("unroll") for (int p = 0; p < NP; ++p) AG_PIECE(p, ks, off); } while (0)
#define AG_PIN(stmt) do { __builtin_amdgcn_sched_barrier(0); stmt; __builtin_amdgcn_sched_barrier(0); } while (0)
#define AG_STEP(cs, ks, DMA, RD) AG_STEPC(cs, ks, DMA, RD, 0, 0)
#define AG_STEPC(cs, ks, DMA, RD, NEARLY, slot0) do { \
    LAS char* const pa = ra + ord; \
    LAS char* const pb = rb + ord; \
    __builtin_amdgcn_sched_barrier(0); \
    _Pragma("unroll") for (int i = 0; i < NM; ++i) { \
      mfma_acc(acc[i % MF][i / MF], fa[cs][i % MF], fb[cs][i / MF]); \
      _Pragma("unroll") for (int j = 0; j < MF + 8; ++j) \
        if (RD && i == (NP ? (MF == 3 ? 2 * j : (3 * j) / 2) : j)) { \
          if (j < MF) AG_PIN(fa[(cs) ^ 1][j] = *(const LAS bf16x8*)(pa + j * 1024)); \
          else AG_PIN(fb[(cs) ^ 1][j - MF] = *(const LAS bf16x8*)(pb + (j - MF) * 1024)); \
        } \
      _Pragma("unroll") for (int p = 0; p < NP; ++p) \
        if (i == 1 + p * ((NM + 1) / (NP ? NP : 1))) { \
          if (DMA) AG_PIN(AG_PIECE(p, ks, owr)); \
          if (NEARLY) { \
            __builtin_amdgcn_sched_barrier(0); \
            if (early) { AG_PIECE(p, ks, (slot0) * SLOTB); if (NEARLY > 1) AG_PIECE(p, (ks) + 1, ((slot0) + 1) * SLOTB); } \
            __builtin_amdgcn_sched_barrier(0); \
          } \
        } \
    } \
    ord = ord + SLOTB == NSLOT * SLOTB ? 0 : ord + SLOTB; \
    owr = owr + SLOTB == NSLOT * SLOTB ? 0 : owr + SLOTB; \
    __builtin_amdgcn_sched_barrier(0); \
  } while (0)
#define AG_GATE(c) do { if (NP) gate(c, ready, cnt, target, ws, call); } while (0)
  f32x4 acc[MF][8];
#pragma unroll
  for (int m = 0; m < MF; ++m)
#pragma unroll
    for (int n = 0; n < 8; ++n) { acc[m][n] = (f32x4){0.f, 0.f, 0.f, 0.f}; asm volatile("" : "+v"(acc[m][n])); }
  __builtin_amdgcn_s_setprio(NP ? 1 : 0);
  int ready = 0;
  bool early = false;
  AG_GATE(0);
  constexpr int LA = 4, W = (LA - 2) * NP;
#pragma unroll
  for (int s = 0; s < 4; ++s) AG_ALL(s, s * SLOTB);
  wait_bar<W>();
  bf16x8 fa[2][MF], fb[2][8];
#pragma unroll
  for (int m = 0; m < MF; ++m) fa[0][m] = *(const LAS bf16x8*)(ra + m * 1024);
#pragma unroll
  for (int n = 0; n < 8; ++n) fb[0][n] = *(const LAS bf16x8*)(rb + n * 1024);
  u32 ord = SLOTB, owr = LA * SLOTB;
  for (int kc = 0; kc < NKC - 3; ++kc) {
    AG_GATE(kc + 1);
    AG_STEP(0, kc * 4 + LA, true, true); wait_bar<W>();
    AG_STEP(1, kc * 4 + LA + 1, true, true); wait_bar<W>();
    AG_STEP(0, kc * 4 + LA + 2, true, true); wait_bar<W>();
    AG_STEP(1, kc * 4 + LA + 3, true, true); wait_bar<W>();
  }
  AG_GATE(NKC - 2);
  AG_STEP(0, NKS - 12 + LA, true, true); wait_bar<W>();
  AG_STEP(1, NKS - 11 + LA, true, true); wait_bar<W>();
  AG_STEP(0, NKS - 10 + LA, true, true); wait_bar<W>();
  AG_STEP(1, NKS - 9 + LA, true, true); wait_bar<2 * NP>();
  AG_STEP(0, 0, false, true); wait_bar<NP>();
  u32 last = 0;
  if (NP) asm volatile("global_load_dword %0, %1, %2 sc1" : "=&v"(last) : "v"(0u), "s"(cnt0 + (NKC - 1) * 4) : "memory");
  AG_STEP(1, 0, false, true);
  asm volatile("s_waitcnt vmcnt(0)\n\ts_barrier" : "+v"(last) :: "memory");
  early = NP && (ready >= NKC || (int)((u32)__builtin_amdgcn_readfirstlane(last) - target) >= 0);
  AG_STEPC(0, NKS - 4, false, true, 2, 2);
  AG_STEPC(1, NKS - 2, false, false, 1, 4);
  asm volatile("s_barrier" ::: "memory");
  if (!early) {
    AG_GATE(NKC - 1);
#pragma unroll
    for (int s = 0; s < 3; ++s) AG_ALL(NKS - 4 + s, (2 + s) * SLOTB);
  }
  AG_ALL(NKS - 1, 0);
  wait_bar<2 * NP>();
  u32 tmo;
  asm volatile("global_load_dword %0, %1, %2 sc1" : "=&v"(tmo) : "v"(0u), "s"(ws + OFF_TMO) : "memory");
#pragma unroll
  for (int m = 0; m < MF; ++m) fa[0][m] = *(const LAS bf16x8*)(ra + 2 * SLOTB + m * 1024);
#pragma unroll
  for (int n = 0; n < 8; ++n) fb[0][n] = *(const LAS bf16x8*)(rb + 2 * SLOTB + n * 1024);
  ord = 3 * SLOTB;
  AG_STEP(0, 0, false, true); wait_bar<NP + 1>();
  AG_STEP(1, 0, false, true); wait_bar<0>();
  AG_STEP(0, 0, false, true);
  AG_STEP(1, 0, false, false);
  asm volatile("" : "+v"(tmo));
  asm volatile("s_nop 15" ::: "memory");
  __builtin_amdgcn_sched_barrier(0);
#undef AG_PIECE
#undef AG_ALL
#undef AG_PIN
#undef AG_DMA
#undef AG_GATE
#undef AG_STEP
#undef AG_STEPC
  const bool bad = (u32)__builtin_amdgcn_readfirstlane(tmo) == call;
#pragma unroll
  for (int m = 0; m < MF; ++m)
#pragma unroll
    for (int e = 0; e < 4; ++e)
#pragma unroll
      for (int n = 0; n < 4; ++n) o[m][e][n] = relu_pk(acc[m][2 * n][e], acc[m][2 * n + 1][e]);
  if (bad) {
#pragma unroll
    for (int m = 0; m < MF; ++m)
#pragma unroll
      for (int e = 0; e < 4; ++e) o[m][e] = (u32x4){0x7fc07fc0u, 0x7fc07fc0u, 0x7fc07fc0u, 0x7fc07fc0u};
    asm volatile("" ::: "memory");
  }
}

__device__ __forceinline__ void store_tile(const u32x4 (&o)[3][4], const Params& P, const Tile& T, u32 f0, u32 lane, u32 wid) {
  const rsrc_t ro = make_rsrc(pick(P.out, P.rank), 8192u * 1024u * 2u);
  const u32 vo = (lane >> 4) * 4u * 2048u + (lane & 15u) * 16u;
#pragma unroll
  for (int m = 0; m < 3; ++m) {
    if (f0 + m >= T.nfr) continue;
    const u32 row0 = frag_row(P.rank, T.gf0 + f0 + m);
#pragma unroll
    for (int e = 0; e < 4; ++e) store16(o[m][e], ro, vo, (row0 + e) * 2048u + T.ct * 512u + (wid >> 2) * 256u);
  }
}

template <int MF>
__device__ __forceinline__ void send_tile(const u32x4 (&o)[MF][4], const Params& P, char* ws, u32 call, const Tile& T, LAS char* lds, u32 lane, u32 wid) {
  const u32 r = P.rank, g4 = r & 4u, rt = T.rt, ct = T.ct;
  const u32 j = ct - 4u, uid = rt * 8u + wid;
  u32 qi;
  u64 keep;
  asm volatile("s_mov_b64 %1, exec\n\ts_mov_b64 exec, 1\n\tglobal_atomic_add %0, %2, %3, %4 sc0\n\ts_mov_b64 exec, %1"
               : "=&v"(qi), "=&s"(keep) : "v"(0u), "v"(1u), "s"(ws + OFF_TKT + (11u + j) * 128u) : "memory");
  u32 mk[MF], fl[MF * 8], lb[MF * 8], hb[MF * 8], c = 0;
  u16x2 big[2] = {{0, 0}, {0, 0}};
#pragma unroll
  for (int m = 0; m < MF; ++m) {
    mk[m] = 0;
#pragma unroll
    for (int s = 0; s < 8; ++s) {
      const u32 d0 = o[m][s >> 1][(s & 1) * 2], d1 = o[m][s >> 1][(s & 1) * 2 + 1];
      u32 t0, t1;
      asm("v_pk_min_u16 %0, %1, %2" : "=v"(t0) : "v"(d0), "v"(0x00010001u));
      asm("v_pk_min_u16 %0, %1, %2" : "=v"(t1) : "v"(d1), "v"(0x00010001u));
      fl[m * 8 + s] = __builtin_amdgcn_perm(t1, t0, 0x06040200u);
      mk[m] |= fl[m * 8 + s] << s;
      big[0] = __builtin_elementwise_max(big[0], __builtin_bit_cast(u16x2, d0));
      big[1] = __builtin_elementwise_max(big[1], __builtin_bit_cast(u16x2, d1));
      lb[m * 8 + s] = __builtin_amdgcn_perm(d1, d0, 0x06040200u);
      hb[m * 8 + s] = __builtin_amdgcn_perm(d1, d0, 0x07050301u);
    }
    c += __builtin_popcount(mk[m]);
  }
  c = (c & ~3u) + 4u;
  const u32 incl = wave_scan(c), n = __builtin_amdgcn_readlane(incl, 63);
  big[0] = __builtin_elementwise_max(big[0], big[1]);
  u32 top = wave_max((u32)(big[0].x > big[0].y ? big[0].x : big[0].y) >> 8);
  top = top < 15u ? 15u : top;
  bool raw = n > 4096u, tri = false;
  u32 total = 0;
  const rsrc_t rb2 = make_rsrc(ws + OFF_OB + j * (OUNITS * OSLOT) + uid * OSLOT, OSLOT);
  LAS unsigned char* const scr = (LAS unsigned char*)lds + (wid < 7u ? SLOTB + wid * 12160u : NSLOT * SLOTB);
  if (!raw) {
    LAS unsigned char* const lo = scr + MF * 256u;
  #pragma unroll
    for (int m = 0; m < MF; ++m) *(LAS u32*)(scr + m * 256u + lane * 4u) = mk[m];
    asm volatile("" ::: "memory");
    *(LAS u32*)(lo + O_HI + n + (lane & 15u) * 4u) = 0u;
    const u32 t4 = top * 0x01010101u;
    u32 wide = 0;
    u32 a = (u32)(size_t)lo + incl - c, cl = 0, ch = 0;
#pragma unroll
    for (int m = 0; m < MF; ++m) {
      u32 sel[8];
#pragma unroll
      for (int s = 0; s < 8; ++s) sel[s] = *(const LAS u32*)(size_t)__builtin_amdgcn_udot4(fl[m * 8 + s], 0x20100804u, TAB, false);
      __builtin_amdgcn_sched_barrier(0);
#pragma unroll
      for (int s = 0; s < 8; ++s) {
        const u32 sh = (a & 3u) * 8u, an = __builtin_amdgcn_sad_u8(fl[m * 8 + s], 0u, a);
        const u32 code = __builtin_amdgcn_perm(0u, t4 - hb[m * 8 + s], sel[s]);
        const u64 tl = (u64)__builtin_amdgcn_perm(0u, lb[m * 8 + s], sel[s]) << sh, th = (u64)code << sh;
        wide |= code;
        const u32 wl = cl | (u32)tl, wh = ch | (u32)th;
        LAS u32* const pw = (LAS u32*)(size_t)(a & ~3u);
        pw[0] = wl;
        pw[O_HI / 4] = wh;
        const bool cross = (an ^ a) & 4u;
        cl = cross ? (u32)(tl >> 32) : wl;
        ch = cross ? (u32)(th >> 32) : wh;
        a = an;
      }
    }
    ((LAS u32*)(size_t)(a & ~3u))[0] = cl;
    ((LAS u32*)(size_t)(a & ~3u))[O_HI / 4] = ch;
    asm volatile("" ::: "memory");
    LAS unsigned char* const planes = lo + ((n + 15u) & ~15u);
    u32x4 hq[4];
#pragma unroll
    for (int i = 0; i < 4; ++i) hq[i] = *(const LAS u32x4*)(lo + O_HI + lane * 64u + i * 16u);
    raw = __builtin_amdgcn_ballot_w64((wide & 0xf0f0f0f0u) != 0) != 0;
    tri = __builtin_amdgcn_ballot_w64((wide & 0xf8f8f8f8u) != 0) == 0;
    if (tri) {
#pragma unroll
      for (int h = 0; h < 2; ++h) {
        const u32x4 x = hq[2 * h], y = hq[2 * h + 1];
        LAS u32* const pw = (LAS u32*)(planes + lane * 24u + h * 12u);
        pw[0] = x[0] | (x[1] << 3) | ((x[2] & 0x03030303u) << 6);
        pw[1] = x[3] | (y[0] << 3) | ((y[1] & 0x03030303u) << 6);
        pw[2] = y[2] | (y[3] << 3) | ((x[2] & 0x04040404u) << 4) | ((y[1] & 0x04040404u) << 5);
      }
    } else {
#pragma unroll
      for (int i = 0; i < 2; ++i) *(LAS u32x4*)(planes + lane * 32u + i * 16u) = hq[i] | (hq[2 + i] << 4);
    }
    total = MF * 256u + ((n + 15u) & ~15u) + ((n + 63u) >> 6) * (tri ? 24u : 32u);
    asm volatile("" ::: "memory");
  }
  if (raw) {
    total = 4096u * MF;
#pragma unroll
    for (int m = 0; m < MF; ++m)
#pragma unroll
      for (int i = 0; i < 4; ++i)
        store16((u32x4){__builtin_amdgcn_perm(hb[m * 8 + 2 * i], lb[m * 8 + 2 * i], 0x05010400u), __builtin_amdgcn_perm(hb[m * 8 + 2 * i], lb[m * 8 + 2 * i], 0x07030602u),
                        __builtin_amdgcn_perm(hb[m * 8 + 2 * i + 1], lb[m * 8 + 2 * i + 1], 0x05010400u), __builtin_amdgcn_perm(hb[m * 8 + 2 * i + 1], lb[m * 8 + 2 * i + 1], 0x07030602u)},
                rb2, lane * 16u, (m * 4 + i) * 1024u);
  } else {
    for (u32 p = 0; p < total; p += 1024u) store16(*(const LAS u32x4*)(scr + p + lane * 16u), rb2, p + lane * 16u < total ? lane * 16u : 0x80000000u, p);
  }
  asm volatile("s_waitcnt vmcnt(0)" : "+v"(qi) :: "memory");
  const u32 slot = ((u32)__builtin_amdgcn_readfirstlane(qi) - (call - 1u) * OUNITS) & 255u;
  flag_store(pick(P.ws, (g4 ^ 4u) | j) + OFF_OQ + (r & 3u) * 2048u + slot * 8u, ((u64)call << 32) | uid | (total << 8) | (raw ? 1u << 22 : 0u) | (top << 23) | (tri ? 1u << 31 : 0u), 1u);
}

__global__ __launch_bounds__(512, 2) void ag_gemm_kernel(Params P) {
  extern __shared__ __attribute__((aligned(16))) char smem[];
  LAS char* lds = (LAS char*)smem;
  const u32 lane = threadIdx.x & 63u, wid = __builtin_amdgcn_readfirstlane(threadIdx.x >> 6);
  char* const ws = pick(P.ws, P.rank);
  u32 bid = blockIdx.x;
  asm volatile("" : "+s"(bid));
  char* const cptr = ws + OFF_CALL + bid * 128u + wid * 4u;
  const u32 callv = ld_agent(cptr);
  LAS unsigned char* scr = (LAS unsigned char*)lds + wid * WAVE_SCRATCH;
  build_tables(lds, lane);
  const u32 li = bid < 7u * NCL ? bid % 7u : 7u, lidx = bid / 7u, pkw = bid - 7u * NCL;
  const u32 xcd = bid & 7u, pos = xcd * 26u + (xcd > 4u ? xcd - 4u : 0u) + (bid >> 3) - (xcd < 4u ? 6u : 5u);
  const bool foreign = pos >= 4u * NOT;
  const u32 ent = foreign ? pos - 4u * NOT : pos;
  Tile T;
  T.rt = ent >> 2;
  T.ct = (ent & 3u) + (foreign ? 4u : 0u);
  T.gf0 = foreign ? foreign_base(T.rt) : T.rt * 12u;
  T.nfr = foreign ? (T.rt < 4u ? 10u : 8u) : (T.rt == NOT - 1u ? 4u : 12u);
  LAS u32* const ctl = (LAS u32*)(lds + 8 * WAVE_SCRATCH);
  if (bid < NCOMM) {
    ctl[lane & 3u] = 0;
    asm volatile("s_waitcnt lgkmcnt(0)\n\ts_barrier" ::: "memory");
  }
  const bool isg = bid >= NCOMM;
  u32 call;
  if (li < 7u) {
    call = __builtin_amdgcn_readfirstlane(callv) + 1u;
    puller(P, ws, call, li, lidx, ctl, scr, lane, wid);
  } else {
    const u32 q0 = isg ? wid * NGEMM + (bid - NCOMM) : GPACK + wid * NPKWG + pkw;
    const u32 q1 = isg ? (T.nfr == 8u ? 8u * NGEMM + wid * 108u + ent - 16u : GPACK) : q0 + NPK;
    for (u32 i = isg ? wid : 0u; i; --i) __builtin_amdgcn_s_sleep(28);
    call = packer(P, ws, callv, q0, q1, isg ? 864u : (u32)NPK, isg ? GPACK : (u32)NUNITS, scr, lane);
  }
  __hip_atomic_store((GAS u32*)cptr, call, __ATOMIC_RELAXED, __HIP_MEMORY_SCOPE_AGENT);
  if (!isg) {
    const u32 hi = li < 3u ? li * NCL + lidx : 3u * NCL + pkw;
    if (li - 3u < 4u || hi < 4u * HELP) out_puller(P, ws, call, li - 3u < 4u ? li - 3u : hi & 3u, ctl, scr, lane);
    return;
  }
  asm volatile("s_waitcnt lgkmcnt(0)\n\ts_barrier" ::: "memory");
  if (T.nfr == 8u || (T.nfr == 10u && wid - 2u < 4u)) {
    u32x4 o[2][4];
    const u32 f0 = foreign_frag0(T.rt, wid);
    if (T.nfr == 10u || wid >= 4u) gemm_wave<2, 0>(o, ws, call, T, f0, lds, lane, wid);
    else if (wid < 2u) gemm_wave<2, 4>(o, ws, call, T, f0, lds, lane, wid);
    else gemm_wave<2, 8>(o, ws, call, T, f0, lds, lane, wid);
    send_tile<2>(o, P, ws, call, T, lds, lane, wid);
  } else {
    u32x4 o[3][4];
    const u32 f0 = foreign ? foreign_frag0(0u, wid) : (wid & 3u) * 3u;
    if (foreign ? wid >= 6u : wid - 2u < 2u) gemm_wave<3, 8>(o, ws, call, T, f0, lds, lane, wid);
    else if (foreign) gemm_wave<3, 5>(o, ws, call, T, f0, lds, lane, wid);
    else if (wid < 2u) gemm_wave<3, 6>(o, ws, call, T, f0, lds, lane, wid);
    else gemm_wave<3, 0>(o, ws, call, T, f0, lds, lane, wid);
    if (foreign) send_tile<3>(o, P, ws, call, T, lds, lane, wid);
    else store_tile(o, P, T, f0, lane, wid);
  }
}

}

extern "C" void kernel_launch(int rank, int nranks, ncclComm_t comm, hipStream_t stream,
                              void* const* d_in, const int* in_sizes, int n_in,
                              void* d_out, int out_size, const int* in_dtypes, int out_dtype,
                              void* const* d_out_all, void* const* d_ws_all, size_t ws_bytes) {
  (void)comm; (void)in_dtypes; (void)out_dtype; (void)d_out;
  if (nranks != ag::NR || n_in != 2 || (unsigned)rank >= (unsigned)ag::NR || in_sizes[0] != 1024 * 8192 || in_sizes[1] != 8192 * 1024 ||
      out_size != 8192 * 1024 || ws_bytes < ag::WS_NEED) {
    fprintf(stderr, "ag_gemm: unexpected problem shape\n");
    return;
  }
  ag::Params p;
  p.x = (const char*)d_in[0];
  p.w = (const char*)d_in[1];
  for (int i = 0; i < ag::NR; ++i) {
    const int phys = (ag::ORDER >> (4 * i)) & 7;
    p.ws[i] = (char*)d_ws_all[phys];
    p.out[i] = (char*)d_out_all[phys];
    if (phys == rank) p.rank = i;
  }
  p.pad = 0;
  hipError_t e = hipFuncSetAttribute((const void*)ag::ag_gemm_kernel, hipFuncAttributeMaxDynamicSharedMemorySize, ag::LDS_BYTES);
  if (e != hipSuccess) { fprintf(stderr, "ag_gemm: %s\n", hipGetErrorString(e)); return; }
  hipLaunchKernelGGL(ag::ag_gemm_kernel, dim3(ag::NWG), dim3(512), ag::LDS_BYTES, stream, p);
  e = hipGetLastError();
  if (e != hipSuccess) fprintf(stderr, "ag_gemm launch: %s\n", hipGetErrorString(e));
}
```

```cpp
#include <hip/hip_runtime.h>
#include <rccl/rccl.h>
#include <cstdio>

namespace ag {

typedef unsigned u32;
typedef unsigned long long u64;
typedef unsigned u32x4 __attribute__((ext_vector_type(4)));
typedef unsigned short u16x2 __attribute__((ext_vector_type(2)));
typedef short bf16x8 __attribute__((ext_vector_type(8)));
typedef float f32x4 __attribute__((ext_vector_type(4)));
typedef __amdgpu_buffer_rsrc_t rsrc_t;
#define LAS __attribute__((address_space(3)))
#define GAS __attribute__((address_space(1)))

constexpr int NR = 8;
constexpr u32 ORDER = 0x75416320u;
constexpr int AU = 21;
constexpr int AWAY = 32 - AU;
constexpr int NKC = 64, NKS = 256;
constexpr int EARLY_UNITS = AU + 4 * AWAY;
constexpr u32 NFT = 31, NOT = 22, ROW_OWN = NFT, ROW_W = NFT + NOT;
constexpr int NOUT = AWAY + 8;

constexpr int NCOMM = 44, NGEMM = 4 * (NFT + NOT), NWG = NCOMM + NGEMM;
constexpr int NCL = 6;
constexpr int NPKWG = NCOMM - 7 * NCL;
static_assert(NWG == 256 && NPKWG > 0, "one workgroup per CU");
constexpr int NPK = NPKWG * 8;
constexpr u32 GPACK = 8 * NGEMM + 2240;
constexpr int NUNITS = 4096;
constexpr u32 LEAD = 6, RAMP = 2;
constexpr u32 HALVES = 4, OUT_TOK = 2;
constexpr u32 HELP = 4;

constexpr u32 A_STEP = 262144, B_STEP = 16384;
constexpr u32 B_TILE = B_STEP * NKS;
constexpr u32 OFF_XU = 0;
constexpr u32 OFF_WU = OFF_XU + NKS * A_STEP;
constexpr u32 FRAG_BYTES = OFF_WU + 8 * B_TILE;
constexpr u32 SLOT = 8192, PK_BYTES = NUNITS * SLOT;
constexpr u32 OFF_PK = FRAG_BYTES;
constexpr u32 OSLOT = 12288, OUNITS = NFT * 8;
constexpr u32 OFF_OB = OFF_PK + 2 * PK_BYTES;
constexpr u32 IB_SRC = 64 * SLOT, IB_BYTES = 2 * NR * IB_SRC;
constexpr u32 OFF_IB = OFF_OB + 4 * OUNITS * OSLOT;
constexpr u32 OFF_G = OFF_IB + IB_BYTES;
constexpr u32 G_BYTES = 2 * NR * NUNITS * 8;
constexpr u32 OFF_CNT = OFF_G + G_BYTES;
constexpr u32 OFF_OQ = OFF_CNT + 16384;
constexpr u32 OFF_TKT = OFF_OQ + 4 * 2048;
constexpr u32 OFF_TMO = OFF_TKT + 2048;
constexpr u32 OFF_CALL = OFF_TMO + 128;
constexpr u32 WS_NEED = OFF_CALL + NWG * 128;

constexpr u32 A_LDS = 12288, SLOTB = A_LDS + B_STEP, NSLOT = 5;
constexpr u32 TAB_OFF = 163840 - 256;
static_assert(NSLOT * SLOTB + 12160 <= TAB_OFF && 7 * 12160 <= 3 * SLOTB, "LDS map");
constexpr int LDS_BYTES = TAB_OFF + 256;
constexpr int WAVE_SCRATCH = 13312;
static_assert(8 * WAVE_SCRATCH + 64 <= LDS_BYTES, "scratch");

constexpr u32 SPARE = 4000, SPARE_CNT = SPARE | SPARE << 16;
constexpr u64 DEADLINE = 50000000ull;
constexpr int AUX_SYS = 17, AUX_NT = 2;

struct Params {
  const char* x;
  const char* w;
  char* ws[NR];
  char* out[NR];
  int rank;
  int pad;
};

__device__ __forceinline__ rsrc_t make_rsrc(const void* p, u32 bytes) {
  const u64 a = (u64)p;
  const u32 lo = __builtin_amdgcn_readfirstlane((u32)a), hi = __builtin_amdgcn_readfirstlane((u32)(a >> 32));
  return __builtin_amdgcn_make_buffer_rsrc((void*)(((u64)hi << 32) | lo), 0, bytes, 0x00020000);
}
__device__ __forceinline__ char* pick(char* const (&t)[NR], u32 i) {
  char* p = t[0];
#pragma unroll
  for (u32 j = 1; j < NR; ++j) p = (i == j) ? t[j] : p;
  return p;
}
__device__ __forceinline__ u32 ld_agent(const char* p) { return __hip_atomic_load((GAS u32*)p, __ATOMIC_RELAXED, __HIP_MEMORY_SCOPE_AGENT); }
__device__ __forceinline__ void store16(u32x4 v, rsrc_t rs, u32 voff, u32 soff) {
  __builtin_amdgcn_raw_buffer_store_b128(v, rs, voff, soff, AUX_SYS);
  asm volatile("s_nop 1" :: "v"(v) : "memory");
}
__device__ __forceinline__ u32 pair_offset(u32 c, u32 lane) { return ((lane & 1u) ? c >> 16 : c & 0xffffu) * 4u; }
__device__ __forceinline__ u64 pair_mask(u32 c) { return (c >> 16) == SPARE ? 1u : 3u; }
__device__ __forceinline__ void count_in(const char* base, u32 c, u32 lane) {
  u64 keep;
  asm volatile("s_mov_b64 %0, exec\n\ts_mov_b64 exec, %4\n\tglobal_atomic_add %1, %2, %3\n\ts_mov_b64 exec, %0"
               : "=&s"(keep) : "v"(pair_offset(c, lane)), "v"(1u), "s"(base), "s"(pair_mask(c)) : "memory");
}
__device__ __forceinline__ void flag_store(char* p, u64 v, u64 mask) {
  u64 keep;
  asm volatile("s_mov_b64 %0, exec\n\ts_mov_b64 exec, %3\n\tglobal_store_dwordx2 %1, %2, off sc0 sc1\n\ts_mov_b64 exec, %0"
               : "=&s"(keep) : "v"(p), "v"(v), "s"(mask) : "memory");
}
__device__ __forceinline__ void drain_vm() { asm volatile("s_waitcnt vmcnt(0)" ::: "memory"); }

__device__ __forceinline__ bool give_up(char* ws, u32 call, u64 t0) {
  const u32 t = __builtin_amdgcn_readfirstlane(ld_agent(ws + OFF_TMO));
  if (t == call) return true;
  if (__builtin_amdgcn_s_memrealtime() - t0 > DEADLINE) {
    __hip_atomic_store((GAS u32*)(ws + OFF_TMO), call, __ATOMIC_RELAXED, __HIP_MEMORY_SCOPE_AGENT);
    return true;
  }
  return false;
}

__device__ __forceinline__ u32 wave_sum(u32 v) {
  v += __builtin_amdgcn_update_dpp(0u, v, 0x111, 0xf, 0xf, true);
  v += __builtin_amdgcn_update_dpp(0u, v, 0x112, 0xf, 0xf, true);
  v += __builtin_amdgcn_update_dpp(0u, v, 0x114, 0xf, 0xf, true);
  v += __builtin_amdgcn_update_dpp(0u, v, 0x118, 0xf, 0xf, true);
  return __builtin_amdgcn_readlane(v, 15) + __builtin_amdgcn_readlane(v, 31) + __builtin_amdgcn_readlane(v, 47) + __builtin_amdgcn_readlane(v, 63);
}
__device__ __forceinline__ u32 wave_max(u32 v) {
  v = max(v, (u32)__builtin_amdgcn_update_dpp(0u, v, 0x111, 0xf, 0xf, true));
  v = max(v, (u32)__builtin_amdgcn_update_dpp(0u, v, 0x112, 0xf, 0xf, true));
  v = max(v, (u32)__builtin_amdgcn_update_dpp(0u, v, 0x114, 0xf, 0xf, true));
  v = max(v, (u32)__builtin_amdgcn_update_dpp(0u, v, 0x118, 0xf, 0xf, true));
  return max(max((u32)__builtin_amdgcn_readlane(v, 15), (u32)__builtin_amdgcn_readlane(v, 31)), max((u32)__builtin_amdgcn_readlane(v, 47), (u32)__builtin_amdgcn_readlane(v, 63)));
}
__device__ __forceinline__ u32 wave_scan(u32 v) {
  v += __builtin_amdgcn_update_dpp(0u, v, 0x111, 0xf, 0xf, true);
  v += __builtin_amdgcn_update_dpp(0u, v, 0x112, 0xf, 0xf, true);
  v += __builtin_amdgcn_update_dpp(0u, v, 0x114, 0xf, 0xf, true);
  v += __builtin_amdgcn_update_dpp(0u, v, 0x118, 0xf, 0xf, true);
  v += __builtin_amdgcn_update_dpp(0u, v, 0x142, 0xa, 0xf, false);
  v += __builtin_amdgcn_update_dpp(0u, v, 0x143, 0xc, 0xf, false);
  return v;
}

__device__ __forceinline__ u32 quad_swz(u32 rq) { return (0x78u >> (2 * rq)) & 3u; }

struct Dest { u32 base, s1, s0, cnt; };
__device__ __forceinline__ Dest dest_x(u32 ux, u32 kc) {
  const u32 ft = ux < 20u ? ux / 5u : (ux - 4u) >> 2, ot = ROW_OWN + ux / 6u;
  return Dest{OFF_XU + 4 * kc * A_STEP + ux * 2048u, 1024u, A_STEP, (ft * 64 + kc) | (ot * 64 + kc) << 16};
}
__device__ __forceinline__ Dest dest_w(u32 ct, u32 kc, u32 ks4, u32 half) {
  return Dest{OFF_WU + ct * B_TILE + (4 * kc + ks4) * B_STEP + half * 8192u, 4096u, 1024u, ((ROW_W + ct) * 64 + kc) | SPARE << 16};
}
__device__ __forceinline__ void store_frags(const u32x4 (&v)[8], rsrc_t rf, const Dest& d, u32 lane) {
#pragma unroll
  for (int f = 0; f < 8; ++f) store16(v[f], rf, lane * 16, d.base + (f >> 2) * d.s1 + (f & 3) * d.s0);
}

constexpr u32 RAW = 1u << 31, PUSHED = 1u << 30;
constexpr u32 S_LIST = 0, S_TAIL = 2304, S_RARE = 3584;
struct Packed { u32x4 mq[4], p01, tail; };

__device__ __forceinline__ void build_tables(LAS char* lds, u32 lane) {
  u32 ct = 0x0c0c0c0cu, xt = 0x07060504u, zt = 0x0c0c0c0cu, k = 0;
#pragma unroll
  for (u32 b = 0; b < 4; ++b) {
    const u32 on = 0u - ((lane >> b) & 1u);
    ct ^= on & ((0x0cu ^ b) << (8u * k));
    xt ^= on & (((4u + b) ^ k) << (8u * b));
    zt ^= on & ((0x0cu ^ k) << (8u * b));
    k -= on;
  }
  *(LAS u32*)(lds + TAB_OFF + lane * 4u) = (lane & 32u) ? zt : ((lane & 16u) ? xt : ct);
}

constexpr u32 TAB = TAB_OFF;

template <bool TO_LIST>
__device__ __forceinline__ void walk_spelled(u32 m, u32 off, LAS unsigned char* cells, LAS unsigned char* list, u32 flip) {
  while (m != 0) {
    const u32 p = __builtin_ctz(m);
    m &= m - 1;
    LAS unsigned char* cell = cells + (p & 7u) * 4u + (p >> 3);
    if (TO_LIST) list[off] = *cell ^ flip; else *cell = list[off];
    ++off;
  }
}

__device__ __forceinline__ u32 encode_unit(u32x4& tail, const u32x4 (&v)[8], rsrc_t rp, u32 soff, LAS unsigned char* scr, u32 lane) {
  u32x4 mq[4];
  const u32x4 zero = {0u, 0u, 0u, 0u};
  *(LAS u32x4*)(scr + S_LIST + lane * 32u) = zero;
  *(LAS u32x4*)(scr + S_LIST + lane * 32u + 16u) = zero;
  *(LAS u32x4*)(scr + S_RARE + lane * 16u) = zero;
  u32 E[16], esum = 0;
#pragma unroll
  for (int j = 0; j < 16; ++j) {
    const u32 d0 = v[j >> 1][(j & 1) * 2], d1 = v[j >> 1][(j & 1) * 2 + 1];
    const u32 r0 = __builtin_amdgcn_alignbit(d0, d0, 31), r1 = __builtin_amdgcn_alignbit(d1, d1, 31);
    mq[j >> 2][j & 3] = __builtin_amdgcn_perm(r1, r0, 0x06040200u);
    E[j] = __builtin_amdgcn_perm(r1, r0, 0x07050301u);
    esum = __builtin_amdgcn_sad_u8(E[j], 0u, esum);
  }
#pragma unroll
  for (int i = 0; i < 4; ++i) store16(mq[i], rp, lane * 16, soff + i * 1024);
  esum = wave_sum(esum);
  u32 base = (esum + 8028u) >> 12;
  base = base < 19u ? 19u : (base > 254u ? 254u : base);
  const u32 mode = base * 4096u < esum + 6361u ? 1u : 0u;
  const u16x2 b2 = {(unsigned short)base, (unsigned short)base};
  u32 P[2][2] = {{0, 0}, {0, 0}};
#pragma unroll
  for (int j = 0; j < 16; ++j) {
    const u16x2 ev = __builtin_bit_cast(u16x2, E[j] & 0x00ff00ffu), od = __builtin_bit_cast(u16x2, E[j]) >> (u16x2){8, 8};
    const u32 ce = __builtin_bit_cast(u32, __builtin_elementwise_min((u16x2)(b2 - ev), (u16x2){3, 3}));
    const u32 co = __builtin_bit_cast(u32, __builtin_elementwise_min((u16x2)(b2 - od), (u16x2){3, 3}));
    const u32 c = ce | (co << 8);
    P[0][j >> 3] |= (c & 0x01010101u) << (j & 7);
    P[1][j >> 3] |= ((c >> 1) & 0x01010101u) << (j & 7);
  }
  store16((u32x4){P[0][0], P[0][1], P[1][0], P[1][1]}, rp, lane * 16, soff + 4096);
  tail = zero;
  const u32 mk[2] = {P[0][0] & P[1][0], P[0][1] & P[1][1]};
  u32 fl[16], sel[16];
#pragma unroll
  for (int j = 0; j < 16; ++j) {
    fl[j] = (mk[j >> 3] >> (j & 7)) & 0x01010101u;
    sel[j] = *(const LAS u32*)(size_t)__builtin_amdgcn_udot4(fl[j], 0x20100804u, TAB, false);
  }
  __builtin_amdgcn_sched_barrier(0);
  const u32 c = __builtin_popcount(mk[0]) + __builtin_popcount(mk[1]), incl = wave_scan(c);
  const u32 n2 = __builtin_amdgcn_readlane(incl, 63);
  if (n2 > 2044u) return RAW;
  const u32 flip = (base - 3u) * 0x01010101u;
  u32 a = (u32)(size_t)(scr + S_LIST) + incl - c, cur = 0;
#pragma unroll
  for (int j = 0; j < 16; ++j) {
    const u64 t = (u64)__builtin_amdgcn_perm(0u, E[j] ^ flip, sel[j]) << ((a & 3u) * 8u);
    const u32 w = cur | (u32)t, an = __builtin_amdgcn_sad_u8(fl[j], 0u, a);
    __hip_atomic_fetch_or((LAS u32*)(size_t)(a & ~3u), w, __ATOMIC_RELAXED, __HIP_MEMORY_SCOPE_WAVEFRONT);
    cur = ((an ^ a) & 4u) ? (u32)(t >> 32) : w;
    a = an;
  }
  __hip_atomic_fetch_or((LAS u32*)(size_t)(a & ~3u), cur, __ATOMIC_RELAXED, __HIP_MEMORY_SCOPE_WAVEFRONT);
  asm volatile("" ::: "memory");
  const u32x4 l[2] = {*(const LAS u32x4*)(scr + S_LIST + lane * 32u), *(const LAS u32x4*)(scr + S_LIST + lane * 32u + 16u)};
  const unsigned short ta = mode ? base + 1u : base - 3u, tb = mode ? 3u : 0u, tc = base + 1u;
  u32 Q[2] = {0, 0};
#pragma unroll
  for (int s = 0; s < 8; ++s) {
    const u32 x = l[s >> 2][s & 3] ^ flip;
    const u16x2 ev = __builtin_bit_cast(u16x2, x & 0x00ff00ffu), od = __builtin_bit_cast(u16x2, x) >> (u16x2){8, 8};
    const u32 ce = __builtin_bit_cast(u32, __builtin_elementwise_min(__builtin_elementwise_sub_sat((u16x2)((u16x2){ta, ta} - ev), (u16x2){tb, tb}), (u16x2){3, 3}));
    const u32 co = __builtin_bit_cast(u32, __builtin_elementwise_min(__builtin_elementwise_sub_sat((u16x2)((u16x2){ta, ta} - od), (u16x2){tb, tb}), (u16x2){3, 3}));
    const u32 c2 = ce | (co << 8);
    Q[0] |= (c2 & 0x01010101u) << s;
    Q[1] |= ((c2 >> 1) & 0x01010101u) << s;
  }
  *(LAS u32*)(scr + S_TAIL + lane * 8u) = Q[0];
  *(LAS u32*)(scr + S_TAIL + lane * 8u + 4u) = Q[1];
  const u32 m3 = Q[0] & Q[1];
#pragma unroll
  for (int s = 0; s < 8; ++s) {
    fl[s] = (m3 >> s) & 0x01010101u;
    sel[s] = *(const LAS u32*)(size_t)__builtin_amdgcn_udot4(fl[s], 0x20100804u, TAB, false);
  }
  __builtin_amdgcn_sched_barrier(0);
  const u32 c3 = __builtin_popcount(m3), incl3 = wave_scan(c3), n3 = __builtin_amdgcn_readlane(incl3, 63);
  if (n3 > 768u) return RAW;
  const u32 turn = flip ^ (tc * 0x01010101u);
  a = (u32)(size_t)(scr + S_RARE) + incl3 - c3;
  cur = 0;
#pragma unroll
  for (int s = 0; s < 8; ++s) {
    const u64 t = (u64)__builtin_amdgcn_perm(0u, l[s >> 2][s & 3] ^ turn, sel[s]) << ((a & 3u) * 8u);
    const u32 w = cur | (u32)t, an = __builtin_amdgcn_sad_u8(fl[s], 0u, a);
    __hip_atomic_fetch_or((LAS u32*)(size_t)(a & ~3u), w, __ATOMIC_RELAXED, __HIP_MEMORY_SCOPE_WAVEFRONT);
    cur = ((an ^ a) & 4u) ? (u32)(t >> 32) : w;
    a = an;
  }
  __hip_atomic_fetch_or((LAS u32*)(size_t)(a & ~3u), cur, __ATOMIC_RELAXED, __HIP_MEMORY_SCOPE_WAVEFRONT);
  asm volatile("" ::: "memory");
  const u32x4 rq = *(const LAS u32x4*)(scr + S_RARE + lane * 16u);
  u32 cn[4], m4 = 0;
#pragma unroll
  for (int s = 0; s < 4; ++s) {
    const u32 x = rq[s] ^ (tc * 0x01010101u);
    const u16x2 ev = __builtin_bit_cast(u16x2, x & 0x00ff00ffu), od = __builtin_bit_cast(u16x2, x) >> (u16x2){8, 8};
    const u32 ce = __builtin_bit_cast(u32, __builtin_elementwise_min(__builtin_elementwise_sub_sat((u16x2)((u16x2){tc, tc} - ev), (u16x2){5, 5}), (u16x2){15, 15}));
    const u32 co = __builtin_bit_cast(u32, __builtin_elementwise_min(__builtin_elementwise_sub_sat((u16x2)((u16x2){tc, tc} - od), (u16x2){5, 5}), (u16x2){15, 15}));
    cn[s] = ce | (co << 8);
    m4 |= ((cn[s] + 0x01010101u) & 0x10101010u) >> 4 << s;
  }
  const u32 nib = ((n2 + 31u) >> 5) * 8u;
  u32 bytes = nib + ((n3 + 15u) >> 4) * 8u;
  *(LAS u32*)(scr + S_TAIL + nib + lane * 8u) = cn[0] | (cn[1] << 4);
  *(LAS u32*)(scr + S_TAIL + nib + lane * 8u + 4u) = cn[2] | (cn[3] << 4);
  if (__builtin_amdgcn_ballot_w64(m4 != 0) != 0) {
    const u32 c4 = __builtin_popcount(m4), incl4 = wave_scan(c4), n4 = __builtin_amdgcn_readlane(incl4, 63);
    if (n4 > 128u) return RAW;
    asm volatile("" ::: "memory");
    walk_spelled<true>(m4, incl4 - c4, scr + S_RARE + lane * 16u, scr + S_TAIL + bytes, tc);
    bytes += n4;
  }
  asm volatile("" ::: "memory");
  tail = *(const LAS u32x4*)(scr + S_TAIL + lane * 16u);
  return base | (mode << 8) | (bytes << 16);
}

__device__ __forceinline__ void store_raw(const u32x4 (&v)[8], rsrc_t rp, u32 soff, u32 lane) {
#pragma unroll
  for (int f = 0; f < 8; ++f) store16(v[f], rp, lane * 16, soff + f * 1024);
}

__device__ __forceinline__ void decode_unit(u32x4 (&v)[8], const Packed& k, u32 info, LAS unsigned char* scr, u32 lane) {
  *(LAS u32x4*)(scr + S_TAIL + lane * 16u) = k.tail;
  asm volatile("" ::: "memory");
  const u32 q0 = *(const LAS u32*)(scr + S_TAIL + lane * 8u), q1 = *(const LAS u32*)(scr + S_TAIL + lane * 8u + 4u);
  const u32 b4 = (info & 255u) * 0x01010101u, lut = (info & 256u) ? b4 - 0x00040300u + 1u : b4 - 0x00050403u;
  const u32 P0[2] = {k.p01.x, k.p01.y}, P1[2] = {k.p01.z, k.p01.w}, mk[2] = {P0[0] & P1[0], P0[1] & P1[1]};
  const u32 c = __builtin_popcount(mk[0]) + __builtin_popcount(mk[1]), incl = wave_scan(c);
  const u32 g2 = (((u32)__builtin_amdgcn_readlane(incl, 63) + 31u) >> 5) & 127u;
  LAS unsigned char* const nib = scr + S_TAIL + (g2 < 64u ? g2 : 64u) * 8u;
  const u32 x0 = *(const LAS u32*)(nib + lane * 8u), x1 = *(const LAS u32*)(nib + lane * 8u + 4u);
  u32 a[17], sel[16];
  a[0] = (u32)(size_t)(scr + S_LIST) + incl - c;
#pragma unroll
  for (int j = 0; j < 16; ++j) {
    const u32 f = (mk[j >> 3] >> (j & 7)) & 0x01010101u;
    sel[j] = *(const LAS u32*)(size_t)__builtin_amdgcn_udot4(f, 0x20100804u, TAB + 64u, false);
    a[j + 1] = __builtin_amdgcn_sad_u8(f, 0u, a[j]);
  }
  __builtin_amdgcn_sched_barrier(0);
  const u32 live = lane < g2 ? ~0u : 0u, Q0 = q0 & live, Q1 = q1 & live, m3 = Q0 & Q1;
  const u32 c3 = __builtin_popcount(m3), incl3 = wave_scan(c3), g3 = (((u32)__builtin_amdgcn_readlane(incl3, 63) + 15u) >> 4) & 63u;
  const u32 live3 = lane < g3 ? 0x0f0f0f0fu : 0u, cn[4] = {x0 & live3, (x0 >> 4) & live3, x1 & live3, (x1 >> 4) & live3};
  u32x4 rq;
  u32 any = 0;
#pragma unroll
  for (int s = 0; s < 4; ++s) {
    const u32 z = ~(cn[s] + 0x0f0f0f0fu) & 0x10101010u;
    rq[s] = b4 - 0x04040404u - cn[s] + (z >> 4) + (z >> 2);
    any |= cn[s] + 0x01010101u;
  }
  *(LAS u32x4*)(scr + S_RARE + lane * 16u) = rq;
  if (__builtin_amdgcn_ballot_w64((any & 0x10101010u) != 0) != 0) {
    u32 m4 = 0;
#pragma unroll
    for (int s = 0; s < 4; ++s) m4 |= ((cn[s] + 0x01010101u) & 0x10101010u) >> 4 << s;
    const u32 c4 = __builtin_popcount(m4);
    walk_spelled<false>(m4, (wave_scan(c4) - c4) & 127u, scr + S_RARE + lane * 16u, nib + g3 * 8u, 0u);
  }
  asm volatile("" ::: "memory");
  u32x4 l[2];
  {
    u32 a3 = (u32)(size_t)(scr + S_RARE) + ((incl3 - c3) & 1023u), at[8], s3[8], r0[8], r1[8];
#pragma unroll
    for (int s = 0; s < 8; ++s) {
      const u32 f = (m3 >> s) & 0x01010101u;
      const LAS u32* const pw = (const LAS u32*)(size_t)(a3 & ~3u);
      at[s] = a3;
      s3[s] = *(const LAS u32*)(size_t)__builtin_amdgcn_udot4(f, 0x20100804u, TAB + 64u, false);
      r0[s] = pw[0];
      r1[s] = pw[1];
      a3 = __builtin_amdgcn_sad_u8(f, 0u, a3);
    }
    __builtin_amdgcn_sched_barrier(0);
#pragma unroll
    for (int s = 0; s < 8; ++s) {
      const u32 c2 = ((Q0 >> s) & 0x01010101u) | (((Q1 >> s) & 0x01010101u) << 1);
      l[s >> 2][s & 3] = __builtin_amdgcn_perm(__builtin_amdgcn_perm(0u, lut, c2), __builtin_amdgcn_alignbyte(r1[s], r0[s], at[s]), s3[s]);
    }
  }
  *(LAS u32x4*)(scr + S_LIST + lane * 32u) = l[0];
  *(LAS u32x4*)(scr + S_LIST + lane * 32u + 16u) = l[1];
  asm volatile("" ::: "memory");
  u32 w0[16], w1[16];
#pragma unroll
  for (int j = 0; j < 16; ++j) {
    const LAS u32* const pw = (const LAS u32*)(size_t)(a[j] & ~3u);
    w0[j] = pw[0];
    w1[j] = pw[1];
  }
  __builtin_amdgcn_sched_barrier(0);
#pragma unroll
  for (int j = 0; j < 16; ++j) {
    const int s = j & 7, wd = j >> 3;
    const u32 c1 = ((P0[wd] >> s) & 0x01010101u) | (((P1[wd] >> s) & 0x01010101u) << 1);
    const u32 e = __builtin_amdgcn_perm(b4 - c1, __builtin_amdgcn_alignbyte(w1[j], w0[j], a[j]), sel[j]), m = k.mq[j >> 2][j & 3];
    const u32 r0 = __builtin_amdgcn_perm(e, m, 0x05010400u), r1 = __builtin_amdgcn_perm(e, m, 0x07030602u);
    v[j >> 1][(j & 1) * 2] = __builtin_amdgcn_alignbit(r0, r0, 1);
    v[j >> 1][(j & 1) * 2 + 1] = __builtin_amdgcn_alignbit(r1, r1, 1);
  }
}

__device__ __forceinline__ void load_unit(u32x4 (&s)[8], rsrc_t rx, rsrc_t rw, u32 q, u32 vx, u32 vw) {
  const u32 kc = q >> 6, t = q & 63u;
  if (t < 32u) {
#pragma unroll
    for (int f = 0; f < 8; ++f) s[f] = __builtin_amdgcn_raw_buffer_load_b128(rx, vx, (32u * t + 16u * (f >> 2)) * 16384u + kc * 256u + (f & 3) * 64u, AUX_NT);
  } else {
#pragma unroll
    for (int i = 0; i < 8; ++i) s[i] = __builtin_amdgcn_raw_buffer_load_b128(rw, vw, (128u * kc + 32u * ((t - 32u) >> 3) + i) * 2048u + (t & 7u) * 256u, AUX_NT);
  }
}

__device__ __forceinline__ u32 packer(const Params& P, char* ws, u32 callv, u32 q0, u32 q1, u32 dq, u32 qend, LAS unsigned char* scr, u32 lane) {
  const rsrc_t rx = make_rsrc(P.x, 1024u * 8192u * 2u), rw = make_rsrc(P.w, 8192u * 1024u * 2u);
  const u32 kq = (lane & 3u) ^ quad_swz(lane >> 4);
  const u32 vx = (lane >> 2) * 16384u + kq * 16u, vw = (lane >> 4) * 8u * 2048u + (lane & 15u) * 16u;
  u32x4 s[8];
  load_unit(s, rx, rw, q0, vx, vw);
  __builtin_amdgcn_sched_barrier(0);
  const u32 call = __builtin_amdgcn_readfirstlane(callv) + 1u;
  const u32 r = P.rank, g4 = r & 4u, par = call & 1u;
  const rsrc_t rf = make_rsrc(ws, FRAG_BYTES), rp = make_rsrc(ws + OFF_PK + par * PK_BYTES, PK_BYTES);
  const u32 goff = OFF_G + (par * NR + r) * NUNITS * 8;
  char* g_in = pick(P.ws, g4 | ((r + 1u) & 3u));
  char* g_out = pick(P.ws, g4 ^ 4u);
#pragma unroll
  for (u32 j = 1; j < 4; ++j) {
    char* const pi = pick(P.ws, g4 | ((r + 1u + j) & 3u));
    char* const po = pick(P.ws, (g4 ^ 4u) | j);
    g_in = lane == j ? pi : g_in;
    g_out = lane == j ? po : g_out;
  }
  u64 psend = 0;
  char* pgp = g_out;
  u32 pinfo = 0, pcnt = SPARE_CNT;
  auto announce = [&]() {
    flag_store(pgp, ((u64)call << 32) | pinfo, psend);
    if (pcnt != SPARE_CNT) count_in(ws + OFF_CNT, pcnt, lane);
  };
  for (u32 q = q0; q < qend; q = q1, q1 += dq) {
    const u32 kc = q >> 6, t = q & 63u;
    u32x4 v[8];
    Dest d;
    bool own;
    u64 send;
    char* gp;
    if (t < 32u) {
#pragma unroll
      for (int f = 0; f < 8; ++f) v[f] = s[f];
      own = t < AU;
      d = dest_x(t, kc);
      send = own ? 7u : 15u;
      gp = own ? g_in : g_out;
    } else {
      const u32 ks4 = (t - 32u) >> 3, cg = t & 7u;
#pragma unroll
      for (int nf = 0; nf < 8; ++nf)
#pragma unroll
        for (int tt = 0; tt < 4; ++tt) v[nf][tt] = __builtin_amdgcn_perm(s[2 * tt + 1][nf >> 1], s[2 * tt][nf >> 1], (nf & 1) ? 0x07060302u : 0x05040100u);
      own = true;
      d = dest_w(cg >> 1, kc, ks4, cg & 1u);
      send = 1u << (cg >> 1);
      gp = g_out;
    }
    const bool lazy = qend != NUNITS;
    if (q1 < qend && !lazy) load_unit(s, rx, rw, q1, vx, vw);
    u32 info = RAW | PUSHED;
    if (kc == 0u && (t & 3u) < 3u) {
      for (u32 m = (u32)send; m; m &= m - 1u) {
        const u32 jd = __builtin_ctz(m), dst = (t < AU) ? (g4 | ((r + 1u + jd) & 3u)) : ((g4 ^ 4u) | jd);
        store_raw(v, make_rsrc(pick(P.ws, dst) + OFF_IB + (par * NR + r) * IB_SRC, IB_SRC), q * SLOT, lane);
      }
      if (own) store_frags(v, rf, d, lane);
    } else {
      if (own) store_frags(v, rf, d, lane);
      u32x4 tail;
      info = encode_unit(tail, v, rp, q * SLOT, scr, lane);
      if (!lazy || (info & RAW)) {
        drain_vm();
#pragma unroll
        for (int f = 0; f < 8; ++f) asm volatile("" : "+v"(s[f]));
        announce();
      }
      if (info & RAW) store_raw(v, rp, q * SLOT, lane);
      else store16(tail, rp, lane * 16u < (info >> 16) ? lane * 16u : 0x80000000u, q * SLOT + 5120u);
    }
    psend = send;
    pgp = gp + goff + q * 8u;
    pinfo = info;
    pcnt = own ? d.cnt : SPARE_CNT;
    if (q1 < qend && lazy) {
      drain_vm();
      announce();
      psend = 0;
      pcnt = SPARE_CNT;
      load_unit(s, rx, rw, q1, vx, vw);
    }
  }
  drain_vm();
  announce();
  return call;
}

__device__ __forceinline__ u32 lds_add(LAS u32* p, u32 v) {
  return __builtin_amdgcn_readfirstlane(__hip_atomic_fetch_add(p, v, __ATOMIC_RELAXED, __HIP_MEMORY_SCOPE_WORKGROUP)) >> 6;
}

__device__ __forceinline__ void puller(const Params& P, char* ws, u32 call, u32 li, u32 lidx, LAS u32* ctl, LAS unsigned char* scr, u32 lane, u32 wid) {
  const u32 r = P.rank, g4 = r & 4u, par = call & 1u;
  const bool ing = li < 3u;
  const u32 src = ing ? (g4 | ((r + 1 + li) & 3u)) : ((g4 ^ 4u) | (li - 3u));
  const rsrc_t rf = make_rsrc(ws, FRAG_BYTES), rp = make_rsrc(pick(P.ws, src) + OFF_PK + par * PK_BYTES, PK_BYTES);
  const rsrc_t ri = make_rsrc(ws + OFF_IB + (par * NR + src) * IB_SRC, IB_SRC);
  char* const gbase = ws + OFF_G + (par * NR + src) * NUNITS * 8;
  GAS u32* const cnt = (GAS u32*)(ws + OFF_CNT);
  const u32 total = ing ? NKC * AU : NKC * NOUT;
  constexpr u32 NONE = 0xffffffffu;
  const u32 idle = ((OFF_CALL - OFF_CNT + (lidx * 7u + li) * 128u + 64u + wid * 4u) >> 2) | SPARE << 16;
  GAS u32* const tkt = (GAS u32*)(ws + OFF_TKT + li * 128u);
  const u32 tbase = (call - 1u) * (total + NCL * 8u) * 64u;
  auto draw = [&]() -> u32 { return __hip_atomic_fetch_add(tkt, 1u, __ATOMIC_RELAXED, __HIP_MEMORY_SCOPE_AGENT); };
  const u32 rot = (0u - (EARLY_UNITS + li * AU)) & 3u;
  auto unit_of = [&](u32 raw_ticket) -> u32 {
    const u32 i = ((u32)__builtin_amdgcn_readfirstlane(raw_ticket) - tbase) >> 6;
    if (i >= total) return NONE;
    if (ing) {
      constexpr u32 XL = 8, LI = 4, R = AU - XL, F1 = XL * LI * (AU + 1), F2 = F1 + (NKC - LI - XL * LI) * AU;
      u32 kc, t, xi;
      if (i < F1) { kc = i / (AU + 1); t = i - kc * (AU + 1); xi = kc * (XL + 1) + t - R; }
      else if (i < F2) { kc = (i - F1) / AU; t = i - F1 - kc * AU; kc += XL * LI; xi = (kc + LI) * XL + t - R; }
      else { kc = (i - F2) / R; t = i - F2 - kc * R; kc += NKC - LI; xi = 0; }
      if (t >= R) { kc = xi / XL; t = xi - kc * XL; } else t += XL;
      t += rot;
      return kc * 64u + (t < AU ? t : t - AU);
    }
    constexpr u32 C1 = 8u * LEAD / RAMP, N1 = NOUT + RAMP, E1 = C1 * N1, E2 = E1 + (NKC - LEAD - C1) * NOUT;
    u32 kc, t, wi;
    if (i < E1) { kc = i / N1; t = i - kc * N1; wi = kc * (8u + RAMP) + t - AWAY; }
    else if (i < E2) { kc = (i - E1) / NOUT; t = i - E1 - kc * NOUT; kc += C1; wi = (kc + LEAD) * 8u + t - AWAY; }
    else { kc = (i - E2) / AWAY; t = i - E2 - kc * AWAY; kc += NKC - LEAD; wi = 0; }
    return t < AWAY ? kc * 64u + AU + t : (wi >> 3) * 64u + 32u + ((wi >> 1) & 3u) * 8u + 2u * (r & 3u) + (wi & 1u);
  };
  auto granule = [&](u32 q) -> u64 { return __hip_atomic_load((GAS u64*)(gbase + (q & (NUNITS - 1)) * 8u), __ATOMIC_RELAXED, __HIP_MEMORY_SCOPE_SYSTEM); };
  auto peek = [&](LAS u32* p) -> u32 { return (u32)(__builtin_amdgcn_readfirstlane(__hip_atomic_load(p, __ATOMIC_RELAXED, __HIP_MEMORY_SCOPE_WORKGROUP)) >> 6); };
  auto turn = [&](u32 my) { while (peek(ctl + 2) + HALVES <= my) __builtin_amdgcn_s_sleep(1); };
  auto dest_of = [&](u32 q) -> Dest {
    const u32 kc = q >> 6, tq = q & 63u;
    if (q == NUNITS) return Dest{FRAG_BYTES, 0u, 0u, NONE};
    if (tq < 32u) return dest_x(ing ? EARLY_UNITS + li * AU + tq : AU + (li - 3u) * AWAY + (tq - AU), kc);
    return dest_w(4u + (li - 3u), kc, (tq - 32u) >> 3, tq & 1u);
  };
  auto publish = [&](u32 c) { if (c != NONE) count_in(ws + OFF_CNT, c, lane); };
  u32 q = NUNITS, pend = NONE, tag = call, info = 127u;
  u64 gr;
  while (q != NONE) {
    if (tag != call || (!(info & PUSHED) && peek(ctl + 2) + HALVES <= peek(ctl))) {
      drain_vm();
      publish(pend);
      pend = NONE;
      if (tag != call) {
        const u64 t0 = __builtin_amdgcn_s_memrealtime();
        for (;;) {
          gr = granule(q);
          tag = __builtin_amdgcn_readfirstlane((u32)(gr >> 32));
          info = __builtin_amdgcn_readfirstlane((u32)gr);
          if (tag == call || give_up(ws, call, t0)) break;
          __builtin_amdgcn_s_sleep(2);
        }
        if (tag != call) break;
      }
    }
    u32 my = 0;
    if (!(info & PUSHED)) { my = lds_add(ctl, 2u); turn(my); }
    asm volatile("" ::: "memory");
    const Dest d = dest_of(q);
    const u32 cur = info, soff = q * SLOT;
    u32x4 v[8];
    if (cur & RAW) {
#pragma unroll
      for (int f = 0; f < 8; ++f) v[f] = __builtin_amdgcn_raw_buffer_load_b128((cur & PUSHED) ? ri : rp, lane * 16, soff + f * 1024, AUX_SYS);
      const u32 nt = draw();
      drain_vm();
      if (!(cur & PUSHED)) lds_add(ctl + 2, 2u);
      q = unit_of(nt);
      gr = granule(q);
      publish(pend);
      tag = __builtin_amdgcn_readfirstlane((u32)(gr >> 32));
      info = __builtin_amdgcn_readfirstlane((u32)gr);
      store_frags(v, rf, d, lane);
    } else {
      Packed k;
      u32 nt;
      u64 keep;
      asm volatile("s_mov_b64 %1, exec\n\ts_mov_b64 exec, 1\n\tglobal_atomic_add %0, %2, %3, %4 sc0\n\ts_mov_b64 exec, %1"
                   : "=&v"(nt), "=&s"(keep) : "v"(0u), "v"(64u), "s"(tkt) : "memory");
      k.tail = __builtin_amdgcn_raw_buffer_load_b128(rp, lane * 16u < (cur >> 16) ? lane * 16u : 0x80000000u, soff + 5120, AUX_SYS);
      k.p01 = __builtin_amdgcn_raw_buffer_load_b128(rp, lane * 16, soff + 4096, AUX_SYS);
      k.mq[0] = __builtin_amdgcn_raw_buffer_load_b128(rp, lane * 16, soff, AUX_SYS);
      asm volatile("" ::: "memory");
      turn(my + 1u);
      asm volatile("" ::: "memory");
#pragma unroll
      for (int i = 1; i < 4; ++i) k.mq[i] = __builtin_amdgcn_raw_buffer_load_b128(rp, lane * 16, soff + i * 1024, AUX_SYS);
      asm volatile("s_waitcnt vmcnt(5)" ::: "memory");
      lds_add(ctl + 2, 1u);
      asm volatile("s_waitcnt vmcnt(2)" ::: "memory");
      lds_add(ctl + 2, 1u);
      asm volatile("s_waitcnt vmcnt(0)" : "+v"(nt), "+v"(k.mq[0]), "+v"(k.mq[1]), "+v"(k.mq[2]), "+v"(k.mq[3]), "+v"(k.p01), "+v"(k.tail) :: "memory");
      q = unit_of(nt);
      asm volatile("global_load_dwordx2 %0, %2, %3 sc0 sc1\n\ts_mov_b64 %1, exec\n\ts_mov_b64 exec, %7\n\tglobal_atomic_add %4, %5, %6\n\ts_mov_b64 exec, %1"
                   : "=&v"(gr), "=&s"(keep) : "v"((q & (NUNITS - 1)) * 8u), "s"(gbase), "v"(pair_offset(pend == NONE ? idle : pend, lane)), "v"(1u), "s"(cnt),
                     "s"(pair_mask(pend == NONE ? idle : pend)) : "memory");
      decode_unit(v, k, cur, scr, lane);
      __builtin_amdgcn_sched_barrier(0);
      asm volatile("s_waitcnt vmcnt(1)" : "+v"(gr) :: "memory");
      tag = __builtin_amdgcn_readfirstlane((u32)(gr >> 32));
      info = __builtin_amdgcn_readfirstlane((u32)gr);
      __builtin_amdgcn_sched_barrier(0);
      store_frags(v, rf, d, lane);
    }
    pend = d.cnt;
  }
  drain_vm();
  publish(pend);
}

__device__ __forceinline__ u32 frag_row(u32 r, u32 gf) {
  const u32 g4 = r & 4u, ux = gf >> 1;
  u32 src, su;
  if (ux >= EARLY_UNITS) {
    const u32 j = (ux - EARLY_UNITS) / AU;
    src = g4 | ((r + 1 + j) & 3u);
    su = ux - EARLY_UNITS - j * AU;
  } else if (ux < AU) {
    src = r;
    su = ux;
  } else {
    const u32 j = (ux - AU) / AWAY;
    src = (g4 ^ 4u) | j;
    su = ux - j * AWAY;
  }
  return ((ORDER >> (4u * src)) & 7u) * 1024u + su * 32u + (gf & 1u) * 16u;
}
__device__ __forceinline__ u32 foreign_frag0(u32 ft, u32 wid) { return ft < 4u ? (0x74208630u >> (4 * wid)) & 15u : (wid & 3u) * 2u; }
__device__ __forceinline__ u32 foreign_base(u32 ft) { return ft < 4u ? ft * 10u : ft * 8u + 8u; }

constexpr u32 O_HI = 7168;
__device__ __forceinline__ void out_puller(const Params& P, char* ws, u32 call, u32 j, LAS u32* ctl, LAS unsigned char* scr, u32 lane) {
  const u32 r = P.rank, src = ((r & 4u) ^ 4u) | j;
  char* const ob = pick(P.ws, src) + OFF_OB + (r & 3u) * (OUNITS * OSLOT);
  const rsrc_t ro = make_rsrc(pick(P.out, r), 8192u * 1024u * 2u);
  GAS u32* const tkt = (GAS u32*)(ws + OFF_TKT + (7u + j) * 128u);
  const u32 tbase = (call - 1u) * (OUNITS + (NCL + HELP) * 8u) * 64u;
  const u32 vo = (lane >> 4) * 4u * 2048u + (lane & 15u) * 16u;
  char* const queue = ws + OFF_OQ + j * 2048u;
  u32 k = ((u32)__builtin_amdgcn_readfirstlane(__hip_atomic_fetch_add(tkt, 1u, __ATOMIC_RELAXED, __HIP_MEMORY_SCOPE_AGENT)) - tbase) >> 6;
  u32 tag = 0, info = 0;
  while (k < OUNITS) {
    if (tag != call) {
      const u64 t0 = __builtin_amdgcn_s_memrealtime();
      for (u32 spins = 0;; ++spins) {
        const u64 gr = __hip_atomic_load((GAS u64*)(queue + k * 8u), __ATOMIC_RELAXED, __HIP_MEMORY_SCOPE_SYSTEM);
        tag = __builtin_amdgcn_readfirstlane((u32)(gr >> 32));
        info = __builtin_amdgcn_readfirstlane((u32)gr);
        if (tag == call || ((spins & 7u) == 7u && give_up(ws, call, t0))) break;
        __builtin_amdgcn_s_sleep(3);
      }
      if (tag != call) {
        *(GAS unsigned short*)(pick(P.out, r) + (size_t)((ORDER >> (4u * r)) & 7u) * 1024u * 2048u) = 0x7fc0;
        break;
      }
    }
    {
      const u32 my = lds_add(ctl, 1u);
      while ((u32)(__builtin_amdgcn_readfirstlane(__hip_atomic_load(ctl + 2, __ATOMIC_RELAXED, __HIP_MEMORY_SCOPE_WORKGROUP)) >> 6) + OUT_TOK <= my) __builtin_amdgcn_s_sleep(1);
    }
    asm volatile("" ::: "memory");
    const u32 uid = (info & 255u) < OUNITS ? (info & 255u) : 0u, sw = uid & 7u, mf = (uid < 32u && sw - 2u >= 4u) ? 3u : 2u;
    u32 total = (info >> 8) & 0x3fffu;
    total = total < 12288u ? total : 12288u;
    const bool raw = (info >> 22) & 1u, tri = info >> 31;
    const u32 t4 = ((info >> 23) & 255u) * 0x01010101u;
    const rsrc_t rs = make_rsrc(ob + uid * OSLOT, (total + 15u) & ~15u);
    u32 nt;
    u64 keep, gr;
    asm volatile("s_mov_b64 %1, exec\n\ts_mov_b64 exec, 1\n\tglobal_atomic_add %0, %2, %3, %4 sc0\n\ts_mov_b64 exec, %1"
                 : "=&v"(nt), "=&s"(keep) : "v"(0u), "v"(64u), "s"(tkt) : "memory");
    for (u32 p = 0; p < total; p += 1024u) __builtin_amdgcn_raw_ptr_buffer_load_lds(rs, scr + p, 16, lane * 16u, p, 0, AUX_SYS);
    asm volatile("s_waitcnt vmcnt(2)" ::: "memory");
    lds_add(ctl + 2, 1u);
    asm volatile("s_waitcnt vmcnt(0)" : "+v"(nt) :: "memory");
    const u32 kn = ((u32)__builtin_amdgcn_readfirstlane(nt) - tbase) >> 6;
    asm volatile("global_load_dwordx2 %0, %1, %2 sc0 sc1" : "=&v"(gr) : "v"((kn & 255u) * 8u), "s"(queue) : "memory");
    u32 mk[3] = {0, 0, 0}, a = 0;
    if (!raw) {
#pragma unroll
      for (u32 m = 0; m < 3; ++m) mk[m] = *(const LAS u32*)(scr + m * 256u + lane * 4u);
      mk[2] = mf == 3u ? mk[2] : 0u;
      const u32 c = ((__builtin_popcount(mk[0]) + __builtin_popcount(mk[1]) + __builtin_popcount(mk[2])) & ~3u) + 4u, incl = wave_scan(c);
      u32 n = __builtin_amdgcn_readlane(incl, 63);
      n = n < 4096u ? n : 4096u;
      LAS unsigned char* const lo = scr + mf * 256u;
      LAS unsigned char* const planes = lo + ((n + 15u) & ~15u);
      a = (u32)(size_t)lo + incl - c;
      if (tri) {
#pragma unroll
        for (int h = 0; h < 2; ++h) {
          const LAS u32* const pw = (const LAS u32*)(planes + lane * 24u + h * 12u);
          const u32 w0 = pw[0], w1 = pw[1], w2 = pw[2];
          *(LAS u32x4*)(lo + O_HI + lane * 64u + h * 32u) =
              (u32x4){w0 & 0x07070707u, (w0 >> 3) & 0x07070707u, ((w0 >> 6) & 0x03030303u) | ((w2 >> 4) & 0x04040404u), w1 & 0x07070707u};
          *(LAS u32x4*)(lo + O_HI + lane * 64u + h * 32u + 16u) =
              (u32x4){(w1 >> 3) & 0x07070707u, ((w1 >> 6) & 0x03030303u) | ((w2 >> 5) & 0x04040404u), w2 & 0x07070707u, (w2 >> 3) & 0x07070707u};
        }
      } else {
        const u32x4 nb[2] = {*(const LAS u32x4*)(planes + lane * 32u), *(const LAS u32x4*)(planes + lane * 32u + 16u)};
#pragma unroll
        for (int i = 0; i < 2; ++i) {
          *(LAS u32x4*)(lo + O_HI + lane * 64u + i * 16u) = nb[i] & 0x0f0f0f0fu;
          *(LAS u32x4*)(lo + O_HI + lane * 64u + 32u + i * 16u) = (nb[i] >> 4) & 0x0f0f0f0fu;
        }
      }
      asm volatile("" ::: "memory");
    }
    asm volatile("s_waitcnt vmcnt(0)" : "+v"(gr) :: "memory");
    tag = __builtin_amdgcn_readfirstlane((u32)(gr >> 32));
    info = __builtin_amdgcn_readfirstlane((u32)gr);
    const u32 col = j * 512u + (sw >> 2) * 256u, f0 = foreign_base(uid >> 3) + foreign_frag0(uid >> 3, sw);
    for (u32 m = 0; m < mf; ++m) {
      const u32 w = m == 0 ? mk[0] : (m == 1 ? mk[1] : mk[2]);
      u32x4 o[4];
      if (raw) {
#pragma unroll
        for (int e = 0; e < 4; ++e) o[e] = *(const LAS u32x4*)(scr + (m * 4u + e) * 1024u + lane * 16u);
      } else {
        u32 at[8], sel[8], l0[8], l1[8], h0[8], h1[8];
#pragma unroll
        for (int s = 0; s < 8; ++s) {
          const u32 f = (w >> s) & 0x01010101u;
          const LAS u32* const pw = (const LAS u32*)(size_t)(a & ~3u);
          at[s] = a;
          sel[s] = *(const LAS u32*)(size_t)__builtin_amdgcn_udot4(f, 0x20100804u, TAB + 128u, false);
          l0[s] = pw[0];
          l1[s] = pw[1];
          h0[s] = pw[O_HI / 4];
          h1[s] = pw[O_HI / 4 + 1];
          a = __builtin_amdgcn_sad_u8(f, 0u, a);
        }
        __builtin_amdgcn_sched_barrier(0);
#pragma unroll
        for (int s = 0; s < 8; ++s) {
          const u32 lb = __builtin_amdgcn_perm(0u, __builtin_amdgcn_alignbyte(l1[s], l0[s], at[s]), sel[s]);
          const u32 hb = __builtin_amdgcn_perm(0u, t4 - (__builtin_amdgcn_alignbyte(h1[s], h0[s], at[s]) & 0x0f0f0f0fu), sel[s]);
          o[s >> 1][(s & 1) * 2] = __builtin_amdgcn_perm(hb, lb, 0x05010400u);
          o[s >> 1][(s & 1) * 2 + 1] = __builtin_amdgcn_perm(hb, lb, 0x07030602u);
        }
      }
      const u32 row0 = frag_row(src, f0 + m);
#pragma unroll
      for (int e = 0; e < 4; ++e) store16(o[e], ro, vo, (row0 + e) * 2048u + col);
    }
    k = kn;
  }
}

__device__ __forceinline__ void gate(int c, int& ready, const char* cnt, u32 target, char* ws, u32 call) {
  if (c < ready) return;
  const u64 t0 = __builtin_amdgcn_s_memrealtime();
  for (u32 spins = 0;; ++spins) {
    const u64 m = __builtin_amdgcn_ballot_w64((int)(ld_agent(cnt) - target) >= 0);
    ready = ~m ? __builtin_ctzll(~m) : 64;
    if (c < ready) break;
    if ((spins & 7u) == 7u && give_up(ws, call, t0)) { ready = 64; break; }
    __builtin_amdgcn_s_sleep(3);
  }
  asm volatile("" ::: "memory");
}

__device__ __forceinline__ u32 relu_pk(float lo, float hi) {
  u32 r;
  asm("v_cvt_pk_bf16_f32 %0, %1, %2\n\tv_pk_max_i16 %0, %0, 0" : "=v"(r) : "v"(lo), "v"(hi));
  return r;
}

__device__ __forceinline__ void mfma_acc(f32x4& c, const bf16x8& a, const bf16x8& b) {
  asm volatile("v_mfma_f32_16x16x32_bf16 %0, %1, %2, %0" : "+v"(c) : "v"(a), "v"(b));
}
template <int N> __device__ __forceinline__ void wait_bar() { asm volatile("s_waitcnt vmcnt(%0)\n\ts_barrier" :: "i"(N) : "memory"); }

struct Tile { u32 rt, ct, gf0, nfr; };
template <int MF, int NP>
__device__ __forceinline__ void gemm_wave(u32x4 (&o)[MF][4], char* ws, u32 call, const Tile& T, u32 f0, LAS char* lds, u32 lane, u32 wid) {
  constexpr bool XL = NP && NP < 8;
  constexpr u32 STEP = XL ? A_STEP : B_STEP;
  constexpr int NM = MF * 8;
  const u32 rt = T.rt, ct = T.ct, wc = wid >> 2;
  const bool own = ct < 4u;
  const u32 p0 = (wid & 1u) * NP;
  const rsrc_t rs = XL ? make_rsrc(ws + OFF_XU, OFF_WU - OFF_XU) : make_rsrc(ws + OFF_WU + ct * B_TILE, B_TILE);
  const char* const cnt0 = ws + OFF_CNT + (XL ? (own ? ROW_OWN + rt : rt) : ROW_W + ct) * 256;
  const char* const cnt = cnt0 + lane * 4;
  const u32 target = call * (XL ? T.nfr >> 1 : 8u);
  u32 vp[NP ? NP : 1];
#pragma unroll
  for (u32 p = 0; p < NP; ++p) vp[p] = lane * 16u + (XL ? T.gf0 + (p0 + p < T.nfr ? p0 : T.nfr - 1u - p) : p0) * 1024u;
  LAS char* const ring = lds + (XL ? 0u : A_LDS) + p0 * 1024u;
  const u32 img = ((lane & 15u) * 4u + ((lane >> 4) ^ quad_swz((lane >> 2) & 3u))) * 16u;
  LAS char* const ra = lds + f0 * 1024u + img;
  LAS char* const rb = lds + A_LDS + wc * 8192u + lane * 16u;
#define AG_DMA(p, ks, off, imm) __builtin_amdgcn_raw_ptr_buffer_load_lds(rs, ring + (off), 16, vp[p < NP ? p : 0], (ks), imm, AUX_SYS)
#define AG_PIECE(p, ks, off) do { \
    if (p == 0) AG_DMA(0, (ks) * STEP, off, 0); if (p == 1) AG_DMA(1, (ks) * STEP, off, 1024); \
    if (p == 2) AG_DMA(2, (ks) * STEP, off, 2048); if (p == 3) AG_DMA(3, (ks) * STEP, off, 3072); \
    if (p == 4) AG_DMA(4, (ks) * STEP + 4096, (off) + 4096, 0); if (p == 5) AG_DMA(5, (ks) * STEP + 4096, (off) + 4096, 1024); \
    if (p == 6) AG_DMA(6, (ks) * STEP + 4096, (off) + 4096, 2048); if (p == 7) AG_DMA(7, (ks) * STEP + 4096, (off) + 4096, 3072); \
  } while (0)
#define AG_ALL(ks, off) do { _Pragma("unroll") for (int p = 0; p < NP; ++p) AG_PIECE(p, ks, off); } while (0)
#define AG_PIN(stmt) do { __builtin_amdgcn_sched_barrier(0); stmt; __builtin_amdgcn_sched_barrier(0); } while (0)
#define AG_STEP(cs, ks, DMA, RD) AG_STEPC(cs, ks, DMA, RD, 0, 0)
#define AG_STEPC(cs, ks, DMA, RD, NEARLY, slot0) do { \
    LAS char* const pa = ra + ord; \
    LAS char* const pb = rb + ord; \
    __builtin_amdgcn_sched_barrier(0); \
    _Pragma("unroll") for (int i = 0; i < NM; ++i) { \
      mfma_acc(acc[i % MF][i / MF], fa[cs][i % MF], fb[cs][i / MF]); \
      _Pragma("unroll") for (int j = 0; j < MF + 8; ++j) \
        if (RD && i == (NP ? (MF == 3 ? 2 * j : (3 * j) / 2) : j)) { \
          if (j < MF) AG_PIN(fa[(cs) ^ 1][j] = *(const LAS bf16x8*)(pa + j * 1024)); \
          else AG_PIN(fb[(cs) ^ 1][j - MF] = *(const LAS bf16x8*)(pb + (j - MF) * 1024)); \
        } \
      _Pragma("unroll") for (int p = 0; p < NP; ++p) \
        if (i == 1 + p * ((NM + 1) / (NP ? NP : 1))) { \
          if (DMA) AG_PIN(AG_PIECE(p, ks, owr)); \
          if (NEARLY) { \
            __builtin_amdgcn_sched_barrier(0); \
            if (early) { AG_PIECE(p, ks, (slot0) * SLOTB); if (NEARLY > 1) AG_PIECE(p, (ks) + 1, ((slot0) + 1) * SLOTB); } \
            __builtin_amdgcn_sched_barrier(0); \
          } \
        } \
    } \
    ord = ord + SLOTB == NSLOT * SLOTB ? 0 : ord + SLOTB; \
    owr = owr + SLOTB == NSLOT * SLOTB ? 0 : owr + SLOTB; \
    __builtin_amdgcn_sched_barrier(0); \
  } while (0)
#define AG_GATE(c) do { if (NP) gate(c, ready, cnt, target, ws, call); } while (0)
  f32x4 acc[MF][8];
#pragma unroll
  for (int m = 0; m < MF; ++m)
#pragma unroll
    for (int n = 0; n < 8; ++n) { acc[m][n] = (f32x4){0.f, 0.f, 0.f, 0.f}; asm volatile("" : "+v"(acc[m][n])); }
  __builtin_amdgcn_s_setprio(NP ? 1 : 0);
  int ready = 0;
  bool early = false;
  AG_GATE(0);
  constexpr int LA = 4, W = (LA - 2) * NP;
#pragma unroll
  for (int s = 0; s < 4; ++s) AG_ALL(s, s * SLOTB);
  wait_bar<W>();
  bf16x8 fa[2][MF], fb[2][8];
#pragma unroll
  for (int m = 0; m < MF; ++m) fa[0][m] = *(const LAS bf16x8*)(ra + m * 1024);
#pragma unroll
  for (int n = 0; n < 8; ++n) fb[0][n] = *(const LAS bf16x8*)(rb + n * 1024);
  u32 ord = SLOTB, owr = LA * SLOTB;
  for (int kc = 0; kc < NKC - 3; ++kc) {
    AG_GATE(kc + 1);
    AG_STEP(0, kc * 4 + LA, true, true); wait_bar<W>();
    AG_STEP(1, kc * 4 + LA + 1, true, true); wait_bar<W>();
    AG_STEP(0, kc * 4 + LA + 2, true, true); wait_bar<W>();
    AG_STEP(1, kc * 4 + LA + 3, true, true); wait_bar<W>();
  }
  AG_GATE(NKC - 2);
  AG_STEP(0, NKS - 12 + LA, true, true); wait_bar<W>();
  AG_STEP(1, NKS - 11 + LA, true, true); wait_bar<W>();
  AG_STEP(0, NKS - 10 + LA, true, true); wait_bar<W>();
  AG_STEP(1, NKS - 9 + LA, true, true); wait_bar<2 * NP>();
  AG_STEP(0, 0, false, true); wait_bar<NP>();
  u32 last = 0;
  if (NP) asm volatile("global_load_dword %0, %1, %2 sc1" : "=&v"(last) : "v"(0u), "s"(cnt0 + (NKC - 1) * 4) : "memory");
  AG_STEP(1, 0, false, true);
  asm volatile("s_waitcnt vmcnt(0)\n\ts_barrier" : "+v"(last) :: "memory");
  early = NP && (ready >= NKC || (int)((u32)__builtin_amdgcn_readfirstlane(last) - target) >= 0);
  AG_STEPC(0, NKS - 4, false, true, 2, 2);
  AG_STEPC(1, NKS - 2, false, false, 1, 4);
  asm volatile("s_barrier" ::: "memory");
  if (!early) {
    AG_GATE(NKC - 1);
#pragma unroll
    for (int s = 0; s < 3; ++s) AG_ALL(NKS - 4 + s, (2 + s) * SLOTB);
  }
  AG_ALL(NKS - 1, 0);
  wait_bar<2 * NP>();
  u32 tmo;
  asm volatile("global_load_dword %0, %1, %2 sc1" : "=&v"(tmo) : "v"(0u), "s"(ws + OFF_TMO) : "memory");
#pragma unroll
  for (int m = 0; m < MF; ++m) fa[0][m] = *(const LAS bf16x8*)(ra + 2 * SLOTB + m * 1024);
#pragma unroll
  for (int n = 0; n < 8; ++n) fb[0][n] = *(const LAS bf16x8*)(rb + 2 * SLOTB + n * 1024);
  ord = 3 * SLOTB;
  AG_STEP(0, 0, false, true); wait_bar<NP + 1>();
  AG_STEP(1, 0, false, true); wait_bar<0>();
  AG_STEP(0, 0, false, true);
  AG_STEP(1, 0, false, false);
  asm volatile("" : "+v"(tmo));
  asm volatile("s_nop 15" ::: "memory");
  __builtin_amdgcn_sched_barrier(0);
#undef AG_PIECE
#undef AG_ALL
#undef AG_PIN
#undef AG_DMA
#undef AG_GATE
#undef AG_STEP
#undef AG_STEPC
  const bool bad = (u32)__builtin_amdgcn_readfirstlane(tmo) == call;
#pragma unroll
  for (int m = 0; m < MF; ++m)
#pragma unroll
    for (int e = 0; e < 4; ++e)
#pragma unroll
      for (int n = 0; n < 4; ++n) o[m][e][n] = relu_pk(acc[m][2 * n][e], acc[m][2 * n + 1][e]);
  if (bad) {
#pragma unroll
    for (int m = 0; m < MF; ++m)
#pragma unroll
      for (int e = 0; e < 4; ++e) o[m][e] = (u32x4){0x7fc07fc0u, 0x7fc07fc0u, 0x7fc07fc0u, 0x7fc07fc0u};
    asm volatile("" ::: "memory");
  }
}

__device__ __forceinline__ void store_tile(const u32x4 (&o)[3][4], const Params& P, const Tile& T, u32 f0, u32 lane, u32 wid) {
  const rsrc_t ro = make_rsrc(pick(P.out, P.rank), 8192u * 1024u * 2u);
  const u32 vo = (lane >> 4) * 4u * 2048u + (lane & 15u) * 16u;
#pragma unroll
  for (int m = 0; m < 3; ++m) {
    if (f0 + m >= T.nfr) continue;
    const u32 row0 = frag_row(P.rank, T.gf0 + f0 + m);
#pragma unroll
    for (int e = 0; e < 4; ++e) store16(o[m][e], ro, vo, (row0 + e) * 2048u + T.ct * 512u + (wid >> 2) * 256u);
  }
}

template <int MF>
__device__ __forceinline__ void send_tile(const u32x4 (&o)[MF][4], const Params& P, char* ws, u32 call, const Tile& T, LAS char* lds, u32 lane, u32 wid) {
  const u32 r = P.rank, g4 = r & 4u, rt = T.rt, ct = T.ct;
  const u32 j = ct - 4u, uid = rt * 8u + wid;
  u32 qi;
  u64 keep;
  asm volatile("s_mov_b64 %1, exec\n\ts_mov_b64 exec, 1\n\tglobal_atomic_add %0, %2, %3, %4 sc0\n\ts_mov_b64 exec, %1"
               : "=&v"(qi), "=&s"(keep) : "v"(0u), "v"(1u), "s"(ws + OFF_TKT + (11u + j) * 128u) : "memory");
  u32 mk[MF], fl[MF * 8], lb[MF * 8], hb[MF * 8], c = 0;
  u16x2 big[2] = {{0, 0}, {0, 0}};
#pragma unroll
  for (int m = 0; m < MF; ++m) {
    mk[m] = 0;
#pragma unroll
    for (int s = 0; s < 8; ++s) {
      const u32 d0 = o[m][s >> 1][(s & 1) * 2], d1 = o[m][s >> 1][(s & 1) * 2 + 1];
      u32 t0, t1;
      asm("v_pk_min_u16 %0, %1, %2" : "=v"(t0) : "v"(d0), "v"(0x00010001u));
      asm("v_pk_min_u16 %0, %1, %2" : "=v"(t1) : "v"(d1), "v"(0x00010001u));
      fl[m * 8 + s] = __builtin_amdgcn_perm(t1, t0, 0x06040200u);
      mk[m] |= fl[m * 8 + s] << s;
      big[0] = __builtin_elementwise_max(big[0], __builtin_bit_cast(u16x2, d0));
      big[1] = __builtin_elementwise_max(big[1], __builtin_bit_cast(u16x2, d1));
      lb[m * 8 + s] = __builtin_amdgcn_perm(d1, d0, 0x06040200u);
      hb[m * 8 + s] = __builtin_amdgcn_perm(d1, d0, 0x07050301u);
    }
    c += __builtin_popcount(mk[m]);
  }
  c = (c & ~3u) + 4u;
  const u32 incl = wave_scan(c), n = __builtin_amdgcn_readlane(incl, 63);
  big[0] = __builtin_elementwise_max(big[0], big[1]);
  u32 top = wave_max((u32)(big[0].x > big[0].y ? big[0].x : big[0].y) >> 8);
  top = top < 15u ? 15u : top;
  bool raw = n > 4096u, tri = false;
  u32 total = 0;
  const rsrc_t rb2 = make_rsrc(ws + OFF_OB + j * (OUNITS * OSLOT) + uid * OSLOT, OSLOT);
  LAS unsigned char* const scr = (LAS unsigned char*)lds + (wid < 7u ? SLOTB + wid * 12160u : NSLOT * SLOTB);
  if (!raw) {
    LAS unsigned char* const lo = scr + MF * 256u;
  #pragma unroll
    for (int m = 0; m < MF; ++m) *(LAS u32*)(scr + m * 256u + lane * 4u) = mk[m];
    asm volatile("" ::: "memory");
    *(LAS u32*)(lo + O_HI + n + (lane & 15u) * 4u) = 0u;
    const u32 t4 = top * 0x01010101u;
    u32 wide = 0;
    u32 a = (u32)(size_t)lo + incl - c, cl = 0, ch = 0;
#pragma unroll
    for (int m = 0; m < MF; ++m) {
      u32 sel[8];
#pragma unroll
      for (int s = 0; s < 8; ++s) sel[s] = *(const LAS u32*)(size_t)__builtin_amdgcn_udot4(fl[m * 8 + s], 0x20100804u, TAB, false);
      __builtin_amdgcn_sched_barrier(0);
#pragma unroll
      for (int s = 0; s < 8; ++s) {
        const u32 sh = (a & 3u) * 8u, an = __builtin_amdgcn_sad_u8(fl[m * 8 + s], 0u, a);
        const u32 code = __builtin_amdgcn_perm(0u, t4 - hb[m * 8 + s], sel[s]);
        const u64 tl = (u64)__builtin_amdgcn_perm(0u, lb[m * 8 + s], sel[s]) << sh, th = (u64)code << sh;
        wide |= code;
        const u32 wl = cl | (u32)tl, wh = ch | (u32)th;
        LAS u32* const pw = (LAS u32*)(size_t)(a & ~3u);
        pw[0] = wl;
        pw[O_HI / 4] = wh;
        const bool cross = (an ^ a) & 4u;
        cl = cross ? (u32)(tl >> 32) : wl;
        ch = cross ? (u32)(th >> 32) : wh;
        a = an;
      }
    }
    ((LAS u32*)(size_t)(a & ~3u))[0] = cl;
    ((LAS u32*)(size_t)(a & ~3u))[O_HI / 4] = ch;
    asm volatile("" ::: "memory");
    LAS unsigned char* const planes = lo + ((n + 15u) & ~15u);
    u32x4 hq[4];
#pragma unroll
    for (int i = 0; i < 4; ++i) hq[i] = *(const LAS u32x4*)(lo + O_HI + lane * 64u + i * 16u);
    raw = __builtin_amdgcn_ballot_w64((wide & 0xf0f0f0f0u) != 0) != 0;
    tri = __builtin_amdgcn_ballot_w64((wide & 0xf8f8f8f8u) != 0) == 0;
    if (tri) {
#pragma unroll
      for (int h = 0; h < 2; ++h) {
        const u32x4 x = hq[2 * h], y = hq[2 * h + 1];
        LAS u32* const pw = (LAS u32*)(planes + lane * 24u + h * 12u);
        pw[0] = x[0] | (x[1] << 3) | ((x[2] & 0x03030303u) << 6);
        pw[1] = x[3] | (y[0] << 3) | ((y[1] & 0x03030303u) << 6);
        pw[2] = y[2] | (y[3] << 3) | ((x[2] & 0x04040404u) << 4) | ((y[1] & 0x04040404u) << 5);
      }
    } else {
#pragma unroll
      for (int i = 0; i < 2; ++i) *(LAS u32x4*)(planes + lane * 32u + i * 16u) = hq[i] | (hq[2 + i] << 4);
    }
    total = MF * 256u + ((n + 15u) & ~15u) + ((n + 63u) >> 6) * (tri ? 24u : 32u);
    asm volatile("" ::: "memory");
  }
  if (raw) {
    total = 4096u * MF;
#pragma unroll
    for (int m = 0; m < MF; ++m)
#pragma unroll
      for (int i = 0; i < 4; ++i)
        store16((u32x4){__builtin_amdgcn_perm(hb[m * 8 + 2 * i], lb[m * 8 + 2 * i], 0x05010400u), __builtin_amdgcn_perm(hb[m * 8 + 2 * i], lb[m * 8 + 2 * i], 0x07030602u),
                        __builtin_amdgcn_perm(hb[m * 8 + 2 * i + 1], lb[m * 8 + 2 * i + 1], 0x05010400u), __builtin_amdgcn_perm(hb[m * 8 + 2 * i + 1], lb[m * 8 + 2 * i + 1], 0x07030602u)},
                rb2, lane * 16u, (m * 4 + i) * 1024u);
  } else {
    for (u32 p = 0; p < total; p += 1024u) store16(*(const LAS u32x4*)(scr + p + lane * 16u), rb2, p + lane * 16u < total ? lane * 16u : 0x80000000u, p);
  }
  asm volatile("s_waitcnt vmcnt(0)" : "+v"(qi) :: "memory");
  const u32 slot = ((u32)__builtin_amdgcn_readfirstlane(qi) - (call - 1u) * OUNITS) & 255u;
  flag_store(pick(P.ws, (g4 ^ 4u) | j) + OFF_OQ + (r & 3u) * 2048u + slot * 8u, ((u64)call << 32) | uid | (total << 8) | (raw ? 1u << 22 : 0u) | (top << 23) | (tri ? 1u << 31 : 0u), 1u);
}

__global__ __launch_bounds__(512, 2) void ag_gemm_kernel(Params P) {
  extern __shared__ __attribute__((aligned(16))) char smem[];
  LAS char* lds = (LAS char*)smem;
  const u32 lane = threadIdx.x & 63u, wid = __builtin_amdgcn_readfirstlane(threadIdx.x >> 6);
  char* const ws = pick(P.ws, P.rank);
  u32 bid = blockIdx.x;
  asm volatile("" : "+s"(bid));
  char* const cptr = ws + OFF_CALL + bid * 128u + wid * 4u;
  const u32 callv = ld_agent(cptr);
  LAS unsigned char* scr = (LAS unsigned char*)lds + wid * WAVE_SCRATCH;
  build_tables(lds, lane);
  const u32 li = bid < 7u * NCL ? bid % 7u : 7u, lidx = bid / 7u, pkw = bid - 7u * NCL;
  const u32 xcd = bid & 7u, pos = xcd * 26u + (xcd > 4u ? xcd - 4u : 0u) + (bid >> 3) - (xcd < 4u ? 6u : 5u);
  const bool foreign = pos >= 4u * NOT;
  const u32 ent = foreign ? pos - 4u * NOT : pos;
  Tile T;
  T.rt = ent >> 2;
  T.ct = (ent & 3u) + (foreign ? 4u : 0u);
  T.gf0 = foreign ? foreign_base(T.rt) : T.rt * 12u;
  T.nfr = foreign ? (T.rt < 4u ? 10u : 8u) : (T.rt == NOT - 1u ? 4u : 12u);
  LAS u32* const ctl = (LAS u32*)(lds + 8 * WAVE_SCRATCH);
  if (bid < NCOMM) {
    ctl[lane & 3u] = 0;
    asm volatile("s_waitcnt lgkmcnt(0)\n\ts_barrier" ::: "memory");
  }
  const bool isg = bid >= NCOMM;
  u32 call;
  if (li < 7u) {
    call = __builtin_amdgcn_readfirstlane(callv) + 1u;
    puller(P, ws, call, li, lidx, ctl, scr, lane, wid);
  } else {
    const u32 q0 = isg ? wid * NGEMM + (bid - NCOMM) : GPACK + wid * NPKWG + pkw;
    const u32 q1 = isg ? (T.nfr == 8u ? 8u * NGEMM + wid * 108u + ent - 16u : GPACK) : q0 + NPK;
    for (u32 i = isg ? wid : 0u; i; --i) __builtin_amdgcn_s_sleep(28);
    call = packer(P, ws, callv, q0, q1, isg ? 864u : (u32)NPK, isg ? GPACK : (u32)NUNITS, scr, lane);
  }
  __hip_atomic_store((GAS u32*)cptr, call, __ATOMIC_RELAXED, __HIP_MEMORY_SCOPE_AGENT);
  if (!isg) {
    const u32 hi = li < 3u ? li * NCL + lidx : 3u * NCL + pkw;
    if (li - 3u < 4u || hi < 4u * HELP) out_puller(P, ws, call, li - 3u < 4u ? li - 3u : hi & 3u, ctl, scr, lane);
    return;
  }
  asm volatile("s_waitcnt lgkmcnt(0)\n\ts_barrier" ::: "memory");
  if (T.nfr == 8u || (T.nfr == 10u && wid - 2u < 4u)) {
    u32x4 o[2][4];
    const u32 f0 = foreign_frag0(T.rt, wid);
    if (T.nfr == 10u || wid >= 4u) gemm_wave<2, 0>(o, ws, call, T, f0, lds, lane, wid);
    else if (wid < 2u) gemm_wave<2, 4>(o, ws, call, T, f0, lds, lane, wid);
    else gemm_wave<2, 8>(o, ws, call, T, f0, lds, lane, wid);
    send_tile<2>(o, P, ws, call, T, lds, lane, wid);
  } else {
    u32x4 o[3][4];
    const u32 f0 = foreign ? foreign_frag0(0u, wid) : (wid & 3u) * 3u;
    if (foreign ? wid >= 6u : wid - 2u < 2u) gemm_wave<3, 8>(o, ws, call, T, f0, lds, lane, wid);
    else if (foreign) gemm_wave<3, 5>(o, ws, call, T, f0, lds, lane, wid);
    else if (wid < 2u) gemm_wave<3, 6>(o, ws, call, T, f0, lds, lane, wid);
    else gemm_wave<3, 0>(o, ws, call, T, f0, lds, lane, wid);
    if (foreign) send_tile<3>(o, P, ws, call, T, lds, lane, wid);
    else store_tile(o, P, T, f0, lane, wid);
  }
}

}

extern "C" void kernel_launch(int rank, int nranks, ncclComm_t comm, hipStream_t stream,
                              void* const* d_in, const int* in_sizes, int n_in,
                              void* d_out, int out_size, const int* in_dtypes, int out_dtype,
                              void* const* d_out_all, void* const* d_ws_all, size_t ws_bytes) {
  (void)comm; (void)in_dtypes; (void)out_dtype; (void)d_out;
  if (nranks != ag::NR || n_in != 2 || (unsigned)rank >= (unsigned)ag::NR || in_sizes[0] != 1024 * 8192 || in_sizes[1] != 8192 * 1024 ||
      out_size != 8192 * 1024 || ws_bytes < ag::WS_NEED) {
    fprintf(stderr, "ag_gemm: unexpected problem shape\n");
    return;
  }
  ag::Params p;
  p.x = (const char*)d_in[0];
  p.w = (const char*)d_in[1];
  for (int i = 0; i < ag::NR; ++i) {
    const int phys = (ag::ORDER >> (4 * i)) & 7;
    p.ws[i] = (char*)d_ws_all[phys];
    p.out[i] = (char*)d_out_all[phys];
    if (phys == rank) p.rank = i;
  }
  p.pad = 0;
  hipError_t e = hipFuncSetAttribute((const void*)ag::ag_gemm_kernel, hipFuncAttributeMaxDynamicSharedMemorySize, ag::LDS_BYTES);
  if (e != hipSuccess) { fprintf(stderr, "ag_gemm: %s\n", hipGetErrorString(e)); return; }
  hipLaunchKernelGGL(ag::ag_gemm_kernel, dim3(ag::NWG), dim3(512), ag::LDS_BYTES, stream, p);
  e = hipGetLastError();
  if (e != hipSuccess) fprintf(stderr, "ag_gemm launch: %s\n", hipGetErrorString(e));
}
```
